```python
import numpy as np

kernel_name = "gpu_a2a_gemm_m4096_k4096_n32768_bf16_gelu"


ABSMAX_RTOL = 0.008

COMPUTE_DTYPE = "bfloat16"
EPILOGUE = "gelu"
IN_QUANT = None
OUT_QUANT = None
IN_DTYPES = {"x": "bfloat16", "w_mat": "bfloat16"}
WORLD_SIZE = 8

def _to_bf16(x):
    u = np.ascontiguousarray(x, dtype=np.float32).view(np.uint32)
    return ((u + 0x7FFF + ((u >> 16) & 1)) & 0xFFFF0000).view(np.float32)

def setup_inputs():

    x = _to_bf16(np.random.randn(4096, 4096).astype(np.float32))
    w_mat = _to_bf16((np.random.randn(4096, 32768) / np.sqrt(4096)).astype(np.float32))
    return {"x": x, "w_mat": w_mat}

def _epilogue(y):
    c = 0.7978845608028654
    return (0.5 * y * (1.0 + np.tanh(c * (y + 0.044715 * y ** 3)))).astype(np.float32)

def reference(x, w_mat):
    y = (x @ w_mat).astype(np.float32)
    return _epilogue(y)
```

```cpp
#include <hip/hip_runtime.h>
#include <rccl/rccl.h>
#include <cstdio>

#define HIP_CHECK(expr) do { hipError_t _e = (expr); if (_e != hipSuccess) { \
    fprintf(stderr, "HIP error %s at %s:%d\n", hipGetErrorString(_e), __FILE__, __LINE__); \
    return; } } while (0)

#define GEMM_ASM_TEXT \
  "s_load_dwordx16 s[72:87], %[karg], 0x0\n\t" \
  "s_load_dwordx8 s[88:95], %[karg], 0x40\n\t" \
  "s_load_dword s58, %[karg], 0x60\n\t" \
  "s_mov_b32 s66, %[wave]\n\t" \
  "s_bfe_u32 s54, %[bid], 0x10003\n\t" \
  "s_and_b32 s55, %[bid], 7\n\t" \
  "s_lshl_b32 s55, s55, 4\n\t" \
  "s_lshr_b32 s56, %[bid], 4\n\t" \
  "s_add_u32 s55, s55, s56\n\t" \
  "s_lshl_b32 s55, s55, 9\n\t" \
  "s_lshl_b32 s29, s54, 18\n\t" \
  "s_lshl_b32 s54, s54, 24\n\t" \
  "s_add_u32 s57, s54, s55\n\t" \
  "s_mov_b32 s38, 0x10000000\n\t" \
  "s_mov_b32 s39, 0x00020000\n\t" \
  "s_mov_b32 s42, 0x00400000\n\t" \
  "s_mov_b32 s43, 0x00020000\n\t" \
  "s_lshl_b32 s62, s66, 12\n\t" \
  "s_add_u32 s62, s62, 65536\n\t" \
  "s_mov_b32 s49, s62\n\t" \
  "s_add_u32 s33, s49, 0x4000\n\t" \
  "s_lshl_b32 s48, s66, 13\n\t" \
  "s_add_u32 s48, s48, 0\n\t" \
  "s_add_u32 s32, s48, 0x1000\n\t" \
  "s_mov_b32 s46, 0\n\t" \
  "v_add_u32 v34, 0, %[voffB]\n\t" \
  "v_add_u32 v35, 261120, %[voffB]\n\t" \
  "v_add_u32 v36, 1046528, %[voffB]\n\t" \
  "v_add_u32 v37, 1307648, %[voffB]\n\t" \
  "v_add_u32 v38, 2097152, %[voffB]\n\t" \
  "v_add_u32 v39, 2358272, %[voffB]\n\t" \
  "v_add_u32 v40, 3143680, %[voffB]\n\t" \
  "v_add_u32 v41, 3404800, %[voffB]\n\t" \
  "v_add_u32 v26, 0, %[voffA]\n\t" \
  "v_add_u32 v27, 31744, %[voffA]\n\t" \
  "v_add_u32 v28, 63488, %[voffA]\n\t" \
  "v_add_u32 v29, 95232, %[voffA]\n\t" \
  "v_add_u32 v30, 131072, %[voffA]\n\t" \
  "v_add_u32 v31, 162816, %[voffA]\n\t" \
  "v_add_u32 v32, 194560, %[voffA]\n\t" \
  "v_add_u32 v33, 226304, %[voffA]\n\t" \
  "v_mov_b32 v51, 0\n\t" \
  "s_lshl_b32 s56, %[bid], 6\n\t" \
  "s_waitcnt lgkmcnt(0)\n\t" \
  "s_add_u32 s36, s74, s55\n\t" \
  "s_addc_u32 s37, s75, 0\n\t" \
  "s_lshl_b32 s47, s58, 25\n\t" \
  "s_add_u32 s34, s78, s56\n\t" \
  "s_addc_u32 s35, s79, 0\n\t" \
  "global_load_dword v55, v51, s[34:35] sc1\n\t" \
  "s_mov_b32 m0, s49\n\t" \
  "s_nop 0\n\t" \
  "buffer_load_dwordx4 v34, s[36:39], s47 offen offset:0 lds\n\t" \
  "buffer_load_dwordx4 v35, s[36:39], s47 offen offset:1024 lds\n\t" \
  "buffer_load_dwordx4 v36, s[36:39], s47 offen offset:2048 lds\n\t" \
  "buffer_load_dwordx4 v37, s[36:39], s47 offen offset:3072 lds\n\t" \
  "s_mov_b32 m0, s33\n\t" \
  "s_nop 0\n\t" \
  "buffer_load_dwordx4 v38, s[36:39], s47 offen offset:0 lds\n\t" \
  "buffer_load_dwordx4 v39, s[36:39], s47 offen offset:1024 lds\n\t" \
  "buffer_load_dwordx4 v40, s[36:39], s47 offen offset:2048 lds\n\t" \
  "buffer_load_dwordx4 v41, s[36:39], s47 offen offset:3072 lds\n\t" \
  "s_lshl_b32 s54, s58, 19\n\t" \
  "s_add_u32 s54, s54, s29\n\t" \
  "s_add_u32 s40, s72, s54\n\t" \
  "s_addc_u32 s41, s73, 0\n\t" \
  "s_mov_b32 m0, s48\n\t" \
  "s_nop 0\n\t" \
  "buffer_load_dwordx4 v26, s[40:43], s46 offen offset:0 sc0 sc1 lds\n\t" \
  "buffer_load_dwordx4 v27, s[40:43], s46 offen offset:1024 sc0 sc1 lds\n\t" \
  "buffer_load_dwordx4 v28, s[40:43], s46 offen offset:2048 sc0 sc1 lds\n\t" \
  "buffer_load_dwordx4 v29, s[40:43], s46 offen offset:3072 sc0 sc1 lds\n\t" \
  "s_mov_b32 m0, s32\n\t" \
  "s_nop 0\n\t" \
  "buffer_load_dwordx4 v30, s[40:43], s46 offen offset:0 sc0 sc1 lds\n\t" \
  "buffer_load_dwordx4 v31, s[40:43], s46 offen offset:1024 sc0 sc1 lds\n\t" \
  "buffer_load_dwordx4 v32, s[40:43], s46 offen offset:2048 sc0 sc1 lds\n\t" \
  "buffer_load_dwordx4 v33, s[40:43], s46 offen offset:3072 sc0 sc1 lds\n\t" \
  "s_add_u32 s30, s76, s57\n\t" \
  "s_addc_u32 s31, s77, 0\n\t" \
  "s_mov_b32 s71, 0x8000\n\t" \
  "s_mov_b32 s61, 0\n\t" \
  "s_mov_b32 s52, s58\n\t" \
  "s_mov_b32 s53, 0\n\t" \
  "s_lshl_b32 s24, 1, s58\n\t" \
  "s_andn2_b32 s25, 0xff, s24\n\t" \
  "s_mov_b32 s21, 0\n\t" \
  "s_mov_b32 s19, 7\n\t" \
  "s_mov_b32 s18, 0\n\t" \
  "s_mov_b32 s16, 2\n\t" \
  "s_mov_b32 s12, 0\n\t" \
  "s_cmp_lt_u32 %[bid], 1\n\t" \
  "s_cselect_b32 s17, -1, 0\n\t" \
  "s_mov_b64 s[26:27], s[78:79]\n\t" \
  "v_writelane_b32 v10, s80, 0\n\t" \
  "v_writelane_b32 v11, s81, 0\n\t" \
  "v_writelane_b32 v10, s82, 1\n\t" \
  "v_writelane_b32 v11, s83, 1\n\t" \
  "v_writelane_b32 v10, s84, 2\n\t" \
  "v_writelane_b32 v11, s85, 2\n\t" \
  "v_writelane_b32 v10, s86, 3\n\t" \
  "v_writelane_b32 v11, s87, 3\n\t" \
  "v_writelane_b32 v10, s88, 4\n\t" \
  "v_writelane_b32 v11, s89, 4\n\t" \
  "v_writelane_b32 v10, s90, 5\n\t" \
  "v_writelane_b32 v11, s91, 5\n\t" \
  "v_writelane_b32 v10, s92, 6\n\t" \
  "v_writelane_b32 v11, s93, 6\n\t" \
  "v_writelane_b32 v10, s94, 7\n\t" \
  "v_writelane_b32 v11, s95, 7\n\t" \
  "s_mov_b64 s[90:91], s[26:27]\n\t" \
  "s_mov_b64 s[92:93], -1\n\t" \
  "v_mbcnt_lo_u32_b32 v54, -1, 0\n\t" \
  "v_mbcnt_hi_u32_b32 v54, -1, v54\n\t" \
  "v_and_b32 v50, 7, v54\n\t" \
  "v_lshlrev_b32 v50, 4, v50\n\t" \
  "v_and_b32 v52, 8, v54\n\t" \
  "v_lshlrev_b32 v52, 7, v52\n\t" \
  "v_add_u32 v50, v50, v52\n\t" \
  "s_lshl_b32 s54, s66, 11\n\t" \
  "v_add_u32 v50, s54, v50\n\t" \
  "v_lshlrev_b32 v54, 3, v54\n\t" \
  "s_lshl_b32 s54, s66, 9\n\t" \
  "v_add_u32 v54, s54, v54\n\t" \
  "s_mov_b32 s67, %[bid]\n\t" \
  "s_mov_b32 s68, 0\n\t" \
  "s_mov_b32 s69, 0\n\t" \
  "s_mov_b32 s70, 0\n\t" \
  "s_mul_i32 s54, s67, 9363\n\t" \
  "s_lshr_b32 s54, s54, 16\n\t" \
  "s_mul_i32 s55, s54, 7\n\t" \
  "s_sub_u32 s55, s67, s55\n\t" \
  "s_add_u32 s55, s55, s58\n\t" \
  "s_add_u32 s55, s55, 1\n\t" \
  "s_and_b32 s55, s55, 7\n\t" \
  "v_readlane_b32 s98, v10, s55\n\t" \
  "v_readlane_b32 s99, v11, s55\n\t" \
  "s_lshl_b32 s56, s54, 13\n\t" \
  "s_lshl_b32 s57, s68, 7\n\t" \
  "s_add_u32 s56, s56, s57\n\t" \
  "s_lshl_b32 s55, s55, 19\n\t" \
  "s_add_u32 s55, s55, s56\n\t" \
  "s_add_u32 s74, s72, s55\n\t" \
  "s_addc_u32 s75, s73, 0\n\t" \
  "s_lshl_b32 s57, s68, 11\n\t" \
  "s_lshl_b32 s54, s54, 2\n\t" \
  "s_add_u32 s57, s57, s54\n\t" \
  "s_add_u32 s57, s57, s70\n\t" \
  "s_add_u32 s82, s98, s57\n\t" \
  "s_addc_u32 s83, s99, 0\n\t" \
  "s_add_u32 s56, s56, s69\n\t" \
  "s_add_u32 s98, s98, s56\n\t" \
  "s_addc_u32 s99, s99, 0\n\t" \
  "s_add_u32 s54, s67, 256\n\t" \
  "s_cmp_ge_u32 s54, 448\n\t" \
  "s_cselect_b32 s55, 448, 0\n\t" \
  "s_cselect_b32 s56, 1, 0\n\t" \
  "s_sub_u32 s54, s54, s55\n\t" \
  "s_add_u32 s56, s68, s56\n\t" \
  "s_cmp_lt_u32 s56, 8\n\t" \
  "s_cselect_b32 s67, s54, s67\n\t" \
  "s_cselect_b32 s68, s56, s68\n\t" \
  "s_mov_b64 exec, 0xffff\n\t" \
  "global_load_dwordx4 v[42:45], v50, s[74:75]\n\t" \
  "s_mov_b64 exec, -1\n\t" \
  "s_mul_i32 s54, s67, 9363\n\t" \
  "s_lshr_b32 s54, s54, 16\n\t" \
  "s_mul_i32 s55, s54, 7\n\t" \
  "s_sub_u32 s55, s67, s55\n\t" \
  "s_add_u32 s55, s55, s58\n\t" \
  "s_add_u32 s55, s55, 1\n\t" \
  "s_and_b32 s55, s55, 7\n\t" \
  "v_readlane_b32 s100, v10, s55\n\t" \
  "v_readlane_b32 s101, v11, s55\n\t" \
  "s_lshl_b32 s56, s54, 13\n\t" \
  "s_lshl_b32 s57, s68, 7\n\t" \
  "s_add_u32 s56, s56, s57\n\t" \
  "s_lshl_b32 s55, s55, 19\n\t" \
  "s_add_u32 s55, s55, s56\n\t" \
  "s_add_u32 s74, s72, s55\n\t" \
  "s_addc_u32 s75, s73, 0\n\t" \
  "s_lshl_b32 s57, s68, 11\n\t" \
  "s_lshl_b32 s54, s54, 2\n\t" \
  "s_add_u32 s57, s57, s54\n\t" \
  "s_add_u32 s57, s57, s70\n\t" \
  "s_add_u32 s84, s100, s57\n\t" \
  "s_addc_u32 s85, s101, 0\n\t" \
  "s_add_u32 s56, s56, s69\n\t" \
  "s_add_u32 s100, s100, s56\n\t" \
  "s_addc_u32 s101, s101, 0\n\t" \
  "s_add_u32 s54, s67, 256\n\t" \
  "s_cmp_ge_u32 s54, 448\n\t" \
  "s_cselect_b32 s55, 448, 0\n\t" \
  "s_cselect_b32 s56, 1, 0\n\t" \
  "s_sub_u32 s54, s54, s55\n\t" \
  "s_add_u32 s56, s68, s56\n\t" \
  "s_cmp_lt_u32 s56, 8\n\t" \
  "s_cselect_b32 s67, s54, s67\n\t" \
  "s_cselect_b32 s68, s56, s68\n\t" \
  "s_mov_b64 exec, 0xffff\n\t" \
  "global_load_dwordx4 v[46:49], v50, s[74:75]\n\t" \
  "s_mov_b64 exec, -1\n\t" \
  "s_mul_i32 s54, s67, 9363\n\t" \
  "s_lshr_b32 s54, s54, 16\n\t" \
  "s_mul_i32 s55, s54, 7\n\t" \
  "s_sub_u32 s55, s67, s55\n\t" \
  "s_add_u32 s55, s55, s58\n\t" \
  "s_add_u32 s55, s55, 1\n\t" \
  "s_and_b32 s55, s55, 7\n\t" \
  "v_readlane_b32 s76, v10, s55\n\t" \
  "v_readlane_b32 s77, v11, s55\n\t" \
  "s_lshl_b32 s56, s54, 13\n\t" \
  "s_lshl_b32 s57, s68, 7\n\t" \
  "s_add_u32 s56, s56, s57\n\t" \
  "s_lshl_b32 s55, s55, 19\n\t" \
  "s_add_u32 s55, s55, s56\n\t" \
  "s_add_u32 s74, s72, s55\n\t" \
  "s_addc_u32 s75, s73, 0\n\t" \
  "s_lshl_b32 s57, s68, 11\n\t" \
  "s_lshl_b32 s54, s54, 2\n\t" \
  "s_add_u32 s57, s57, s54\n\t" \
  "s_add_u32 s57, s57, s70\n\t" \
  "s_add_u32 s86, s76, s57\n\t" \
  "s_addc_u32 s87, s77, 0\n\t" \
  "s_add_u32 s56, s56, s69\n\t" \
  "s_add_u32 s76, s76, s56\n\t" \
  "s_addc_u32 s77, s77, 0\n\t" \
  "s_add_u32 s54, s67, 256\n\t" \
  "s_cmp_ge_u32 s54, 448\n\t" \
  "s_cselect_b32 s55, 448, 0\n\t" \
  "s_cselect_b32 s56, 1, 0\n\t" \
  "s_sub_u32 s54, s54, s55\n\t" \
  "s_add_u32 s56, s68, s56\n\t" \
  "s_cmp_lt_u32 s56, 8\n\t" \
  "s_cselect_b32 s67, s54, s67\n\t" \
  "s_cselect_b32 s68, s56, s68\n\t" \
  "s_mov_b64 exec, 0xffff\n\t" \
  "global_load_dwordx4 v[12:15], v50, s[74:75]\n\t" \
  "s_mov_b64 exec, -1\n\t" \
  "s_add_u32 s61, s61, 1\n\t" \
  "s_lshl_b32 s54, -2, s52\n\t" \
  "s_and_b32 s54, s54, s24\n\t" \
  "s_cselect_b32 s63, 0, 1\n\t" \
  "s_cselect_b32 s54, s54, s24\n\t" \
  "s_add_u32 s53, s53, s63\n\t" \
  "s_bcnt1_i32_b32 s55, s21\n\t" \
  "s_lshl_b32 s55, s55, 3\n\t" \
  "s_cmp_eq_u32 s53, 8\n\t" \
  "s_cselect_b32 s24, s21, s24\n\t" \
  "s_cselect_b32 s54, s21, s54\n\t" \
  "s_cselect_b32 s53, 0, s53\n\t" \
  "s_cselect_b32 s63, 0, s63\n\t" \
  "s_cselect_b32 s19, s55, s19\n\t" \
  "s_sub_u32 s19, s19, 1\n\t" \
  "s_cmp_eq_u32 s19, 0\n\t" \
  "s_cselect_b32 s23, s25, 0\n\t" \
  "s_cmp_le_u32 s19, s16\n\t" \
  "s_cselect_b32 s22, s25, 0\n\t" \
  "s_cmp_eq_u32 s12, s18\n\t" \
  "s_cselect_b32 s22, s22, 0\n\t" \
  "s_and_b32 s22, s22, s17\n\t" \
  "s_ff1_i32_b32 s52, s54\n\t" \
  "s_and_b32 s52, s52, 7\n\t" \
  "s_lshl_b32 s56, s52, 19\n\t" \
  "s_cmp_lt_u32 s61, 8\n\t" \
  "s_cselect_b32 s63, 0, s63\n\t" \
  "s_cselect_b32 s56, 0, s56\n\t" \
  "s_lshl_b32 s54, s52, 25\n\t" \
  "s_lshl_b32 s55, s53, 22\n\t" \
  "s_or_b32 s47, s54, s55\n\t" \
  "s_lshl_b32 s55, s53, 7\n\t" \
  "s_or_b32 s59, s56, s55\n\t" \
  "s_mov_b32 s46, s59\n\t" \
  "s_cmp_lt_u32 s61, 8\n\t" \
  "s_cselect_b32 s40, s40, s44\n\t" \
  "s_cselect_b32 s41, s41, s45\n\t" \
  "s_add_u32 s49, s62, 0x8000\n\t" \
  "s_add_u32 s33, s49, 0x4000\n\t" \
  "s_xor_b32 s48, s48, 0x8000\n\t" \
  "s_add_u32 s32, s48, 0x1000\n\t" \
  "s_mov_b32 m0, s49\n\t" \
  "s_nop 0\n\t" \
  "buffer_load_dwordx4 v34, s[36:39], s47 offen offset:0 lds\n\t" \
  "buffer_load_dwordx4 v35, s[36:39], s47 offen offset:1024 lds\n\t" \
  "buffer_load_dwordx4 v36, s[36:39], s47 offen offset:2048 lds\n\t" \
  "buffer_load_dwordx4 v37, s[36:39], s47 offen offset:3072 lds\n\t" \
  "s_mov_b32 m0, s33\n\t" \
  "s_nop 0\n\t" \
  "buffer_load_dwordx4 v38, s[36:39], s47 offen offset:0 lds\n\t" \
  "buffer_load_dwordx4 v39, s[36:39], s47 offen offset:1024 lds\n\t" \
  "buffer_load_dwordx4 v40, s[36:39], s47 offen offset:2048 lds\n\t" \
  "buffer_load_dwordx4 v41, s[36:39], s47 offen offset:3072 lds\n\t" \
  "s_add_u32 s61, s61, 1\n\t" \
  "s_lshl_b32 s54, -2, s52\n\t" \
  "s_and_b32 s54, s54, s24\n\t" \
  "s_cselect_b32 s63, 0, 1\n\t" \
  "s_cselect_b32 s54, s54, s24\n\t" \
  "s_add_u32 s53, s53, s63\n\t" \
  "s_bcnt1_i32_b32 s55, s21\n\t" \
  "s_lshl_b32 s55, s55, 3\n\t" \
  "s_cmp_eq_u32 s53, 8\n\t" \
  "s_cselect_b32 s24, s21, s24\n\t" \
  "s_cselect_b32 s54, s21, s54\n\t" \
  "s_cselect_b32 s53, 0, s53\n\t" \
  "s_cselect_b32 s63, 0, s63\n\t" \
  "s_cselect_b32 s19, s55, s19\n\t" \
  "s_sub_u32 s19, s19, 1\n\t" \
  "s_cmp_eq_u32 s19, 0\n\t" \
  "s_cselect_b32 s23, s25, 0\n\t" \
  "s_cmp_le_u32 s19, s16\n\t" \
  "s_cselect_b32 s22, s25, 0\n\t" \
  "s_cmp_eq_u32 s12, s18\n\t" \
  "s_cselect_b32 s22, s22, 0\n\t" \
  "s_and_b32 s22, s22, s17\n\t" \
  "s_ff1_i32_b32 s52, s54\n\t" \
  "s_and_b32 s52, s52, 7\n\t" \
  "s_lshl_b32 s56, s52, 19\n\t" \
  "s_cmp_lt_u32 s61, 8\n\t" \
  "s_cselect_b32 s63, 0, s63\n\t" \
  "s_cselect_b32 s56, 0, s56\n\t" \
  "s_lshl_b32 s54, s52, 25\n\t" \
  "s_lshl_b32 s55, s53, 22\n\t" \
  "s_or_b32 s47, s54, s55\n\t" \
  "s_lshl_b32 s55, s53, 7\n\t" \
  "s_or_b32 s59, s56, s55\n\t" \
  "s_add_u32 s49, s62, 0x10000\n\t" \
  "s_add_u32 s33, s49, 0x4000\n\t" \
  "v_mov_b32 v16, %[rdA0]\n\t" \
  "v_xor_b32 v17, 64, %[rdA0]\n\t" \
  "v_xor_b32 v18, 0, %[f16]\n\t" \
  "v_add_u32 v18, v18, %[rdB]\n\t" \
  "v_xor_b32 v19, 32, %[f16]\n\t" \
  "v_add_u32 v19, v19, %[rdB]\n\t" \
  "v_xor_b32 v20, 64, %[f16]\n\t" \
  "v_add_u32 v20, v20, %[rdB]\n\t" \
  "v_xor_b32 v21, 96, %[f16]\n\t" \
  "v_add_u32 v21, v21, %[rdB]\n\t" \
  "v_xor_b32 v22, 128, %[f16]\n\t" \
  "v_add_u32 v22, v22, %[rdB]\n\t" \
  "v_xor_b32 v23, 160, %[f16]\n\t" \
  "v_add_u32 v23, v23, %[rdB]\n\t" \
  "v_xor_b32 v24, 192, %[f16]\n\t" \
  "v_add_u32 v24, v24, %[rdB]\n\t" \
  "v_xor_b32 v25, 224, %[f16]\n\t" \
  "v_add_u32 v25, v25, %[rdB]\n\t" \
  "s_waitcnt vmcnt(11)\n\t" \
  "s_barrier\n\t" \
  "ds_read_b64_tr_b16 v[64:65], v18 offset:0\n\t" \
  "ds_read_b64_tr_b16 v[66:67], v18 offset:1024\n\t" \
  "ds_read_b64_tr_b16 v[68:69], v18 offset:16384\n\t" \
  "ds_read_b64_tr_b16 v[70:71], v18 offset:17408\n\t" \
  "ds_read_b128 v[128:131], v16 offset:0\n\t" \
  "ds_read_b128 v[132:135], v17 offset:0\n\t" \
  "ds_read_b128 v[136:139], v16 offset:16384\n\t" \
  "ds_read_b128 v[140:143], v17 offset:16384\n\t" \
  "ds_read_b128 v[144:147], v16 offset:1024\n\t" \
  "ds_read_b128 v[148:151], v17 offset:1024\n\t" \
  "ds_read_b128 v[152:155], v16 offset:17408\n\t" \
  "ds_read_b128 v[156:159], v17 offset:17408\n\t" \
  "ds_read_b128 v[160:163], v16 offset:2048\n\t" \
  "ds_read_b128 v[164:167], v17 offset:2048\n\t" \
  "ds_read_b128 v[168:171], v16 offset:18432\n\t" \
  "ds_read_b128 v[172:175], v17 offset:18432\n\t" \
  "ds_read_b128 v[176:179], v16 offset:3072\n\t" \
  "ds_read_b128 v[180:183], v17 offset:3072\n\t" \
  "ds_read_b128 v[184:187], v16 offset:19456\n\t" \
  "ds_read_b128 v[188:191], v17 offset:19456\n\t" \
  "ds_read_b64_tr_b16 v[72:73], v19 offset:0\n\t" \
  "ds_read_b64_tr_b16 v[74:75], v19 offset:1024\n\t" \
  "ds_read_b64_tr_b16 v[76:77], v19 offset:16384\n\t" \
  "ds_read_b64_tr_b16 v[78:79], v19 offset:17408\n\t" \
  "ds_read_b64_tr_b16 v[80:81], v20 offset:0\n\t" \
  "ds_read_b64_tr_b16 v[82:83], v20 offset:1024\n\t" \
  "ds_read_b64_tr_b16 v[84:85], v20 offset:16384\n\t" \
  "ds_read_b64_tr_b16 v[86:87], v20 offset:17408\n\t" \
  "ds_read_b64_tr_b16 v[88:89], v21 offset:0\n\t" \
  "ds_read_b64_tr_b16 v[90:91], v21 offset:1024\n\t" \
  "ds_read_b64_tr_b16 v[92:93], v21 offset:16384\n\t" \
  "ds_read_b64_tr_b16 v[94:95], v21 offset:17408\n\t" \
  "ds_read_b64_tr_b16 v[96:97], v22 offset:0\n\t" \
  "ds_read_b64_tr_b16 v[98:99], v22 offset:1024\n\t" \
  "ds_read_b64_tr_b16 v[100:101], v22 offset:16384\n\t" \
  "ds_read_b64_tr_b16 v[102:103], v22 offset:17408\n\t" \
  "ds_read_b64_tr_b16 v[104:105], v23 offset:0\n\t" \
  "ds_read_b64_tr_b16 v[106:107], v23 offset:1024\n\t" \
  "ds_read_b64_tr_b16 v[108:109], v23 offset:16384\n\t" \
  "ds_read_b64_tr_b16 v[110:111], v23 offset:17408\n\t" \
  "ds_read_b64_tr_b16 v[112:113], v24 offset:0\n\t" \
  "ds_read_b64_tr_b16 v[114:115], v24 offset:1024\n\t" \
  "ds_read_b64_tr_b16 v[116:117], v24 offset:16384\n\t" \
  "ds_read_b64_tr_b16 v[118:119], v24 offset:17408\n\t" \
  "ds_read_b64_tr_b16 v[120:121], v25 offset:0\n\t" \
  "ds_read_b64_tr_b16 v[122:123], v25 offset:1024\n\t" \
  "ds_read_b64_tr_b16 v[124:125], v25 offset:16384\n\t" \
  "ds_read_b64_tr_b16 v[126:127], v25 offset:17408\n\t" \
  "s_mov_b32 m0, s48\n\t" \
  "s_nop 0\n\t" \
  "buffer_load_dwordx4 v26, s[40:43], s46 offen offset:0 sc0 sc1 lds\n\t" \
  "buffer_load_dwordx4 v27, s[40:43], s46 offen offset:1024 sc0 sc1 lds\n\t" \
  "buffer_load_dwordx4 v28, s[40:43], s46 offen offset:2048 sc0 sc1 lds\n\t" \
  "buffer_load_dwordx4 v29, s[40:43], s46 offen offset:3072 sc0 sc1 lds\n\t" \
  "s_mov_b32 m0, s32\n\t" \
  "s_nop 0\n\t" \
  "buffer_load_dwordx4 v30, s[40:43], s46 offen offset:0 sc0 sc1 lds\n\t" \
  "buffer_load_dwordx4 v31, s[40:43], s46 offen offset:1024 sc0 sc1 lds\n\t" \
  "buffer_load_dwordx4 v32, s[40:43], s46 offen offset:2048 sc0 sc1 lds\n\t" \
  "buffer_load_dwordx4 v33, s[40:43], s46 offen offset:3072 sc0 sc1 lds\n\t" \
  "s_mov_b32 s46, s59\n\t" \
  "s_cmp_lt_u32 s61, 8\n\t" \
  "s_cselect_b32 s40, s40, s44\n\t" \
  "s_cselect_b32 s41, s41, s45\n\t" \
  "s_xor_b32 s48, s48, 0x8000\n\t" \
  "s_add_u32 s32, s48, 0x1000\n\t" \
  "v_readfirstlane_b32 s65, v55\n\t" \
  "s_nop 3\n\t" \
  "s_add_u32 s65, s65, 1\n\t" \
  "v_mov_b32 v55, s65\n\t" \
  "s_and_b32 s54, s65, 1\n\t" \
  "s_lshl_b32 s54, s54, 22\n\t" \
  "s_add_u32 s54, s54, %[wsdata]\n\t" \
  "s_lshl_b32 s55, s58, 19\n\t" \
  "s_add_u32 s69, s54, s55\n\t" \
  "s_add_u32 s98, s98, s69\n\t" \
  "s_addc_u32 s99, s99, 0\n\t" \
  "s_add_u32 s100, s100, s69\n\t" \
  "s_addc_u32 s101, s101, 0\n\t" \
  "s_waitcnt vmcnt(16)\n\t" \
  "s_mov_b64 exec, 0xffff\n\t" \
  "global_store_dwordx4 v50, v[42:45], s[98:99] sc0 sc1\n\t" \
  "s_mov_b64 exec, -1\n\t" \
  "s_mov_b64 exec, 0xffff\n\t" \
  "global_store_dwordx4 v50, v[46:49], s[100:101] sc0 sc1\n\t" \
  "s_mov_b64 exec, -1\n\t" \
  "v_add_u32 v18, 0x8000, v18\n\t" \
  "v_add_u32 v19, 0x8000, v19\n\t" \
  "v_add_u32 v20, 0x8000, v20\n\t" \
  "v_add_u32 v21, 0x8000, v21\n\t" \
  "v_add_u32 v22, 0x8000, v22\n\t" \
  "v_add_u32 v23, 0x8000, v23\n\t" \
  "v_add_u32 v24, 0x8000, v24\n\t" \
  "v_add_u32 v25, 0x8000, v25\n\t" \
  "v_xor_b32 v16, 0x8000, v16\n\t" \
  "v_xor_b32 v17, 0x8000, v17\n\t" \
  "s_mov_b32 s60, 0x8000\n\t" \
  "s_mov_b32 s51, 0x8000\n\t" \
  "s_waitcnt lgkmcnt(0)\n\t" \
  "s_barrier\n\t" \
  "v_mfma_f32_16x16x32_bf16 a[0:3], v[64:67], v[128:131], 0\n\t" \
  "s_mov_b32 m0, s49\n\t" \
  "s_lshl_b32 s55, s58, 19\n\t" \
  "s_sub_u32 s54, s69, s55\n\t" \
  "v_mfma_f32_16x16x32_bf16 a[0:3], v[68:71], v[132:135], a[0:3]\n\t" \
  "buffer_load_dwordx4 v34, s[36:39], s47 offen offset:0 lds\n\t" \
  "s_lshl_b32 s55, s58, 8\n\t" \
  "v_mfma_f32_16x16x32_bf16 a[4:7], v[64:67], v[136:139], 0\n\t" \
  "s_add_u32 s70, s55, %[wsflags]\n\t" \
  "s_add_u32 s54, s54, s29\n\t" \
  "v_mfma_f32_16x16x32_bf16 a[4:7], v[68:71], v[140:143], a[4:7]\n\t" \
  "s_add_u32 s44, s90, s54\n\t" \
  "s_addc_u32 s45, s91, 0\n\t" \
  "v_mfma_f32_16x16x32_bf16 a[8:11], v[64:67], v[144:147], 0\n\t" \
  "buffer_load_dwordx4 v35, s[36:39], s47 offen offset:1024 lds\n\t" \
  "s_add_u32 s14, s90, %[wsdec]\n\t" \
  "v_mfma_f32_16x16x32_bf16 a[8:11], v[68:71], v[148:151], a[8:11]\n\t" \
  "s_addc_u32 s15, s91, 0\n\t" \
  "s_add_u32 s94, s90, %[wstmo]\n\t" \
  "v_mfma_f32_16x16x32_bf16 a[12:15], v[64:67], v[152:155], 0\n\t" \
  "s_addc_u32 s95, s91, 0\n\t" \
  "s_add_u32 s90, s90, %[wsflags]\n\t" \
  "v_mfma_f32_16x16x32_bf16 a[12:15], v[68:71], v[156:159], a[12:15]\n\t" \
  "buffer_load_dwordx4 v36, s[36:39], s47 offen offset:2048 lds\n\t" \
  "s_addc_u32 s91, s91, 0\n\t" \
  "v_mfma_f32_16x16x32_bf16 a[16:19], v[64:67], v[160:163], 0\n\t" \
  "s_add_u32 s76, s76, s69\n\t" \
  "s_addc_u32 s77, s77, 0\n\t" \
  "v_mfma_f32_16x16x32_bf16 a[16:19], v[68:71], v[164:167], a[16:19]\n\t" \
  "s_add_u32 s82, s82, s70\n\t" \
  "s_addc_u32 s83, s83, 0\n\t" \
  "v_mfma_f32_16x16x32_bf16 a[20:23], v[64:67], v[168:171], 0\n\t" \
  "buffer_load_dwordx4 v37, s[36:39], s47 offen offset:3072 lds\n\t" \
  "s_add_u32 s84, s84, s70\n\t" \
  "v_mfma_f32_16x16x32_bf16 a[20:23], v[68:71], v[172:175], a[20:23]\n\t" \
  "s_addc_u32 s85, s85, 0\n\t" \
  "s_add_u32 s86, s86, s70\n\t" \
  "v_mfma_f32_16x16x32_bf16 a[24:27], v[64:67], v[176:179], 0\n\t" \
  "s_mov_b32 m0, s33\n\t" \
  "s_addc_u32 s87, s87, 0\n\t" \
  "v_mfma_f32_16x16x32_bf16 a[24:27], v[68:71], v[180:183], a[24:27]\n\t" \
  "buffer_load_dwordx4 v38, s[36:39], s47 offen offset:0 lds\n\t" \
  "s_mul_i32 s54, s67, 9363\n\t" \
  "v_mfma_f32_16x16x32_bf16 a[28:31], v[64:67], v[184:187], 0\n\t" \
  "s_lshr_b32 s54, s54, 16\n\t" \
  "s_mul_i32 s55, s54, 7\n\t" \
  "v_mfma_f32_16x16x32_bf16 a[28:31], v[68:71], v[188:191], a[28:31]\n\t" \
  "s_sub_u32 s55, s67, s55\n\t" \
  "s_add_u32 s55, s55, s58\n\t" \
  "v_mfma_f32_16x16x32_bf16 a[32:35], v[72:75], v[128:131], 0\n\t" \
  "buffer_load_dwordx4 v39, s[36:39], s47 offen offset:1024 lds\n\t" \
  "s_add_u32 s55, s55, 1\n\t" \
  "v_mfma_f32_16x16x32_bf16 a[32:35], v[76:79], v[132:135], a[32:35]\n\t" \
  "s_and_b32 s55, s55, 7\n\t" \
  "v_readlane_b32 s78, v10, s55\n\t" \
  "v_mfma_f32_16x16x32_bf16 a[36:39], v[72:75], v[136:139], 0\n\t" \
  "v_readlane_b32 s79, v11, s55\n\t" \
  "s_lshl_b32 s56, s54, 13\n\t" \
  "v_mfma_f32_16x16x32_bf16 a[36:39], v[76:79], v[140:143], a[36:39]\n\t" \
  "buffer_load_dwordx4 v40, s[36:39], s47 offen offset:2048 lds\n\t" \
  "s_lshl_b32 s57, s68, 7\n\t" \
  "v_mfma_f32_16x16x32_bf16 a[40:43], v[72:75], v[144:147], 0\n\t" \
  "s_add_u32 s56, s56, s57\n\t" \
  "s_lshl_b32 s55, s55, 19\n\t" \
  "v_mfma_f32_16x16x32_bf16 a[40:43], v[76:79], v[148:151], a[40:43]\n\t" \
  "s_add_u32 s55, s55, s56\n\t" \
  "s_add_u32 s74, s72, s55\n\t" \
  "v_mfma_f32_16x16x32_bf16 a[44:47], v[72:75], v[152:155], 0\n\t" \
  "buffer_load_dwordx4 v41, s[36:39], s47 offen offset:3072 lds\n\t" \
  "s_addc_u32 s75, s73, 0\n\t" \
  "v_mfma_f32_16x16x32_bf16 a[44:47], v[76:79], v[156:159], a[44:47]\n\t" \
  "s_lshl_b32 s57, s68, 11\n\t" \
  "s_lshl_b32 s54, s54, 2\n\t" \
  "v_mfma_f32_16x16x32_bf16 a[48:51], v[72:75], v[160:163], 0\n\t" \
  "s_mov_b32 m0, s48\n\t" \
  "s_add_u32 s57, s57, s54\n\t" \
  "v_mfma_f32_16x16x32_bf16 a[48:51], v[76:79], v[164:167], a[48:51]\n\t" \
  "buffer_load_dwordx4 v26, s[40:43], s46 offen offset:0 sc0 sc1 lds\n\t" \
  "s_add_u32 s57, s57, s70\n\t" \
  "v_mfma_f32_16x16x32_bf16 a[52:55], v[72:75], v[168:171], 0\n\t" \
  "s_add_u32 s88, s78, s57\n\t" \
  "s_addc_u32 s89, s79, 0\n\t" \
  "v_mfma_f32_16x16x32_bf16 a[52:55], v[76:79], v[172:175], a[52:55]\n\t" \
  "s_add_u32 s56, s56, s69\n\t" \
  "s_add_u32 s78, s78, s56\n\t" \
  "v_mfma_f32_16x16x32_bf16 a[56:59], v[72:75], v[176:179], 0\n\t" \
  "buffer_load_dwordx4 v27, s[40:43], s46 offen offset:1024 sc0 sc1 lds\n\t" \
  "s_addc_u32 s79, s79, 0\n\t" \
  "v_mfma_f32_16x16x32_bf16 a[56:59], v[76:79], v[180:183], a[56:59]\n\t" \
  "s_add_u32 s54, s67, 256\n\t" \
  "s_cmp_ge_u32 s54, 448\n\t" \
  "v_mfma_f32_16x16x32_bf16 a[60:63], v[72:75], v[184:187], 0\n\t" \
  "s_cselect_b32 s55, 448, 0\n\t" \
  "s_cselect_b32 s56, 1, 0\n\t" \
  "v_mfma_f32_16x16x32_bf16 a[60:63], v[76:79], v[188:191], a[60:63]\n\t" \
  "buffer_load_dwordx4 v28, s[40:43], s46 offen offset:2048 sc0 sc1 lds\n\t" \
  "s_sub_u32 s54, s54, s55\n\t" \
  "v_mfma_f32_16x16x32_bf16 a[64:67], v[80:83], v[128:131], 0\n\t" \
  "s_add_u32 s56, s68, s56\n\t" \
  "s_cmp_lt_u32 s56, 8\n\t" \
  "v_mfma_f32_16x16x32_bf16 a[64:67], v[84:87], v[132:135], a[64:67]\n\t" \
  "s_cselect_b32 s67, s54, s67\n\t" \
  "s_cselect_b32 s68, s56, s68\n\t" \
  "v_mfma_f32_16x16x32_bf16 a[68:71], v[80:83], v[136:139], 0\n\t" \
  "buffer_load_dwordx4 v29, s[40:43], s46 offen offset:3072 sc0 sc1 lds\n\t" \
  "s_cmp_eq_u32 s66, 0\n\t" \
  "v_mfma_f32_16x16x32_bf16 a[68:71], v[84:87], v[140:143], a[68:71]\n\t" \
  "s_cselect_b32 s26, 1, 0\n\t" \
  "s_mov_b32 s27, 0\n\t" \
  "v_mfma_f32_16x16x32_bf16 a[72:75], v[80:83], v[144:147], 0\n\t" \
  "s_mov_b32 m0, s32\n\t" \
  "s_mov_b32 s28, 0\n\t" \
  "v_mfma_f32_16x16x32_bf16 a[72:75], v[84:87], v[148:151], a[72:75]\n\t" \
  "buffer_load_dwordx4 v30, s[40:43], s46 offen offset:0 sc0 sc1 lds\n\t" \
  "s_add_u32 s61, s61, 1\n\t" \
  "s_lshl_b32 s54, -2, s52\n\t" \
  "v_mfma_f32_16x16x32_bf16 a[76:79], v[80:83], v[152:155], 0\n\t" \
  "s_and_b32 s54, s54, s24\n\t" \
  "s_cselect_b32 s63, 0, 1\n\t" \
  "v_mfma_f32_16x16x32_bf16 a[76:79], v[84:87], v[156:159], a[76:79]\n\t" \
  "s_cselect_b32 s54, s54, s24\n\t" \
  "s_add_u32 s53, s53, s63\n\t" \
  "v_mfma_f32_16x16x32_bf16 a[80:83], v[80:83], v[160:163], 0\n\t" \
  "buffer_load_dwordx4 v31, s[40:43], s46 offen offset:1024 sc0 sc1 lds\n\t" \
  "s_bcnt1_i32_b32 s55, s21\n\t" \
  "s_lshl_b32 s55, s55, 3\n\t" \
  "v_mfma_f32_16x16x32_bf16 a[80:83], v[84:87], v[164:167], a[80:83]\n\t" \
  "s_cmp_eq_u32 s53, 8\n\t" \
  "s_cselect_b32 s24, s21, s24\n\t" \
  "v_mfma_f32_16x16x32_bf16 a[84:87], v[80:83], v[168:171], 0\n\t" \
  "s_cselect_b32 s54, s21, s54\n\t" \
  "s_cselect_b32 s53, 0, s53\n\t" \
  "v_mfma_f32_16x16x32_bf16 a[84:87], v[84:87], v[172:175], a[84:87]\n\t" \
  "buffer_load_dwordx4 v32, s[40:43], s46 offen offset:2048 sc0 sc1 lds\n\t" \
  "s_cselect_b32 s63, 0, s63\n\t" \
  "s_cselect_b32 s19, s55, s19\n\t" \
  "v_mfma_f32_16x16x32_bf16 a[88:91], v[80:83], v[176:179], 0\n\t" \
  "s_sub_u32 s19, s19, 1\n\t" \
  "s_cmp_eq_u32 s19, 0\n\t" \
  "v_mfma_f32_16x16x32_bf16 a[88:91], v[84:87], v[180:183], a[88:91]\n\t" \
  "s_cselect_b32 s23, s25, 0\n\t" \
  "s_cmp_le_u32 s19, s16\n\t" \
  "v_mfma_f32_16x16x32_bf16 a[92:95], v[80:83], v[184:187], 0\n\t" \
  "buffer_load_dwordx4 v33, s[40:43], s46 offen offset:3072 sc0 sc1 lds\n\t" \
  "s_cselect_b32 s22, s25, 0\n\t" \
  "s_cmp_eq_u32 s12, s18\n\t" \
  "v_mfma_f32_16x16x32_bf16 a[92:95], v[84:87], v[188:191], a[92:95]\n\t" \
  "s_waitcnt vmcnt(18)\n\t" \
  "s_barrier\n\t" \
  "v_mfma_f32_16x16x32_bf16 a[96:99], v[88:91], v[128:131], 0\n\t" \
  "ds_read_b64_tr_b16 v[56:57], v25 offset:0\n\t" \
  "s_cselect_b32 s22, s22, 0\n\t" \
  "v_mfma_f32_16x16x32_bf16 a[96:99], v[92:95], v[132:135], a[96:99]\n\t" \
  "ds_read_b64_tr_b16 v[58:59], v25 offset:1024\n\t" \
  "s_and_b32 s22, s22, s17\n\t" \
  "s_ff1_i32_b32 s52, s54\n\t" \
  "v_mfma_f32_16x16x32_bf16 a[100:103], v[88:91], v[136:139], 0\n\t" \
  "ds_read_b64_tr_b16 v[60:61], v25 offset:16384\n\t" \
  "s_mov_b64 exec, 0xffff\n\t" \
  "global_load_dwordx4 v[46:49], v50, s[74:75]\n\t" \
  "s_mov_b64 exec, -1\n\t" \
  "v_mfma_f32_16x16x32_bf16 a[100:103], v[92:95], v[140:143], a[100:103]\n\t" \
  "ds_read_b64_tr_b16 v[62:63], v25 offset:17408\n\t" \
  "s_and_b32 s52, s52, 7\n\t" \
  "v_mfma_f32_16x16x32_bf16 a[104:107], v[88:91], v[144:147], 0\n\t" \
  "ds_read_b64_tr_b16 v[64:65], v18 offset:0\n\t" \
  "s_mov_b64 exec, 0xffff\n\t" \
  "global_store_dwordx4 v50, v[12:15], s[76:77] sc0 sc1\n\t" \
  "s_mov_b64 exec, -1\n\t" \
  "v_mfma_f32_16x16x32_bf16 a[104:107], v[92:95], v[148:151], a[104:107]\n\t" \
  "ds_read_b64_tr_b16 v[66:67], v18 offset:1024\n\t" \
  "s_lshl_b32 s56, s52, 19\n\t" \
  "v_mfma_f32_16x16x32_bf16 a[108:111], v[88:91], v[152:155], 0\n\t" \
  "ds_read_b64_tr_b16 v[68:69], v18 offset:16384\n\t" \
  "s_cmp_lt_u32 s61, 8\n\t" \
  "s_cselect_b32 s63, 0, s63\n\t" \
  "v_mfma_f32_16x16x32_bf16 a[108:111], v[92:95], v[156:159], a[108:111]\n\t" \
  "ds_read_b64_tr_b16 v[70:71], v18 offset:17408\n\t" \
  "s_cselect_b32 s56, 0, s56\n\t" \
  "v_mfma_f32_16x16x32_bf16 a[112:115], v[88:91], v[160:163], 0\n\t" \
  "ds_read_b64_tr_b16 v[72:73], v19 offset:0\n\t" \
  "s_lshl_b32 s54, s52, 25\n\t" \
  "v_mfma_f32_16x16x32_bf16 a[112:115], v[92:95], v[164:167], a[112:115]\n\t" \
  "ds_read_b64_tr_b16 v[74:75], v19 offset:1024\n\t" \
  "s_lshl_b32 s55, s53, 22\n\t" \
  "s_or_b32 s47, s54, s55\n\t" \
  "v_mfma_f32_16x16x32_bf16 a[116:119], v[88:91], v[168:171], 0\n\t" \
  "ds_read_b64_tr_b16 v[76:77], v19 offset:16384\n\t" \
  "s_lshl_b32 s55, s53, 7\n\t" \
  "v_mfma_f32_16x16x32_bf16 a[116:119], v[92:95], v[172:175], a[116:119]\n\t" \
  "ds_read_b64_tr_b16 v[78:79], v19 offset:17408\n\t" \
  "s_or_b32 s59, s56, s55\n\t" \
  "v_mfma_f32_16x16x32_bf16 a[120:123], v[88:91], v[176:179], 0\n\t" \
  "ds_read_b128 v[192:195], v16 offset:0\n\t" \
  "s_mov_b32 s49, s62\n\t" \
  "s_add_u32 s33, s49, 0x4000\n\t" \
  "v_mfma_f32_16x16x32_bf16 a[120:123], v[92:95], v[180:183], a[120:123]\n\t" \
  "ds_read_b128 v[196:199], v17 offset:0\n\t" \
  "v_add_u32 v25, s51, v25\n\t" \
  "v_mfma_f32_16x16x32_bf16 a[124:127], v[88:91], v[184:187], 0\n\t" \
  "ds_read_b128 v[200:203], v16 offset:16384\n\t" \
  "v_add_u32 v18, s51, v18\n\t" \
  "v_mfma_f32_16x16x32_bf16 a[124:127], v[92:95], v[188:191], a[124:127]\n\t" \
  "ds_read_b128 v[204:207], v17 offset:16384\n\t" \
  "v_add_u32 v19, s51, v19\n\t" \
  "v_mfma_f32_16x16x32_bf16 a[128:131], v[96:99], v[128:131], 0\n\t" \
  "ds_read_b64_tr_b16 v[80:81], v20 offset:0\n\t" \
  "v_mfma_f32_16x16x32_bf16 a[128:131], v[100:103], v[132:135], a[128:131]\n\t" \
  "ds_read_b64_tr_b16 v[82:83], v20 offset:1024\n\t" \
  "s_mov_b32 m0, s49\n\t" \
  "v_mfma_f32_16x16x32_bf16 a[132:135], v[96:99], v[136:139], 0\n\t" \
  "ds_read_b64_tr_b16 v[84:85], v20 offset:16384\n\t" \
  "s_mov_b64 s[76:77], s[78:79]\n\t" \
  "v_mfma_f32_16x16x32_bf16 a[132:135], v[100:103], v[140:143], a[132:135]\n\t" \
  "ds_read_b64_tr_b16 v[86:87], v20 offset:17408\n\t" \
  "buffer_load_dwordx4 v34, s[36:39], s47 offen offset:0 lds\n\t" \
  "v_mfma_f32_16x16x32_bf16 a[136:139], v[96:99], v[144:147], 0\n\t" \
  "ds_read_b64_tr_b16 v[88:89], v21 offset:0\n\t" \
  "s_mov_b64 s[80:81], s[82:83]\n\t" \
  "v_mfma_f32_16x16x32_bf16 a[136:139], v[100:103], v[148:151], a[136:139]\n\t" \
  "ds_read_b64_tr_b16 v[90:91], v21 offset:1024\n\t" \
  "s_mov_b64 s[82:83], s[84:85]\n\t" \
  "v_mfma_f32_16x16x32_bf16 a[140:143], v[96:99], v[152:155], 0\n\t" \
  "ds_read_b64_tr_b16 v[92:93], v21 offset:16384\n\t" \
  "s_mov_b64 s[84:85], s[86:87]\n\t" \
  "v_mfma_f32_16x16x32_bf16 a[140:143], v[100:103], v[156:159], a[140:143]\n\t" \
  "ds_read_b64_tr_b16 v[94:95], v21 offset:17408\n\t" \
  "s_mov_b64 s[86:87], s[88:89]\n\t" \
  "v_mfma_f32_16x16x32_bf16 a[144:147], v[96:99], v[160:163], 0\n\t" \
  "ds_read_b128 v[208:211], v16 offset:1024\n\t" \
  "v_mfma_f32_16x16x32_bf16 a[144:147], v[100:103], v[164:167], a[144:147]\n\t" \
  "ds_read_b128 v[212:215], v17 offset:1024\n\t" \
  "buffer_load_dwordx4 v35, s[36:39], s47 offen offset:1024 lds\n\t" \
  "v_mfma_f32_16x16x32_bf16 a[148:151], v[96:99], v[168:171], 0\n\t" \
  "ds_read_b128 v[216:219], v16 offset:17408\n\t" \
  "s_mul_i32 s54, s67, 9363\n\t" \
  "v_mfma_f32_16x16x32_bf16 a[148:151], v[100:103], v[172:175], a[148:151]\n\t" \
  "ds_read_b128 v[220:223], v17 offset:17408\n\t" \
  "s_lshr_b32 s54, s54, 16\n\t" \
  "v_mfma_f32_16x16x32_bf16 a[152:155], v[96:99], v[176:179], 0\n\t" \
  "ds_read_b128 v[224:227], v16 offset:2048\n\t" \
  "s_mul_i32 s55, s54, 7\n\t" \
  "v_mfma_f32_16x16x32_bf16 a[152:155], v[100:103], v[180:183], a[152:155]\n\t" \
  "ds_read_b128 v[228:231], v17 offset:2048\n\t" \
  "v_add_u32 v20, s51, v20\n\t" \
  "v_mfma_f32_16x16x32_bf16 a[156:159], v[96:99], v[184:187], 0\n\t" \
  "ds_read_b128 v[232:235], v16 offset:18432\n\t" \
  "v_add_u32 v21, s51, v21\n\t" \
  "v_mfma_f32_16x16x32_bf16 a[156:159], v[100:103], v[188:191], a[156:159]\n\t" \
  "ds_read_b128 v[236:239], v17 offset:18432\n\t" \
  "buffer_load_dwordx4 v36, s[36:39], s47 offen offset:2048 lds\n\t" \
  "v_mfma_f32_16x16x32_bf16 a[160:163], v[104:107], v[128:131], 0\n\t" \
  "ds_read_b64_tr_b16 v[96:97], v22 offset:0\n\t" \
  "s_sub_u32 s55, s67, s55\n\t" \
  "v_mfma_f32_16x16x32_bf16 a[160:163], v[108:111], v[132:135], a[160:163]\n\t" \
  "s_add_u32 s55, s55, s58\n\t" \
  "v_mfma_f32_16x16x32_bf16 a[164:167], v[104:107], v[136:139], 0\n\t" \
  "ds_read_b64_tr_b16 v[98:99], v22 offset:1024\n\t" \
  "s_add_u32 s55, s55, 1\n\t" \
  "v_mfma_f32_16x16x32_bf16 a[164:167], v[108:111], v[140:143], a[164:167]\n\t" \
  "s_and_b32 s55, s55, 7\n\t" \
  "v_readlane_b32 s78, v10, s55\n\t" \
  "v_mfma_f32_16x16x32_bf16 a[168:171], v[104:107], v[144:147], 0\n\t" \
  "ds_read_b64_tr_b16 v[100:101], v22 offset:16384\n\t" \
  "v_readlane_b32 s79, v11, s55\n\t" \
  "v_mfma_f32_16x16x32_bf16 a[168:171], v[108:111], v[148:151], a[168:171]\n\t" \
  "buffer_load_dwordx4 v37, s[36:39], s47 offen offset:3072 lds\n\t" \
  "s_lshl_b32 s56, s54, 13\n\t" \
  "v_mfma_f32_16x16x32_bf16 a[172:175], v[104:107], v[152:155], 0\n\t" \
  "ds_read_b64_tr_b16 v[102:103], v22 offset:17408\n\t" \
  "s_lshl_b32 s57, s68, 7\n\t" \
  "v_mfma_f32_16x16x32_bf16 a[172:175], v[108:111], v[156:159], a[172:175]\n\t" \
  "s_add_u32 s56, s56, s57\n\t" \
  "v_mfma_f32_16x16x32_bf16 a[176:179], v[104:107], v[160:163], 0\n\t" \
  "ds_read_b128 v[240:243], v16 offset:3072\n\t" \
  "s_lshl_b32 s55, s55, 19\n\t" \
  "v_mfma_f32_16x16x32_bf16 a[176:179], v[108:111], v[164:167], a[176:179]\n\t" \
  "s_mov_b32 m0, s33\n\t" \
  "s_add_u32 s55, s55, s56\n\t" \
  "v_mfma_f32_16x16x32_bf16 a[180:183], v[104:107], v[168:171], 0\n\t" \
  "ds_read_b128 v[244:247], v17 offset:3072\n\t" \
  "s_add_u32 s74, s72, s55\n\t" \
  "v_mfma_f32_16x16x32_bf16 a[180:183], v[108:111], v[172:175], a[180:183]\n\t" \
  "buffer_load_dwordx4 v38, s[36:39], s47 offen offset:0 lds\n\t" \
  "s_addc_u32 s75, s73, 0\n\t" \
  "v_mfma_f32_16x16x32_bf16 a[184:187], v[104:107], v[176:179], 0\n\t" \
  "ds_read_b128 v[248:251], v16 offset:19456\n\t" \
  "v_mfma_f32_16x16x32_bf16 a[184:187], v[108:111], v[180:183], a[184:187]\n\t" \
  "s_lshl_b32 s57, s68, 11\n\t" \
  "s_lshl_b32 s54, s54, 2\n\t" \
  "v_mfma_f32_16x16x32_bf16 a[188:191], v[104:107], v[184:187], 0\n\t" \
  "ds_read_b128 v[252:255], v17 offset:19456\n\t" \
  "s_add_u32 s57, s57, s54\n\t" \
  "v_mfma_f32_16x16x32_bf16 a[188:191], v[108:111], v[188:191], a[188:191]\n\t" \
  "v_add_u32 v22, s51, v22\n\t" \
  "s_add_u32 s57, s57, s70\n\t" \
  "v_mfma_f32_16x16x32_bf16 a[192:195], v[112:115], v[128:131], 0\n\t" \
  "ds_read_b64_tr_b16 v[104:105], v23 offset:0\n\t" \
  "s_add_u32 s88, s78, s57\n\t" \
  "v_mfma_f32_16x16x32_bf16 a[192:195], v[116:119], v[132:135], a[192:195]\n\t" \
  "buffer_load_dwordx4 v39, s[36:39], s47 offen offset:1024 lds\n\t" \
  "s_addc_u32 s89, s79, 0\n\t" \
  "v_mfma_f32_16x16x32_bf16 a[196:199], v[112:115], v[136:139], 0\n\t" \
  "ds_read_b64_tr_b16 v[106:107], v23 offset:1024\n\t" \
  "v_mfma_f32_16x16x32_bf16 a[196:199], v[116:119], v[140:143], a[196:199]\n\t" \
  "s_add_u32 s56, s56, s69\n\t" \
  "s_add_u32 s78, s78, s56\n\t" \
  "v_mfma_f32_16x16x32_bf16 a[200:203], v[112:115], v[144:147], 0\n\t" \
  "ds_read_b64_tr_b16 v[108:109], v23 offset:16384\n\t" \
  "s_addc_u32 s79, s79, 0\n\t" \
  "v_mfma_f32_16x16x32_bf16 a[200:203], v[116:119], v[148:151], a[200:203]\n\t" \
  "s_add_u32 s54, s67, 256\n\t" \
  "s_cmp_ge_u32 s54, 448\n\t" \
  "v_mfma_f32_16x16x32_bf16 a[204:207], v[112:115], v[152:155], 0\n\t" \
  "ds_read_b64_tr_b16 v[110:111], v23 offset:17408\n\t" \
  "s_cselect_b32 s55, 448, 0\n\t" \
  "v_mfma_f32_16x16x32_bf16 a[204:207], v[116:119], v[156:159], a[204:207]\n\t" \
  "buffer_load_dwordx4 v40, s[36:39], s47 offen offset:2048 lds\n\t" \
  "v_mfma_f32_16x16x32_bf16 a[208:211], v[112:115], v[160:163], 0\n\t" \
  "v_add_u32 v23, s51, v23\n\t" \
  "s_cselect_b32 s56, 1, 0\n\t" \
  "v_mfma_f32_16x16x32_bf16 a[208:211], v[116:119], v[164:167], a[208:211]\n\t" \
  "v_xor_b32 v16, 0x8000, v16\n\t" \
  "s_sub_u32 s54, s54, s55\n\t" \
  "v_mfma_f32_16x16x32_bf16 a[212:215], v[112:115], v[168:171], 0\n\t" \
  "v_xor_b32 v17, 0x8000, v17\n\t" \
  "s_add_u32 s56, s68, s56\n\t" \
  "v_mfma_f32_16x16x32_bf16 a[212:215], v[116:119], v[172:175], a[212:215]\n\t" \
  "s_cmp_lt_u32 s56, 8\n\t" \
  "s_cselect_b32 s67, s54, s67\n\t" \
  "v_mfma_f32_16x16x32_bf16 a[216:219], v[112:115], v[176:179], 0\n\t" \
  "s_cselect_b32 s68, s56, s68\n\t" \
  "v_mfma_f32_16x16x32_bf16 a[216:219], v[116:119], v[180:183], a[216:219]\n\t" \
  "buffer_load_dwordx4 v41, s[36:39], s47 offen offset:3072 lds\n\t" \
  "v_mfma_f32_16x16x32_bf16 a[220:223], v[112:115], v[184:187], 0\n\t" \
  "v_mfma_f32_16x16x32_bf16 a[220:223], v[116:119], v[188:191], a[220:223]\n\t" \
  "v_mfma_f32_16x16x32_bf16 a[224:227], v[120:123], v[128:131], 0\n\t" \
  "ds_read_b64_tr_b16 v[112:113], v24 offset:0\n\t" \
  "s_mov_b32 s46, s59\n\t" \
  "v_mfma_f32_16x16x32_bf16 a[224:227], v[124:127], v[132:135], a[224:227]\n\t" \
  "ds_read_b64_tr_b16 v[114:115], v24 offset:1024\n\t" \
  "s_cmp_lt_u32 s61, 8\n\t" \
  "v_mfma_f32_16x16x32_bf16 a[228:231], v[120:123], v[136:139], 0\n\t" \
  "ds_read_b64_tr_b16 v[116:117], v24 offset:16384\n\t" \
  "s_cselect_b32 s40, s40, s44\n\t" \
  "v_mfma_f32_16x16x32_bf16 a[228:231], v[124:127], v[140:143], a[228:231]\n\t" \
  "ds_read_b64_tr_b16 v[118:119], v24 offset:17408\n\t" \
  "s_cselect_b32 s41, s41, s45\n\t" \
  "v_mfma_f32_16x16x32_bf16 a[232:235], v[120:123], v[144:147], 0\n\t" \
  "s_add_u32 s49, s62, s60\n\t" \
  "v_mfma_f32_16x16x32_bf16 a[232:235], v[124:127], v[148:151], a[232:235]\n\t" \
  "v_add_u32 v24, s51, v24\n\t" \
  "s_add_u32 s33, s49, 0x4000\n\t" \
  "v_mfma_f32_16x16x32_bf16 a[236:239], v[120:123], v[152:155], 0\n\t" \
  "s_add_u32 s60, s60, s51\n\t" \
  "v_mfma_f32_16x16x32_bf16 a[236:239], v[124:127], v[156:159], a[236:239]\n\t" \
  "s_cmp_eq_u32 s60, 0x10000\n\t" \
  "v_mfma_f32_16x16x32_bf16 a[240:243], v[120:123], v[160:163], 0\n\t" \
  "s_cselect_b32 s51, 0xffff0000, s71\n\t" \
  "v_mfma_f32_16x16x32_bf16 a[240:243], v[124:127], v[164:167], a[240:243]\n\t" \
  "s_xor_b32 s48, s48, 0x8000\n\t" \
  "v_mfma_f32_16x16x32_bf16 a[244:247], v[120:123], v[168:171], 0\n\t" \
  "s_add_u32 s32, s48, 0x1000\n\t" \
  "v_mfma_f32_16x16x32_bf16 a[244:247], v[124:127], v[172:175], a[244:247]\n\t" \
  "v_mfma_f32_16x16x32_bf16 a[248:251], v[120:123], v[176:179], 0\n\t" \
  "v_mfma_f32_16x16x32_bf16 a[248:251], v[124:127], v[180:183], a[248:251]\n\t" \
  "v_mfma_f32_16x16x32_bf16 a[252:255], v[120:123], v[184:187], 0\n\t" \
  "v_mfma_f32_16x16x32_bf16 a[252:255], v[124:127], v[188:191], a[252:255]\n\t" \
  "s_waitcnt vmcnt(10) lgkmcnt(0)\n\t" \
  "s_barrier\n\t" \
  "s_mov_b32 s50, 7\n\t" \
  "LOOP1_%=:\n\t" \
  "s_mov_b64 exec, s[26:27]\n\t" \
  "global_store_dword v51, v55, s[80:81] sc0 sc1\n\t" \
  "s_mov_b64 exec, -1\n\t" \
  "v_mfma_f32_16x16x32_bf16 a[0:3], v[64:67], v[192:195], a[0:3]\n\t" \
  "ds_read_b64_tr_b16 v[120:121], v25 offset:0\n\t" \
  "s_add_u32 s61, s61, 1\n\t" \
  "v_mfma_f32_16x16x32_bf16 a[0:3], v[68:71], v[196:199], a[0:3]\n\t" \
  "s_mov_b32 m0, s48\n\t" \
  "s_lshl_b32 s54, -2, s52\n\t" \
  "v_mfma_f32_16x16x32_bf16 a[4:7], v[68:71], v[204:207], a[4:7]\n\t" \
  "ds_read_b64_tr_b16 v[122:123], v25 offset:1024\n\t" \
  "s_and_b32 s54, s54, s24\n\t" \
  "v_mfma_f32_16x16x32_bf16 a[4:7], v[64:67], v[200:203], a[4:7]\n\t" \
  "buffer_load_dwordx4 v26, s[40:43], s46 offen offset:0 sc0 sc1 lds\n\t" \
  "v_mfma_f32_16x16x32_bf16 a[8:11], v[64:67], v[208:211], a[8:11]\n\t" \
  "ds_read_b64_tr_b16 v[124:125], v25 offset:16384\n\t" \
  "s_cselect_b32 s63, 0, 1\n\t" \
  "v_mfma_f32_16x16x32_bf16 a[8:11], v[68:71], v[212:215], a[8:11]\n\t" \
  "s_cselect_b32 s54, s54, s24\n\t" \
  "s_add_u32 s53, s53, s63\n\t" \
  "v_mfma_f32_16x16x32_bf16 a[12:15], v[68:71], v[220:223], a[12:15]\n\t" \
  "ds_read_b64_tr_b16 v[126:127], v25 offset:17408\n\t" \
  "s_bcnt1_i32_b32 s55, s21\n\t" \
  "v_mfma_f32_16x16x32_bf16 a[12:15], v[64:67], v[216:219], a[12:15]\n\t" \
  "s_lshl_b32 s55, s55, 3\n\t" \
  "v_mfma_f32_16x16x32_bf16 a[16:19], v[64:67], v[224:227], a[16:19]\n\t" \
  "ds_read_b128 v[128:131], v16 offset:0\n\t" \
  "s_cmp_eq_u32 s53, 8\n\t" \
  "v_mfma_f32_16x16x32_bf16 a[16:19], v[68:71], v[228:231], a[16:19]\n\t" \
  "v_add_u32 v25, s51, v25\n\t" \
  "buffer_load_dwordx4 v27, s[40:43], s46 offen offset:1024 sc0 sc1 lds\n\t" \
  "v_mfma_f32_16x16x32_bf16 a[20:23], v[68:71], v[236:239], a[20:23]\n\t" \
  "s_cselect_b32 s24, s21, s24\n\t" \
  "s_cselect_b32 s54, s21, s54\n\t" \
  "v_mfma_f32_16x16x32_bf16 a[20:23], v[64:67], v[232:235], a[20:23]\n\t" \
  "ds_read_b128 v[132:135], v17 offset:0\n\t" \
  "v_mfma_f32_16x16x32_bf16 a[24:27], v[64:67], v[240:243], a[24:27]\n\t" \
  "s_cselect_b32 s53, 0, s53\n\t" \
  "s_cselect_b32 s63, 0, s63\n\t" \
  "v_mfma_f32_16x16x32_bf16 a[24:27], v[68:71], v[244:247], a[24:27]\n\t" \
  "s_cselect_b32 s19, s55, s19\n\t" \
  "s_sub_u32 s19, s19, 1\n\t" \
  "v_mfma_f32_16x16x32_bf16 a[28:31], v[68:71], v[252:255], a[28:31]\n\t" \
  "ds_read_b128 v[136:139], v16 offset:16384\n\t" \
  "s_cmp_eq_u32 s19, 0\n\t" \
  "v_mfma_f32_16x16x32_bf16 a[28:31], v[64:67], v[248:251], a[28:31]\n\t" \
  "buffer_load_dwordx4 v28, s[40:43], s46 offen offset:2048 sc0 sc1 lds\n\t" \
  "v_mfma_f32_16x16x32_bf16 a[60:63], v[72:75], v[248:251], a[60:63]\n\t" \
  "ds_read_b64_tr_b16 v[64:65], v18 offset:0\n\t" \
  "s_cselect_b32 s23, s25, 0\n\t" \
  "v_mfma_f32_16x16x32_bf16 a[60:63], v[76:79], v[252:255], a[60:63]\n\t" \
  "s_cmp_le_u32 s19, s16\n\t" \
  "s_cselect_b32 s22, s25, 0\n\t" \
  "v_mfma_f32_16x16x32_bf16 a[56:59], v[76:79], v[244:247], a[56:59]\n\t" \
  "ds_read_b64_tr_b16 v[66:67], v18 offset:1024\n\t" \
  "v_mfma_f32_16x16x32_bf16 a[56:59], v[72:75], v[240:243], a[56:59]\n\t" \
  "s_cmp_eq_u32 s12, s18\n\t" \
  "s_cselect_b32 s22, s22, 0\n\t" \
  "v_mfma_f32_16x16x32_bf16 a[52:55], v[72:75], v[232:235], a[52:55]\n\t" \
  "ds_read_b64_tr_b16 v[68:69], v18 offset:16384\n\t" \
  "s_and_b32 s22, s22, s17\n\t" \
  "v_mfma_f32_16x16x32_bf16 a[52:55], v[76:79], v[236:239], a[52:55]\n\t" \
  "buffer_load_dwordx4 v29, s[40:43], s46 offen offset:3072 sc0 sc1 lds\n\t" \
  "s_ff1_i32_b32 s52, s54\n\t" \
  "v_mfma_f32_16x16x32_bf16 a[48:51], v[76:79], v[228:231], a[48:51]\n\t" \
  "ds_read_b64_tr_b16 v[70:71], v18 offset:17408\n\t" \
  "v_mfma_f32_16x16x32_bf16 a[48:51], v[72:75], v[224:227], a[48:51]\n\t" \
  "s_and_b32 s52, s52, 7\n\t" \
  "s_lshl_b32 s56, s52, 19\n\t" \
  "v_mfma_f32_16x16x32_bf16 a[44:47], v[72:75], v[216:219], a[44:47]\n\t" \
  "ds_read_b128 v[140:143], v17 offset:16384\n\t" \
  "s_cmp_lt_u32 s61, 8\n\t" \
  "v_mfma_f32_16x16x32_bf16 a[44:47], v[76:79], v[220:223], a[44:47]\n\t" \
  "v_add_u32 v18, s51, v18\n\t" \
  "s_mov_b32 m0, s32\n\t" \
  "v_mfma_f32_16x16x32_bf16 a[40:43], v[76:79], v[212:215], a[40:43]\n\t" \
  "s_cselect_b32 s63, 0, s63\n\t" \
  "s_cselect_b32 s56, 0, s56\n\t" \
  "v_mfma_f32_16x16x32_bf16 a[40:43], v[72:75], v[208:211], a[40:43]\n\t" \
  "ds_read_b128 v[144:147], v16 offset:1024\n\t" \
  "buffer_load_dwordx4 v30, s[40:43], s46 offen offset:0 sc0 sc1 lds\n\t" \
  "v_mfma_f32_16x16x32_bf16 a[36:39], v[72:75], v[200:203], a[36:39]\n\t" \
  "s_lshl_b32 s54, s52, 25\n\t" \
  "v_mfma_f32_16x16x32_bf16 a[36:39], v[76:79], v[204:207], a[36:39]\n\t" \
  "s_lshl_b32 s55, s53, 22\n\t" \
  "s_or_b32 s47, s54, s55\n\t" \
  "v_mfma_f32_16x16x32_bf16 a[32:35], v[76:79], v[196:199], a[32:35]\n\t" \
  "ds_read_b128 v[148:151], v17 offset:1024\n\t" \
  "s_lshl_b32 s55, s53, 7\n\t" \
  "v_mfma_f32_16x16x32_bf16 a[32:35], v[72:75], v[192:195], a[32:35]\n\t" \
  "s_or_b32 s59, s56, s55\n\t" \
  "v_mfma_f32_16x16x32_bf16 a[64:67], v[80:83], v[192:195], a[64:67]\n\t" \
  "ds_read_b64_tr_b16 v[72:73], v19 offset:0\n\t" \
  "v_cmp_le_u32 vcc, s65, v52\n\t" \
  "v_mfma_f32_16x16x32_bf16 a[64:67], v[84:87], v[196:199], a[64:67]\n\t" \
  "buffer_load_dwordx4 v31, s[40:43], s46 offen offset:1024 sc0 sc1 lds\n\t" \
  "v_cmp_le_u32 s[54:55], s65, v53\n\t" \
  "v_mfma_f32_16x16x32_bf16 a[68:71], v[84:87], v[204:207], a[68:71]\n\t" \
  "ds_read_b64_tr_b16 v[74:75], v19 offset:1024\n\t" \
  "s_nop 0\n\t" \
  "v_mfma_f32_16x16x32_bf16 a[68:71], v[80:83], v[200:203], a[68:71]\n\t" \
  "s_and_b64 vcc, vcc, s[54:55]\n\t" \
  "v_mfma_f32_16x16x32_bf16 a[72:75], v[80:83], v[208:211], a[72:75]\n\t" \
  "ds_read_b64_tr_b16 v[76:77], v19 offset:16384\n\t" \
  "s_or_b64 vcc, vcc, s[92:93]\n\t" \
  "v_mfma_f32_16x16x32_bf16 a[72:75], v[84:87], v[212:215], a[72:75]\n\t" \
  "s_cmp_eq_u64 vcc, -1\n\t" \
  "s_cselect_b32 s64, 0, s63\n\t" \
  "v_mfma_f32_16x16x32_bf16 a[76:79], v[84:87], v[220:223], a[76:79]\n\t" \
  "ds_read_b64_tr_b16 v[78:79], v19 offset:17408\n\t" \
  "v_mfma_f32_16x16x32_bf16 a[76:79], v[80:83], v[216:219], a[76:79]\n\t" \
  "buffer_load_dwordx4 v32, s[40:43], s46 offen offset:2048 sc0 sc1 lds\n\t" \
  "s_or_b32 s13, s23, s22\n\t" \
  "v_mfma_f32_16x16x32_bf16 a[80:83], v[80:83], v[224:227], a[80:83]\n\t" \
  "ds_read_b128 v[152:155], v16 offset:17408\n\t" \
  "s_or_b32 s28, s13, s64\n\t" \
  "v_mfma_f32_16x16x32_bf16 a[80:83], v[84:87], v[228:231], a[80:83]\n\t" \
  "v_add_u32 v19, s51, v19\n\t" \
  "s_add_u32 s57, s53, 1\n\t" \
  "v_mfma_f32_16x16x32_bf16 a[84:87], v[84:87], v[236:239], a[84:87]\n\t" \
  "s_and_b32 s57, s57, 7\n\t" \
  "v_mfma_f32_16x16x32_bf16 a[84:87], v[80:83], v[232:235], a[84:87]\n\t" \
  "ds_read_b128 v[156:159], v17 offset:17408\n\t" \
  "s_lshl_b32 s57, s57, 11\n\t" \
  "v_mfma_f32_16x16x32_bf16 a[88:91], v[80:83], v[240:243], a[88:91]\n\t" \
  "s_add_u32 s96, s90, s57\n\t" \
  "s_addc_u32 s97, s91, 0\n\t" \
  "v_mfma_f32_16x16x32_bf16 a[88:91], v[84:87], v[244:247], a[88:91]\n\t" \
  "buffer_load_dwordx4 v33, s[40:43], s46 offen offset:3072 sc0 sc1 lds\n\t" \
  "v_mfma_f32_16x16x32_bf16 a[92:95], v[84:87], v[252:255], a[92:95]\n\t" \
  "ds_read_b128 v[160:163], v16 offset:2048\n\t" \
  "global_load_dwordx2 v[52:53], v54, s[96:97] sc0 sc1\n\t" \
  "s_cmp_lg_u32 s13, 0\n\t" \
  "s_cbranch_scc1 OP2_%=\n\t" \
  "OPB2_%=:\n\t" \
  "v_mfma_f32_16x16x32_bf16 a[92:95], v[80:83], v[248:251], a[92:95]\n\t" \
  "s_waitcnt vmcnt(18)\n\t" \
  "v_mfma_f32_16x16x32_bf16 a[124:127], v[88:91], v[248:251], a[124:127]\n\t" \
  "ds_read_b64_tr_b16 v[80:81], v20 offset:0\n\t" \
  "v_mfma_f32_16x16x32_bf16 a[124:127], v[92:95], v[252:255], a[124:127]\n\t" \
  "s_mov_b64 exec, 0xffff\n\t" \
  "global_load_dwordx4 v[42:45], v50, s[74:75]\n\t" \
  "s_mov_b64 exec, -1\n\t" \
  "v_mfma_f32_16x16x32_bf16 a[120:123], v[92:95], v[244:247], a[120:123]\n\t" \
  "ds_read_b64_tr_b16 v[82:83], v20 offset:1024\n\t" \
  "v_mfma_f32_16x16x32_bf16 a[120:123], v[88:91], v[240:243], a[120:123]\n\t" \
  "s_mov_b64 exec, 0xffff\n\t" \
  "global_store_dwordx4 v50, v[46:49], s[76:77] sc0 sc1\n\t" \
  "s_mov_b64 exec, -1\n\t" \
  "v_mfma_f32_16x16x32_bf16 a[116:119], v[88:91], v[232:235], a[116:119]\n\t" \
  "ds_read_b64_tr_b16 v[84:85], v20 offset:16384\n\t" \
  "v_mfma_f32_16x16x32_bf16 a[116:119], v[92:95], v[236:239], a[116:119]\n\t" \
  "s_mov_b64 s[76:77], s[78:79]\n\t" \
  "s_mov_b64 s[80:81], s[82:83]\n\t" \
  "v_mfma_f32_16x16x32_bf16 a[112:115], v[92:95], v[228:231], a[112:115]\n\t" \
  "ds_read_b64_tr_b16 v[86:87], v20 offset:17408\n\t" \
  "v_mfma_f32_16x16x32_bf16 a[112:115], v[88:91], v[224:227], a[112:115]\n\t" \
  "s_mov_b64 s[82:83], s[84:85]\n\t" \
  "s_mov_b64 s[84:85], s[86:87]\n\t" \
  "v_mfma_f32_16x16x32_bf16 a[108:111], v[88:91], v[216:219], a[108:111]\n\t" \
  "ds_read_b128 v[164:167], v17 offset:2048\n\t" \
  "v_mfma_f32_16x16x32_bf16 a[108:111], v[92:95], v[220:223], a[108:111]\n\t" \
  "v_add_u32 v20, s51, v20\n\t" \
  "s_mov_b64 s[86:87], s[88:89]\n\t" \
  "v_mfma_f32_16x16x32_bf16 a[104:107], v[92:95], v[212:215], a[104:107]\n\t" \
  "s_mul_i32 s54, s67, 9363\n\t" \
  "v_mfma_f32_16x16x32_bf16 a[104:107], v[88:91], v[208:211], a[104:107]\n\t" \
  "ds_read_b128 v[168:171], v16 offset:18432\n\t" \
  "v_mfma_f32_16x16x32_bf16 a[100:103], v[88:91], v[200:203], a[100:103]\n\t" \
  "s_lshr_b32 s54, s54, 16\n\t" \
  "s_mul_i32 s55, s54, 7\n\t" \
  "v_mfma_f32_16x16x32_bf16 a[100:103], v[92:95], v[204:207], a[100:103]\n\t" \
  "s_sub_u32 s55, s67, s55\n\t" \
  "v_mfma_f32_16x16x32_bf16 a[96:99], v[92:95], v[196:199], a[96:99]\n\t" \
  "ds_read_b128 v[172:175], v17 offset:18432\n\t" \
  "s_add_u32 s55, s55, s58\n\t" \
  "v_mfma_f32_16x16x32_bf16 a[96:99], v[88:91], v[192:195], a[96:99]\n\t" \
  "s_add_u32 s55, s55, 1\n\t" \
  "v_mfma_f32_16x16x32_bf16 a[128:131], v[96:99], v[192:195], a[128:131]\n\t" \
  "ds_read_b64_tr_b16 v[88:89], v21 offset:0\n\t" \
  "v_mfma_f32_16x16x32_bf16 a[128:131], v[100:103], v[196:199], a[128:131]\n\t" \
  "s_mov_b32 m0, s49\n\t" \
  "s_and_b32 s55, s55, 7\n\t" \
  "v_mfma_f32_16x16x32_bf16 a[132:135], v[100:103], v[204:207], a[132:135]\n\t" \
  "ds_read_b64_tr_b16 v[90:91], v21 offset:1024\n\t" \
  "v_mfma_f32_16x16x32_bf16 a[132:135], v[96:99], v[200:203], a[132:135]\n\t" \
  "buffer_load_dwordx4 v34, s[36:39], s47 offen offset:0 lds\n\t" \
  "v_readlane_b32 s78, v10, s55\n\t" \
  "v_mfma_f32_16x16x32_bf16 a[136:139], v[96:99], v[208:211], a[136:139]\n\t" \
  "ds_read_b64_tr_b16 v[92:93], v21 offset:16384\n\t" \
  "v_mfma_f32_16x16x32_bf16 a[136:139], v[100:103], v[212:215], a[136:139]\n\t" \
  "v_readlane_b32 s79, v11, s55\n\t" \
  "s_lshl_b32 s56, s54, 13\n\t" \
  "v_mfma_f32_16x16x32_bf16 a[140:143], v[100:103], v[220:223], a[140:143]\n\t" \
  "ds_read_b64_tr_b16 v[94:95], v21 offset:17408\n\t" \
  "v_mfma_f32_16x16x32_bf16 a[140:143], v[96:99], v[216:219], a[140:143]\n\t" \
  "s_lshl_b32 s57, s68, 7\n\t" \
  "v_mfma_f32_16x16x32_bf16 a[144:147], v[96:99], v[224:227], a[144:147]\n\t" \
  "ds_read_b128 v[176:179], v16 offset:3072\n\t" \
  "s_add_u32 s56, s56, s57\n\t" \
  "v_mfma_f32_16x16x32_bf16 a[144:147], v[100:103], v[228:231], a[144:147]\n\t" \
  "v_add_u32 v21, s51, v21\n\t" \
  "buffer_load_dwordx4 v35, s[36:39], s47 offen offset:1024 lds\n\t" \
  "v_mfma_f32_16x16x32_bf16 a[148:151], v[100:103], v[236:239], a[148:151]\n\t" \
  "s_lshl_b32 s55, s55, 19\n\t" \
  "v_mfma_f32_16x16x32_bf16 a[148:151], v[96:99], v[232:235], a[148:151]\n\t" \
  "ds_read_b128 v[180:183], v17 offset:3072\n\t" \
  "s_add_u32 s55, s55, s56\n\t" \
  "v_mfma_f32_16x16x32_bf16 a[152:155], v[96:99], v[240:243], a[152:155]\n\t" \
  "s_add_u32 s74, s72, s55\n\t" \
  "v_mfma_f32_16x16x32_bf16 a[152:155], v[100:103], v[244:247], a[152:155]\n\t" \
  "s_addc_u32 s75, s73, 0\n\t" \
  "v_mfma_f32_16x16x32_bf16 a[156:159], v[100:103], v[252:255], a[156:159]\n\t" \
  "ds_read_b128 v[184:187], v16 offset:19456\n\t" \
  "s_lshl_b32 s57, s68, 11\n\t" \
  "v_mfma_f32_16x16x32_bf16 a[156:159], v[96:99], v[248:251], a[156:159]\n\t" \
  "buffer_load_dwordx4 v36, s[36:39], s47 offen offset:2048 lds\n\t" \
  "v_mfma_f32_16x16x32_bf16 a[188:191], v[104:107], v[248:251], a[188:191]\n\t" \
  "ds_read_b64_tr_b16 v[96:97], v22 offset:0\n\t" \
  "s_lshl_b32 s54, s54, 2\n\t" \
  "v_mfma_f32_16x16x32_bf16 a[188:191], v[108:111], v[252:255], a[188:191]\n\t" \
  "s_add_u32 s57, s57, s54\n\t" \
  "v_mfma_f32_16x16x32_bf16 a[184:187], v[108:111], v[244:247], a[184:187]\n\t" \
  "ds_read_b64_tr_b16 v[98:99], v22 offset:1024\n\t" \
  "v_mfma_f32_16x16x32_bf16 a[184:187], v[104:107], v[240:243], a[184:187]\n\t" \
  "s_add_u32 s57, s57, s70\n\t" \
  "s_add_u32 s88, s78, s57\n\t" \
  "v_mfma_f32_16x16x32_bf16 a[180:183], v[104:107], v[232:235], a[180:183]\n\t" \
  "ds_read_b64_tr_b16 v[100:101], v22 offset:16384\n\t" \
  "v_mfma_f32_16x16x32_bf16 a[180:183], v[108:111], v[236:239], a[180:183]\n\t" \
  "buffer_load_dwordx4 v37, s[36:39], s47 offen offset:3072 lds\n\t" \
  "s_addc_u32 s89, s79, 0\n\t" \
  "v_mfma_f32_16x16x32_bf16 a[176:179], v[108:111], v[228:231], a[176:179]\n\t" \
  "ds_read_b64_tr_b16 v[102:103], v22 offset:17408\n\t" \
  "v_mfma_f32_16x16x32_bf16 a[176:179], v[104:107], v[224:227], a[176:179]\n\t" \
  "s_add_u32 s56, s56, s69\n\t" \
  "s_add_u32 s78, s78, s56\n\t" \
  "v_mfma_f32_16x16x32_bf16 a[172:175], v[104:107], v[216:219], a[172:175]\n\t" \
  "ds_read_b128 v[188:191], v17 offset:19456\n\t" \
  "v_mfma_f32_16x16x32_bf16 a[172:175], v[108:111], v[220:223], a[172:175]\n\t" \
  "v_add_u32 v22, s51, v22\n\t" \
  "s_mov_b32 m0, s33\n\t" \
  "v_mfma_f32_16x16x32_bf16 a[168:171], v[108:111], v[212:215], a[168:171]\n\t" \
  "s_addc_u32 s79, s79, 0\n\t" \
  "v_mfma_f32_16x16x32_bf16 a[168:171], v[104:107], v[208:211], a[168:171]\n\t" \
  "buffer_load_dwordx4 v38, s[36:39], s47 offen offset:0 lds\n\t" \
  "s_add_u32 s54, s67, 256\n\t" \
  "v_mfma_f32_16x16x32_bf16 a[164:167], v[104:107], v[200:203], a[164:167]\n\t" \
  "s_cmp_ge_u32 s54, 448\n\t" \
  "v_mfma_f32_16x16x32_bf16 a[164:167], v[108:111], v[204:207], a[164:167]\n\t" \
  "s_cselect_b32 s55, 448, 0\n\t" \
  "s_cselect_b32 s56, 1, 0\n\t" \
  "v_mfma_f32_16x16x32_bf16 a[160:163], v[108:111], v[196:199], a[160:163]\n\t" \
  "s_sub_u32 s54, s54, s55\n\t" \
  "v_mfma_f32_16x16x32_bf16 a[160:163], v[104:107], v[192:195], a[160:163]\n\t" \
  "s_add_u32 s56, s68, s56\n\t" \
  "v_mfma_f32_16x16x32_bf16 a[192:195], v[112:115], v[192:195], a[192:195]\n\t" \
  "ds_read_b64_tr_b16 v[104:105], v23 offset:0\n\t" \
  "s_cmp_lt_u32 s56, 8\n\t" \
  "v_mfma_f32_16x16x32_bf16 a[192:195], v[116:119], v[196:199], a[192:195]\n\t" \
  "buffer_load_dwordx4 v39, s[36:39], s47 offen offset:1024 lds\n\t" \
  "v_mfma_f32_16x16x32_bf16 a[196:199], v[116:119], v[204:207], a[196:199]\n\t" \
  "ds_read_b64_tr_b16 v[106:107], v23 offset:1024\n\t" \
  "s_cselect_b32 s67, s54, s67\n\t" \
  "v_mfma_f32_16x16x32_bf16 a[196:199], v[112:115], v[200:203], a[196:199]\n\t" \
  "s_cselect_b32 s68, s56, s68\n\t" \
  "v_mfma_f32_16x16x32_bf16 a[200:203], v[112:115], v[208:211], a[200:203]\n\t" \
  "ds_read_b64_tr_b16 v[108:109], v23 offset:16384\n\t" \
  "v_mfma_f32_16x16x32_bf16 a[200:203], v[116:119], v[212:215], a[200:203]\n\t" \
  "v_mfma_f32_16x16x32_bf16 a[204:207], v[116:119], v[220:223], a[204:207]\n\t" \
  "ds_read_b64_tr_b16 v[110:111], v23 offset:17408\n\t" \
  "s_mov_b32 s46, s59\n\t" \
  "v_mfma_f32_16x16x32_bf16 a[204:207], v[112:115], v[216:219], a[204:207]\n\t" \
  "buffer_load_dwordx4 v40, s[36:39], s47 offen offset:2048 lds\n\t" \
  "v_mfma_f32_16x16x32_bf16 a[208:211], v[112:115], v[224:227], a[208:211]\n\t" \
  "s_cmp_lt_u32 s61, 8\n\t" \
  "v_mfma_f32_16x16x32_bf16 a[208:211], v[116:119], v[228:231], a[208:211]\n\t" \
  "v_add_u32 v23, s51, v23\n\t" \
  "v_mfma_f32_16x16x32_bf16 a[212:215], v[116:119], v[236:239], a[212:215]\n\t" \
  "s_cselect_b32 s40, s40, s44\n\t" \
  "v_mfma_f32_16x16x32_bf16 a[212:215], v[112:115], v[232:235], a[212:215]\n\t" \
  "v_xor_b32 v16, 0x8000, v16\n\t" \
  "s_cselect_b32 s41, s41, s45\n\t" \
  "v_mfma_f32_16x16x32_bf16 a[216:219], v[112:115], v[240:243], a[216:219]\n\t" \
  "v_xor_b32 v17, 0x8000, v17\n\t" \
  "v_mfma_f32_16x16x32_bf16 a[216:219], v[116:119], v[244:247], a[216:219]\n\t" \
  "buffer_load_dwordx4 v41, s[36:39], s47 offen offset:3072 lds\n\t" \
  "v_mfma_f32_16x16x32_bf16 a[220:223], v[116:119], v[252:255], a[220:223]\n\t" \
  "s_add_u32 s49, s62, s60\n\t" \
  "v_mfma_f32_16x16x32_bf16 a[220:223], v[112:115], v[248:251], a[220:223]\n\t" \
  "s_add_u32 s33, s49, 0x4000\n\t" \
  "v_mfma_f32_16x16x32_bf16 a[252:255], v[56:59], v[248:251], a[252:255]\n\t" \
  "ds_read_b64_tr_b16 v[112:113], v24 offset:0\n\t" \
  "s_xor_b32 s48, s48, 0x8000\n\t" \
  "v_mfma_f32_16x16x32_bf16 a[252:255], v[60:63], v[252:255], a[252:255]\n\t" \
  "ds_read_b64_tr_b16 v[114:115], v24 offset:1024\n\t" \
  "v_mfma_f32_16x16x32_bf16 a[248:251], v[60:63], v[244:247], a[248:251]\n\t" \
  "ds_read_b64_tr_b16 v[116:117], v24 offset:16384\n\t" \
  "v_mfma_f32_16x16x32_bf16 a[248:251], v[56:59], v[240:243], a[248:251]\n\t" \
  "ds_read_b64_tr_b16 v[118:119], v24 offset:17408\n\t" \
  "v_mfma_f32_16x16x32_bf16 a[244:247], v[56:59], v[232:235], a[244:247]\n\t" \
  "s_add_u32 s32, s48, 0x1000\n\t" \
  "v_mfma_f32_16x16x32_bf16 a[244:247], v[60:63], v[236:239], a[244:247]\n\t" \
  "v_add_u32 v24, s51, v24\n\t" \
  "v_mfma_f32_16x16x32_bf16 a[240:243], v[60:63], v[228:231], a[240:243]\n\t" \
  "s_add_u32 s60, s60, s51\n\t" \
  "v_mfma_f32_16x16x32_bf16 a[240:243], v[56:59], v[224:227], a[240:243]\n\t" \
  "v_mfma_f32_16x16x32_bf16 a[236:239], v[56:59], v[216:219], a[236:239]\n\t" \
  "s_cmp_eq_u32 s60, 0x10000\n\t" \
  "v_mfma_f32_16x16x32_bf16 a[236:239], v[60:63], v[220:223], a[236:239]\n\t" \
  "v_mfma_f32_16x16x32_bf16 a[232:235], v[60:63], v[212:215], a[232:235]\n\t" \
  "v_mfma_f32_16x16x32_bf16 a[232:235], v[56:59], v[208:211], a[232:235]\n\t" \
  "s_cselect_b32 s51, 0xffff0000, s71\n\t" \
  "v_mfma_f32_16x16x32_bf16 a[228:231], v[56:59], v[200:203], a[228:231]\n\t" \
  "v_mfma_f32_16x16x32_bf16 a[228:231], v[60:63], v[204:207], a[228:231]\n\t" \
  "s_cmp_lg_u32 s28, 0\n\t" \
  "v_mfma_f32_16x16x32_bf16 a[224:227], v[60:63], v[196:199], a[224:227]\n\t" \
  "v_mfma_f32_16x16x32_bf16 a[224:227], v[56:59], v[192:195], a[224:227]\n\t" \
  "s_waitcnt vmcnt(10) lgkmcnt(0)\n\t" \
  "s_cbranch_scc1 OE2_%=\n\t" \
  "OEB2_%=:\n\t" \
  "s_barrier\n\t" \
  "s_mov_b64 exec, s[26:27]\n\t" \
  "global_store_dword v51, v55, s[80:81] sc0 sc1\n\t" \
  "s_mov_b64 exec, -1\n\t" \
  "v_mfma_f32_16x16x32_bf16 a[0:3], v[64:67], v[128:131], a[0:3]\n\t" \
  "ds_read_b64_tr_b16 v[56:57], v25 offset:0\n\t" \
  "s_add_u32 s61, s61, 1\n\t" \
  "v_mfma_f32_16x16x32_bf16 a[0:3], v[68:71], v[132:135], a[0:3]\n\t" \
  "s_mov_b32 m0, s48\n\t" \
  "s_lshl_b32 s54, -2, s52\n\t" \
  "v_mfma_f32_16x16x32_bf16 a[4:7], v[68:71], v[140:143], a[4:7]\n\t" \
  "ds_read_b64_tr_b16 v[58:59], v25 offset:1024\n\t" \
  "s_and_b32 s54, s54, s24\n\t" \
  "v_mfma_f32_16x16x32_bf16 a[4:7], v[64:67], v[136:139], a[4:7]\n\t" \
  "buffer_load_dwordx4 v26, s[40:43], s46 offen offset:0 sc0 sc1 lds\n\t" \
  "v_mfma_f32_16x16x32_bf16 a[8:11], v[64:67], v[144:147], a[8:11]\n\t" \
  "ds_read_b64_tr_b16 v[60:61], v25 offset:16384\n\t" \
  "s_cselect_b32 s63, 0, 1\n\t" \
  "v_mfma_f32_16x16x32_bf16 a[8:11], v[68:71], v[148:151], a[8:11]\n\t" \
  "s_cselect_b32 s54, s54, s24\n\t" \
  "s_add_u32 s53, s53, s63\n\t" \
  "v_mfma_f32_16x16x32_bf16 a[12:15], v[68:71], v[156:159], a[12:15]\n\t" \
  "ds_read_b64_tr_b16 v[62:63], v25 offset:17408\n\t" \
  "s_bcnt1_i32_b32 s55, s21\n\t" \
  "v_mfma_f32_16x16x32_bf16 a[12:15], v[64:67], v[152:155], a[12:15]\n\t" \
  "s_lshl_b32 s55, s55, 3\n\t" \
  "v_mfma_f32_16x16x32_bf16 a[16:19], v[64:67], v[160:163], a[16:19]\n\t" \
  "ds_read_b128 v[192:195], v16 offset:0\n\t" \
  "s_cmp_eq_u32 s53, 8\n\t" \
  "v_mfma_f32_16x16x32_bf16 a[16:19], v[68:71], v[164:167], a[16:19]\n\t" \
  "v_add_u32 v25, s51, v25\n\t" \
  "buffer_load_dwordx4 v27, s[40:43], s46 offen offset:1024 sc0 sc1 lds\n\t" \
  "v_mfma_f32_16x16x32_bf16 a[20:23], v[68:71], v[172:175], a[20:23]\n\t" \
  "s_cselect_b32 s24, s21, s24\n\t" \
  "s_cselect_b32 s54, s21, s54\n\t" \
  "v_mfma_f32_16x16x32_bf16 a[20:23], v[64:67], v[168:171], a[20:23]\n\t" \
  "ds_read_b128 v[196:199], v17 offset:0\n\t" \
  "v_mfma_f32_16x16x32_bf16 a[24:27], v[64:67], v[176:179], a[24:27]\n\t" \
  "s_cselect_b32 s53, 0, s53\n\t" \
  "s_cselect_b32 s63, 0, s63\n\t" \
  "v_mfma_f32_16x16x32_bf16 a[24:27], v[68:71], v[180:183], a[24:27]\n\t" \
  "s_cselect_b32 s19, s55, s19\n\t" \
  "s_sub_u32 s19, s19, 1\n\t" \
  "v_mfma_f32_16x16x32_bf16 a[28:31], v[68:71], v[188:191], a[28:31]\n\t" \
  "ds_read_b128 v[200:203], v16 offset:16384\n\t" \
  "s_cmp_eq_u32 s19, 0\n\t" \
  "v_mfma_f32_16x16x32_bf16 a[28:31], v[64:67], v[184:187], a[28:31]\n\t" \
  "buffer_load_dwordx4 v28, s[40:43], s46 offen offset:2048 sc0 sc1 lds\n\t" \
  "v_mfma_f32_16x16x32_bf16 a[60:63], v[72:75], v[184:187], a[60:63]\n\t" \
  "ds_read_b64_tr_b16 v[64:65], v18 offset:0\n\t" \
  "s_cselect_b32 s23, s25, 0\n\t" \
  "v_mfma_f32_16x16x32_bf16 a[60:63], v[76:79], v[188:191], a[60:63]\n\t" \
  "s_cmp_le_u32 s19, s16\n\t" \
  "s_cselect_b32 s22, s25, 0\n\t" \
  "v_mfma_f32_16x16x32_bf16 a[56:59], v[76:79], v[180:183], a[56:59]\n\t" \
  "ds_read_b64_tr_b16 v[66:67], v18 offset:1024\n\t" \
  "v_mfma_f32_16x16x32_bf16 a[56:59], v[72:75], v[176:179], a[56:59]\n\t" \
  "s_cmp_eq_u32 s12, s18\n\t" \
  "s_cselect_b32 s22, s22, 0\n\t" \
  "v_mfma_f32_16x16x32_bf16 a[52:55], v[72:75], v[168:171], a[52:55]\n\t" \
  "ds_read_b64_tr_b16 v[68:69], v18 offset:16384\n\t" \
  "s_and_b32 s22, s22, s17\n\t" \
  "v_mfma_f32_16x16x32_bf16 a[52:55], v[76:79], v[172:175], a[52:55]\n\t" \
  "buffer_load_dwordx4 v29, s[40:43], s46 offen offset:3072 sc0 sc1 lds\n\t" \
  "s_ff1_i32_b32 s52, s54\n\t" \
  "v_mfma_f32_16x16x32_bf16 a[48:51], v[76:79], v[164:167], a[48:51]\n\t" \
  "ds_read_b64_tr_b16 v[70:71], v18 offset:17408\n\t" \
  "v_mfma_f32_16x16x32_bf16 a[48:51], v[72:75], v[160:163], a[48:51]\n\t" \
  "s_and_b32 s52, s52, 7\n\t" \
  "s_lshl_b32 s56, s52, 19\n\t" \
  "v_mfma_f32_16x16x32_bf16 a[44:47], v[72:75], v[152:155], a[44:47]\n\t" \
  "ds_read_b128 v[204:207], v17 offset:16384\n\t" \
  "s_cmp_lt_u32 s61, 8\n\t" \
  "v_mfma_f32_16x16x32_bf16 a[44:47], v[76:79], v[156:159], a[44:47]\n\t" \
  "v_add_u32 v18, s51, v18\n\t" \
  "s_mov_b32 m0, s32\n\t" \
  "v_mfma_f32_16x16x32_bf16 a[40:43], v[76:79], v[148:151], a[40:43]\n\t" \
  "s_cselect_b32 s63, 0, s63\n\t" \
  "s_cselect_b32 s56, 0, s56\n\t" \
  "v_mfma_f32_16x16x32_bf16 a[40:43], v[72:75], v[144:147], a[40:43]\n\t" \
  "ds_read_b128 v[208:211], v16 offset:1024\n\t" \
  "buffer_load_dwordx4 v30, s[40:43], s46 offen offset:0 sc0 sc1 lds\n\t" \
  "v_mfma_f32_16x16x32_bf16 a[36:39], v[72:75], v[136:139], a[36:39]\n\t" \
  "s_lshl_b32 s54, s52, 25\n\t" \
  "v_mfma_f32_16x16x32_bf16 a[36:39], v[76:79], v[140:143], a[36:39]\n\t" \
  "s_lshl_b32 s55, s53, 22\n\t" \
  "s_or_b32 s47, s54, s55\n\t" \
  "v_mfma_f32_16x16x32_bf16 a[32:35], v[76:79], v[132:135], a[32:35]\n\t" \
  "ds_read_b128 v[212:215], v17 offset:1024\n\t" \
  "s_lshl_b32 s55, s53, 7\n\t" \
  "v_mfma_f32_16x16x32_bf16 a[32:35], v[72:75], v[128:131], a[32:35]\n\t" \
  "s_or_b32 s59, s56, s55\n\t" \
  "v_mfma_f32_16x16x32_bf16 a[64:67], v[80:83], v[128:131], a[64:67]\n\t" \
  "ds_read_b64_tr_b16 v[72:73], v19 offset:0\n\t" \
  "v_cmp_le_u32 vcc, s65, v52\n\t" \
  "v_mfma_f32_16x16x32_bf16 a[64:67], v[84:87], v[132:135], a[64:67]\n\t" \
  "buffer_load_dwordx4 v31, s[40:43], s46 offen offset:1024 sc0 sc1 lds\n\t" \
  "v_cmp_le_u32 s[54:55], s65, v53\n\t" \
  "v_mfma_f32_16x16x32_bf16 a[68:71], v[84:87], v[140:143], a[68:71]\n\t" \
  "ds_read_b64_tr_b16 v[74:75], v19 offset:1024\n\t" \
  "s_nop 0\n\t" \
  "v_mfma_f32_16x16x32_bf16 a[68:71], v[80:83], v[136:139], a[68:71]\n\t" \
  "s_and_b64 vcc, vcc, s[54:55]\n\t" \
  "v_mfma_f32_16x16x32_bf16 a[72:75], v[80:83], v[144:147], a[72:75]\n\t" \
  "ds_read_b64_tr_b16 v[76:77], v19 offset:16384\n\t" \
  "s_or_b64 vcc, vcc, s[92:93]\n\t" \
  "v_mfma_f32_16x16x32_bf16 a[72:75], v[84:87], v[148:151], a[72:75]\n\t" \
  "s_cmp_eq_u64 vcc, -1\n\t" \
  "s_cselect_b32 s64, 0, s63\n\t" \
  "v_mfma_f32_16x16x32_bf16 a[76:79], v[84:87], v[156:159], a[76:79]\n\t" \
  "ds_read_b64_tr_b16 v[78:79], v19 offset:17408\n\t" \
  "v_mfma_f32_16x16x32_bf16 a[76:79], v[80:83], v[152:155], a[76:79]\n\t" \
  "buffer_load_dwordx4 v32, s[40:43], s46 offen offset:2048 sc0 sc1 lds\n\t" \
  "s_or_b32 s13, s23, s22\n\t" \
  "v_mfma_f32_16x16x32_bf16 a[80:83], v[80:83], v[160:163], a[80:83]\n\t" \
  "ds_read_b128 v[216:219], v16 offset:17408\n\t" \
  "s_or_b32 s28, s13, s64\n\t" \
  "v_mfma_f32_16x16x32_bf16 a[80:83], v[84:87], v[164:167], a[80:83]\n\t" \
  "v_add_u32 v19, s51, v19\n\t" \
  "s_add_u32 s57, s53, 1\n\t" \
  "v_mfma_f32_16x16x32_bf16 a[84:87], v[84:87], v[172:175], a[84:87]\n\t" \
  "s_and_b32 s57, s57, 7\n\t" \
  "v_mfma_f32_16x16x32_bf16 a[84:87], v[80:83], v[168:171], a[84:87]\n\t" \
  "ds_read_b128 v[220:223], v17 offset:17408\n\t" \
  "s_lshl_b32 s57, s57, 11\n\t" \
  "v_mfma_f32_16x16x32_bf16 a[88:91], v[80:83], v[176:179], a[88:91]\n\t" \
  "s_add_u32 s96, s90, s57\n\t" \
  "s_addc_u32 s97, s91, 0\n\t" \
  "v_mfma_f32_16x16x32_bf16 a[88:91], v[84:87], v[180:183], a[88:91]\n\t" \
  "buffer_load_dwordx4 v33, s[40:43], s46 offen offset:3072 sc0 sc1 lds\n\t" \
  "v_mfma_f32_16x16x32_bf16 a[92:95], v[84:87], v[188:191], a[92:95]\n\t" \
  "ds_read_b128 v[224:227], v16 offset:2048\n\t" \
  "global_load_dwordx2 v[52:53], v54, s[96:97] sc0 sc1\n\t" \
  "s_cmp_lg_u32 s13, 0\n\t" \
  "s_cbranch_scc1 OP3_%=\n\t" \
  "OPB3_%=:\n\t" \
  "v_mfma_f32_16x16x32_bf16 a[92:95], v[80:83], v[184:187], a[92:95]\n\t" \
  "s_waitcnt vmcnt(18)\n\t" \
  "v_mfma_f32_16x16x32_bf16 a[124:127], v[88:91], v[184:187], a[124:127]\n\t" \
  "ds_read_b64_tr_b16 v[80:81], v20 offset:0\n\t" \
  "v_mfma_f32_16x16x32_bf16 a[124:127], v[92:95], v[188:191], a[124:127]\n\t" \
  "s_mov_b64 exec, 0xffff\n\t" \
  "global_load_dwordx4 v[46:49], v50, s[74:75]\n\t" \
  "s_mov_b64 exec, -1\n\t" \
  "v_mfma_f32_16x16x32_bf16 a[120:123], v[92:95], v[180:183], a[120:123]\n\t" \
  "ds_read_b64_tr_b16 v[82:83], v20 offset:1024\n\t" \
  "v_mfma_f32_16x16x32_bf16 a[120:123], v[88:91], v[176:179], a[120:123]\n\t" \
  "s_mov_b64 exec, 0xffff\n\t" \
  "global_store_dwordx4 v50, v[42:45], s[76:77] sc0 sc1\n\t" \
  "s_mov_b64 exec, -1\n\t" \
  "v_mfma_f32_16x16x32_bf16 a[116:119], v[88:91], v[168:171], a[116:119]\n\t" \
  "ds_read_b64_tr_b16 v[84:85], v20 offset:16384\n\t" \
  "v_mfma_f32_16x16x32_bf16 a[116:119], v[92:95], v[172:175], a[116:119]\n\t" \
  "s_mov_b64 s[76:77], s[78:79]\n\t" \
  "s_mov_b64 s[80:81], s[82:83]\n\t" \
  "v_mfma_f32_16x16x32_bf16 a[112:115], v[92:95], v[164:167], a[112:115]\n\t" \
  "ds_read_b64_tr_b16 v[86:87], v20 offset:17408\n\t" \
  "v_mfma_f32_16x16x32_bf16 a[112:115], v[88:91], v[160:163], a[112:115]\n\t" \
  "s_mov_b64 s[82:83], s[84:85]\n\t" \
  "s_mov_b64 s[84:85], s[86:87]\n\t" \
  "v_mfma_f32_16x16x32_bf16 a[108:111], v[88:91], v[152:155], a[108:111]\n\t" \
  "ds_read_b128 v[228:231], v17 offset:2048\n\t" \
  "v_mfma_f32_16x16x32_bf16 a[108:111], v[92:95], v[156:159], a[108:111]\n\t" \
  "v_add_u32 v20, s51, v20\n\t" \
  "s_mov_b64 s[86:87], s[88:89]\n\t" \
  "v_mfma_f32_16x16x32_bf16 a[104:107], v[92:95], v[148:151], a[104:107]\n\t" \
  "s_mul_i32 s54, s67, 9363\n\t" \
  "v_mfma_f32_16x16x32_bf16 a[104:107], v[88:91], v[144:147], a[104:107]\n\t" \
  "ds_read_b128 v[232:235], v16 offset:18432\n\t" \
  "v_mfma_f32_16x16x32_bf16 a[100:103], v[88:91], v[136:139], a[100:103]\n\t" \
  "s_lshr_b32 s54, s54, 16\n\t" \
  "s_mul_i32 s55, s54, 7\n\t" \
  "v_mfma_f32_16x16x32_bf16 a[100:103], v[92:95], v[140:143], a[100:103]\n\t" \
  "s_sub_u32 s55, s67, s55\n\t" \
  "v_mfma_f32_16x16x32_bf16 a[96:99], v[92:95], v[132:135], a[96:99]\n\t" \
  "ds_read_b128 v[236:239], v17 offset:18432\n\t" \
  "s_add_u32 s55, s55, s58\n\t" \
  "v_mfma_f32_16x16x32_bf16 a[96:99], v[88:91], v[128:131], a[96:99]\n\t" \
  "s_add_u32 s55, s55, 1\n\t" \
  "v_mfma_f32_16x16x32_bf16 a[128:131], v[96:99], v[128:131], a[128:131]\n\t" \
  "ds_read_b64_tr_b16 v[88:89], v21 offset:0\n\t" \
  "v_mfma_f32_16x16x32_bf16 a[128:131], v[100:103], v[132:135], a[128:131]\n\t" \
  "s_mov_b32 m0, s49\n\t" \
  "s_and_b32 s55, s55, 7\n\t" \
  "v_mfma_f32_16x16x32_bf16 a[132:135], v[100:103], v[140:143], a[132:135]\n\t" \
  "ds_read_b64_tr_b16 v[90:91], v21 offset:1024\n\t" \
  "v_mfma_f32_16x16x32_bf16 a[132:135], v[96:99], v[136:139], a[132:135]\n\t" \
  "buffer_load_dwordx4 v34, s[36:39], s47 offen offset:0 lds\n\t" \
  "v_readlane_b32 s78, v10, s55\n\t" \
  "v_mfma_f32_16x16x32_bf16 a[136:139], v[96:99], v[144:147], a[136:139]\n\t" \
  "ds_read_b64_tr_b16 v[92:93], v21 offset:16384\n\t" \
  "v_mfma_f32_16x16x32_bf16 a[136:139], v[100:103], v[148:151], a[136:139]\n\t" \
  "v_readlane_b32 s79, v11, s55\n\t" \
  "s_lshl_b32 s56, s54, 13\n\t" \
  "v_mfma_f32_16x16x32_bf16 a[140:143], v[100:103], v[156:159], a[140:143]\n\t" \
  "ds_read_b64_tr_b16 v[94:95], v21 offset:17408\n\t" \
  "v_mfma_f32_16x16x32_bf16 a[140:143], v[96:99], v[152:155], a[140:143]\n\t" \
  "s_lshl_b32 s57, s68, 7\n\t" \
  "v_mfma_f32_16x16x32_bf16 a[144:147], v[96:99], v[160:163], a[144:147]\n\t" \
  "ds_read_b128 v[240:243], v16 offset:3072\n\t" \
  "s_add_u32 s56, s56, s57\n\t" \
  "v_mfma_f32_16x16x32_bf16 a[144:147], v[100:103], v[164:167], a[144:147]\n\t" \
  "v_add_u32 v21, s51, v21\n\t" \
  "buffer_load_dwordx4 v35, s[36:39], s47 offen offset:1024 lds\n\t" \
  "v_mfma_f32_16x16x32_bf16 a[148:151], v[100:103], v[172:175], a[148:151]\n\t" \
  "s_lshl_b32 s55, s55, 19\n\t" \
  "v_mfma_f32_16x16x32_bf16 a[148:151], v[96:99], v[168:171], a[148:151]\n\t" \
  "ds_read_b128 v[244:247], v17 offset:3072\n\t" \
  "s_add_u32 s55, s55, s56\n\t" \
  "v_mfma_f32_16x16x32_bf16 a[152:155], v[96:99], v[176:179], a[152:155]\n\t" \
  "s_add_u32 s74, s72, s55\n\t" \
  "v_mfma_f32_16x16x32_bf16 a[152:155], v[100:103], v[180:183], a[152:155]\n\t" \
  "s_addc_u32 s75, s73, 0\n\t" \
  "v_mfma_f32_16x16x32_bf16 a[156:159], v[100:103], v[188:191], a[156:159]\n\t" \
  "ds_read_b128 v[248:251], v16 offset:19456\n\t" \
  "s_lshl_b32 s57, s68, 11\n\t" \
  "v_mfma_f32_16x16x32_bf16 a[156:159], v[96:99], v[184:187], a[156:159]\n\t" \
  "buffer_load_dwordx4 v36, s[36:39], s47 offen offset:2048 lds\n\t" \
  "v_mfma_f32_16x16x32_bf16 a[188:191], v[104:107], v[184:187], a[188:191]\n\t" \
  "ds_read_b64_tr_b16 v[96:97], v22 offset:0\n\t" \
  "s_lshl_b32 s54, s54, 2\n\t" \
  "v_mfma_f32_16x16x32_bf16 a[188:191], v[108:111], v[188:191], a[188:191]\n\t" \
  "s_add_u32 s57, s57, s54\n\t" \
  "v_mfma_f32_16x16x32_bf16 a[184:187], v[108:111], v[180:183], a[184:187]\n\t" \
  "ds_read_b64_tr_b16 v[98:99], v22 offset:1024\n\t" \
  "v_mfma_f32_16x16x32_bf16 a[184:187], v[104:107], v[176:179], a[184:187]\n\t" \
  "s_add_u32 s57, s57, s70\n\t" \
  "s_add_u32 s88, s78, s57\n\t" \
  "v_mfma_f32_16x16x32_bf16 a[180:183], v[104:107], v[168:171], a[180:183]\n\t" \
  "ds_read_b64_tr_b16 v[100:101], v22 offset:16384\n\t" \
  "v_mfma_f32_16x16x32_bf16 a[180:183], v[108:111], v[172:175], a[180:183]\n\t" \
  "buffer_load_dwordx4 v37, s[36:39], s47 offen offset:3072 lds\n\t" \
  "s_addc_u32 s89, s79, 0\n\t" \
  "v_mfma_f32_16x16x32_bf16 a[176:179], v[108:111], v[164:167], a[176:179]\n\t" \
  "ds_read_b64_tr_b16 v[102:103], v22 offset:17408\n\t" \
  "v_mfma_f32_16x16x32_bf16 a[176:179], v[104:107], v[160:163], a[176:179]\n\t" \
  "s_add_u32 s56, s56, s69\n\t" \
  "s_add_u32 s78, s78, s56\n\t" \
  "v_mfma_f32_16x16x32_bf16 a[172:175], v[104:107], v[152:155], a[172:175]\n\t" \
  "ds_read_b128 v[252:255], v17 offset:19456\n\t" \
  "v_mfma_f32_16x16x32_bf16 a[172:175], v[108:111], v[156:159], a[172:175]\n\t" \
  "v_add_u32 v22, s51, v22\n\t" \
  "s_mov_b32 m0, s33\n\t" \
  "v_mfma_f32_16x16x32_bf16 a[168:171], v[108:111], v[148:151], a[168:171]\n\t" \
  "s_addc_u32 s79, s79, 0\n\t" \
  "v_mfma_f32_16x16x32_bf16 a[168:171], v[104:107], v[144:147], a[168:171]\n\t" \
  "buffer_load_dwordx4 v38, s[36:39], s47 offen offset:0 lds\n\t" \
  "s_add_u32 s54, s67, 256\n\t" \
  "v_mfma_f32_16x16x32_bf16 a[164:167], v[104:107], v[136:139], a[164:167]\n\t" \
  "s_cmp_ge_u32 s54, 448\n\t" \
  "v_mfma_f32_16x16x32_bf16 a[164:167], v[108:111], v[140:143], a[164:167]\n\t" \
  "s_cselect_b32 s55, 448, 0\n\t" \
  "s_cselect_b32 s56, 1, 0\n\t" \
  "v_mfma_f32_16x16x32_bf16 a[160:163], v[108:111], v[132:135], a[160:163]\n\t" \
  "s_sub_u32 s54, s54, s55\n\t" \
  "v_mfma_f32_16x16x32_bf16 a[160:163], v[104:107], v[128:131], a[160:163]\n\t" \
  "s_add_u32 s56, s68, s56\n\t" \
  "v_mfma_f32_16x16x32_bf16 a[192:195], v[112:115], v[128:131], a[192:195]\n\t" \
  "ds_read_b64_tr_b16 v[104:105], v23 offset:0\n\t" \
  "s_cmp_lt_u32 s56, 8\n\t" \
  "v_mfma_f32_16x16x32_bf16 a[192:195], v[116:119], v[132:135], a[192:195]\n\t" \
  "buffer_load_dwordx4 v39, s[36:39], s47 offen offset:1024 lds\n\t" \
  "v_mfma_f32_16x16x32_bf16 a[196:199], v[116:119], v[140:143], a[196:199]\n\t" \
  "ds_read_b64_tr_b16 v[106:107], v23 offset:1024\n\t" \
  "s_cselect_b32 s67, s54, s67\n\t" \
  "v_mfma_f32_16x16x32_bf16 a[196:199], v[112:115], v[136:139], a[196:199]\n\t" \
  "s_cselect_b32 s68, s56, s68\n\t" \
  "v_mfma_f32_16x16x32_bf16 a[200:203], v[112:115], v[144:147], a[200:203]\n\t" \
  "ds_read_b64_tr_b16 v[108:109], v23 offset:16384\n\t" \
  "v_mfma_f32_16x16x32_bf16 a[200:203], v[116:119], v[148:151], a[200:203]\n\t" \
  "v_mfma_f32_16x16x32_bf16 a[204:207], v[116:119], v[156:159], a[204:207]\n\t" \
  "ds_read_b64_tr_b16 v[110:111], v23 offset:17408\n\t" \
  "s_mov_b32 s46, s59\n\t" \
  "v_mfma_f32_16x16x32_bf16 a[204:207], v[112:115], v[152:155], a[204:207]\n\t" \
  "buffer_load_dwordx4 v40, s[36:39], s47 offen offset:2048 lds\n\t" \
  "v_mfma_f32_16x16x32_bf16 a[208:211], v[112:115], v[160:163], a[208:211]\n\t" \
  "s_cmp_lt_u32 s61, 8\n\t" \
  "v_mfma_f32_16x16x32_bf16 a[208:211], v[116:119], v[164:167], a[208:211]\n\t" \
  "v_add_u32 v23, s51, v23\n\t" \
  "v_mfma_f32_16x16x32_bf16 a[212:215], v[116:119], v[172:175], a[212:215]\n\t" \
  "s_cselect_b32 s40, s40, s44\n\t" \
  "v_mfma_f32_16x16x32_bf16 a[212:215], v[112:115], v[168:171], a[212:215]\n\t" \
  "v_xor_b32 v16, 0x8000, v16\n\t" \
  "s_cselect_b32 s41, s41, s45\n\t" \
  "v_mfma_f32_16x16x32_bf16 a[216:219], v[112:115], v[176:179], a[216:219]\n\t" \
  "v_xor_b32 v17, 0x8000, v17\n\t" \
  "v_mfma_f32_16x16x32_bf16 a[216:219], v[116:119], v[180:183], a[216:219]\n\t" \
  "buffer_load_dwordx4 v41, s[36:39], s47 offen offset:3072 lds\n\t" \
  "v_mfma_f32_16x16x32_bf16 a[220:223], v[116:119], v[188:191], a[220:223]\n\t" \
  "s_add_u32 s49, s62, s60\n\t" \
  "v_mfma_f32_16x16x32_bf16 a[220:223], v[112:115], v[184:187], a[220:223]\n\t" \
  "s_add_u32 s33, s49, 0x4000\n\t" \
  "v_mfma_f32_16x16x32_bf16 a[252:255], v[120:123], v[184:187], a[252:255]\n\t" \
  "ds_read_b64_tr_b16 v[112:113], v24 offset:0\n\t" \
  "s_xor_b32 s48, s48, 0x8000\n\t" \
  "v_mfma_f32_16x16x32_bf16 a[252:255], v[124:127], v[188:191], a[252:255]\n\t" \
  "ds_read_b64_tr_b16 v[114:115], v24 offset:1024\n\t" \
  "v_mfma_f32_16x16x32_bf16 a[248:251], v[124:127], v[180:183], a[248:251]\n\t" \
  "ds_read_b64_tr_b16 v[116:117], v24 offset:16384\n\t" \
  "v_mfma_f32_16x16x32_bf16 a[248:251], v[120:123], v[176:179], a[248:251]\n\t" \
  "ds_read_b64_tr_b16 v[118:119], v24 offset:17408\n\t" \
  "v_mfma_f32_16x16x32_bf16 a[244:247], v[120:123], v[168:171], a[244:247]\n\t" \
  "s_add_u32 s32, s48, 0x1000\n\t" \
  "v_mfma_f32_16x16x32_bf16 a[244:247], v[124:127], v[172:175], a[244:247]\n\t" \
  "v_add_u32 v24, s51, v24\n\t" \
  "v_mfma_f32_16x16x32_bf16 a[240:243], v[124:127], v[164:167], a[240:243]\n\t" \
  "s_add_u32 s60, s60, s51\n\t" \
  "v_mfma_f32_16x16x32_bf16 a[240:243], v[120:123], v[160:163], a[240:243]\n\t" \
  "s_cmp_eq_u32 s60, 0x10000\n\t" \
  "v_mfma_f32_16x16x32_bf16 a[236:239], v[120:123], v[152:155], a[236:239]\n\t" \
  "s_cselect_b32 s51, 0xffff0000, s71\n\t" \
  "v_mfma_f32_16x16x32_bf16 a[236:239], v[124:127], v[156:159], a[236:239]\n\t" \
  "v_mfma_f32_16x16x32_bf16 a[232:235], v[124:127], v[148:151], a[232:235]\n\t" \
  "s_sub_u32 s50, s50, 1\n\t" \
  "v_mfma_f32_16x16x32_bf16 a[232:235], v[120:123], v[144:147], a[232:235]\n\t" \
  "s_cmp_lg_u32 s50, 0\n\t" \
  "v_mfma_f32_16x16x32_bf16 a[228:231], v[120:123], v[136:139], a[228:231]\n\t" \
  "v_mfma_f32_16x16x32_bf16 a[228:231], v[124:127], v[140:143], a[228:231]\n\t" \
  "s_cselect_b64 vcc, -1, 0\n\t" \
  "v_mfma_f32_16x16x32_bf16 a[224:227], v[124:127], v[132:135], a[224:227]\n\t" \
  "s_cmp_lg_u32 s28, 0\n\t" \
  "v_mfma_f32_16x16x32_bf16 a[224:227], v[120:123], v[128:131], a[224:227]\n\t" \
  "s_waitcnt vmcnt(10) lgkmcnt(0)\n\t" \
  "s_cbranch_scc1 OE3_%=\n\t" \
  "OEB3_%=:\n\t" \
  "s_barrier\n\t" \
  "s_cbranch_vccnz LOOP1_%=\n\t" \
  "s_mov_b32 s50, 23\n\t" \
  "LOOP2_%=:\n\t" \
  "v_mfma_f32_16x16x32_bf16 a[0:3], v[64:67], v[192:195], a[0:3]\n\t" \
  "ds_read_b64_tr_b16 v[120:121], v25 offset:0\n\t" \
  "s_add_u32 s61, s61, 1\n\t" \
  "v_mfma_f32_16x16x32_bf16 a[0:3], v[68:71], v[196:199], a[0:3]\n\t" \
  "s_mov_b32 m0, s48\n\t" \
  "v_mfma_f32_16x16x32_bf16 a[4:7], v[68:71], v[204:207], a[4:7]\n\t" \
  "ds_read_b64_tr_b16 v[122:123], v25 offset:1024\n\t" \
  "s_lshl_b32 s54, -2, s52\n\t" \
  "v_mfma_f32_16x16x32_bf16 a[4:7], v[64:67], v[200:203], a[4:7]\n\t" \
  "buffer_load_dwordx4 v26, s[40:43], s46 offen offset:0 sc0 sc1 lds\n\t" \
  "v_mfma_f32_16x16x32_bf16 a[8:11], v[64:67], v[208:211], a[8:11]\n\t" \
  "ds_read_b64_tr_b16 v[124:125], v25 offset:16384\n\t" \
  "v_mfma_f32_16x16x32_bf16 a[8:11], v[68:71], v[212:215], a[8:11]\n\t" \
  "s_and_b32 s54, s54, s24\n\t" \
  "s_cselect_b32 s63, 0, 1\n\t" \
  "v_mfma_f32_16x16x32_bf16 a[12:15], v[68:71], v[220:223], a[12:15]\n\t" \
  "ds_read_b64_tr_b16 v[126:127], v25 offset:17408\n\t" \
  "v_mfma_f32_16x16x32_bf16 a[12:15], v[64:67], v[216:219], a[12:15]\n\t" \
  "s_cselect_b32 s54, s54, s24\n\t" \
  "s_add_u32 s53, s53, s63\n\t" \
  "v_mfma_f32_16x16x32_bf16 a[16:19], v[64:67], v[224:227], a[16:19]\n\t" \
  "ds_read_b128 v[128:131], v16 offset:0\n\t" \
  "v_mfma_f32_16x16x32_bf16 a[16:19], v[68:71], v[228:231], a[16:19]\n\t" \
  "v_add_u32 v25, s51, v25\n\t" \
  "buffer_load_dwordx4 v27, s[40:43], s46 offen offset:1024 sc0 sc1 lds\n\t" \
  "v_mfma_f32_16x16x32_bf16 a[20:23], v[68:71], v[236:239], a[20:23]\n\t" \
  "s_bcnt1_i32_b32 s55, s21\n\t" \
  "v_mfma_f32_16x16x32_bf16 a[20:23], v[64:67], v[232:235], a[20:23]\n\t" \
  "ds_read_b128 v[132:135], v17 offset:0\n\t" \
  "s_lshl_b32 s55, s55, 3\n\t" \
  "v_mfma_f32_16x16x32_bf16 a[24:27], v[64:67], v[240:243], a[24:27]\n\t" \
  "s_cmp_eq_u32 s53, 8\n\t" \
  "v_mfma_f32_16x16x32_bf16 a[24:27], v[68:71], v[244:247], a[24:27]\n\t" \
  "s_cselect_b32 s24, s21, s24\n\t" \
  "s_cselect_b32 s54, s21, s54\n\t" \
  "v_mfma_f32_16x16x32_bf16 a[28:31], v[68:71], v[252:255], a[28:31]\n\t" \
  "ds_read_b128 v[136:139], v16 offset:16384\n\t" \
  "v_mfma_f32_16x16x32_bf16 a[28:31], v[64:67], v[248:251], a[28:31]\n\t" \
  "buffer_load_dwordx4 v28, s[40:43], s46 offen offset:2048 sc0 sc1 lds\n\t" \
  "v_mfma_f32_16x16x32_bf16 a[60:63], v[72:75], v[248:251], a[60:63]\n\t" \
  "ds_read_b64_tr_b16 v[64:65], v18 offset:0\n\t" \
  "s_cselect_b32 s53, 0, s53\n\t" \
  "v_mfma_f32_16x16x32_bf16 a[60:63], v[76:79], v[252:255], a[60:63]\n\t" \
  "s_cselect_b32 s63, 0, s63\n\t" \
  "v_mfma_f32_16x16x32_bf16 a[56:59], v[76:79], v[244:247], a[56:59]\n\t" \
  "ds_read_b64_tr_b16 v[66:67], v18 offset:1024\n\t" \
  "s_cselect_b32 s19, s55, s19\n\t" \
  "v_mfma_f32_16x16x32_bf16 a[56:59], v[72:75], v[240:243], a[56:59]\n\t" \
  "s_sub_u32 s19, s19, 1\n\t" \
  "v_mfma_f32_16x16x32_bf16 a[52:55], v[72:75], v[232:235], a[52:55]\n\t" \
  "ds_read_b64_tr_b16 v[68:69], v18 offset:16384\n\t" \
  "v_mfma_f32_16x16x32_bf16 a[52:55], v[76:79], v[236:239], a[52:55]\n\t" \
  "buffer_load_dwordx4 v29, s[40:43], s46 offen offset:3072 sc0 sc1 lds\n\t" \
  "s_cmp_eq_u32 s19, 0\n\t" \
  "v_mfma_f32_16x16x32_bf16 a[48:51], v[76:79], v[228:231], a[48:51]\n\t" \
  "ds_read_b64_tr_b16 v[70:71], v18 offset:17408\n\t" \
  "v_mfma_f32_16x16x32_bf16 a[48:51], v[72:75], v[224:227], a[48:51]\n\t" \
  "s_cselect_b32 s23, s25, 0\n\t" \
  "v_mfma_f32_16x16x32_bf16 a[44:47], v[72:75], v[216:219], a[44:47]\n\t" \
  "ds_read_b128 v[140:143], v17 offset:16384\n\t" \
  "s_cmp_le_u32 s19, s16\n\t" \
  "v_mfma_f32_16x16x32_bf16 a[44:47], v[76:79], v[220:223], a[44:47]\n\t" \
  "v_add_u32 v18, s51, v18\n\t" \
  "s_mov_b32 m0, s32\n\t" \
  "v_mfma_f32_16x16x32_bf16 a[40:43], v[76:79], v[212:215], a[40:43]\n\t" \
  "s_cselect_b32 s22, s25, 0\n\t" \
  "v_mfma_f32_16x16x32_bf16 a[40:43], v[72:75], v[208:211], a[40:43]\n\t" \
  "ds_read_b128 v[144:147], v16 offset:1024\n\t" \
  "buffer_load_dwordx4 v30, s[40:43], s46 offen offset:0 sc0 sc1 lds\n\t" \
  "v_mfma_f32_16x16x32_bf16 a[36:39], v[72:75], v[200:203], a[36:39]\n\t" \
  "s_cmp_eq_u32 s12, s18\n\t" \
  "s_cselect_b32 s22, s22, 0\n\t" \
  "v_mfma_f32_16x16x32_bf16 a[36:39], v[76:79], v[204:207], a[36:39]\n\t" \
  "s_and_b32 s22, s22, s17\n\t" \
  "v_mfma_f32_16x16x32_bf16 a[32:35], v[76:79], v[196:199], a[32:35]\n\t" \
  "ds_read_b128 v[148:151], v17 offset:1024\n\t" \
  "v_mfma_f32_16x16x32_bf16 a[32:35], v[72:75], v[192:195], a[32:35]\n\t" \
  "s_ff1_i32_b32 s52, s54\n\t" \
  "s_and_b32 s52, s52, 7\n\t" \
  "v_mfma_f32_16x16x32_bf16 a[64:67], v[80:83], v[192:195], a[64:67]\n\t" \
  "ds_read_b64_tr_b16 v[72:73], v19 offset:0\n\t" \
  "v_mfma_f32_16x16x32_bf16 a[64:67], v[84:87], v[196:199], a[64:67]\n\t" \
  "buffer_load_dwordx4 v31, s[40:43], s46 offen offset:1024 sc0 sc1 lds\n\t" \
  "s_lshl_b32 s56, s52, 19\n\t" \
  "v_mfma_f32_16x16x32_bf16 a[68:71], v[84:87], v[204:207], a[68:71]\n\t" \
  "ds_read_b64_tr_b16 v[74:75], v19 offset:1024\n\t" \
  "v_mfma_f32_16x16x32_bf16 a[68:71], v[80:83], v[200:203], a[68:71]\n\t" \
  "s_cmp_lt_u32 s61, 8\n\t" \
  "v_mfma_f32_16x16x32_bf16 a[72:75], v[80:83], v[208:211], a[72:75]\n\t" \
  "ds_read_b64_tr_b16 v[76:77], v19 offset:16384\n\t" \
  "s_cselect_b32 s63, 0, s63\n\t" \
  "v_mfma_f32_16x16x32_bf16 a[72:75], v[84:87], v[212:215], a[72:75]\n\t" \
  "s_cselect_b32 s56, 0, s56\n\t" \
  "v_mfma_f32_16x16x32_bf16 a[76:79], v[84:87], v[220:223], a[76:79]\n\t" \
  "ds_read_b64_tr_b16 v[78:79], v19 offset:17408\n\t" \
  "s_lshl_b32 s54, s52, 25\n\t" \
  "v_mfma_f32_16x16x32_bf16 a[76:79], v[80:83], v[216:219], a[76:79]\n\t" \
  "buffer_load_dwordx4 v32, s[40:43], s46 offen offset:2048 sc0 sc1 lds\n\t" \
  "v_mfma_f32_16x16x32_bf16 a[80:83], v[80:83], v[224:227], a[80:83]\n\t" \
  "ds_read_b128 v[152:155], v16 offset:17408\n\t" \
  "v_mfma_f32_16x16x32_bf16 a[80:83], v[84:87], v[228:231], a[80:83]\n\t" \
  "v_add_u32 v19, s51, v19\n\t" \
  "s_lshl_b32 s55, s53, 22\n\t" \
  "v_mfma_f32_16x16x32_bf16 a[84:87], v[84:87], v[236:239], a[84:87]\n\t" \
  "s_or_b32 s47, s54, s55\n\t" \
  "v_mfma_f32_16x16x32_bf16 a[84:87], v[80:83], v[232:235], a[84:87]\n\t" \
  "ds_read_b128 v[156:159], v17 offset:17408\n\t" \
  "v_mfma_f32_16x16x32_bf16 a[88:91], v[80:83], v[240:243], a[88:91]\n\t" \
  "s_lshl_b32 s55, s53, 7\n\t" \
  "s_or_b32 s59, s56, s55\n\t" \
  "v_mfma_f32_16x16x32_bf16 a[88:91], v[84:87], v[244:247], a[88:91]\n\t" \
  "buffer_load_dwordx4 v33, s[40:43], s46 offen offset:3072 sc0 sc1 lds\n\t" \
  "v_mfma_f32_16x16x32_bf16 a[92:95], v[84:87], v[252:255], a[92:95]\n\t" \
  "ds_read_b128 v[160:163], v16 offset:2048\n\t" \
  "v_cmp_le_u32 vcc, s65, v52\n\t" \
  "v_mfma_f32_16x16x32_bf16 a[92:95], v[80:83], v[248:251], a[92:95]\n\t" \
  "v_cmp_le_u32 s[54:55], s65, v53\n\t" \
  "v_mfma_f32_16x16x32_bf16 a[124:127], v[88:91], v[248:251], a[124:127]\n\t" \
  "ds_read_b64_tr_b16 v[80:81], v20 offset:0\n\t" \
  "v_mfma_f32_16x16x32_bf16 a[124:127], v[92:95], v[252:255], a[124:127]\n\t" \
  "s_nop 0\n\t" \
  "s_and_b64 vcc, vcc, s[54:55]\n\t" \
  "v_mfma_f32_16x16x32_bf16 a[120:123], v[92:95], v[244:247], a[120:123]\n\t" \
  "ds_read_b64_tr_b16 v[82:83], v20 offset:1024\n\t" \
  "v_mfma_f32_16x16x32_bf16 a[120:123], v[88:91], v[240:243], a[120:123]\n\t" \
  "s_or_b64 vcc, vcc, s[92:93]\n\t" \
  "s_cmp_eq_u64 vcc, -1\n\t" \
  "v_mfma_f32_16x16x32_bf16 a[116:119], v[88:91], v[232:235], a[116:119]\n\t" \
  "ds_read_b64_tr_b16 v[84:85], v20 offset:16384\n\t" \
  "v_mfma_f32_16x16x32_bf16 a[116:119], v[92:95], v[236:239], a[116:119]\n\t" \
  "s_cselect_b32 s64, 0, s63\n\t" \
  "v_mfma_f32_16x16x32_bf16 a[112:115], v[92:95], v[228:231], a[112:115]\n\t" \
  "ds_read_b64_tr_b16 v[86:87], v20 offset:17408\n\t" \
  "s_or_b32 s13, s23, s22\n\t" \
  "v_mfma_f32_16x16x32_bf16 a[112:115], v[88:91], v[224:227], a[112:115]\n\t" \
  "s_or_b32 s28, s13, s64\n\t" \
  "v_mfma_f32_16x16x32_bf16 a[108:111], v[88:91], v[216:219], a[108:111]\n\t" \
  "ds_read_b128 v[164:167], v17 offset:2048\n\t" \
  "s_add_u32 s57, s53, 1\n\t" \
  "v_mfma_f32_16x16x32_bf16 a[108:111], v[92:95], v[220:223], a[108:111]\n\t" \
  "v_add_u32 v20, s51, v20\n\t" \
  "v_mfma_f32_16x16x32_bf16 a[104:107], v[92:95], v[212:215], a[104:107]\n\t" \
  "s_and_b32 s57, s57, 7\n\t" \
  "v_mfma_f32_16x16x32_bf16 a[104:107], v[88:91], v[208:211], a[104:107]\n\t" \
  "ds_read_b128 v[168:171], v16 offset:18432\n\t" \
  "s_lshl_b32 s57, s57, 11\n\t" \
  "v_mfma_f32_16x16x32_bf16 a[100:103], v[88:91], v[200:203], a[100:103]\n\t" \
  "s_add_u32 s96, s90, s57\n\t" \
  "v_mfma_f32_16x16x32_bf16 a[100:103], v[92:95], v[204:207], a[100:103]\n\t" \
  "s_addc_u32 s97, s91, 0\n\t" \
  "v_mfma_f32_16x16x32_bf16 a[96:99], v[92:95], v[196:199], a[96:99]\n\t" \
  "ds_read_b128 v[172:175], v17 offset:18432\n\t" \
  "global_load_dwordx2 v[52:53], v54, s[96:97] sc0 sc1\n\t" \
  "v_mfma_f32_16x16x32_bf16 a[96:99], v[88:91], v[192:195], a[96:99]\n\t" \
  "s_cmp_lg_u32 s13, 0\n\t" \
  "s_cbranch_scc1 OP4_%=\n\t" \
  "OPB4_%=:\n\t" \
  "v_mfma_f32_16x16x32_bf16 a[128:131], v[96:99], v[192:195], a[128:131]\n\t" \
  "ds_read_b64_tr_b16 v[88:89], v21 offset:0\n\t" \
  "v_mfma_f32_16x16x32_bf16 a[128:131], v[100:103], v[196:199], a[128:131]\n\t" \
  "s_mov_b32 m0, s49\n\t" \
  "v_mfma_f32_16x16x32_bf16 a[132:135], v[100:103], v[204:207], a[132:135]\n\t" \
  "ds_read_b64_tr_b16 v[90:91], v21 offset:1024\n\t" \
  "v_mfma_f32_16x16x32_bf16 a[132:135], v[96:99], v[200:203], a[132:135]\n\t" \
  "buffer_load_dwordx4 v34, s[36:39], s47 offen offset:0 lds\n\t" \
  "v_mfma_f32_16x16x32_bf16 a[136:139], v[96:99], v[208:211], a[136:139]\n\t" \
  "ds_read_b64_tr_b16 v[92:93], v21 offset:16384\n\t" \
  "v_mfma_f32_16x16x32_bf16 a[136:139], v[100:103], v[212:215], a[136:139]\n\t" \
  "v_mfma_f32_16x16x32_bf16 a[140:143], v[100:103], v[220:223], a[140:143]\n\t" \
  "ds_read_b64_tr_b16 v[94:95], v21 offset:17408\n\t" \
  "v_mfma_f32_16x16x32_bf16 a[140:143], v[96:99], v[216:219], a[140:143]\n\t" \
  "v_mfma_f32_16x16x32_bf16 a[144:147], v[96:99], v[224:227], a[144:147]\n\t" \
  "ds_read_b128 v[176:179], v16 offset:3072\n\t" \
  "v_mfma_f32_16x16x32_bf16 a[144:147], v[100:103], v[228:231], a[144:147]\n\t" \
  "v_add_u32 v21, s51, v21\n\t" \
  "buffer_load_dwordx4 v35, s[36:39], s47 offen offset:1024 lds\n\t" \
  "v_mfma_f32_16x16x32_bf16 a[148:151], v[100:103], v[236:239], a[148:151]\n\t" \
  "v_mfma_f32_16x16x32_bf16 a[148:151], v[96:99], v[232:235], a[148:151]\n\t" \
  "ds_read_b128 v[180:183], v17 offset:3072\n\t" \
  "v_mfma_f32_16x16x32_bf16 a[152:155], v[96:99], v[240:243], a[152:155]\n\t" \
  "v_mfma_f32_16x16x32_bf16 a[152:155], v[100:103], v[244:247], a[152:155]\n\t" \
  "v_mfma_f32_16x16x32_bf16 a[156:159], v[100:103], v[252:255], a[156:159]\n\t" \
  "ds_read_b128 v[184:187], v16 offset:19456\n\t" \
  "v_mfma_f32_16x16x32_bf16 a[156:159], v[96:99], v[248:251], a[156:159]\n\t" \
  "buffer_load_dwordx4 v36, s[36:39], s47 offen offset:2048 lds\n\t" \
  "v_mfma_f32_16x16x32_bf16 a[188:191], v[104:107], v[248:251], a[188:191]\n\t" \
  "ds_read_b64_tr_b16 v[96:97], v22 offset:0\n\t" \
  "v_mfma_f32_16x16x32_bf16 a[188:191], v[108:111], v[252:255], a[188:191]\n\t" \
  "v_mfma_f32_16x16x32_bf16 a[184:187], v[108:111], v[244:247], a[184:187]\n\t" \
  "ds_read_b64_tr_b16 v[98:99], v22 offset:1024\n\t" \
  "v_mfma_f32_16x16x32_bf16 a[184:187], v[104:107], v[240:243], a[184:187]\n\t" \
  "v_mfma_f32_16x16x32_bf16 a[180:183], v[104:107], v[232:235], a[180:183]\n\t" \
  "ds_read_b64_tr_b16 v[100:101], v22 offset:16384\n\t" \
  "v_mfma_f32_16x16x32_bf16 a[180:183], v[108:111], v[236:239], a[180:183]\n\t" \
  "buffer_load_dwordx4 v37, s[36:39], s47 offen offset:3072 lds\n\t" \
  "v_mfma_f32_16x16x32_bf16 a[176:179], v[108:111], v[228:231], a[176:179]\n\t" \
  "ds_read_b64_tr_b16 v[102:103], v22 offset:17408\n\t" \
  "v_mfma_f32_16x16x32_bf16 a[176:179], v[104:107], v[224:227], a[176:179]\n\t" \
  "v_mfma_f32_16x16x32_bf16 a[172:175], v[104:107], v[216:219], a[172:175]\n\t" \
  "ds_read_b128 v[188:191], v17 offset:19456\n\t" \
  "v_mfma_f32_16x16x32_bf16 a[172:175], v[108:111], v[220:223], a[172:175]\n\t" \
  "v_add_u32 v22, s51, v22\n\t" \
  "s_mov_b32 m0, s33\n\t" \
  "v_mfma_f32_16x16x32_bf16 a[168:171], v[108:111], v[212:215], a[168:171]\n\t" \
  "v_mfma_f32_16x16x32_bf16 a[168:171], v[104:107], v[208:211], a[168:171]\n\t" \
  "buffer_load_dwordx4 v38, s[36:39], s47 offen offset:0 lds\n\t" \
  "v_mfma_f32_16x16x32_bf16 a[164:167], v[104:107], v[200:203], a[164:167]\n\t" \
  "s_mov_b32 s46, s59\n\t" \
  "v_mfma_f32_16x16x32_bf16 a[164:167], v[108:111], v[204:207], a[164:167]\n\t" \
  "s_cmp_lt_u32 s61, 8\n\t" \
  "v_mfma_f32_16x16x32_bf16 a[160:163], v[108:111], v[196:199], a[160:163]\n\t" \
  "s_cselect_b32 s40, s40, s44\n\t" \
  "v_mfma_f32_16x16x32_bf16 a[160:163], v[104:107], v[192:195], a[160:163]\n\t" \
  "v_mfma_f32_16x16x32_bf16 a[192:195], v[112:115], v[192:195], a[192:195]\n\t" \
  "ds_read_b64_tr_b16 v[104:105], v23 offset:0\n\t" \
  "v_mfma_f32_16x16x32_bf16 a[192:195], v[116:119], v[196:199], a[192:195]\n\t" \
  "buffer_load_dwordx4 v39, s[36:39], s47 offen offset:1024 lds\n\t" \
  "v_mfma_f32_16x16x32_bf16 a[196:199], v[116:119], v[204:207], a[196:199]\n\t" \
  "ds_read_b64_tr_b16 v[106:107], v23 offset:1024\n\t" \
  "v_mfma_f32_16x16x32_bf16 a[196:199], v[112:115], v[200:203], a[196:199]\n\t" \
  "s_cselect_b32 s41, s41, s45\n\t" \
  "v_mfma_f32_16x16x32_bf16 a[200:203], v[112:115], v[208:211], a[200:203]\n\t" \
  "ds_read_b64_tr_b16 v[108:109], v23 offset:16384\n\t" \
  "v_mfma_f32_16x16x32_bf16 a[200:203], v[116:119], v[212:215], a[200:203]\n\t" \
  "s_add_u32 s49, s62, s60\n\t" \
  "v_mfma_f32_16x16x32_bf16 a[204:207], v[116:119], v[220:223], a[204:207]\n\t" \
  "ds_read_b64_tr_b16 v[110:111], v23 offset:17408\n\t" \
  "v_mfma_f32_16x16x32_bf16 a[204:207], v[112:115], v[216:219], a[204:207]\n\t" \
  "buffer_load_dwordx4 v40, s[36:39], s47 offen offset:2048 lds\n\t" \
  "v_mfma_f32_16x16x32_bf16 a[208:211], v[112:115], v[224:227], a[208:211]\n\t" \
  "s_add_u32 s33, s49, 0x4000\n\t" \
  "v_mfma_f32_16x16x32_bf16 a[208:211], v[116:119], v[228:231], a[208:211]\n\t" \
  "v_add_u32 v23, s51, v23\n\t" \
  "v_mfma_f32_16x16x32_bf16 a[212:215], v[116:119], v[236:239], a[212:215]\n\t" \
  "v_mfma_f32_16x16x32_bf16 a[212:215], v[112:115], v[232:235], a[212:215]\n\t" \
  "v_xor_b32 v16, 0x8000, v16\n\t" \
  "v_mfma_f32_16x16x32_bf16 a[216:219], v[112:115], v[240:243], a[216:219]\n\t" \
  "v_xor_b32 v17, 0x8000, v17\n\t" \
  "v_mfma_f32_16x16x32_bf16 a[216:219], v[116:119], v[244:247], a[216:219]\n\t" \
  "buffer_load_dwordx4 v41, s[36:39], s47 offen offset:3072 lds\n\t" \
  "v_mfma_f32_16x16x32_bf16 a[220:223], v[116:119], v[252:255], a[220:223]\n\t" \
  "s_xor_b32 s48, s48, 0x8000\n\t" \
  "v_mfma_f32_16x16x32_bf16 a[220:223], v[112:115], v[248:251], a[220:223]\n\t" \
  "s_add_u32 s32, s48, 0x1000\n\t" \
  "v_mfma_f32_16x16x32_bf16 a[252:255], v[56:59], v[248:251], a[252:255]\n\t" \
  "ds_read_b64_tr_b16 v[112:113], v24 offset:0\n\t" \
  "v_mfma_f32_16x16x32_bf16 a[252:255], v[60:63], v[252:255], a[252:255]\n\t" \
  "ds_read_b64_tr_b16 v[114:115], v24 offset:1024\n\t" \
  "v_mfma_f32_16x16x32_bf16 a[248:251], v[60:63], v[244:247], a[248:251]\n\t" \
  "ds_read_b64_tr_b16 v[116:117], v24 offset:16384\n\t" \
  "v_mfma_f32_16x16x32_bf16 a[248:251], v[56:59], v[240:243], a[248:251]\n\t" \
  "ds_read_b64_tr_b16 v[118:119], v24 offset:17408\n\t" \
  "v_mfma_f32_16x16x32_bf16 a[244:247], v[56:59], v[232:235], a[244:247]\n\t" \
  "v_mfma_f32_16x16x32_bf16 a[244:247], v[60:63], v[236:239], a[244:247]\n\t" \
  "v_add_u32 v24, s51, v24\n\t" \
  "v_mfma_f32_16x16x32_bf16 a[240:243], v[60:63], v[228:231], a[240:243]\n\t" \
  "s_add_u32 s60, s60, s51\n\t" \
  "v_mfma_f32_16x16x32_bf16 a[240:243], v[56:59], v[224:227], a[240:243]\n\t" \
  "v_mfma_f32_16x16x32_bf16 a[236:239], v[56:59], v[216:219], a[236:239]\n\t" \
  "s_cmp_eq_u32 s60, 0x10000\n\t" \
  "v_mfma_f32_16x16x32_bf16 a[236:239], v[60:63], v[220:223], a[236:239]\n\t" \
  "v_mfma_f32_16x16x32_bf16 a[232:235], v[60:63], v[212:215], a[232:235]\n\t" \
  "v_mfma_f32_16x16x32_bf16 a[232:235], v[56:59], v[208:211], a[232:235]\n\t" \
  "s_cselect_b32 s51, 0xffff0000, s71\n\t" \
  "v_mfma_f32_16x16x32_bf16 a[228:231], v[56:59], v[200:203], a[228:231]\n\t" \
  "v_mfma_f32_16x16x32_bf16 a[228:231], v[60:63], v[204:207], a[228:231]\n\t" \
  "s_cmp_lg_u32 s28, 0\n\t" \
  "v_mfma_f32_16x16x32_bf16 a[224:227], v[60:63], v[196:199], a[224:227]\n\t" \
  "v_mfma_f32_16x16x32_bf16 a[224:227], v[56:59], v[192:195], a[224:227]\n\t" \
  "s_waitcnt vmcnt(8) lgkmcnt(0)\n\t" \
  "s_cbranch_scc1 OE4_%=\n\t" \
  "OEB4_%=:\n\t" \
  "s_barrier\n\t" \
  "v_mfma_f32_16x16x32_bf16 a[0:3], v[64:67], v[128:131], a[0:3]\n\t" \
  "ds_read_b64_tr_b16 v[56:57], v25 offset:0\n\t" \
  "s_add_u32 s61, s61, 1\n\t" \
  "v_mfma_f32_16x16x32_bf16 a[0:3], v[68:71], v[132:135], a[0:3]\n\t" \
  "s_mov_b32 m0, s48\n\t" \
  "v_mfma_f32_16x16x32_bf16 a[4:7], v[68:71], v[140:143], a[4:7]\n\t" \
  "ds_read_b64_tr_b16 v[58:59], v25 offset:1024\n\t" \
  "s_lshl_b32 s54, -2, s52\n\t" \
  "v_mfma_f32_16x16x32_bf16 a[4:7], v[64:67], v[136:139], a[4:7]\n\t" \
  "buffer_load_dwordx4 v26, s[40:43], s46 offen offset:0 sc0 sc1 lds\n\t" \
  "v_mfma_f32_16x16x32_bf16 a[8:11], v[64:67], v[144:147], a[8:11]\n\t" \
  "ds_read_b64_tr_b16 v[60:61], v25 offset:16384\n\t" \
  "v_mfma_f32_16x16x32_bf16 a[8:11], v[68:71], v[148:151], a[8:11]\n\t" \
  "s_and_b32 s54, s54, s24\n\t" \
  "s_cselect_b32 s63, 0, 1\n\t" \
  "v_mfma_f32_16x16x32_bf16 a[12:15], v[68:71], v[156:159], a[12:15]\n\t" \
  "ds_read_b64_tr_b16 v[62:63], v25 offset:17408\n\t" \
  "v_mfma_f32_16x16x32_bf16 a[12:15], v[64:67], v[152:155], a[12:15]\n\t" \
  "s_cselect_b32 s54, s54, s24\n\t" \
  "s_add_u32 s53, s53, s63\n\t" \
  "v_mfma_f32_16x16x32_bf16 a[16:19], v[64:67], v[160:163], a[16:19]\n\t" \
  "ds_read_b128 v[192:195], v16 offset:0\n\t" \
  "v_mfma_f32_16x16x32_bf16 a[16:19], v[68:71], v[164:167], a[16:19]\n\t" \
  "v_add_u32 v25, s51, v25\n\t" \
  "buffer_load_dwordx4 v27, s[40:43], s46 offen offset:1024 sc0 sc1 lds\n\t" \
  "v_mfma_f32_16x16x32_bf16 a[20:23], v[68:71], v[172:175], a[20:23]\n\t" \
  "s_bcnt1_i32_b32 s55, s21\n\t" \
  "v_mfma_f32_16x16x32_bf16 a[20:23], v[64:67], v[168:171], a[20:23]\n\t" \
  "ds_read_b128 v[196:199], v17 offset:0\n\t" \
  "s_lshl_b32 s55, s55, 3\n\t" \
  "v_mfma_f32_16x16x32_bf16 a[24:27], v[64:67], v[176:179], a[24:27]\n\t" \
  "s_cmp_eq_u32 s53, 8\n\t" \
  "v_mfma_f32_16x16x32_bf16 a[24:27], v[68:71], v[180:183], a[24:27]\n\t" \
  "s_cselect_b32 s24, s21, s24\n\t" \
  "s_cselect_b32 s54, s21, s54\n\t" \
  "v_mfma_f32_16x16x32_bf16 a[28:31], v[68:71], v[188:191], a[28:31]\n\t" \
  "ds_read_b128 v[200:203], v16 offset:16384\n\t" \
  "v_mfma_f32_16x16x32_bf16 a[28:31], v[64:67], v[184:187], a[28:31]\n\t" \
  "buffer_load_dwordx4 v28, s[40:43], s46 offen offset:2048 sc0 sc1 lds\n\t" \
  "v_mfma_f32_16x16x32_bf16 a[60:63], v[72:75], v[184:187], a[60:63]\n\t" \
  "ds_read_b64_tr_b16 v[64:65], v18 offset:0\n\t" \
  "s_cselect_b32 s53, 0, s53\n\t" \
  "v_mfma_f32_16x16x32_bf16 a[60:63], v[76:79], v[188:191], a[60:63]\n\t" \
  "s_cselect_b32 s63, 0, s63\n\t" \
  "v_mfma_f32_16x16x32_bf16 a[56:59], v[76:79], v[180:183], a[56:59]\n\t" \
  "ds_read_b64_tr_b16 v[66:67], v18 offset:1024\n\t" \
  "s_cselect_b32 s19, s55, s19\n\t" \
  "v_mfma_f32_16x16x32_bf16 a[56:59], v[72:75], v[176:179], a[56:59]\n\t" \
  "s_sub_u32 s19, s19, 1\n\t" \
  "v_mfma_f32_16x16x32_bf16 a[52:55], v[72:75], v[168:171], a[52:55]\n\t" \
  "ds_read_b64_tr_b16 v[68:69], v18 offset:16384\n\t" \
  "v_mfma_f32_16x16x32_bf16 a[52:55], v[76:79], v[172:175], a[52:55]\n\t" \
  "buffer_load_dwordx4 v29, s[40:43], s46 offen offset:3072 sc0 sc1 lds\n\t" \
  "s_cmp_eq_u32 s19, 0\n\t" \
  "v_mfma_f32_16x16x32_bf16 a[48:51], v[76:79], v[164:167], a[48:51]\n\t" \
  "ds_read_b64_tr_b16 v[70:71], v18 offset:17408\n\t" \
  "v_mfma_f32_16x16x32_bf16 a[48:51], v[72:75], v[160:163], a[48:51]\n\t" \
  "s_cselect_b32 s23, s25, 0\n\t" \
  "v_mfma_f32_16x16x32_bf16 a[44:47], v[72:75], v[152:155], a[44:47]\n\t" \
  "ds_read_b128 v[204:207], v17 offset:16384\n\t" \
  "s_cmp_le_u32 s19, s16\n\t" \
  "v_mfma_f32_16x16x32_bf16 a[44:47], v[76:79], v[156:159], a[44:47]\n\t" \
  "v_add_u32 v18, s51, v18\n\t" \
  "s_mov_b32 m0, s32\n\t" \
  "v_mfma_f32_16x16x32_bf16 a[40:43], v[76:79], v[148:151], a[40:43]\n\t" \
  "s_cselect_b32 s22, s25, 0\n\t" \
  "v_mfma_f32_16x16x32_bf16 a[40:43], v[72:75], v[144:147], a[40:43]\n\t" \
  "ds_read_b128 v[208:211], v16 offset:1024\n\t" \
  "buffer_load_dwordx4 v30, s[40:43], s46 offen offset:0 sc0 sc1 lds\n\t" \
  "v_mfma_f32_16x16x32_bf16 a[36:39], v[72:75], v[136:139], a[36:39]\n\t" \
  "s_cmp_eq_u32 s12, s18\n\t" \
  "s_cselect_b32 s22, s22, 0\n\t" \
  "v_mfma_f32_16x16x32_bf16 a[36:39], v[76:79], v[140:143], a[36:39]\n\t" \
  "s_and_b32 s22, s22, s17\n\t" \
  "v_mfma_f32_16x16x32_bf16 a[32:35], v[76:79], v[132:135], a[32:35]\n\t" \
  "ds_read_b128 v[212:215], v17 offset:1024\n\t" \
  "v_mfma_f32_16x16x32_bf16 a[32:35], v[72:75], v[128:131], a[32:35]\n\t" \
  "s_ff1_i32_b32 s52, s54\n\t" \
  "s_and_b32 s52, s52, 7\n\t" \
  "v_mfma_f32_16x16x32_bf16 a[64:67], v[80:83], v[128:131], a[64:67]\n\t" \
  "ds_read_b64_tr_b16 v[72:73], v19 offset:0\n\t" \
  "v_mfma_f32_16x16x32_bf16 a[64:67], v[84:87], v[132:135], a[64:67]\n\t" \
  "buffer_load_dwordx4 v31, s[40:43], s46 offen offset:1024 sc0 sc1 lds\n\t" \
  "s_lshl_b32 s56, s52, 19\n\t" \
  "v_mfma_f32_16x16x32_bf16 a[68:71], v[84:87], v[140:143], a[68:71]\n\t" \
  "ds_read_b64_tr_b16 v[74:75], v19 offset:1024\n\t" \
  "v_mfma_f32_16x16x32_bf16 a[68:71], v[80:83], v[136:139], a[68:71]\n\t" \
  "s_cmp_lt_u32 s61, 8\n\t" \
  "v_mfma_f32_16x16x32_bf16 a[72:75], v[80:83], v[144:147], a[72:75]\n\t" \
  "ds_read_b64_tr_b16 v[76:77], v19 offset:16384\n\t" \
  "s_cselect_b32 s63, 0, s63\n\t" \
  "v_mfma_f32_16x16x32_bf16 a[72:75], v[84:87], v[148:151], a[72:75]\n\t" \
  "s_cselect_b32 s56, 0, s56\n\t" \
  "v_mfma_f32_16x16x32_bf16 a[76:79], v[84:87], v[156:159], a[76:79]\n\t" \
  "ds_read_b64_tr_b16 v[78:79], v19 offset:17408\n\t" \
  "s_lshl_b32 s54, s52, 25\n\t" \
  "v_mfma_f32_16x16x32_bf16 a[76:79], v[80:83], v[152:155], a[76:79]\n\t" \
  "buffer_load_dwordx4 v32, s[40:43], s46 offen offset:2048 sc0 sc1 lds\n\t" \
  "v_mfma_f32_16x16x32_bf16 a[80:83], v[80:83], v[160:163], a[80:83]\n\t" \
  "ds_read_b128 v[216:219], v16 offset:17408\n\t" \
  "v_mfma_f32_16x16x32_bf16 a[80:83], v[84:87], v[164:167], a[80:83]\n\t" \
  "v_add_u32 v19, s51, v19\n\t" \
  "s_lshl_b32 s55, s53, 22\n\t" \
  "v_mfma_f32_16x16x32_bf16 a[84:87], v[84:87], v[172:175], a[84:87]\n\t" \
  "s_or_b32 s47, s54, s55\n\t" \
  "v_mfma_f32_16x16x32_bf16 a[84:87], v[80:83], v[168:171], a[84:87]\n\t" \
  "ds_read_b128 v[220:223], v17 offset:17408\n\t" \
  "v_mfma_f32_16x16x32_bf16 a[88:91], v[80:83], v[176:179], a[88:91]\n\t" \
  "s_lshl_b32 s55, s53, 7\n\t" \
  "s_or_b32 s59, s56, s55\n\t" \
  "v_mfma_f32_16x16x32_bf16 a[88:91], v[84:87], v[180:183], a[88:91]\n\t" \
  "buffer_load_dwordx4 v33, s[40:43], s46 offen offset:3072 sc0 sc1 lds\n\t" \
  "v_mfma_f32_16x16x32_bf16 a[92:95], v[84:87], v[188:191], a[92:95]\n\t" \
  "ds_read_b128 v[224:227], v16 offset:2048\n\t" \
  "v_cmp_le_u32 vcc, s65, v52\n\t" \
  "v_mfma_f32_16x16x32_bf16 a[92:95], v[80:83], v[184:187], a[92:95]\n\t" \
  "v_cmp_le_u32 s[54:55], s65, v53\n\t" \
  "v_mfma_f32_16x16x32_bf16 a[124:127], v[88:91], v[184:187], a[124:127]\n\t" \
  "ds_read_b64_tr_b16 v[80:81], v20 offset:0\n\t" \
  "v_mfma_f32_16x16x32_bf16 a[124:127], v[92:95], v[188:191], a[124:127]\n\t" \
  "s_nop 0\n\t" \
  "s_and_b64 vcc, vcc, s[54:55]\n\t" \
  "v_mfma_f32_16x16x32_bf16 a[120:123], v[92:95], v[180:183], a[120:123]\n\t" \
  "ds_read_b64_tr_b16 v[82:83], v20 offset:1024\n\t" \
  "v_mfma_f32_16x16x32_bf16 a[120:123], v[88:91], v[176:179], a[120:123]\n\t" \
  "s_or_b64 vcc, vcc, s[92:93]\n\t" \
  "s_cmp_eq_u64 vcc, -1\n\t" \
  "v_mfma_f32_16x16x32_bf16 a[116:119], v[88:91], v[168:171], a[116:119]\n\t" \
  "ds_read_b64_tr_b16 v[84:85], v20 offset:16384\n\t" \
  "v_mfma_f32_16x16x32_bf16 a[116:119], v[92:95], v[172:175], a[116:119]\n\t" \
  "s_cselect_b32 s64, 0, s63\n\t" \
  "v_mfma_f32_16x16x32_bf16 a[112:115], v[92:95], v[164:167], a[112:115]\n\t" \
  "ds_read_b64_tr_b16 v[86:87], v20 offset:17408\n\t" \
  "s_or_b32 s13, s23, s22\n\t" \
  "v_mfma_f32_16x16x32_bf16 a[112:115], v[88:91], v[160:163], a[112:115]\n\t" \
  "s_or_b32 s28, s13, s64\n\t" \
  "v_mfma_f32_16x16x32_bf16 a[108:111], v[88:91], v[152:155], a[108:111]\n\t" \
  "ds_read_b128 v[228:231], v17 offset:2048\n\t" \
  "s_add_u32 s57, s53, 1\n\t" \
  "v_mfma_f32_16x16x32_bf16 a[108:111], v[92:95], v[156:159], a[108:111]\n\t" \
  "v_add_u32 v20, s51, v20\n\t" \
  "v_mfma_f32_16x16x32_bf16 a[104:107], v[92:95], v[148:151], a[104:107]\n\t" \
  "s_and_b32 s57, s57, 7\n\t" \
  "v_mfma_f32_16x16x32_bf16 a[104:107], v[88:91], v[144:147], a[104:107]\n\t" \
  "ds_read_b128 v[232:235], v16 offset:18432\n\t" \
  "s_lshl_b32 s57, s57, 11\n\t" \
  "v_mfma_f32_16x16x32_bf16 a[100:103], v[88:91], v[136:139], a[100:103]\n\t" \
  "s_add_u32 s96, s90, s57\n\t" \
  "v_mfma_f32_16x16x32_bf16 a[100:103], v[92:95], v[140:143], a[100:103]\n\t" \
  "s_addc_u32 s97, s91, 0\n\t" \
  "v_mfma_f32_16x16x32_bf16 a[96:99], v[92:95], v[132:135], a[96:99]\n\t" \
  "ds_read_b128 v[236:239], v17 offset:18432\n\t" \
  "global_load_dwordx2 v[52:53], v54, s[96:97] sc0 sc1\n\t" \
  "v_mfma_f32_16x16x32_bf16 a[96:99], v[88:91], v[128:131], a[96:99]\n\t" \
  "s_cmp_lg_u32 s13, 0\n\t" \
  "s_cbranch_scc1 OP5_%=\n\t" \
  "OPB5_%=:\n\t" \
  "v_mfma_f32_16x16x32_bf16 a[128:131], v[96:99], v[128:131], a[128:131]\n\t" \
  "ds_read_b64_tr_b16 v[88:89], v21 offset:0\n\t" \
  "v_mfma_f32_16x16x32_bf16 a[128:131], v[100:103], v[132:135], a[128:131]\n\t" \
  "s_mov_b32 m0, s49\n\t" \
  "v_mfma_f32_16x16x32_bf16 a[132:135], v[100:103], v[140:143], a[132:135]\n\t" \
  "ds_read_b64_tr_b16 v[90:91], v21 offset:1024\n\t" \
  "v_mfma_f32_16x16x32_bf16 a[132:135], v[96:99], v[136:139], a[132:135]\n\t" \
  "buffer_load_dwordx4 v34, s[36:39], s47 offen offset:0 lds\n\t" \
  "v_mfma_f32_16x16x32_bf16 a[136:139], v[96:99], v[144:147], a[136:139]\n\t" \
  "ds_read_b64_tr_b16 v[92:93], v21 offset:16384\n\t" \
  "v_mfma_f32_16x16x32_bf16 a[136:139], v[100:103], v[148:151], a[136:139]\n\t" \
  "v_mfma_f32_16x16x32_bf16 a[140:143], v[100:103], v[156:159], a[140:143]\n\t" \
  "ds_read_b64_tr_b16 v[94:95], v21 offset:17408\n\t" \
  "v_mfma_f32_16x16x32_bf16 a[140:143], v[96:99], v[152:155], a[140:143]\n\t" \
  "v_mfma_f32_16x16x32_bf16 a[144:147], v[96:99], v[160:163], a[144:147]\n\t" \
  "ds_read_b128 v[240:243], v16 offset:3072\n\t" \
  "v_mfma_f32_16x16x32_bf16 a[144:147], v[100:103], v[164:167], a[144:147]\n\t" \
  "v_add_u32 v21, s51, v21\n\t" \
  "buffer_load_dwordx4 v35, s[36:39], s47 offen offset:1024 lds\n\t" \
  "v_mfma_f32_16x16x32_bf16 a[148:151], v[100:103], v[172:175], a[148:151]\n\t" \
  "v_mfma_f32_16x16x32_bf16 a[148:151], v[96:99], v[168:171], a[148:151]\n\t" \
  "ds_read_b128 v[244:247], v17 offset:3072\n\t" \
  "v_mfma_f32_16x16x32_bf16 a[152:155], v[96:99], v[176:179], a[152:155]\n\t" \
  "v_mfma_f32_16x16x32_bf16 a[152:155], v[100:103], v[180:183], a[152:155]\n\t" \
  "v_mfma_f32_16x16x32_bf16 a[156:159], v[100:103], v[188:191], a[156:159]\n\t" \
  "ds_read_b128 v[248:251], v16 offset:19456\n\t" \
  "v_mfma_f32_16x16x32_bf16 a[156:159], v[96:99], v[184:187], a[156:159]\n\t" \
  "buffer_load_dwordx4 v36, s[36:39], s47 offen offset:2048 lds\n\t" \
  "v_mfma_f32_16x16x32_bf16 a[188:191], v[104:107], v[184:187], a[188:191]\n\t" \
  "ds_read_b64_tr_b16 v[96:97], v22 offset:0\n\t" \
  "v_mfma_f32_16x16x32_bf16 a[188:191], v[108:111], v[188:191], a[188:191]\n\t" \
  "v_mfma_f32_16x16x32_bf16 a[184:187], v[108:111], v[180:183], a[184:187]\n\t" \
  "ds_read_b64_tr_b16 v[98:99], v22 offset:1024\n\t" \
  "v_mfma_f32_16x16x32_bf16 a[184:187], v[104:107], v[176:179], a[184:187]\n\t" \
  "v_mfma_f32_16x16x32_bf16 a[180:183], v[104:107], v[168:171], a[180:183]\n\t" \
  "ds_read_b64_tr_b16 v[100:101], v22 offset:16384\n\t" \
  "v_mfma_f32_16x16x32_bf16 a[180:183], v[108:111], v[172:175], a[180:183]\n\t" \
  "buffer_load_dwordx4 v37, s[36:39], s47 offen offset:3072 lds\n\t" \
  "v_mfma_f32_16x16x32_bf16 a[176:179], v[108:111], v[164:167], a[176:179]\n\t" \
  "ds_read_b64_tr_b16 v[102:103], v22 offset:17408\n\t" \
  "v_mfma_f32_16x16x32_bf16 a[176:179], v[104:107], v[160:163], a[176:179]\n\t" \
  "v_mfma_f32_16x16x32_bf16 a[172:175], v[104:107], v[152:155], a[172:175]\n\t" \
  "ds_read_b128 v[252:255], v17 offset:19456\n\t" \
  "v_mfma_f32_16x16x32_bf16 a[172:175], v[108:111], v[156:159], a[172:175]\n\t" \
  "v_add_u32 v22, s51, v22\n\t" \
  "s_mov_b32 m0, s33\n\t" \
  "v_mfma_f32_16x16x32_bf16 a[168:171], v[108:111], v[148:151], a[168:171]\n\t" \
  "v_mfma_f32_16x16x32_bf16 a[168:171], v[104:107], v[144:147], a[168:171]\n\t" \
  "buffer_load_dwordx4 v38, s[36:39], s47 offen offset:0 lds\n\t" \
  "v_mfma_f32_16x16x32_bf16 a[164:167], v[104:107], v[136:139], a[164:167]\n\t" \
  "s_mov_b32 s46, s59\n\t" \
  "v_mfma_f32_16x16x32_bf16 a[164:167], v[108:111], v[140:143], a[164:167]\n\t" \
  "s_cmp_lt_u32 s61, 8\n\t" \
  "v_mfma_f32_16x16x32_bf16 a[160:163], v[108:111], v[132:135], a[160:163]\n\t" \
  "s_cselect_b32 s40, s40, s44\n\t" \
  "v_mfma_f32_16x16x32_bf16 a[160:163], v[104:107], v[128:131], a[160:163]\n\t" \
  "v_mfma_f32_16x16x32_bf16 a[192:195], v[112:115], v[128:131], a[192:195]\n\t" \
  "ds_read_b64_tr_b16 v[104:105], v23 offset:0\n\t" \
  "v_mfma_f32_16x16x32_bf16 a[192:195], v[116:119], v[132:135], a[192:195]\n\t" \
  "buffer_load_dwordx4 v39, s[36:39], s47 offen offset:1024 lds\n\t" \
  "v_mfma_f32_16x16x32_bf16 a[196:199], v[116:119], v[140:143], a[196:199]\n\t" \
  "ds_read_b64_tr_b16 v[106:107], v23 offset:1024\n\t" \
  "v_mfma_f32_16x16x32_bf16 a[196:199], v[112:115], v[136:139], a[196:199]\n\t" \
  "s_cselect_b32 s41, s41, s45\n\t" \
  "v_mfma_f32_16x16x32_bf16 a[200:203], v[112:115], v[144:147], a[200:203]\n\t" \
  "ds_read_b64_tr_b16 v[108:109], v23 offset:16384\n\t" \
  "v_mfma_f32_16x16x32_bf16 a[200:203], v[116:119], v[148:151], a[200:203]\n\t" \
  "s_add_u32 s49, s62, s60\n\t" \
  "v_mfma_f32_16x16x32_bf16 a[204:207], v[116:119], v[156:159], a[204:207]\n\t" \
  "ds_read_b64_tr_b16 v[110:111], v23 offset:17408\n\t" \
  "v_mfma_f32_16x16x32_bf16 a[204:207], v[112:115], v[152:155], a[204:207]\n\t" \
  "buffer_load_dwordx4 v40, s[36:39], s47 offen offset:2048 lds\n\t" \
  "v_mfma_f32_16x16x32_bf16 a[208:211], v[112:115], v[160:163], a[208:211]\n\t" \
  "s_add_u32 s33, s49, 0x4000\n\t" \
  "v_mfma_f32_16x16x32_bf16 a[208:211], v[116:119], v[164:167], a[208:211]\n\t" \
  "v_add_u32 v23, s51, v23\n\t" \
  "v_mfma_f32_16x16x32_bf16 a[212:215], v[116:119], v[172:175], a[212:215]\n\t" \
  "v_mfma_f32_16x16x32_bf16 a[212:215], v[112:115], v[168:171], a[212:215]\n\t" \
  "v_xor_b32 v16, 0x8000, v16\n\t" \
  "v_mfma_f32_16x16x32_bf16 a[216:219], v[112:115], v[176:179], a[216:219]\n\t" \
  "v_xor_b32 v17, 0x8000, v17\n\t" \
  "v_mfma_f32_16x16x32_bf16 a[216:219], v[116:119], v[180:183], a[216:219]\n\t" \
  "buffer_load_dwordx4 v41, s[36:39], s47 offen offset:3072 lds\n\t" \
  "v_mfma_f32_16x16x32_bf16 a[220:223], v[116:119], v[188:191], a[220:223]\n\t" \
  "s_xor_b32 s48, s48, 0x8000\n\t" \
  "v_mfma_f32_16x16x32_bf16 a[220:223], v[112:115], v[184:187], a[220:223]\n\t" \
  "s_add_u32 s32, s48, 0x1000\n\t" \
  "v_mfma_f32_16x16x32_bf16 a[252:255], v[120:123], v[184:187], a[252:255]\n\t" \
  "ds_read_b64_tr_b16 v[112:113], v24 offset:0\n\t" \
  "v_mfma_f32_16x16x32_bf16 a[252:255], v[124:127], v[188:191], a[252:255]\n\t" \
  "ds_read_b64_tr_b16 v[114:115], v24 offset:1024\n\t" \
  "v_mfma_f32_16x16x32_bf16 a[248:251], v[124:127], v[180:183], a[248:251]\n\t" \
  "ds_read_b64_tr_b16 v[116:117], v24 offset:16384\n\t" \
  "v_mfma_f32_16x16x32_bf16 a[248:251], v[120:123], v[176:179], a[248:251]\n\t" \
  "ds_read_b64_tr_b16 v[118:119], v24 offset:17408\n\t" \
  "v_mfma_f32_16x16x32_bf16 a[244:247], v[120:123], v[168:171], a[244:247]\n\t" \
  "v_mfma_f32_16x16x32_bf16 a[244:247], v[124:127], v[172:175], a[244:247]\n\t" \
  "v_add_u32 v24, s51, v24\n\t" \
  "v_mfma_f32_16x16x32_bf16 a[240:243], v[124:127], v[164:167], a[240:243]\n\t" \
  "s_add_u32 s60, s60, s51\n\t" \
  "v_mfma_f32_16x16x32_bf16 a[240:243], v[120:123], v[160:163], a[240:243]\n\t" \
  "s_cmp_eq_u32 s60, 0x10000\n\t" \
  "v_mfma_f32_16x16x32_bf16 a[236:239], v[120:123], v[152:155], a[236:239]\n\t" \
  "s_cselect_b32 s51, 0xffff0000, s71\n\t" \
  "v_mfma_f32_16x16x32_bf16 a[236:239], v[124:127], v[156:159], a[236:239]\n\t" \
  "v_mfma_f32_16x16x32_bf16 a[232:235], v[124:127], v[148:151], a[232:235]\n\t" \
  "s_sub_u32 s50, s50, 1\n\t" \
  "v_mfma_f32_16x16x32_bf16 a[232:235], v[120:123], v[144:147], a[232:235]\n\t" \
  "s_cmp_lg_u32 s50, 0\n\t" \
  "v_mfma_f32_16x16x32_bf16 a[228:231], v[120:123], v[136:139], a[228:231]\n\t" \
  "v_mfma_f32_16x16x32_bf16 a[228:231], v[124:127], v[140:143], a[228:231]\n\t" \
  "s_cselect_b64 vcc, -1, 0\n\t" \
  "v_mfma_f32_16x16x32_bf16 a[224:227], v[124:127], v[132:135], a[224:227]\n\t" \
  "s_cmp_lg_u32 s28, 0\n\t" \
  "v_mfma_f32_16x16x32_bf16 a[224:227], v[120:123], v[128:131], a[224:227]\n\t" \
  "s_waitcnt vmcnt(8) lgkmcnt(0)\n\t" \
  "s_cbranch_scc1 OE5_%=\n\t" \
  "OEB5_%=:\n\t" \
  "s_barrier\n\t" \
  "s_cbranch_vccnz LOOP2_%=\n\t" \
  "v_mfma_f32_16x16x32_bf16 a[0:3], v[64:67], v[192:195], a[0:3]\n\t" \
  "ds_read_b64_tr_b16 v[120:121], v25 offset:0\n\t" \
  "v_mfma_f32_16x16x32_bf16 a[0:3], v[68:71], v[196:199], a[0:3]\n\t" \
  "s_mov_b32 m0, s48\n\t" \
  "v_mfma_f32_16x16x32_bf16 a[4:7], v[68:71], v[204:207], a[4:7]\n\t" \
  "ds_read_b64_tr_b16 v[122:123], v25 offset:1024\n\t" \
  "v_mfma_f32_16x16x32_bf16 a[4:7], v[64:67], v[200:203], a[4:7]\n\t" \
  "buffer_load_dwordx4 v26, s[40:43], s46 offen offset:0 sc0 sc1 lds\n\t" \
  "v_mfma_f32_16x16x32_bf16 a[8:11], v[64:67], v[208:211], a[8:11]\n\t" \
  "ds_read_b64_tr_b16 v[124:125], v25 offset:16384\n\t" \
  "v_mfma_f32_16x16x32_bf16 a[8:11], v[68:71], v[212:215], a[8:11]\n\t" \
  "v_mfma_f32_16x16x32_bf16 a[12:15], v[68:71], v[220:223], a[12:15]\n\t" \
  "ds_read_b64_tr_b16 v[126:127], v25 offset:17408\n\t" \
  "v_mfma_f32_16x16x32_bf16 a[12:15], v[64:67], v[216:219], a[12:15]\n\t" \
  "v_mfma_f32_16x16x32_bf16 a[16:19], v[64:67], v[224:227], a[16:19]\n\t" \
  "ds_read_b128 v[128:131], v16 offset:0\n\t" \
  "v_mfma_f32_16x16x32_bf16 a[16:19], v[68:71], v[228:231], a[16:19]\n\t" \
  "v_add_u32 v25, s51, v25\n\t" \
  "buffer_load_dwordx4 v27, s[40:43], s46 offen offset:1024 sc0 sc1 lds\n\t" \
  "v_mfma_f32_16x16x32_bf16 a[20:23], v[68:71], v[236:239], a[20:23]\n\t" \
  "v_mfma_f32_16x16x32_bf16 a[20:23], v[64:67], v[232:235], a[20:23]\n\t" \
  "ds_read_b128 v[132:135], v17 offset:0\n\t" \
  "v_mfma_f32_16x16x32_bf16 a[24:27], v[64:67], v[240:243], a[24:27]\n\t" \
  "v_mfma_f32_16x16x32_bf16 a[24:27], v[68:71], v[244:247], a[24:27]\n\t" \
  "v_mfma_f32_16x16x32_bf16 a[28:31], v[68:71], v[252:255], a[28:31]\n\t" \
  "ds_read_b128 v[136:139], v16 offset:16384\n\t" \
  "v_mfma_f32_16x16x32_bf16 a[28:31], v[64:67], v[248:251], a[28:31]\n\t" \
  "buffer_load_dwordx4 v28, s[40:43], s46 offen offset:2048 sc0 sc1 lds\n\t" \
  "v_mfma_f32_16x16x32_bf16 a[60:63], v[72:75], v[248:251], a[60:63]\n\t" \
  "ds_read_b64_tr_b16 v[64:65], v18 offset:0\n\t" \
  "v_mfma_f32_16x16x32_bf16 a[60:63], v[76:79], v[252:255], a[60:63]\n\t" \
  "v_mfma_f32_16x16x32_bf16 a[56:59], v[76:79], v[244:247], a[56:59]\n\t" \
  "ds_read_b64_tr_b16 v[66:67], v18 offset:1024\n\t" \
  "v_mfma_f32_16x16x32_bf16 a[56:59], v[72:75], v[240:243], a[56:59]\n\t" \
  "v_mfma_f32_16x16x32_bf16 a[52:55], v[72:75], v[232:235], a[52:55]\n\t" \
  "ds_read_b64_tr_b16 v[68:69], v18 offset:16384\n\t" \
  "v_mfma_f32_16x16x32_bf16 a[52:55], v[76:79], v[236:239], a[52:55]\n\t" \
  "buffer_load_dwordx4 v29, s[40:43], s46 offen offset:3072 sc0 sc1 lds\n\t" \
  "v_mfma_f32_16x16x32_bf16 a[48:51], v[76:79], v[228:231], a[48:51]\n\t" \
  "ds_read_b64_tr_b16 v[70:71], v18 offset:17408\n\t" \
  "v_mfma_f32_16x16x32_bf16 a[48:51], v[72:75], v[224:227], a[48:51]\n\t" \
  "v_mfma_f32_16x16x32_bf16 a[44:47], v[72:75], v[216:219], a[44:47]\n\t" \
  "ds_read_b128 v[140:143], v17 offset:16384\n\t" \
  "v_mfma_f32_16x16x32_bf16 a[44:47], v[76:79], v[220:223], a[44:47]\n\t" \
  "v_add_u32 v18, s51, v18\n\t" \
  "s_mov_b32 m0, s32\n\t" \
  "v_mfma_f32_16x16x32_bf16 a[40:43], v[76:79], v[212:215], a[40:43]\n\t" \
  "v_mfma_f32_16x16x32_bf16 a[40:43], v[72:75], v[208:211], a[40:43]\n\t" \
  "ds_read_b128 v[144:147], v16 offset:1024\n\t" \
  "buffer_load_dwordx4 v30, s[40:43], s46 offen offset:0 sc0 sc1 lds\n\t" \
  "v_mfma_f32_16x16x32_bf16 a[36:39], v[72:75], v[200:203], a[36:39]\n\t" \
  "v_mfma_f32_16x16x32_bf16 a[36:39], v[76:79], v[204:207], a[36:39]\n\t" \
  "v_mfma_f32_16x16x32_bf16 a[32:35], v[76:79], v[196:199], a[32:35]\n\t" \
  "ds_read_b128 v[148:151], v17 offset:1024\n\t" \
  "v_mfma_f32_16x16x32_bf16 a[32:35], v[72:75], v[192:195], a[32:35]\n\t" \
  "v_mfma_f32_16x16x32_bf16 a[64:67], v[80:83], v[192:195], a[64:67]\n\t" \
  "ds_read_b64_tr_b16 v[72:73], v19 offset:0\n\t" \
  "v_mfma_f32_16x16x32_bf16 a[64:67], v[84:87], v[196:199], a[64:67]\n\t" \
  "buffer_load_dwordx4 v31, s[40:43], s46 offen offset:1024 sc0 sc1 lds\n\t" \
  "v_mfma_f32_16x16x32_bf16 a[68:71], v[84:87], v[204:207], a[68:71]\n\t" \
  "ds_read_b64_tr_b16 v[74:75], v19 offset:1024\n\t" \
  "v_mfma_f32_16x16x32_bf16 a[68:71], v[80:83], v[200:203], a[68:71]\n\t" \
  "v_mfma_f32_16x16x32_bf16 a[72:75], v[80:83], v[208:211], a[72:75]\n\t" \
  "ds_read_b64_tr_b16 v[76:77], v19 offset:16384\n\t" \
  "v_mfma_f32_16x16x32_bf16 a[72:75], v[84:87], v[212:215], a[72:75]\n\t" \
  "v_mfma_f32_16x16x32_bf16 a[76:79], v[84:87], v[220:223], a[76:79]\n\t" \
  "ds_read_b64_tr_b16 v[78:79], v19 offset:17408\n\t" \
  "v_mfma_f32_16x16x32_bf16 a[76:79], v[80:83], v[216:219], a[76:79]\n\t" \
  "buffer_load_dwordx4 v32, s[40:43], s46 offen offset:2048 sc0 sc1 lds\n\t" \
  "v_mfma_f32_16x16x32_bf16 a[80:83], v[80:83], v[224:227], a[80:83]\n\t" \
  "ds_read_b128 v[152:155], v16 offset:17408\n\t" \
  "v_mfma_f32_16x16x32_bf16 a[80:83], v[84:87], v[228:231], a[80:83]\n\t" \
  "v_add_u32 v19, s51, v19\n\t" \
  "v_mfma_f32_16x16x32_bf16 a[84:87], v[84:87], v[236:239], a[84:87]\n\t" \
  "v_mfma_f32_16x16x32_bf16 a[84:87], v[80:83], v[232:235], a[84:87]\n\t" \
  "ds_read_b128 v[156:159], v17 offset:17408\n\t" \
  "v_mfma_f32_16x16x32_bf16 a[88:91], v[80:83], v[240:243], a[88:91]\n\t" \
  "v_mfma_f32_16x16x32_bf16 a[88:91], v[84:87], v[244:247], a[88:91]\n\t" \
  "buffer_load_dwordx4 v33, s[40:43], s46 offen offset:3072 sc0 sc1 lds\n\t" \
  "v_mfma_f32_16x16x32_bf16 a[92:95], v[84:87], v[252:255], a[92:95]\n\t" \
  "ds_read_b128 v[160:163], v16 offset:2048\n\t" \
  "v_mfma_f32_16x16x32_bf16 a[92:95], v[80:83], v[248:251], a[92:95]\n\t" \
  "v_mfma_f32_16x16x32_bf16 a[124:127], v[88:91], v[248:251], a[124:127]\n\t" \
  "ds_read_b64_tr_b16 v[80:81], v20 offset:0\n\t" \
  "v_mfma_f32_16x16x32_bf16 a[124:127], v[92:95], v[252:255], a[124:127]\n\t" \
  "v_mfma_f32_16x16x32_bf16 a[120:123], v[92:95], v[244:247], a[120:123]\n\t" \
  "ds_read_b64_tr_b16 v[82:83], v20 offset:1024\n\t" \
  "v_mfma_f32_16x16x32_bf16 a[120:123], v[88:91], v[240:243], a[120:123]\n\t" \
  "v_mfma_f32_16x16x32_bf16 a[116:119], v[88:91], v[232:235], a[116:119]\n\t" \
  "ds_read_b64_tr_b16 v[84:85], v20 offset:16384\n\t" \
  "v_mfma_f32_16x16x32_bf16 a[116:119], v[92:95], v[236:239], a[116:119]\n\t" \
  "v_mfma_f32_16x16x32_bf16 a[112:115], v[92:95], v[228:231], a[112:115]\n\t" \
  "ds_read_b64_tr_b16 v[86:87], v20 offset:17408\n\t" \
  "v_mfma_f32_16x16x32_bf16 a[112:115], v[88:91], v[224:227], a[112:115]\n\t" \
  "v_mfma_f32_16x16x32_bf16 a[108:111], v[88:91], v[216:219], a[108:111]\n\t" \
  "ds_read_b128 v[164:167], v17 offset:2048\n\t" \
  "v_mfma_f32_16x16x32_bf16 a[108:111], v[92:95], v[220:223], a[108:111]\n\t" \
  "v_add_u32 v20, s51, v20\n\t" \
  "v_mfma_f32_16x16x32_bf16 a[104:107], v[92:95], v[212:215], a[104:107]\n\t" \
  "v_mfma_f32_16x16x32_bf16 a[104:107], v[88:91], v[208:211], a[104:107]\n\t" \
  "ds_read_b128 v[168:171], v16 offset:18432\n\t" \
  "v_mfma_f32_16x16x32_bf16 a[100:103], v[88:91], v[200:203], a[100:103]\n\t" \
  "v_mfma_f32_16x16x32_bf16 a[100:103], v[92:95], v[204:207], a[100:103]\n\t" \
  "v_mfma_f32_16x16x32_bf16 a[96:99], v[92:95], v[196:199], a[96:99]\n\t" \
  "ds_read_b128 v[172:175], v17 offset:18432\n\t" \
  "v_mfma_f32_16x16x32_bf16 a[96:99], v[88:91], v[192:195], a[96:99]\n\t" \
  "v_mfma_f32_16x16x32_bf16 a[128:131], v[96:99], v[192:195], a[128:131]\n\t" \
  "ds_read_b64_tr_b16 v[88:89], v21 offset:0\n\t" \
  "v_mfma_f32_16x16x32_bf16 a[128:131], v[100:103], v[196:199], a[128:131]\n\t" \
  "v_mfma_f32_16x16x32_bf16 a[132:135], v[100:103], v[204:207], a[132:135]\n\t" \
  "ds_read_b64_tr_b16 v[90:91], v21 offset:1024\n\t" \
  "v_mfma_f32_16x16x32_bf16 a[132:135], v[96:99], v[200:203], a[132:135]\n\t" \
  "v_mfma_f32_16x16x32_bf16 a[136:139], v[96:99], v[208:211], a[136:139]\n\t" \
  "ds_read_b64_tr_b16 v[92:93], v21 offset:16384\n\t" \
  "v_mfma_f32_16x16x32_bf16 a[136:139], v[100:103], v[212:215], a[136:139]\n\t" \
  "v_mfma_f32_16x16x32_bf16 a[140:143], v[100:103], v[220:223], a[140:143]\n\t" \
  "ds_read_b64_tr_b16 v[94:95], v21 offset:17408\n\t" \
  "v_mfma_f32_16x16x32_bf16 a[140:143], v[96:99], v[216:219], a[140:143]\n\t" \
  "v_mfma_f32_16x16x32_bf16 a[144:147], v[96:99], v[224:227], a[144:147]\n\t" \
  "ds_read_b128 v[176:179], v16 offset:3072\n\t" \
  "v_mfma_f32_16x16x32_bf16 a[144:147], v[100:103], v[228:231], a[144:147]\n\t" \
  "v_add_u32 v21, s51, v21\n\t" \
  "v_mfma_f32_16x16x32_bf16 a[148:151], v[100:103], v[236:239], a[148:151]\n\t" \
  "v_mfma_f32_16x16x32_bf16 a[148:151], v[96:99], v[232:235], a[148:151]\n\t" \
  "ds_read_b128 v[180:183], v17 offset:3072\n\t" \
  "v_mfma_f32_16x16x32_bf16 a[152:155], v[96:99], v[240:243], a[152:155]\n\t" \
  "v_mfma_f32_16x16x32_bf16 a[152:155], v[100:103], v[244:247], a[152:155]\n\t" \
  "v_mfma_f32_16x16x32_bf16 a[156:159], v[100:103], v[252:255], a[156:159]\n\t" \
  "ds_read_b128 v[184:187], v16 offset:19456\n\t" \
  "v_mfma_f32_16x16x32_bf16 a[156:159], v[96:99], v[248:251], a[156:159]\n\t" \
  "v_mfma_f32_16x16x32_bf16 a[188:191], v[104:107], v[248:251], a[188:191]\n\t" \
  "ds_read_b64_tr_b16 v[96:97], v22 offset:0\n\t" \
  "v_mfma_f32_16x16x32_bf16 a[188:191], v[108:111], v[252:255], a[188:191]\n\t" \
  "v_mfma_f32_16x16x32_bf16 a[184:187], v[108:111], v[244:247], a[184:187]\n\t" \
  "ds_read_b64_tr_b16 v[98:99], v22 offset:1024\n\t" \
  "v_mfma_f32_16x16x32_bf16 a[184:187], v[104:107], v[240:243], a[184:187]\n\t" \
  "v_mfma_f32_16x16x32_bf16 a[180:183], v[104:107], v[232:235], a[180:183]\n\t" \
  "ds_read_b64_tr_b16 v[100:101], v22 offset:16384\n\t" \
  "v_mfma_f32_16x16x32_bf16 a[180:183], v[108:111], v[236:239], a[180:183]\n\t" \
  "v_mfma_f32_16x16x32_bf16 a[176:179], v[108:111], v[228:231], a[176:179]\n\t" \
  "ds_read_b64_tr_b16 v[102:103], v22 offset:17408\n\t" \
  "v_mfma_f32_16x16x32_bf16 a[176:179], v[104:107], v[224:227], a[176:179]\n\t" \
  "v_mfma_f32_16x16x32_bf16 a[172:175], v[104:107], v[216:219], a[172:175]\n\t" \
  "ds_read_b128 v[188:191], v17 offset:19456\n\t" \
  "v_mfma_f32_16x16x32_bf16 a[172:175], v[108:111], v[220:223], a[172:175]\n\t" \
  "v_add_u32 v22, s51, v22\n\t" \
  "v_mfma_f32_16x16x32_bf16 a[168:171], v[108:111], v[212:215], a[168:171]\n\t" \
  "v_mfma_f32_16x16x32_bf16 a[168:171], v[104:107], v[208:211], a[168:171]\n\t" \
  "v_mfma_f32_16x16x32_bf16 a[164:167], v[104:107], v[200:203], a[164:167]\n\t" \
  "v_mfma_f32_16x16x32_bf16 a[164:167], v[108:111], v[204:207], a[164:167]\n\t" \
  "v_mfma_f32_16x16x32_bf16 a[160:163], v[108:111], v[196:199], a[160:163]\n\t" \
  "v_mfma_f32_16x16x32_bf16 a[160:163], v[104:107], v[192:195], a[160:163]\n\t" \
  "v_mfma_f32_16x16x32_bf16 a[192:195], v[112:115], v[192:195], a[192:195]\n\t" \
  "ds_read_b64_tr_b16 v[104:105], v23 offset:0\n\t" \
  "v_mfma_f32_16x16x32_bf16 a[192:195], v[116:119], v[196:199], a[192:195]\n\t" \
  "v_mfma_f32_16x16x32_bf16 a[196:199], v[116:119], v[204:207], a[196:199]\n\t" \
  "ds_read_b64_tr_b16 v[106:107], v23 offset:1024\n\t" \
  "v_mfma_f32_16x16x32_bf16 a[196:199], v[112:115], v[200:203], a[196:199]\n\t" \
  "v_mfma_f32_16x16x32_bf16 a[200:203], v[112:115], v[208:211], a[200:203]\n\t" \
  "ds_read_b64_tr_b16 v[108:109], v23 offset:16384\n\t" \
  "v_mfma_f32_16x16x32_bf16 a[200:203], v[116:119], v[212:215], a[200:203]\n\t" \
  "v_mfma_f32_16x16x32_bf16 a[204:207], v[116:119], v[220:223], a[204:207]\n\t" \
  "ds_read_b64_tr_b16 v[110:111], v23 offset:17408\n\t" \
  "v_mfma_f32_16x16x32_bf16 a[204:207], v[112:115], v[216:219], a[204:207]\n\t" \
  "v_mfma_f32_16x16x32_bf16 a[208:211], v[112:115], v[224:227], a[208:211]\n\t" \
  "v_mfma_f32_16x16x32_bf16 a[208:211], v[116:119], v[228:231], a[208:211]\n\t" \
  "v_add_u32 v23, s51, v23\n\t" \
  "v_mfma_f32_16x16x32_bf16 a[212:215], v[116:119], v[236:239], a[212:215]\n\t" \
  "v_mfma_f32_16x16x32_bf16 a[212:215], v[112:115], v[232:235], a[212:215]\n\t" \
  "v_xor_b32 v16, 0x8000, v16\n\t" \
  "v_mfma_f32_16x16x32_bf16 a[216:219], v[112:115], v[240:243], a[216:219]\n\t" \
  "v_xor_b32 v17, 0x8000, v17\n\t" \
  "v_mfma_f32_16x16x32_bf16 a[216:219], v[116:119], v[244:247], a[216:219]\n\t" \
  "v_mfma_f32_16x16x32_bf16 a[220:223], v[116:119], v[252:255], a[220:223]\n\t" \
  "v_mfma_f32_16x16x32_bf16 a[220:223], v[112:115], v[248:251], a[220:223]\n\t" \
  "v_mfma_f32_16x16x32_bf16 a[252:255], v[56:59], v[248:251], a[252:255]\n\t" \
  "ds_read_b64_tr_b16 v[112:113], v24 offset:0\n\t" \
  "v_mfma_f32_16x16x32_bf16 a[252:255], v[60:63], v[252:255], a[252:255]\n\t" \
  "ds_read_b64_tr_b16 v[114:115], v24 offset:1024\n\t" \
  "v_mfma_f32_16x16x32_bf16 a[248:251], v[60:63], v[244:247], a[248:251]\n\t" \
  "ds_read_b64_tr_b16 v[116:117], v24 offset:16384\n\t" \
  "v_mfma_f32_16x16x32_bf16 a[248:251], v[56:59], v[240:243], a[248:251]\n\t" \
  "ds_read_b64_tr_b16 v[118:119], v24 offset:17408\n\t" \
  "v_mfma_f32_16x16x32_bf16 a[244:247], v[56:59], v[232:235], a[244:247]\n\t" \
  "v_mfma_f32_16x16x32_bf16 a[244:247], v[60:63], v[236:239], a[244:247]\n\t" \
  "v_add_u32 v24, s51, v24\n\t" \
  "v_mfma_f32_16x16x32_bf16 a[240:243], v[60:63], v[228:231], a[240:243]\n\t" \
  "v_mfma_f32_16x16x32_bf16 a[240:243], v[56:59], v[224:227], a[240:243]\n\t" \
  "v_mfma_f32_16x16x32_bf16 a[236:239], v[56:59], v[216:219], a[236:239]\n\t" \
  "v_mfma_f32_16x16x32_bf16 a[236:239], v[60:63], v[220:223], a[236:239]\n\t" \
  "v_mfma_f32_16x16x32_bf16 a[232:235], v[60:63], v[212:215], a[232:235]\n\t" \
  "v_mfma_f32_16x16x32_bf16 a[232:235], v[56:59], v[208:211], a[232:235]\n\t" \
  "v_mfma_f32_16x16x32_bf16 a[228:231], v[56:59], v[200:203], a[228:231]\n\t" \
  "v_mfma_f32_16x16x32_bf16 a[228:231], v[60:63], v[204:207], a[228:231]\n\t" \
  "v_mfma_f32_16x16x32_bf16 a[224:227], v[60:63], v[196:199], a[224:227]\n\t" \
  "v_mfma_f32_16x16x32_bf16 a[224:227], v[56:59], v[192:195], a[224:227]\n\t" \
  "s_waitcnt vmcnt(0) lgkmcnt(0)\n\t" \
  "s_barrier\n\t" \
  "global_load_dword v12, v51, s[94:95] sc1\n\t" \
  "s_mov_b64 exec, s[26:27]\n\t" \
  "global_store_dword v51, v55, s[34:35] sc1\n\t" \
  "s_mov_b64 exec, -1\n\t" \
  "v_mfma_f32_16x16x32_bf16 a[0:3], v[64:67], v[128:131], a[0:3]\n\t" \
  "ds_read_b64_tr_b16 v[56:57], v25 offset:0\n\t" \
  "v_mfma_f32_16x16x32_bf16 a[0:3], v[68:71], v[132:135], a[0:3]\n\t" \
  "v_mfma_f32_16x16x32_bf16 a[4:7], v[68:71], v[140:143], a[4:7]\n\t" \
  "ds_read_b64_tr_b16 v[58:59], v25 offset:1024\n\t" \
  "v_mfma_f32_16x16x32_bf16 a[4:7], v[64:67], v[136:139], a[4:7]\n\t" \
  "v_mfma_f32_16x16x32_bf16 a[8:11], v[64:67], v[144:147], a[8:11]\n\t" \
  "ds_read_b64_tr_b16 v[60:61], v25 offset:16384\n\t" \
  "v_mfma_f32_16x16x32_bf16 a[8:11], v[68:71], v[148:151], a[8:11]\n\t" \
  "v_mfma_f32_16x16x32_bf16 a[12:15], v[68:71], v[156:159], a[12:15]\n\t" \
  "ds_read_b64_tr_b16 v[62:63], v25 offset:17408\n\t" \
  "v_mfma_f32_16x16x32_bf16 a[12:15], v[64:67], v[152:155], a[12:15]\n\t" \
  "v_mfma_f32_16x16x32_bf16 a[16:19], v[64:67], v[160:163], a[16:19]\n\t" \
  "ds_read_b128 v[192:195], v16 offset:0\n\t" \
  "v_mfma_f32_16x16x32_bf16 a[16:19], v[68:71], v[164:167], a[16:19]\n\t" \
  "v_add_u32 v25, s51, v25\n\t" \
  "v_mfma_f32_16x16x32_bf16 a[20:23], v[68:71], v[172:175], a[20:23]\n\t" \
  "v_mfma_f32_16x16x32_bf16 a[20:23], v[64:67], v[168:171], a[20:23]\n\t" \
  "ds_read_b128 v[196:199], v17 offset:0\n\t" \
  "v_mfma_f32_16x16x32_bf16 a[24:27], v[64:67], v[176:179], a[24:27]\n\t" \
  "v_mfma_f32_16x16x32_bf16 a[24:27], v[68:71], v[180:183], a[24:27]\n\t" \
  "v_mfma_f32_16x16x32_bf16 a[28:31], v[68:71], v[188:191], a[28:31]\n\t" \
  "ds_read_b128 v[200:203], v16 offset:16384\n\t" \
  "v_mfma_f32_16x16x32_bf16 a[28:31], v[64:67], v[184:187], a[28:31]\n\t" \
  "v_mfma_f32_16x16x32_bf16 a[60:63], v[72:75], v[184:187], a[60:63]\n\t" \
  "ds_read_b64_tr_b16 v[64:65], v18 offset:0\n\t" \
  "v_mfma_f32_16x16x32_bf16 a[60:63], v[76:79], v[188:191], a[60:63]\n\t" \
  "v_mfma_f32_16x16x32_bf16 a[56:59], v[76:79], v[180:183], a[56:59]\n\t" \
  "ds_read_b64_tr_b16 v[66:67], v18 offset:1024\n\t" \
  "v_mfma_f32_16x16x32_bf16 a[56:59], v[72:75], v[176:179], a[56:59]\n\t" \
  "v_mfma_f32_16x16x32_bf16 a[52:55], v[72:75], v[168:171], a[52:55]\n\t" \
  "ds_read_b64_tr_b16 v[68:69], v18 offset:16384\n\t" \
  "v_mfma_f32_16x16x32_bf16 a[52:55], v[76:79], v[172:175], a[52:55]\n\t" \
  "v_mfma_f32_16x16x32_bf16 a[48:51], v[76:79], v[164:167], a[48:51]\n\t" \
  "ds_read_b64_tr_b16 v[70:71], v18 offset:17408\n\t" \
  "v_mfma_f32_16x16x32_bf16 a[48:51], v[72:75], v[160:163], a[48:51]\n\t" \
  "v_mfma_f32_16x16x32_bf16 a[44:47], v[72:75], v[152:155], a[44:47]\n\t" \
  "ds_read_b128 v[204:207], v17 offset:16384\n\t" \
  "v_mfma_f32_16x16x32_bf16 a[44:47], v[76:79], v[156:159], a[44:47]\n\t" \
  "v_add_u32 v18, s51, v18\n\t" \
  "v_mfma_f32_16x16x32_bf16 a[40:43], v[76:79], v[148:151], a[40:43]\n\t" \
  "v_mfma_f32_16x16x32_bf16 a[40:43], v[72:75], v[144:147], a[40:43]\n\t" \
  "ds_read_b128 v[208:211], v16 offset:1024\n\t" \
  "v_mfma_f32_16x16x32_bf16 a[36:39], v[72:75], v[136:139], a[36:39]\n\t" \
  "v_mfma_f32_16x16x32_bf16 a[36:39], v[76:79], v[140:143], a[36:39]\n\t" \
  "v_mfma_f32_16x16x32_bf16 a[32:35], v[76:79], v[132:135], a[32:35]\n\t" \
  "ds_read_b128 v[212:215], v17 offset:1024\n\t" \
  "v_mfma_f32_16x16x32_bf16 a[32:35], v[72:75], v[128:131], a[32:35]\n\t" \
  "v_mfma_f32_16x16x32_bf16 a[64:67], v[80:83], v[128:131], a[64:67]\n\t" \
  "ds_read_b64_tr_b16 v[72:73], v19 offset:0\n\t" \
  "v_mfma_f32_16x16x32_bf16 a[64:67], v[84:87], v[132:135], a[64:67]\n\t" \
  "v_mfma_f32_16x16x32_bf16 a[68:71], v[84:87], v[140:143], a[68:71]\n\t" \
  "ds_read_b64_tr_b16 v[74:75], v19 offset:1024\n\t" \
  "v_mfma_f32_16x16x32_bf16 a[68:71], v[80:83], v[136:139], a[68:71]\n\t" \
  "v_mfma_f32_16x16x32_bf16 a[72:75], v[80:83], v[144:147], a[72:75]\n\t" \
  "ds_read_b64_tr_b16 v[76:77], v19 offset:16384\n\t" \
  "v_mfma_f32_16x16x32_bf16 a[72:75], v[84:87], v[148:151], a[72:75]\n\t" \
  "v_mfma_f32_16x16x32_bf16 a[76:79], v[84:87], v[156:159], a[76:79]\n\t" \
  "ds_read_b64_tr_b16 v[78:79], v19 offset:17408\n\t" \
  "v_mfma_f32_16x16x32_bf16 a[76:79], v[80:83], v[152:155], a[76:79]\n\t" \
  "v_mfma_f32_16x16x32_bf16 a[80:83], v[80:83], v[160:163], a[80:83]\n\t" \
  "ds_read_b128 v[216:219], v16 offset:17408\n\t" \
  "v_mfma_f32_16x16x32_bf16 a[80:83], v[84:87], v[164:167], a[80:83]\n\t" \
  "v_add_u32 v19, s51, v19\n\t" \
  "v_mfma_f32_16x16x32_bf16 a[84:87], v[84:87], v[172:175], a[84:87]\n\t" \
  "v_mfma_f32_16x16x32_bf16 a[84:87], v[80:83], v[168:171], a[84:87]\n\t" \
  "ds_read_b128 v[220:223], v17 offset:17408\n\t" \
  "v_mfma_f32_16x16x32_bf16 a[88:91], v[80:83], v[176:179], a[88:91]\n\t" \
  "v_mfma_f32_16x16x32_bf16 a[88:91], v[84:87], v[180:183], a[88:91]\n\t" \
  "v_mfma_f32_16x16x32_bf16 a[92:95], v[84:87], v[188:191], a[92:95]\n\t" \
  "ds_read_b128 v[224:227], v16 offset:2048\n\t" \
  "v_mfma_f32_16x16x32_bf16 a[92:95], v[80:83], v[184:187], a[92:95]\n\t" \
  "v_mfma_f32_16x16x32_bf16 a[124:127], v[88:91], v[184:187], a[124:127]\n\t" \
  "ds_read_b64_tr_b16 v[80:81], v20 offset:0\n\t" \
  "v_mfma_f32_16x16x32_bf16 a[124:127], v[92:95], v[188:191], a[124:127]\n\t" \
  "v_mfma_f32_16x16x32_bf16 a[120:123], v[92:95], v[180:183], a[120:123]\n\t" \
  "ds_read_b64_tr_b16 v[82:83], v20 offset:1024\n\t" \
  "v_mfma_f32_16x16x32_bf16 a[120:123], v[88:91], v[176:179], a[120:123]\n\t" \
  "v_mfma_f32_16x16x32_bf16 a[116:119], v[88:91], v[168:171], a[116:119]\n\t" \
  "ds_read_b64_tr_b16 v[84:85], v20 offset:16384\n\t" \
  "v_mfma_f32_16x16x32_bf16 a[116:119], v[92:95], v[172:175], a[116:119]\n\t" \
  "v_mfma_f32_16x16x32_bf16 a[112:115], v[92:95], v[164:167], a[112:115]\n\t" \
  "ds_read_b64_tr_b16 v[86:87], v20 offset:17408\n\t" \
  "v_mfma_f32_16x16x32_bf16 a[112:115], v[88:91], v[160:163], a[112:115]\n\t" \
  "v_mfma_f32_16x16x32_bf16 a[108:111], v[88:91], v[152:155], a[108:111]\n\t" \
  "ds_read_b128 v[228:231], v17 offset:2048\n\t" \
  "v_mfma_f32_16x16x32_bf16 a[108:111], v[92:95], v[156:159], a[108:111]\n\t" \
  "v_add_u32 v20, s51, v20\n\t" \
  "v_mfma_f32_16x16x32_bf16 a[104:107], v[92:95], v[148:151], a[104:107]\n\t" \
  "v_mfma_f32_16x16x32_bf16 a[104:107], v[88:91], v[144:147], a[104:107]\n\t" \
  "ds_read_b128 v[232:235], v16 offset:18432\n\t" \
  "v_mfma_f32_16x16x32_bf16 a[100:103], v[88:91], v[136:139], a[100:103]\n\t" \
  "v_mfma_f32_16x16x32_bf16 a[100:103], v[92:95], v[140:143], a[100:103]\n\t" \
  "v_mfma_f32_16x16x32_bf16 a[96:99], v[92:95], v[132:135], a[96:99]\n\t" \
  "ds_read_b128 v[236:239], v17 offset:18432\n\t" \
  "v_mfma_f32_16x16x32_bf16 a[96:99], v[88:91], v[128:131], a[96:99]\n\t" \
  "v_mfma_f32_16x16x32_bf16 a[128:131], v[96:99], v[128:131], a[128:131]\n\t" \
  "ds_read_b64_tr_b16 v[88:89], v21 offset:0\n\t" \
  "v_mfma_f32_16x16x32_bf16 a[128:131], v[100:103], v[132:135], a[128:131]\n\t" \
  "v_mfma_f32_16x16x32_bf16 a[132:135], v[100:103], v[140:143], a[132:135]\n\t" \
  "ds_read_b64_tr_b16 v[90:91], v21 offset:1024\n\t" \
  "v_mfma_f32_16x16x32_bf16 a[132:135], v[96:99], v[136:139], a[132:135]\n\t" \
  "v_mfma_f32_16x16x32_bf16 a[136:139], v[96:99], v[144:147], a[136:139]\n\t" \
  "ds_read_b64_tr_b16 v[92:93], v21 offset:16384\n\t" \
  "v_mfma_f32_16x16x32_bf16 a[136:139], v[100:103], v[148:151], a[136:139]\n\t" \
  "v_mfma_f32_16x16x32_bf16 a[140:143], v[100:103], v[156:159], a[140:143]\n\t" \
  "ds_read_b64_tr_b16 v[94:95], v21 offset:17408\n\t" \
  "v_mfma_f32_16x16x32_bf16 a[140:143], v[96:99], v[152:155], a[140:143]\n\t" \
  "v_mfma_f32_16x16x32_bf16 a[144:147], v[96:99], v[160:163], a[144:147]\n\t" \
  "ds_read_b128 v[240:243], v16 offset:3072\n\t" \
  "v_mfma_f32_16x16x32_bf16 a[144:147], v[100:103], v[164:167], a[144:147]\n\t" \
  "v_add_u32 v21, s51, v21\n\t" \
  "v_mfma_f32_16x16x32_bf16 a[148:151], v[100:103], v[172:175], a[148:151]\n\t" \
  "v_mfma_f32_16x16x32_bf16 a[148:151], v[96:99], v[168:171], a[148:151]\n\t" \
  "ds_read_b128 v[244:247], v17 offset:3072\n\t" \
  "v_mfma_f32_16x16x32_bf16 a[152:155], v[96:99], v[176:179], a[152:155]\n\t" \
  "v_mfma_f32_16x16x32_bf16 a[152:155], v[100:103], v[180:183], a[152:155]\n\t" \
  "v_mfma_f32_16x16x32_bf16 a[156:159], v[100:103], v[188:191], a[156:159]\n\t" \
  "ds_read_b128 v[248:251], v16 offset:19456\n\t" \
  "v_mfma_f32_16x16x32_bf16 a[156:159], v[96:99], v[184:187], a[156:159]\n\t" \
  "v_mfma_f32_16x16x32_bf16 a[188:191], v[104:107], v[184:187], a[188:191]\n\t" \
  "ds_read_b64_tr_b16 v[96:97], v22 offset:0\n\t" \
  "v_mfma_f32_16x16x32_bf16 a[188:191], v[108:111], v[188:191], a[188:191]\n\t" \
  "v_mfma_f32_16x16x32_bf16 a[184:187], v[108:111], v[180:183], a[184:187]\n\t" \
  "ds_read_b64_tr_b16 v[98:99], v22 offset:1024\n\t" \
  "v_mfma_f32_16x16x32_bf16 a[184:187], v[104:107], v[176:179], a[184:187]\n\t" \
  "v_mfma_f32_16x16x32_bf16 a[180:183], v[104:107], v[168:171], a[180:183]\n\t" \
  "ds_read_b64_tr_b16 v[100:101], v22 offset:16384\n\t" \
  "v_mfma_f32_16x16x32_bf16 a[180:183], v[108:111], v[172:175], a[180:183]\n\t" \
  "v_mfma_f32_16x16x32_bf16 a[176:179], v[108:111], v[164:167], a[176:179]\n\t" \
  "ds_read_b64_tr_b16 v[102:103], v22 offset:17408\n\t" \
  "v_mfma_f32_16x16x32_bf16 a[176:179], v[104:107], v[160:163], a[176:179]\n\t" \
  "v_mfma_f32_16x16x32_bf16 a[172:175], v[104:107], v[152:155], a[172:175]\n\t" \
  "ds_read_b128 v[252:255], v17 offset:19456\n\t" \
  "v_mfma_f32_16x16x32_bf16 a[172:175], v[108:111], v[156:159], a[172:175]\n\t" \
  "v_add_u32 v22, s51, v22\n\t" \
  "v_mfma_f32_16x16x32_bf16 a[168:171], v[108:111], v[148:151], a[168:171]\n\t" \
  "v_mfma_f32_16x16x32_bf16 a[168:171], v[104:107], v[144:147], a[168:171]\n\t" \
  "v_mfma_f32_16x16x32_bf16 a[164:167], v[104:107], v[136:139], a[164:167]\n\t" \
  "v_mfma_f32_16x16x32_bf16 a[164:167], v[108:111], v[140:143], a[164:167]\n\t" \
  "v_mfma_f32_16x16x32_bf16 a[160:163], v[108:111], v[132:135], a[160:163]\n\t" \
  "v_mfma_f32_16x16x32_bf16 a[160:163], v[104:107], v[128:131], a[160:163]\n\t" \
  "v_mfma_f32_16x16x32_bf16 a[192:195], v[112:115], v[128:131], a[192:195]\n\t" \
  "ds_read_b64_tr_b16 v[104:105], v23 offset:0\n\t" \
  "v_mfma_f32_16x16x32_bf16 a[192:195], v[116:119], v[132:135], a[192:195]\n\t" \
  "v_mfma_f32_16x16x32_bf16 a[196:199], v[116:119], v[140:143], a[196:199]\n\t" \
  "ds_read_b64_tr_b16 v[106:107], v23 offset:1024\n\t" \
  "v_mfma_f32_16x16x32_bf16 a[196:199], v[112:115], v[136:139], a[196:199]\n\t" \
  "v_mfma_f32_16x16x32_bf16 a[200:203], v[112:115], v[144:147], a[200:203]\n\t" \
  "ds_read_b64_tr_b16 v[108:109], v23 offset:16384\n\t" \
  "v_mfma_f32_16x16x32_bf16 a[200:203], v[116:119], v[148:151], a[200:203]\n\t" \
  "v_mfma_f32_16x16x32_bf16 a[204:207], v[116:119], v[156:159], a[204:207]\n\t" \
  "ds_read_b64_tr_b16 v[110:111], v23 offset:17408\n\t" \
  "v_mfma_f32_16x16x32_bf16 a[204:207], v[112:115], v[152:155], a[204:207]\n\t" \
  "v_mfma_f32_16x16x32_bf16 a[208:211], v[112:115], v[160:163], a[208:211]\n\t" \
  "v_mfma_f32_16x16x32_bf16 a[208:211], v[116:119], v[164:167], a[208:211]\n\t" \
  "v_add_u32 v23, s51, v23\n\t" \
  "v_mfma_f32_16x16x32_bf16 a[212:215], v[116:119], v[172:175], a[212:215]\n\t" \
  "v_mfma_f32_16x16x32_bf16 a[212:215], v[112:115], v[168:171], a[212:215]\n\t" \
  "v_xor_b32 v16, 0x8000, v16\n\t" \
  "v_mfma_f32_16x16x32_bf16 a[216:219], v[112:115], v[176:179], a[216:219]\n\t" \
  "v_xor_b32 v17, 0x8000, v17\n\t" \
  "v_mfma_f32_16x16x32_bf16 a[216:219], v[116:119], v[180:183], a[216:219]\n\t" \
  "v_mfma_f32_16x16x32_bf16 a[220:223], v[116:119], v[188:191], a[220:223]\n\t" \
  "v_mfma_f32_16x16x32_bf16 a[220:223], v[112:115], v[184:187], a[220:223]\n\t" \
  "v_mfma_f32_16x16x32_bf16 a[252:255], v[120:123], v[184:187], a[252:255]\n\t" \
  "ds_read_b64_tr_b16 v[112:113], v24 offset:0\n\t" \
  "v_mfma_f32_16x16x32_bf16 a[252:255], v[124:127], v[188:191], a[252:255]\n\t" \
  "ds_read_b64_tr_b16 v[114:115], v24 offset:1024\n\t" \
  "v_mfma_f32_16x16x32_bf16 a[248:251], v[124:127], v[180:183], a[248:251]\n\t" \
  "ds_read_b64_tr_b16 v[116:117], v24 offset:16384\n\t" \
  "v_mfma_f32_16x16x32_bf16 a[248:251], v[120:123], v[176:179], a[248:251]\n\t" \
  "ds_read_b64_tr_b16 v[118:119], v24 offset:17408\n\t" \
  "v_mfma_f32_16x16x32_bf16 a[244:247], v[120:123], v[168:171], a[244:247]\n\t" \
  "v_mfma_f32_16x16x32_bf16 a[244:247], v[124:127], v[172:175], a[244:247]\n\t" \
  "v_add_u32 v24, s51, v24\n\t" \
  "v_mfma_f32_16x16x32_bf16 a[240:243], v[124:127], v[164:167], a[240:243]\n\t" \
  "v_mfma_f32_16x16x32_bf16 a[240:243], v[120:123], v[160:163], a[240:243]\n\t" \
  "v_mfma_f32_16x16x32_bf16 a[236:239], v[120:123], v[152:155], a[236:239]\n\t" \
  "v_mfma_f32_16x16x32_bf16 a[236:239], v[124:127], v[156:159], a[236:239]\n\t" \
  "v_mfma_f32_16x16x32_bf16 a[232:235], v[124:127], v[148:151], a[232:235]\n\t" \
  "v_mfma_f32_16x16x32_bf16 a[232:235], v[120:123], v[144:147], a[232:235]\n\t" \
  "v_mfma_f32_16x16x32_bf16 a[228:231], v[120:123], v[136:139], a[228:231]\n\t" \
  "v_mfma_f32_16x16x32_bf16 a[228:231], v[124:127], v[140:143], a[228:231]\n\t" \
  "v_mfma_f32_16x16x32_bf16 a[224:227], v[124:127], v[132:135], a[224:227]\n\t" \
  "v_mfma_f32_16x16x32_bf16 a[224:227], v[120:123], v[128:131], a[224:227]\n\t" \
  "s_waitcnt lgkmcnt(0)\n\t" \
  "s_waitcnt vmcnt(0)\n\t" \
  "v_readfirstlane_b32 s54, v12\n\t" \
  "s_nop 3\n\t" \
  "s_cmp_eq_u32 s54, s65\n\t" \
  "s_cbranch_scc1 NANOUT_%=\n\t" \
  "v_mfma_f32_16x16x32_bf16 a[0:3], v[64:67], v[192:195], a[0:3]\n\t" \
  "v_mfma_f32_16x16x32_bf16 a[0:3], v[68:71], v[196:199], a[0:3]\n\t" \
  "v_mfma_f32_16x16x32_bf16 a[4:7], v[64:67], v[200:203], a[4:7]\n\t" \
  "v_mov_b32 v16, 0xc0135761\n\t" \
  "v_mfma_f32_16x16x32_bf16 a[4:7], v[68:71], v[204:207], a[4:7]\n\t" \
  "v_mov_b32 v17, 0xc0135761\n\t" \
  "v_mfma_f32_16x16x32_bf16 a[8:11], v[64:67], v[208:211], a[8:11]\n\t" \
  "v_mov_b32 v18, 0xbdd2d3e8\n\t" \
  "v_mfma_f32_16x16x32_bf16 a[8:11], v[68:71], v[212:215], a[8:11]\n\t" \
  "v_mov_b32 v19, 0xbdd2d3e8\n\t" \
  "v_mfma_f32_16x16x32_bf16 a[12:15], v[64:67], v[216:219], a[12:15]\n\t" \
  "v_mov_b32 v20, 1.0\n\t" \
  "v_mfma_f32_16x16x32_bf16 a[12:15], v[68:71], v[220:223], a[12:15]\n\t" \
  "v_mov_b32 v21, 1.0\n\t" \
  "v_mfma_f32_16x16x32_bf16 a[16:19], v[64:67], v[224:227], a[16:19]\n\t" \
  "v_mbcnt_lo_u32_b32 v30, -1, 0\n\t" \
  "v_mfma_f32_16x16x32_bf16 a[16:19], v[68:71], v[228:231], a[16:19]\n\t" \
  "v_mbcnt_hi_u32_b32 v30, -1, v30\n\t" \
  "v_mfma_f32_16x16x32_bf16 a[20:23], v[64:67], v[232:235], a[20:23]\n\t" \
  "s_lshl_b32 s54, s66, 14\n\t" \
  "v_mfma_f32_16x16x32_bf16 a[20:23], v[68:71], v[236:239], a[20:23]\n\t" \
  "s_add_u32 s54, s54, 98304\n\t" \
  "v_mfma_f32_16x16x32_bf16 a[24:27], v[64:67], v[240:243], a[24:27]\n\t" \
  "v_and_b32 v31, 15, v30\n\t" \
  "v_mfma_f32_16x16x32_bf16 a[24:27], v[68:71], v[244:247], a[24:27]\n\t" \
  "v_lshrrev_b32 v30, 4, v30\n\t" \
  "v_mfma_f32_16x16x32_bf16 a[28:31], v[64:67], v[248:251], a[28:31]\n\t" \
  "v_mul_u32_u24 v22, 272, v31\n\t" \
  "v_mfma_f32_16x16x32_bf16 a[28:31], v[68:71], v[252:255], a[28:31]\n\t" \
  "v_lshl_add_u32 v22, v30, 3, v22\n\t" \
  "v_mfma_f32_16x16x32_bf16 a[32:35], v[72:75], v[192:195], a[32:35]\n\t" \
  "v_add_u32 v22, s54, v22\n\t" \
  "v_mfma_f32_16x16x32_bf16 a[32:35], v[76:79], v[196:199], a[32:35]\n\t" \
  "v_mul_u32_u24 v23, 272, v30\n\t" \
  "v_mfma_f32_16x16x32_bf16 a[36:39], v[72:75], v[200:203], a[36:39]\n\t" \
  "v_lshl_add_u32 v23, v31, 4, v23\n\t" \
  "v_mfma_f32_16x16x32_bf16 a[36:39], v[76:79], v[204:207], a[36:39]\n\t" \
  "v_add_u32 v23, s54, v23\n\t" \
  "v_mfma_f32_16x16x32_bf16 a[40:43], v[72:75], v[208:211], a[40:43]\n\t" \
  "v_mov_b32 v24, %[ooff]\n\t" \
  "v_mfma_f32_16x16x32_bf16 a[40:43], v[76:79], v[212:215], a[40:43]\n\t" \
  "v_mfma_f32_16x16x32_bf16 a[44:47], v[72:75], v[216:219], a[44:47]\n\t" \
  "v_mfma_f32_16x16x32_bf16 a[44:47], v[76:79], v[220:223], a[44:47]\n\t" \
  "v_mfma_f32_16x16x32_bf16 a[48:51], v[72:75], v[224:227], a[48:51]\n\t" \
  "v_mfma_f32_16x16x32_bf16 a[48:51], v[76:79], v[228:231], a[48:51]\n\t" \
  "v_mfma_f32_16x16x32_bf16 a[52:55], v[72:75], v[232:235], a[52:55]\n\t" \
  "v_mfma_f32_16x16x32_bf16 a[52:55], v[76:79], v[236:239], a[52:55]\n\t" \
  "v_mfma_f32_16x16x32_bf16 a[56:59], v[72:75], v[240:243], a[56:59]\n\t" \
  "v_mfma_f32_16x16x32_bf16 a[56:59], v[76:79], v[244:247], a[56:59]\n\t" \
  "v_mfma_f32_16x16x32_bf16 a[60:63], v[72:75], v[248:251], a[60:63]\n\t" \
  "v_mfma_f32_16x16x32_bf16 a[60:63], v[76:79], v[252:255], a[60:63]\n\t" \
  "v_mfma_f32_16x16x32_bf16 a[64:67], v[80:83], v[192:195], a[64:67]\n\t" \
  "v_mfma_f32_16x16x32_bf16 a[64:67], v[84:87], v[196:199], a[64:67]\n\t" \
  "v_mfma_f32_16x16x32_bf16 a[68:71], v[80:83], v[200:203], a[68:71]\n\t" \
  "v_accvgpr_read_b32 v128, a0\n\t" \
  "v_accvgpr_read_b32 v129, a1\n\t" \
  "v_mfma_f32_16x16x32_bf16 a[68:71], v[84:87], v[204:207], a[68:71]\n\t" \
  "v_accvgpr_read_b32 v130, a2\n\t" \
  "v_accvgpr_read_b32 v131, a3\n\t" \
  "v_mfma_f32_16x16x32_bf16 a[72:75], v[80:83], v[208:211], a[72:75]\n\t" \
  "v_accvgpr_read_b32 v132, a32\n\t" \
  "v_mfma_f32_16x16x32_bf16 a[72:75], v[84:87], v[212:215], a[72:75]\n\t" \
  "v_accvgpr_read_b32 v133, a33\n\t" \
  "v_accvgpr_read_b32 v134, a34\n\t" \
  "v_mfma_f32_16x16x32_bf16 a[76:79], v[80:83], v[216:219], a[76:79]\n\t" \
  "v_accvgpr_read_b32 v135, a35\n\t" \
  "v_pk_mul_f32 v[136:137], v[128:129], v[128:129]\n\t" \
  "v_mfma_f32_16x16x32_bf16 a[76:79], v[84:87], v[220:223], a[76:79]\n\t" \
  "v_pk_mul_f32 v[138:139], v[130:131], v[130:131]\n\t" \
  "v_mfma_f32_16x16x32_bf16 a[80:83], v[80:83], v[224:227], a[80:83]\n\t" \
  "v_pk_mul_f32 v[140:141], v[132:133], v[132:133]\n\t" \
  "v_pk_mul_f32 v[142:143], v[134:135], v[134:135]\n\t" \
  "v_mfma_f32_16x16x32_bf16 a[80:83], v[84:87], v[228:231], a[80:83]\n\t" \
  "v_pk_fma_f32 v[136:137], v[136:137], v[18:19], v[16:17]\n\t" \
  "v_pk_fma_f32 v[138:139], v[138:139], v[18:19], v[16:17]\n\t" \
  "v_mfma_f32_16x16x32_bf16 a[84:87], v[80:83], v[232:235], a[84:87]\n\t" \
  "v_pk_fma_f32 v[140:141], v[140:141], v[18:19], v[16:17]\n\t" \
  "v_mfma_f32_16x16x32_bf16 a[84:87], v[84:87], v[236:239], a[84:87]\n\t" \
  "v_pk_fma_f32 v[142:143], v[142:143], v[18:19], v[16:17]\n\t" \
  "v_pk_mul_f32 v[136:137], v[136:137], v[128:129]\n\t" \
  "v_mfma_f32_16x16x32_bf16 a[88:91], v[80:83], v[240:243], a[88:91]\n\t" \
  "v_pk_mul_f32 v[138:139], v[138:139], v[130:131]\n\t" \
  "v_pk_mul_f32 v[140:141], v[140:141], v[132:133]\n\t" \
  "v_mfma_f32_16x16x32_bf16 a[88:91], v[84:87], v[244:247], a[88:91]\n\t" \
  "v_pk_mul_f32 v[142:143], v[142:143], v[134:135]\n\t" \
  "v_mfma_f32_16x16x32_bf16 a[92:95], v[80:83], v[248:251], a[92:95]\n\t" \
  "v_exp_f32 v136, v136\n\t" \
  "v_exp_f32 v137, v137\n\t" \
  "v_mfma_f32_16x16x32_bf16 a[92:95], v[84:87], v[252:255], a[92:95]\n\t" \
  "v_exp_f32 v138, v138\n\t" \
  "v_exp_f32 v139, v139\n\t" \
  "v_mfma_f32_16x16x32_bf16 a[96:99], v[88:91], v[192:195], a[96:99]\n\t" \
  "v_exp_f32 v140, v140\n\t" \
  "v_mfma_f32_16x16x32_bf16 a[96:99], v[92:95], v[196:199], a[96:99]\n\t" \
  "v_exp_f32 v141, v141\n\t" \
  "v_exp_f32 v142, v142\n\t" \
  "v_mfma_f32_16x16x32_bf16 a[100:103], v[88:91], v[200:203], a[100:103]\n\t" \
  "v_exp_f32 v143, v143\n\t" \
  "v_pk_add_f32 v[136:137], v[136:137], v[20:21]\n\t" \
  "v_mfma_f32_16x16x32_bf16 a[100:103], v[92:95], v[204:207], a[100:103]\n\t" \
  "v_pk_add_f32 v[138:139], v[138:139], v[20:21]\n\t" \
  "v_mfma_f32_16x16x32_bf16 a[104:107], v[88:91], v[208:211], a[104:107]\n\t" \
  "v_pk_add_f32 v[140:141], v[140:141], v[20:21]\n\t" \
  "v_pk_add_f32 v[142:143], v[142:143], v[20:21]\n\t" \
  "v_mfma_f32_16x16x32_bf16 a[104:107], v[92:95], v[212:215], a[104:107]\n\t" \
  "v_rcp_f32 v136, v136\n\t" \
  "v_rcp_f32 v137, v137\n\t" \
  "v_mfma_f32_16x16x32_bf16 a[108:111], v[88:91], v[216:219], a[108:111]\n\t" \
  "v_rcp_f32 v138, v138\n\t" \
  "v_mfma_f32_16x16x32_bf16 a[108:111], v[92:95], v[220:223], a[108:111]\n\t" \
  "v_rcp_f32 v139, v139\n\t" \
  "v_rcp_f32 v140, v140\n\t" \
  "v_mfma_f32_16x16x32_bf16 a[112:115], v[88:91], v[224:227], a[112:115]\n\t" \
  "v_rcp_f32 v141, v141\n\t" \
  "v_rcp_f32 v142, v142\n\t" \
  "v_mfma_f32_16x16x32_bf16 a[112:115], v[92:95], v[228:231], a[112:115]\n\t" \
  "v_rcp_f32 v143, v143\n\t" \
  "v_mfma_f32_16x16x32_bf16 a[116:119], v[88:91], v[232:235], a[116:119]\n\t" \
  "v_pk_mul_f32 v[136:137], v[136:137], v[128:129]\n\t" \
  "v_pk_mul_f32 v[138:139], v[138:139], v[130:131]\n\t" \
  "v_mfma_f32_16x16x32_bf16 a[116:119], v[92:95], v[236:239], a[116:119]\n\t" \
  "v_pk_mul_f32 v[140:141], v[140:141], v[132:133]\n\t" \
  "v_pk_mul_f32 v[142:143], v[142:143], v[134:135]\n\t" \
  "v_mfma_f32_16x16x32_bf16 a[120:123], v[88:91], v[240:243], a[120:123]\n\t" \
  "v_cvt_pk_bf16_f32 v144, v136, v137\n\t" \
  "v_mfma_f32_16x16x32_bf16 a[120:123], v[92:95], v[244:247], a[120:123]\n\t" \
  "v_cvt_pk_bf16_f32 v145, v138, v139\n\t" \
  "v_cvt_pk_bf16_f32 v146, v140, v141\n\t" \
  "v_mfma_f32_16x16x32_bf16 a[124:127], v[88:91], v[248:251], a[124:127]\n\t" \
  "v_cvt_pk_bf16_f32 v147, v142, v143\n\t" \
  "ds_write_b64 v22, v[144:145] offset:0\n\t" \
  "v_mfma_f32_16x16x32_bf16 a[124:127], v[92:95], v[252:255], a[124:127]\n\t" \
  "ds_write_b64 v22, v[146:147] offset:32\n\t" \
  "v_mfma_f32_16x16x32_bf16 a[128:131], v[96:99], v[192:195], a[128:131]\n\t" \
  "v_mfma_f32_16x16x32_bf16 a[128:131], v[100:103], v[196:199], a[128:131]\n\t" \
  "v_mfma_f32_16x16x32_bf16 a[132:135], v[96:99], v[200:203], a[132:135]\n\t" \
  "v_accvgpr_read_b32 v152, a64\n\t" \
  "v_accvgpr_read_b32 v153, a65\n\t" \
  "v_mfma_f32_16x16x32_bf16 a[132:135], v[100:103], v[204:207], a[132:135]\n\t" \
  "v_accvgpr_read_b32 v154, a66\n\t" \
  "v_accvgpr_read_b32 v155, a67\n\t" \
  "v_mfma_f32_16x16x32_bf16 a[136:139], v[96:99], v[208:211], a[136:139]\n\t" \
  "v_accvgpr_read_b32 v156, a96\n\t" \
  "v_mfma_f32_16x16x32_bf16 a[136:139], v[100:103], v[212:215], a[136:139]\n\t" \
  "v_accvgpr_read_b32 v157, a97\n\t" \
  "v_accvgpr_read_b32 v158, a98\n\t" \
  "v_mfma_f32_16x16x32_bf16 a[140:143], v[96:99], v[216:219], a[140:143]\n\t" \
  "v_accvgpr_read_b32 v159, a99\n\t" \
  "v_pk_mul_f32 v[160:161], v[152:153], v[152:153]\n\t" \
  "v_mfma_f32_16x16x32_bf16 a[140:143], v[100:103], v[220:223], a[140:143]\n\t" \
  "v_pk_mul_f32 v[162:163], v[154:155], v[154:155]\n\t" \
  "v_mfma_f32_16x16x32_bf16 a[144:147], v[96:99], v[224:227], a[144:147]\n\t" \
  "v_pk_mul_f32 v[164:165], v[156:157], v[156:157]\n\t" \
  "v_pk_mul_f32 v[166:167], v[158:159], v[158:159]\n\t" \
  "v_mfma_f32_16x16x32_bf16 a[144:147], v[100:103], v[228:231], a[144:147]\n\t" \
  "v_pk_fma_f32 v[160:161], v[160:161], v[18:19], v[16:17]\n\t" \
  "v_pk_fma_f32 v[162:163], v[162:163], v[18:19], v[16:17]\n\t" \
  "v_mfma_f32_16x16x32_bf16 a[148:151], v[96:99], v[232:235], a[148:151]\n\t" \
  "v_pk_fma_f32 v[164:165], v[164:165], v[18:19], v[16:17]\n\t" \
  "v_mfma_f32_16x16x32_bf16 a[148:151], v[100:103], v[236:239], a[148:151]\n\t" \
  "v_pk_fma_f32 v[166:167], v[166:167], v[18:19], v[16:17]\n\t" \
  "v_pk_mul_f32 v[160:161], v[160:161], v[152:153]\n\t" \
  "v_mfma_f32_16x16x32_bf16 a[152:155], v[96:99], v[240:243], a[152:155]\n\t" \
  "v_pk_mul_f32 v[162:163], v[162:163], v[154:155]\n\t" \
  "v_pk_mul_f32 v[164:165], v[164:165], v[156:157]\n\t" \
  "v_mfma_f32_16x16x32_bf16 a[152:155], v[100:103], v[244:247], a[152:155]\n\t" \
  "v_pk_mul_f32 v[166:167], v[166:167], v[158:159]\n\t" \
  "v_mfma_f32_16x16x32_bf16 a[156:159], v[96:99], v[248:251], a[156:159]\n\t" \
  "v_exp_f32 v160, v160\n\t" \
  "v_exp_f32 v161, v161\n\t" \
  "v_mfma_f32_16x16x32_bf16 a[156:159], v[100:103], v[252:255], a[156:159]\n\t" \
  "v_exp_f32 v162, v162\n\t" \
  "v_exp_f32 v163, v163\n\t" \
  "v_mfma_f32_16x16x32_bf16 a[160:163], v[104:107], v[192:195], a[160:163]\n\t" \
  "v_exp_f32 v164, v164\n\t" \
  "v_mfma_f32_16x16x32_bf16 a[160:163], v[108:111], v[196:199], a[160:163]\n\t" \
  "v_exp_f32 v165, v165\n\t" \
  "v_exp_f32 v166, v166\n\t" \
  "v_mfma_f32_16x16x32_bf16 a[164:167], v[104:107], v[200:203], a[164:167]\n\t" \
  "v_exp_f32 v167, v167\n\t" \
  "v_pk_add_f32 v[160:161], v[160:161], v[20:21]\n\t" \
  "v_mfma_f32_16x16x32_bf16 a[164:167], v[108:111], v[204:207], a[164:167]\n\t" \
  "v_pk_add_f32 v[162:163], v[162:163], v[20:21]\n\t" \
  "v_mfma_f32_16x16x32_bf16 a[168:171], v[104:107], v[208:211], a[168:171]\n\t" \
  "v_pk_add_f32 v[164:165], v[164:165], v[20:21]\n\t" \
  "v_pk_add_f32 v[166:167], v[166:167], v[20:21]\n\t" \
  "v_mfma_f32_16x16x32_bf16 a[168:171], v[108:111], v[212:215], a[168:171]\n\t" \
  "v_rcp_f32 v160, v160\n\t" \
  "v_rcp_f32 v161, v161\n\t" \
  "v_mfma_f32_16x16x32_bf16 a[172:175], v[104:107], v[216:219], a[172:175]\n\t" \
  "v_rcp_f32 v162, v162\n\t" \
  "v_mfma_f32_16x16x32_bf16 a[172:175], v[108:111], v[220:223], a[172:175]\n\t" \
  "v_rcp_f32 v163, v163\n\t" \
  "v_rcp_f32 v164, v164\n\t" \
  "v_mfma_f32_16x16x32_bf16 a[176:179], v[104:107], v[224:227], a[176:179]\n\t" \
  "v_rcp_f32 v165, v165\n\t" \
  "v_rcp_f32 v166, v166\n\t" \
  "v_mfma_f32_16x16x32_bf16 a[176:179], v[108:111], v[228:231], a[176:179]\n\t" \
  "v_rcp_f32 v167, v167\n\t" \
  "v_mfma_f32_16x16x32_bf16 a[180:183], v[104:107], v[232:235], a[180:183]\n\t" \
  "v_pk_mul_f32 v[160:161], v[160:161], v[152:153]\n\t" \
  "v_pk_mul_f32 v[162:163], v[162:163], v[154:155]\n\t" \
  "v_mfma_f32_16x16x32_bf16 a[180:183], v[108:111], v[236:239], a[180:183]\n\t" \
  "v_pk_mul_f32 v[164:165], v[164:165], v[156:157]\n\t" \
  "v_pk_mul_f32 v[166:167], v[166:167], v[158:159]\n\t" \
  "v_mfma_f32_16x16x32_bf16 a[184:187], v[104:107], v[240:243], a[184:187]\n\t" \
  "v_cvt_pk_bf16_f32 v168, v160, v161\n\t" \
  "v_mfma_f32_16x16x32_bf16 a[184:187], v[108:111], v[244:247], a[184:187]\n\t" \
  "v_cvt_pk_bf16_f32 v169, v162, v163\n\t" \
  "v_cvt_pk_bf16_f32 v170, v164, v165\n\t" \
  "v_mfma_f32_16x16x32_bf16 a[188:191], v[104:107], v[248:251], a[188:191]\n\t" \
  "v_cvt_pk_bf16_f32 v171, v166, v167\n\t" \
  "ds_write_b64 v22, v[168:169] offset:64\n\t" \
  "v_mfma_f32_16x16x32_bf16 a[188:191], v[108:111], v[252:255], a[188:191]\n\t" \
  "ds_write_b64 v22, v[170:171] offset:96\n\t" \
  "v_mfma_f32_16x16x32_bf16 a[192:195], v[112:115], v[192:195], a[192:195]\n\t" \
  "v_mfma_f32_16x16x32_bf16 a[192:195], v[116:119], v[196:199], a[192:195]\n\t" \
  "v_mfma_f32_16x16x32_bf16 a[196:199], v[112:115], v[200:203], a[196:199]\n\t" \
  "v_accvgpr_read_b32 v128, a128\n\t" \
  "v_accvgpr_read_b32 v129, a129\n\t" \
  "v_mfma_f32_16x16x32_bf16 a[196:199], v[116:119], v[204:207], a[196:199]\n\t" \
  "v_accvgpr_read_b32 v130, a130\n\t" \
  "v_accvgpr_read_b32 v131, a131\n\t" \
  "v_mfma_f32_16x16x32_bf16 a[200:203], v[112:115], v[208:211], a[200:203]\n\t" \
  "v_accvgpr_read_b32 v132, a160\n\t" \
  "v_mfma_f32_16x16x32_bf16 a[200:203], v[116:119], v[212:215], a[200:203]\n\t" \
  "v_accvgpr_read_b32 v133, a161\n\t" \
  "v_accvgpr_read_b32 v134, a162\n\t" \
  "v_mfma_f32_16x16x32_bf16 a[204:207], v[112:115], v[216:219], a[204:207]\n\t" \
  "v_accvgpr_read_b32 v135, a163\n\t" \
  "v_pk_mul_f32 v[136:137], v[128:129], v[128:129]\n\t" \
  "v_mfma_f32_16x16x32_bf16 a[204:207], v[116:119], v[220:223], a[204:207]\n\t" \
  "v_pk_mul_f32 v[138:139], v[130:131], v[130:131]\n\t" \
  "v_mfma_f32_16x16x32_bf16 a[208:211], v[112:115], v[224:227], a[208:211]\n\t" \
  "v_pk_mul_f32 v[140:141], v[132:133], v[132:133]\n\t" \
  "v_pk_mul_f32 v[142:143], v[134:135], v[134:135]\n\t" \
  "v_mfma_f32_16x16x32_bf16 a[208:211], v[116:119], v[228:231], a[208:211]\n\t" \
  "v_pk_fma_f32 v[136:137], v[136:137], v[18:19], v[16:17]\n\t" \
  "v_pk_fma_f32 v[138:139], v[138:139], v[18:19], v[16:17]\n\t" \
  "v_mfma_f32_16x16x32_bf16 a[212:215], v[112:115], v[232:235], a[212:215]\n\t" \
  "v_pk_fma_f32 v[140:141], v[140:141], v[18:19], v[16:17]\n\t" \
  "v_mfma_f32_16x16x32_bf16 a[212:215], v[116:119], v[236:239], a[212:215]\n\t" \
  "v_pk_fma_f32 v[142:143], v[142:143], v[18:19], v[16:17]\n\t" \
  "v_pk_mul_f32 v[136:137], v[136:137], v[128:129]\n\t" \
  "v_mfma_f32_16x16x32_bf16 a[216:219], v[112:115], v[240:243], a[216:219]\n\t" \
  "v_pk_mul_f32 v[138:139], v[138:139], v[130:131]\n\t" \
  "v_pk_mul_f32 v[140:141], v[140:141], v[132:133]\n\t" \
  "v_mfma_f32_16x16x32_bf16 a[216:219], v[116:119], v[244:247], a[216:219]\n\t" \
  "v_pk_mul_f32 v[142:143], v[142:143], v[134:135]\n\t" \
  "v_mfma_f32_16x16x32_bf16 a[220:223], v[112:115], v[248:251], a[220:223]\n\t" \
  "v_exp_f32 v136, v136\n\t" \
  "v_exp_f32 v137, v137\n\t" \
  "v_mfma_f32_16x16x32_bf16 a[220:223], v[116:119], v[252:255], a[220:223]\n\t" \
  "v_exp_f32 v138, v138\n\t" \
  "v_exp_f32 v139, v139\n\t" \
  "v_mfma_f32_16x16x32_bf16 a[224:227], v[56:59], v[192:195], a[224:227]\n\t" \
  "v_exp_f32 v140, v140\n\t" \
  "v_mfma_f32_16x16x32_bf16 a[224:227], v[60:63], v[196:199], a[224:227]\n\t" \
  "v_exp_f32 v141, v141\n\t" \
  "v_exp_f32 v142, v142\n\t" \
  "v_mfma_f32_16x16x32_bf16 a[228:231], v[56:59], v[200:203], a[228:231]\n\t" \
  "v_exp_f32 v143, v143\n\t" \
  "v_pk_add_f32 v[136:137], v[136:137], v[20:21]\n\t" \
  "v_mfma_f32_16x16x32_bf16 a[228:231], v[60:63], v[204:207], a[228:231]\n\t" \
  "v_pk_add_f32 v[138:139], v[138:139], v[20:21]\n\t" \
  "v_mfma_f32_16x16x32_bf16 a[232:235], v[56:59], v[208:211], a[232:235]\n\t" \
  "v_pk_add_f32 v[140:141], v[140:141], v[20:21]\n\t" \
  "v_pk_add_f32 v[142:143], v[142:143], v[20:21]\n\t" \
  "v_mfma_f32_16x16x32_bf16 a[232:235], v[60:63], v[212:215], a[232:235]\n\t" \
  "v_rcp_f32 v136, v136\n\t" \
  "v_rcp_f32 v137, v137\n\t" \
  "v_mfma_f32_16x16x32_bf16 a[236:239], v[56:59], v[216:219], a[236:239]\n\t" \
  "v_rcp_f32 v138, v138\n\t" \
  "v_mfma_f32_16x16x32_bf16 a[236:239], v[60:63], v[220:223], a[236:239]\n\t" \
  "v_rcp_f32 v139, v139\n\t" \
  "v_rcp_f32 v140, v140\n\t" \
  "v_mfma_f32_16x16x32_bf16 a[240:243], v[56:59], v[224:227], a[240:243]\n\t" \
  "v_rcp_f32 v141, v141\n\t" \
  "v_rcp_f32 v142, v142\n\t" \
  "v_mfma_f32_16x16x32_bf16 a[240:243], v[60:63], v[228:231], a[240:243]\n\t" \
  "v_rcp_f32 v143, v143\n\t" \
  "v_mfma_f32_16x16x32_bf16 a[244:247], v[56:59], v[232:235], a[244:247]\n\t" \
  "v_pk_mul_f32 v[136:137], v[136:137], v[128:129]\n\t" \
  "v_pk_mul_f32 v[138:139], v[138:139], v[130:131]\n\t" \
  "v_mfma_f32_16x16x32_bf16 a[244:247], v[60:63], v[236:239], a[244:247]\n\t" \
  "v_pk_mul_f32 v[140:141], v[140:141], v[132:133]\n\t" \
  "v_pk_mul_f32 v[142:143], v[142:143], v[134:135]\n\t" \
  "v_mfma_f32_16x16x32_bf16 a[248:251], v[56:59], v[240:243], a[248:251]\n\t" \
  "v_cvt_pk_bf16_f32 v144, v136, v137\n\t" \
  "v_mfma_f32_16x16x32_bf16 a[248:251], v[60:63], v[244:247], a[248:251]\n\t" \
  "v_cvt_pk_bf16_f32 v145, v138, v139\n\t" \
  "v_cvt_pk_bf16_f32 v146, v140, v141\n\t" \
  "v_mfma_f32_16x16x32_bf16 a[252:255], v[56:59], v[248:251], a[252:255]\n\t" \
  "v_cvt_pk_bf16_f32 v147, v142, v143\n\t" \
  "ds_write_b64 v22, v[144:145] offset:128\n\t" \
  "v_mfma_f32_16x16x32_bf16 a[252:255], v[60:63], v[252:255], a[252:255]\n\t" \
  "ds_write_b64 v22, v[146:147] offset:160\n\t" \
  "s_waitcnt lgkmcnt(0)\n\t" \
  "s_nop 7\n\t" \
  "v_accvgpr_read_b32 v128, a192\n\t" \
  "v_accvgpr_read_b32 v129, a193\n\t" \
  "v_accvgpr_read_b32 v130, a194\n\t" \
  "v_accvgpr_read_b32 v131, a195\n\t" \
  "v_accvgpr_read_b32 v132, a224\n\t" \
  "v_accvgpr_read_b32 v133, a225\n\t" \
  "v_accvgpr_read_b32 v134, a226\n\t" \
  "v_accvgpr_read_b32 v135, a227\n\t" \
  "v_pk_mul_f32 v[136:137], v[128:129], v[128:129]\n\t" \
  "v_pk_mul_f32 v[138:139], v[130:131], v[130:131]\n\t" \
  "v_pk_mul_f32 v[140:141], v[132:133], v[132:133]\n\t" \
  "v_pk_mul_f32 v[142:143], v[134:135], v[134:135]\n\t" \
  "v_pk_fma_f32 v[136:137], v[136:137], v[18:19], v[16:17]\n\t" \
  "v_pk_fma_f32 v[138:139], v[138:139], v[18:19], v[16:17]\n\t" \
  "v_pk_fma_f32 v[140:141], v[140:141], v[18:19], v[16:17]\n\t" \
  "v_pk_fma_f32 v[142:143], v[142:143], v[18:19], v[16:17]\n\t" \
  "v_pk_mul_f32 v[136:137], v[136:137], v[128:129]\n\t" \
  "v_pk_mul_f32 v[138:139], v[138:139], v[130:131]\n\t" \
  "v_pk_mul_f32 v[140:141], v[140:141], v[132:133]\n\t" \
  "v_pk_mul_f32 v[142:143], v[142:143], v[134:135]\n\t" \
  "v_exp_f32 v136, v136\n\t" \
  "v_exp_f32 v137, v137\n\t" \
  "v_accvgpr_read_b32 v152, a4\n\t" \
  "v_exp_f32 v138, v138\n\t" \
  "v_exp_f32 v139, v139\n\t" \
  "v_accvgpr_read_b32 v153, a5\n\t" \
  "v_exp_f32 v140, v140\n\t" \
  "v_accvgpr_read_b32 v154, a6\n\t" \
  "v_exp_f32 v141, v141\n\t" \
  "v_exp_f32 v142, v142\n\t" \
  "v_accvgpr_read_b32 v155, a7\n\t" \
  "v_exp_f32 v143, v143\n\t" \
  "v_accvgpr_read_b32 v156, a36\n\t" \
  "v_pk_add_f32 v[136:137], v[136:137], v[20:21]\n\t" \
  "v_pk_add_f32 v[138:139], v[138:139], v[20:21]\n\t" \
  "v_pk_add_f32 v[140:141], v[140:141], v[20:21]\n\t" \
  "v_pk_add_f32 v[142:143], v[142:143], v[20:21]\n\t" \
  "v_rcp_f32 v136, v136\n\t" \
  "v_accvgpr_read_b32 v157, a37\n\t" \
  "v_rcp_f32 v137, v137\n\t" \
  "v_accvgpr_read_b32 v158, a38\n\t" \
  "v_accvgpr_read_b32 v159, a39\n\t" \
  "v_rcp_f32 v138, v138\n\t" \
  "v_pk_mul_f32 v[160:161], v[152:153], v[152:153]\n\t" \
  "v_pk_mul_f32 v[162:163], v[154:155], v[154:155]\n\t" \
  "v_rcp_f32 v139, v139\n\t" \
  "v_pk_mul_f32 v[164:165], v[156:157], v[156:157]\n\t" \
  "v_pk_mul_f32 v[166:167], v[158:159], v[158:159]\n\t" \
  "v_rcp_f32 v140, v140\n\t" \
  "v_pk_fma_f32 v[160:161], v[160:161], v[18:19], v[16:17]\n\t" \
  "v_pk_fma_f32 v[162:163], v[162:163], v[18:19], v[16:17]\n\t" \
  "v_rcp_f32 v141, v141\n\t" \
  "v_pk_fma_f32 v[164:165], v[164:165], v[18:19], v[16:17]\n\t" \
  "v_pk_fma_f32 v[166:167], v[166:167], v[18:19], v[16:17]\n\t" \
  "v_rcp_f32 v142, v142\n\t" \
  "v_pk_mul_f32 v[160:161], v[160:161], v[152:153]\n\t" \
  "v_pk_mul_f32 v[162:163], v[162:163], v[154:155]\n\t" \
  "v_rcp_f32 v143, v143\n\t" \
  "v_pk_mul_f32 v[164:165], v[164:165], v[156:157]\n\t" \
  "v_pk_mul_f32 v[166:167], v[166:167], v[158:159]\n\t" \
  "v_exp_f32 v160, v160\n\t" \
  "v_pk_mul_f32 v[136:137], v[136:137], v[128:129]\n\t" \
  "v_pk_mul_f32 v[138:139], v[138:139], v[130:131]\n\t" \
  "v_exp_f32 v161, v161\n\t" \
  "v_pk_mul_f32 v[140:141], v[140:141], v[132:133]\n\t" \
  "v_pk_mul_f32 v[142:143], v[142:143], v[134:135]\n\t" \
  "v_exp_f32 v162, v162\n\t" \
  "v_cvt_pk_bf16_f32 v144, v136, v137\n\t" \
  "v_cvt_pk_bf16_f32 v145, v138, v139\n\t" \
  "v_cvt_pk_bf16_f32 v146, v140, v141\n\t" \
  "v_exp_f32 v163, v163\n\t" \
  "v_cvt_pk_bf16_f32 v147, v142, v143\n\t" \
  "ds_write_b64 v22, v[144:145] offset:192\n\t" \
  "v_exp_f32 v164, v164\n\t" \
  "ds_write_b64 v22, v[146:147] offset:224\n\t" \
  "ds_read_b128 v[32:35], v23 offset:0\n\t" \
  "v_exp_f32 v165, v165\n\t" \
  "ds_read_b128 v[36:39], v23 offset:1088\n\t" \
  "ds_read_b128 v[40:43], v23 offset:2176\n\t" \
  "ds_read_b128 v[44:47], v23 offset:3264\n\t" \
  "v_exp_f32 v166, v166\n\t" \
  "v_accvgpr_read_b32 v176, a68\n\t" \
  "v_accvgpr_read_b32 v177, a69\n\t" \
  "v_exp_f32 v167, v167\n\t" \
  "v_accvgpr_read_b32 v178, a70\n\t" \
  "v_accvgpr_read_b32 v179, a71\n\t" \
  "v_accvgpr_read_b32 v180, a100\n\t" \
  "v_pk_add_f32 v[160:161], v[160:161], v[20:21]\n\t" \
  "v_pk_add_f32 v[162:163], v[162:163], v[20:21]\n\t" \
  "v_pk_add_f32 v[164:165], v[164:165], v[20:21]\n\t" \
  "v_pk_add_f32 v[166:167], v[166:167], v[20:21]\n\t" \
  "v_rcp_f32 v160, v160\n\t" \
  "v_accvgpr_read_b32 v181, a101\n\t" \
  "v_accvgpr_read_b32 v182, a102\n\t" \
  "v_accvgpr_read_b32 v183, a103\n\t" \
  "v_rcp_f32 v161, v161\n\t" \
  "v_pk_mul_f32 v[184:185], v[176:177], v[176:177]\n\t" \
  "v_pk_mul_f32 v[186:187], v[178:179], v[178:179]\n\t" \
  "v_pk_mul_f32 v[188:189], v[180:181], v[180:181]\n\t" \
  "v_pk_mul_f32 v[190:191], v[182:183], v[182:183]\n\t" \
  "v_rcp_f32 v162, v162\n\t" \
  "v_pk_fma_f32 v[184:185], v[184:185], v[18:19], v[16:17]\n\t" \
  "v_pk_fma_f32 v[186:187], v[186:187], v[18:19], v[16:17]\n\t" \
  "v_pk_fma_f32 v[188:189], v[188:189], v[18:19], v[16:17]\n\t" \
  "v_rcp_f32 v163, v163\n\t" \
  "v_pk_fma_f32 v[190:191], v[190:191], v[18:19], v[16:17]\n\t" \
  "v_pk_mul_f32 v[184:185], v[184:185], v[176:177]\n\t" \
  "v_pk_mul_f32 v[186:187], v[186:187], v[178:179]\n\t" \
  "v_pk_mul_f32 v[188:189], v[188:189], v[180:181]\n\t" \
  "v_rcp_f32 v164, v164\n\t" \
  "v_pk_mul_f32 v[190:191], v[190:191], v[182:183]\n\t" \
  "s_waitcnt lgkmcnt(0)\n\t" \
  "global_store_dwordx4 v24, v[32:35], s[30:31] nt\n\t" \
  "v_rcp_f32 v165, v165\n\t" \
  "s_add_u32 s30, s30, 0x40000\n\t" \
  "s_addc_u32 s31, s31, 0\n\t" \
  "global_store_dwordx4 v24, v[36:39], s[30:31] nt\n\t" \
  "s_add_u32 s30, s30, 0x40000\n\t" \
  "v_rcp_f32 v166, v166\n\t" \
  "s_addc_u32 s31, s31, 0\n\t" \
  "global_store_dwordx4 v24, v[40:43], s[30:31] nt\n\t" \
  "s_add_u32 s30, s30, 0x40000\n\t" \
  "v_rcp_f32 v167, v167\n\t" \
  "s_addc_u32 s31, s31, 0\n\t" \
  "global_store_dwordx4 v24, v[44:47], s[30:31] nt\n\t" \
  "s_add_u32 s30, s30, 0x40000\n\t" \
  "s_addc_u32 s31, s31, 0\n\t" \
  "v_exp_f32 v184, v184\n\t" \
  "v_pk_mul_f32 v[160:161], v[160:161], v[152:153]\n\t" \
  "v_exp_f32 v185, v185\n\t" \
  "v_pk_mul_f32 v[162:163], v[162:163], v[154:155]\n\t" \
  "v_pk_mul_f32 v[164:165], v[164:165], v[156:157]\n\t" \
  "v_exp_f32 v186, v186\n\t" \
  "v_pk_mul_f32 v[166:167], v[166:167], v[158:159]\n\t" \
  "v_cvt_pk_bf16_f32 v168, v160, v161\n\t" \
  "v_exp_f32 v187, v187\n\t" \
  "v_cvt_pk_bf16_f32 v169, v162, v163\n\t" \
  "v_cvt_pk_bf16_f32 v170, v164, v165\n\t" \
  "v_exp_f32 v188, v188\n\t" \
  "v_cvt_pk_bf16_f32 v171, v166, v167\n\t" \
  "ds_write_b64 v22, v[168:169] offset:4352\n\t" \
  "v_exp_f32 v189, v189\n\t" \
  "ds_write_b64 v22, v[170:171] offset:4384\n\t" \
  "v_accvgpr_read_b32 v200, a132\n\t" \
  "v_exp_f32 v190, v190\n\t" \
  "v_accvgpr_read_b32 v201, a133\n\t" \
  "v_accvgpr_read_b32 v202, a134\n\t" \
  "v_exp_f32 v191, v191\n\t" \
  "v_accvgpr_read_b32 v203, a135\n\t" \
  "v_accvgpr_read_b32 v204, a164\n\t" \
  "v_pk_add_f32 v[184:185], v[184:185], v[20:21]\n\t" \
  "v_pk_add_f32 v[186:187], v[186:187], v[20:21]\n\t" \
  "v_pk_add_f32 v[188:189], v[188:189], v[20:21]\n\t" \
  "v_pk_add_f32 v[190:191], v[190:191], v[20:21]\n\t" \
  "v_rcp_f32 v184, v184\n\t" \
  "v_accvgpr_read_b32 v205, a165\n\t" \
  "v_rcp_f32 v185, v185\n\t" \
  "v_accvgpr_read_b32 v206, a166\n\t" \
  "v_accvgpr_read_b32 v207, a167\n\t" \
  "v_rcp_f32 v186, v186\n\t" \
  "v_pk_mul_f32 v[208:209], v[200:201], v[200:201]\n\t" \
  "v_pk_mul_f32 v[210:211], v[202:203], v[202:203]\n\t" \
  "v_rcp_f32 v187, v187\n\t" \
  "v_pk_mul_f32 v[212:213], v[204:205], v[204:205]\n\t" \
  "v_pk_mul_f32 v[214:215], v[206:207], v[206:207]\n\t" \
  "v_rcp_f32 v188, v188\n\t" \
  "v_pk_fma_f32 v[208:209], v[208:209], v[18:19], v[16:17]\n\t" \
  "v_pk_fma_f32 v[210:211], v[210:211], v[18:19], v[16:17]\n\t" \
  "v_rcp_f32 v189, v189\n\t" \
  "v_pk_fma_f32 v[212:213], v[212:213], v[18:19], v[16:17]\n\t" \
  "v_pk_fma_f32 v[214:215], v[214:215], v[18:19], v[16:17]\n\t" \
  "v_rcp_f32 v190, v190\n\t" \
  "v_pk_mul_f32 v[208:209], v[208:209], v[200:201]\n\t" \
  "v_pk_mul_f32 v[210:211], v[210:211], v[202:203]\n\t" \
  "v_rcp_f32 v191, v191\n\t" \
  "v_pk_mul_f32 v[212:213], v[212:213], v[204:205]\n\t" \
  "v_pk_mul_f32 v[214:215], v[214:215], v[206:207]\n\t" \
  "v_exp_f32 v208, v208\n\t" \
  "v_pk_mul_f32 v[184:185], v[184:185], v[176:177]\n\t" \
  "v_exp_f32 v209, v209\n\t" \
  "v_pk_mul_f32 v[186:187], v[186:187], v[178:179]\n\t" \
  "v_pk_mul_f32 v[188:189], v[188:189], v[180:181]\n\t" \
  "v_exp_f32 v210, v210\n\t" \
  "v_pk_mul_f32 v[190:191], v[190:191], v[182:183]\n\t" \
  "v_cvt_pk_bf16_f32 v192, v184, v185\n\t" \
  "v_exp_f32 v211, v211\n\t" \
  "v_cvt_pk_bf16_f32 v193, v186, v187\n\t" \
  "v_cvt_pk_bf16_f32 v194, v188, v189\n\t" \
  "v_exp_f32 v212, v212\n\t" \
  "v_cvt_pk_bf16_f32 v195, v190, v191\n\t" \
  "ds_write_b64 v22, v[192:193] offset:4416\n\t" \
  "v_exp_f32 v213, v213\n\t" \
  "ds_write_b64 v22, v[194:195] offset:4448\n\t" \
  "v_accvgpr_read_b32 v128, a196\n\t" \
  "v_exp_f32 v214, v214\n\t" \
  "v_accvgpr_read_b32 v129, a197\n\t" \
  "v_accvgpr_read_b32 v130, a198\n\t" \
  "v_exp_f32 v215, v215\n\t" \
  "v_accvgpr_read_b32 v131, a199\n\t" \
  "v_accvgpr_read_b32 v132, a228\n\t" \
  "v_pk_add_f32 v[208:209], v[208:209], v[20:21]\n\t" \
  "v_pk_add_f32 v[210:211], v[210:211], v[20:21]\n\t" \
  "v_pk_add_f32 v[212:213], v[212:213], v[20:21]\n\t" \
  "v_pk_add_f32 v[214:215], v[214:215], v[20:21]\n\t" \
  "v_rcp_f32 v208, v208\n\t" \
  "v_accvgpr_read_b32 v133, a229\n\t" \
  "v_rcp_f32 v209, v209\n\t" \
  "v_accvgpr_read_b32 v134, a230\n\t" \
  "v_accvgpr_read_b32 v135, a231\n\t" \
  "v_rcp_f32 v210, v210\n\t" \
  "v_pk_mul_f32 v[136:137], v[128:129], v[128:129]\n\t" \
  "v_pk_mul_f32 v[138:139], v[130:131], v[130:131]\n\t" \
  "v_rcp_f32 v211, v211\n\t" \
  "v_pk_mul_f32 v[140:141], v[132:133], v[132:133]\n\t" \
  "v_pk_mul_f32 v[142:143], v[134:135], v[134:135]\n\t" \
  "v_rcp_f32 v212, v212\n\t" \
  "v_pk_fma_f32 v[136:137], v[136:137], v[18:19], v[16:17]\n\t" \
  "v_pk_fma_f32 v[138:139], v[138:139], v[18:19], v[16:17]\n\t" \
  "v_rcp_f32 v213, v213\n\t" \
  "v_pk_fma_f32 v[140:141], v[140:141], v[18:19], v[16:17]\n\t" \
  "v_pk_fma_f32 v[142:143], v[142:143], v[18:19], v[16:17]\n\t" \
  "v_rcp_f32 v214, v214\n\t" \
  "v_pk_mul_f32 v[136:137], v[136:137], v[128:129]\n\t" \
  "v_pk_mul_f32 v[138:139], v[138:139], v[130:131]\n\t" \
  "v_rcp_f32 v215, v215\n\t" \
  "v_pk_mul_f32 v[140:141], v[140:141], v[132:133]\n\t" \
  "v_pk_mul_f32 v[142:143], v[142:143], v[134:135]\n\t" \
  "v_exp_f32 v136, v136\n\t" \
  "v_pk_mul_f32 v[208:209], v[208:209], v[200:201]\n\t" \
  "v_exp_f32 v137, v137\n\t" \
  "v_pk_mul_f32 v[210:211], v[210:211], v[202:203]\n\t" \
  "v_pk_mul_f32 v[212:213], v[212:213], v[204:205]\n\t" \
  "v_exp_f32 v138, v138\n\t" \
  "v_pk_mul_f32 v[214:215], v[214:215], v[206:207]\n\t" \
  "v_cvt_pk_bf16_f32 v216, v208, v209\n\t" \
  "v_exp_f32 v139, v139\n\t" \
  "v_cvt_pk_bf16_f32 v217, v210, v211\n\t" \
  "v_cvt_pk_bf16_f32 v218, v212, v213\n\t" \
  "v_exp_f32 v140, v140\n\t" \
  "v_cvt_pk_bf16_f32 v219, v214, v215\n\t" \
  "ds_write_b64 v22, v[216:217] offset:4480\n\t" \
  "v_exp_f32 v141, v141\n\t" \
  "ds_write_b64 v22, v[218:219] offset:4512\n\t" \
  "v_accvgpr_read_b32 v152, a8\n\t" \
  "v_exp_f32 v142, v142\n\t" \
  "v_accvgpr_read_b32 v153, a9\n\t" \
  "v_accvgpr_read_b32 v154, a10\n\t" \
  "v_exp_f32 v143, v143\n\t" \
  "v_accvgpr_read_b32 v155, a11\n\t" \
  "v_accvgpr_read_b32 v156, a40\n\t" \
  "v_pk_add_f32 v[136:137], v[136:137], v[20:21]\n\t" \
  "v_pk_add_f32 v[138:139], v[138:139], v[20:21]\n\t" \
  "v_pk_add_f32 v[140:141], v[140:141], v[20:21]\n\t" \
  "v_pk_add_f32 v[142:143], v[142:143], v[20:21]\n\t" \
  "v_rcp_f32 v136, v136\n\t" \
  "v_accvgpr_read_b32 v157, a41\n\t" \
  "v_rcp_f32 v137, v137\n\t" \
  "v_accvgpr_read_b32 v158, a42\n\t" \
  "v_accvgpr_read_b32 v159, a43\n\t" \
  "v_rcp_f32 v138, v138\n\t" \
  "v_pk_mul_f32 v[160:161], v[152:153], v[152:153]\n\t" \
  "v_pk_mul_f32 v[162:163], v[154:155], v[154:155]\n\t" \
  "v_rcp_f32 v139, v139\n\t" \
  "v_pk_mul_f32 v[164:165], v[156:157], v[156:157]\n\t" \
  "v_pk_mul_f32 v[166:167], v[158:159], v[158:159]\n\t" \
  "v_rcp_f32 v140, v140\n\t" \
  "v_pk_fma_f32 v[160:161], v[160:161], v[18:19], v[16:17]\n\t" \
  "v_pk_fma_f32 v[162:163], v[162:163], v[18:19], v[16:17]\n\t" \
  "v_rcp_f32 v141, v141\n\t" \
  "v_pk_fma_f32 v[164:165], v[164:165], v[18:19], v[16:17]\n\t" \
  "v_pk_fma_f32 v[166:167], v[166:167], v[18:19], v[16:17]\n\t" \
  "v_rcp_f32 v142, v142\n\t" \
  "v_pk_mul_f32 v[160:161], v[160:161], v[152:153]\n\t" \
  "v_pk_mul_f32 v[162:163], v[162:163], v[154:155]\n\t" \
  "v_rcp_f32 v143, v143\n\t" \
  "v_pk_mul_f32 v[164:165], v[164:165], v[156:157]\n\t" \
  "v_pk_mul_f32 v[166:167], v[166:167], v[158:159]\n\t" \
  "v_exp_f32 v160, v160\n\t" \
  "v_pk_mul_f32 v[136:137], v[136:137], v[128:129]\n\t" \
  "v_pk_mul_f32 v[138:139], v[138:139], v[130:131]\n\t" \
  "v_exp_f32 v161, v161\n\t" \
  "v_pk_mul_f32 v[140:141], v[140:141], v[132:133]\n\t" \
  "v_pk_mul_f32 v[142:143], v[142:143], v[134:135]\n\t" \
  "v_exp_f32 v162, v162\n\t" \
  "v_cvt_pk_bf16_f32 v144, v136, v137\n\t" \
  "v_cvt_pk_bf16_f32 v145, v138, v139\n\t" \
  "v_cvt_pk_bf16_f32 v146, v140, v141\n\t" \
  "v_exp_f32 v163, v163\n\t" \
  "v_cvt_pk_bf16_f32 v147, v142, v143\n\t" \
  "ds_write_b64 v22, v[144:145] offset:4544\n\t" \
  "v_exp_f32 v164, v164\n\t" \
  "ds_write_b64 v22, v[146:147] offset:4576\n\t" \
  "ds_read_b128 v[32:35], v23 offset:4352\n\t" \
  "v_exp_f32 v165, v165\n\t" \
  "ds_read_b128 v[36:39], v23 offset:5440\n\t" \
  "ds_read_b128 v[40:43], v23 offset:6528\n\t" \
  "ds_read_b128 v[44:47], v23 offset:7616\n\t" \
  "v_exp_f32 v166, v166\n\t" \
  "v_accvgpr_read_b32 v176, a72\n\t" \
  "v_accvgpr_read_b32 v177, a73\n\t" \
  "v_exp_f32 v167, v167\n\t" \
  "v_accvgpr_read_b32 v178, a74\n\t" \
  "v_accvgpr_read_b32 v179, a75\n\t" \
  "v_accvgpr_read_b32 v180, a104\n\t" \
  "v_pk_add_f32 v[160:161], v[160:161], v[20:21]\n\t" \
  "v_pk_add_f32 v[162:163], v[162:163], v[20:21]\n\t" \
  "v_pk_add_f32 v[164:165], v[164:165], v[20:21]\n\t" \
  "v_pk_add_f32 v[166:167], v[166:167], v[20:21]\n\t" \
  "v_rcp_f32 v160, v160\n\t" \
  "v_accvgpr_read_b32 v181, a105\n\t" \
  "v_accvgpr_read_b32 v182, a106\n\t" \
  "v_accvgpr_read_b32 v183, a107\n\t" \
  "v_rcp_f32 v161, v161\n\t" \
  "v_pk_mul_f32 v[184:185], v[176:177], v[176:177]\n\t" \
  "v_pk_mul_f32 v[186:187], v[178:179], v[178:179]\n\t" \
  "v_pk_mul_f32 v[188:189], v[180:181], v[180:181]\n\t" \
  "v_pk_mul_f32 v[190:191], v[182:183], v[182:183]\n\t" \
  "v_rcp_f32 v162, v162\n\t" \
  "v_pk_fma_f32 v[184:185], v[184:185], v[18:19], v[16:17]\n\t" \
  "v_pk_fma_f32 v[186:187], v[186:187], v[18:19], v[16:17]\n\t" \
  "v_pk_fma_f32 v[188:189], v[188:189], v[18:19], v[16:17]\n\t" \
  "v_rcp_f32 v163, v163\n\t" \
  "v_pk_fma_f32 v[190:191], v[190:191], v[18:19], v[16:17]\n\t" \
  "v_pk_mul_f32 v[184:185], v[184:185], v[176:177]\n\t" \
  "v_pk_mul_f32 v[186:187], v[186:187], v[178:179]\n\t" \
  "v_pk_mul_f32 v[188:189], v[188:189], v[180:181]\n\t" \
  "v_rcp_f32 v164, v164\n\t" \
  "v_pk_mul_f32 v[190:191], v[190:191], v[182:183]\n\t" \
  "s_waitcnt lgkmcnt(0)\n\t" \
  "global_store_dwordx4 v24, v[32:35], s[30:31] nt\n\t" \
  "v_rcp_f32 v165, v165\n\t" \
  "s_add_u32 s30, s30, 0x40000\n\t" \
  "s_addc_u32 s31, s31, 0\n\t" \
  "global_store_dwordx4 v24, v[36:39], s[30:31] nt\n\t" \
  "s_add_u32 s30, s30, 0x40000\n\t" \
  "v_rcp_f32 v166, v166\n\t" \
  "s_addc_u32 s31, s31, 0\n\t" \
  "global_store_dwordx4 v24, v[40:43], s[30:31] nt\n\t" \
  "s_add_u32 s30, s30, 0x40000\n\t" \
  "v_rcp_f32 v167, v167\n\t" \
  "s_addc_u32 s31, s31, 0\n\t" \
  "global_store_dwordx4 v24, v[44:47], s[30:31] nt\n\t" \
  "s_add_u32 s30, s30, 0x40000\n\t" \
  "s_addc_u32 s31, s31, 0\n\t" \
  "v_exp_f32 v184, v184\n\t" \
  "v_pk_mul_f32 v[160:161], v[160:161], v[152:153]\n\t" \
  "v_exp_f32 v185, v185\n\t" \
  "v_pk_mul_f32 v[162:163], v[162:163], v[154:155]\n\t" \
  "v_pk_mul_f32 v[164:165], v[164:165], v[156:157]\n\t" \
  "v_exp_f32 v186, v186\n\t" \
  "v_pk_mul_f32 v[166:167], v[166:167], v[158:159]\n\t" \
  "v_cvt_pk_bf16_f32 v168, v160, v161\n\t" \
  "v_exp_f32 v187, v187\n\t" \
  "v_cvt_pk_bf16_f32 v169, v162, v163\n\t" \
  "v_cvt_pk_bf16_f32 v170, v164, v165\n\t" \
  "v_exp_f32 v188, v188\n\t" \
  "v_cvt_pk_bf16_f32 v171, v166, v167\n\t" \
  "ds_write_b64 v22, v[168:169] offset:0\n\t" \
  "v_exp_f32 v189, v189\n\t" \
  "ds_write_b64 v22, v[170:171] offset:32\n\t" \
  "v_accvgpr_read_b32 v200, a136\n\t" \
  "v_exp_f32 v190, v190\n\t" \
  "v_accvgpr_read_b32 v201, a137\n\t" \
  "v_accvgpr_read_b32 v202, a138\n\t" \
  "v_exp_f32 v191, v191\n\t" \
  "v_accvgpr_read_b32 v203, a139\n\t" \
  "v_accvgpr_read_b32 v204, a168\n\t" \
  "v_pk_add_f32 v[184:185], v[184:185], v[20:21]\n\t" \
  "v_pk_add_f32 v[186:187], v[186:187], v[20:21]\n\t" \
  "v_pk_add_f32 v[188:189], v[188:189], v[20:21]\n\t" \
  "v_pk_add_f32 v[190:191], v[190:191], v[20:21]\n\t" \
  "v_rcp_f32 v184, v184\n\t" \
  "v_accvgpr_read_b32 v205, a169\n\t" \
  "v_rcp_f32 v185, v185\n\t" \
  "v_accvgpr_read_b32 v206, a170\n\t" \
  "v_accvgpr_read_b32 v207, a171\n\t" \
  "v_rcp_f32 v186, v186\n\t" \
  "v_pk_mul_f32 v[208:209], v[200:201], v[200:201]\n\t" \
  "v_pk_mul_f32 v[210:211], v[202:203], v[202:203]\n\t" \
  "v_rcp_f32 v187, v187\n\t" \
  "v_pk_mul_f32 v[212:213], v[204:205], v[204:205]\n\t" \
  "v_pk_mul_f32 v[214:215], v[206:207], v[206:207]\n\t" \
  "v_rcp_f32 v188, v188\n\t" \
  "v_pk_fma_f32 v[208:209], v[208:209], v[18:19], v[16:17]\n\t" \
  "v_pk_fma_f32 v[210:211], v[210:211], v[18:19], v[16:17]\n\t" \
  "v_rcp_f32 v189, v189\n\t" \
  "v_pk_fma_f32 v[212:213], v[212:213], v[18:19], v[16:17]\n\t" \
  "v_pk_fma_f32 v[214:215], v[214:215], v[18:19], v[16:17]\n\t" \
  "v_rcp_f32 v190, v190\n\t" \
  "v_pk_mul_f32 v[208:209], v[208:209], v[200:201]\n\t" \
  "v_pk_mul_f32 v[210:211], v[210:211], v[202:203]\n\t" \
  "v_rcp_f32 v191, v191\n\t" \
  "v_pk_mul_f32 v[212:213], v[212:213], v[204:205]\n\t" \
  "v_pk_mul_f32 v[214:215], v[214:215], v[206:207]\n\t" \
  "v_exp_f32 v208, v208\n\t" \
  "v_pk_mul_f32 v[184:185], v[184:185], v[176:177]\n\t" \
  "v_exp_f32 v209, v209\n\t" \
  "v_pk_mul_f32 v[186:187], v[186:187], v[178:179]\n\t" \
  "v_pk_mul_f32 v[188:189], v[188:189], v[180:181]\n\t" \
  "v_exp_f32 v210, v210\n\t" \
  "v_pk_mul_f32 v[190:191], v[190:191], v[182:183]\n\t" \
  "v_cvt_pk_bf16_f32 v192, v184, v185\n\t" \
  "v_exp_f32 v211, v211\n\t" \
  "v_cvt_pk_bf16_f32 v193, v186, v187\n\t" \
  "v_cvt_pk_bf16_f32 v194, v188, v189\n\t" \
  "v_exp_f32 v212, v212\n\t" \
  "v_cvt_pk_bf16_f32 v195, v190, v191\n\t" \
  "ds_write_b64 v22, v[192:193] offset:64\n\t" \
  "v_exp_f32 v213, v213\n\t" \
  "ds_write_b64 v22, v[194:195] offset:96\n\t" \
  "v_accvgpr_read_b32 v128, a200\n\t" \
  "v_exp_f32 v214, v214\n\t" \
  "v_accvgpr_read_b32 v129, a201\n\t" \
  "v_accvgpr_read_b32 v130, a202\n\t" \
  "v_exp_f32 v215, v215\n\t" \
  "v_accvgpr_read_b32 v131, a203\n\t" \
  "v_accvgpr_read_b32 v132, a232\n\t" \
  "v_pk_add_f32 v[208:209], v[208:209], v[20:21]\n\t" \
  "v_pk_add_f32 v[210:211], v[210:211], v[20:21]\n\t" \
  "v_pk_add_f32 v[212:213], v[212:213], v[20:21]\n\t" \
  "v_pk_add_f32 v[214:215], v[214:215], v[20:21]\n\t" \
  "v_rcp_f32 v208, v208\n\t" \
  "v_accvgpr_read_b32 v133, a233\n\t" \
  "v_rcp_f32 v209, v209\n\t" \
  "v_accvgpr_read_b32 v134, a234\n\t" \
  "v_accvgpr_read_b32 v135, a235\n\t" \
  "v_rcp_f32 v210, v210\n\t" \
  "v_pk_mul_f32 v[136:137], v[128:129], v[128:129]\n\t" \
  "v_pk_mul_f32 v[138:139], v[130:131], v[130:131]\n\t" \
  "v_rcp_f32 v211, v211\n\t" \
  "v_pk_mul_f32 v[140:141], v[132:133], v[132:133]\n\t" \
  "v_pk_mul_f32 v[142:143], v[134:135], v[134:135]\n\t" \
  "v_rcp_f32 v212, v212\n\t" \
  "v_pk_fma_f32 v[136:137], v[136:137], v[18:19], v[16:17]\n\t" \
  "v_pk_fma_f32 v[138:139], v[138:139], v[18:19], v[16:17]\n\t" \
  "v_rcp_f32 v213, v213\n\t" \
  "v_pk_fma_f32 v[140:141], v[140:141], v[18:19], v[16:17]\n\t" \
  "v_pk_fma_f32 v[142:143], v[142:143], v[18:19], v[16:17]\n\t" \
  "v_rcp_f32 v214, v214\n\t" \
  "v_pk_mul_f32 v[136:137], v[136:137], v[128:129]\n\t" \
  "v_pk_mul_f32 v[138:139], v[138:139], v[130:131]\n\t" \
  "v_rcp_f32 v215, v215\n\t" \
  "v_pk_mul_f32 v[140:141], v[140:141], v[132:133]\n\t" \
  "v_pk_mul_f32 v[142:143], v[142:143], v[134:135]\n\t" \
  "v_exp_f32 v136, v136\n\t" \
  "v_pk_mul_f32 v[208:209], v[208:209], v[200:201]\n\t" \
  "v_exp_f32 v137, v137\n\t" \
  "v_pk_mul_f32 v[210:211], v[210:211], v[202:203]\n\t" \
  "v_pk_mul_f32 v[212:213], v[212:213], v[204:205]\n\t" \
  "v_exp_f32 v138, v138\n\t" \
  "v_pk_mul_f32 v[214:215], v[214:215], v[206:207]\n\t" \
  "v_cvt_pk_bf16_f32 v216, v208, v209\n\t" \
  "v_exp_f32 v139, v139\n\t" \
  "v_cvt_pk_bf16_f32 v217, v210, v211\n\t" \
  "v_cvt_pk_bf16_f32 v218, v212, v213\n\t" \
  "v_exp_f32 v140, v140\n\t" \
  "v_cvt_pk_bf16_f32 v219, v214, v215\n\t" \
  "ds_write_b64 v22, v[216:217] offset:128\n\t" \
  "v_exp_f32 v141, v141\n\t" \
  "ds_write_b64 v22, v[218:219] offset:160\n\t" \
  "v_accvgpr_read_b32 v152, a12\n\t" \
  "v_exp_f32 v142, v142\n\t" \
  "v_accvgpr_read_b32 v153, a13\n\t" \
  "v_accvgpr_read_b32 v154, a14\n\t" \
  "v_exp_f32 v143, v143\n\t" \
  "v_accvgpr_read_b32 v155, a15\n\t" \
  "v_accvgpr_read_b32 v156, a44\n\t" \
  "v_pk_add_f32 v[136:137], v[136:137], v[20:21]\n\t" \
  "v_pk_add_f32 v[138:139], v[138:139], v[20:21]\n\t" \
  "v_pk_add_f32 v[140:141], v[140:141], v[20:21]\n\t" \
  "v_pk_add_f32 v[142:143], v[142:143], v[20:21]\n\t" \
  "v_rcp_f32 v136, v136\n\t" \
  "v_accvgpr_read_b32 v157, a45\n\t" \
  "v_rcp_f32 v137, v137\n\t" \
  "v_accvgpr_read_b32 v158, a46\n\t" \
  "v_accvgpr_read_b32 v159, a47\n\t" \
  "v_rcp_f32 v138, v138\n\t" \
  "v_pk_mul_f32 v[160:161], v[152:153], v[152:153]\n\t" \
  "v_pk_mul_f32 v[162:163], v[154:155], v[154:155]\n\t" \
  "v_rcp_f32 v139, v139\n\t" \
  "v_pk_mul_f32 v[164:165], v[156:157], v[156:157]\n\t" \
  "v_pk_mul_f32 v[166:167], v[158:159], v[158:159]\n\t" \
  "v_rcp_f32 v140, v140\n\t" \
  "v_pk_fma_f32 v[160:161], v[160:161], v[18:19], v[16:17]\n\t" \
  "v_pk_fma_f32 v[162:163], v[162:163], v[18:19], v[16:17]\n\t" \
  "v_rcp_f32 v141, v141\n\t" \
  "v_pk_fma_f32 v[164:165], v[164:165], v[18:19], v[16:17]\n\t" \
  "v_pk_fma_f32 v[166:167], v[166:167], v[18:19], v[16:17]\n\t" \
  "v_rcp_f32 v142, v142\n\t" \
  "v_pk_mul_f32 v[160:161], v[160:161], v[152:153]\n\t" \
  "v_pk_mul_f32 v[162:163], v[162:163], v[154:155]\n\t" \
  "v_rcp_f32 v143, v143\n\t" \
  "v_pk_mul_f32 v[164:165], v[164:165], v[156:157]\n\t" \
  "v_pk_mul_f32 v[166:167], v[166:167], v[158:159]\n\t" \
  "v_exp_f32 v160, v160\n\t" \
  "v_pk_mul_f32 v[136:137], v[136:137], v[128:129]\n\t" \
  "v_pk_mul_f32 v[138:139], v[138:139], v[130:131]\n\t" \
  "v_exp_f32 v161, v161\n\t" \
  "v_pk_mul_f32 v[140:141], v[140:141], v[132:133]\n\t" \
  "v_pk_mul_f32 v[142:143], v[142:143], v[134:135]\n\t" \
  "v_exp_f32 v162, v162\n\t" \
  "v_cvt_pk_bf16_f32 v144, v136, v137\n\t" \
  "v_cvt_pk_bf16_f32 v145, v138, v139\n\t" \
  "v_cvt_pk_bf16_f32 v146, v140, v141\n\t" \
  "v_exp_f32 v163, v163\n\t" \
  "v_cvt_pk_bf16_f32 v147, v142, v143\n\t" \
  "ds_write_b64 v22, v[144:145] offset:192\n\t" \
  "v_exp_f32 v164, v164\n\t" \
  "ds_write_b64 v22, v[146:147] offset:224\n\t" \
  "ds_read_b128 v[32:35], v23 offset:0\n\t" \
  "v_exp_f32 v165, v165\n\t" \
  "ds_read_b128 v[36:39], v23 offset:1088\n\t" \
  "ds_read_b128 v[40:43], v23 offset:2176\n\t" \
  "ds_read_b128 v[44:47], v23 offset:3264\n\t" \
  "v_exp_f32 v166, v166\n\t" \
  "v_accvgpr_read_b32 v176, a76\n\t" \
  "v_accvgpr_read_b32 v177, a77\n\t" \
  "v_exp_f32 v167, v167\n\t" \
  "v_accvgpr_read_b32 v178, a78\n\t" \
  "v_accvgpr_read_b32 v179, a79\n\t" \
  "v_accvgpr_read_b32 v180, a108\n\t" \
  "v_pk_add_f32 v[160:161], v[160:161], v[20:21]\n\t" \
  "v_pk_add_f32 v[162:163], v[162:163], v[20:21]\n\t" \
  "v_pk_add_f32 v[164:165], v[164:165], v[20:21]\n\t" \
  "v_pk_add_f32 v[166:167], v[166:167], v[20:21]\n\t" \
  "v_rcp_f32 v160, v160\n\t" \
  "v_accvgpr_read_b32 v181, a109\n\t" \
  "v_accvgpr_read_b32 v182, a110\n\t" \
  "v_accvgpr_read_b32 v183, a111\n\t" \
  "v_rcp_f32 v161, v161\n\t" \
  "v_pk_mul_f32 v[184:185], v[176:177], v[176:177]\n\t" \
  "v_pk_mul_f32 v[186:187], v[178:179], v[178:179]\n\t" \
  "v_pk_mul_f32 v[188:189], v[180:181], v[180:181]\n\t" \
  "v_pk_mul_f32 v[190:191], v[182:183], v[182:183]\n\t" \
  "v_rcp_f32 v162, v162\n\t" \
  "v_pk_fma_f32 v[184:185], v[184:185], v[18:19], v[16:17]\n\t" \
  "v_pk_fma_f32 v[186:187], v[186:187], v[18:19], v[16:17]\n\t" \
  "v_pk_fma_f32 v[188:189], v[188:189], v[18:19], v[16:17]\n\t" \
  "v_rcp_f32 v163, v163\n\t" \
  "v_pk_fma_f32 v[190:191], v[190:191], v[18:19], v[16:17]\n\t" \
  "v_pk_mul_f32 v[184:185], v[184:185], v[176:177]\n\t" \
  "v_pk_mul_f32 v[186:187], v[186:187], v[178:179]\n\t" \
  "v_pk_mul_f32 v[188:189], v[188:189], v[180:181]\n\t" \
  "v_rcp_f32 v164, v164\n\t" \
  "v_pk_mul_f32 v[190:191], v[190:191], v[182:183]\n\t" \
  "s_waitcnt lgkmcnt(0)\n\t" \
  "global_store_dwordx4 v24, v[32:35], s[30:31] nt\n\t" \
  "v_rcp_f32 v165, v165\n\t" \
  "s_add_u32 s30, s30, 0x40000\n\t" \
  "s_addc_u32 s31, s31, 0\n\t" \
  "global_store_dwordx4 v24, v[36:39], s[30:31] nt\n\t" \
  "s_add_u32 s30, s30, 0x40000\n\t" \
  "v_rcp_f32 v166, v166\n\t" \
  "s_addc_u32 s31, s31, 0\n\t" \
  "global_store_dwordx4 v24, v[40:43], s[30:31] nt\n\t" \
  "s_add_u32 s30, s30, 0x40000\n\t" \
  "v_rcp_f32 v167, v167\n\t" \
  "s_addc_u32 s31, s31, 0\n\t" \
  "global_store_dwordx4 v24, v[44:47], s[30:31] nt\n\t" \
  "s_add_u32 s30, s30, 0x40000\n\t" \
  "s_addc_u32 s31, s31, 0\n\t" \
  "v_exp_f32 v184, v184\n\t" \
  "v_pk_mul_f32 v[160:161], v[160:161], v[152:153]\n\t" \
  "v_exp_f32 v185, v185\n\t" \
  "v_pk_mul_f32 v[162:163], v[162:163], v[154:155]\n\t" \
  "v_pk_mul_f32 v[164:165], v[164:165], v[156:157]\n\t" \
  "v_exp_f32 v186, v186\n\t" \
  "v_pk_mul_f32 v[166:167], v[166:167], v[158:159]\n\t" \
  "v_cvt_pk_bf16_f32 v168, v160, v161\n\t" \
  "v_exp_f32 v187, v187\n\t" \
  "v_cvt_pk_bf16_f32 v169, v162, v163\n\t" \
  "v_cvt_pk_bf16_f32 v170, v164, v165\n\t" \
  "v_exp_f32 v188, v188\n\t" \
  "v_cvt_pk_bf16_f32 v171, v166, v167\n\t" \
  "ds_write_b64 v22, v[168:169] offset:4352\n\t" \
  "v_exp_f32 v189, v189\n\t" \
  "ds_write_b64 v22, v[170:171] offset:4384\n\t" \
  "v_accvgpr_read_b32 v200, a140\n\t" \
  "v_exp_f32 v190, v190\n\t" \
  "v_accvgpr_read_b32 v201, a141\n\t" \
  "v_accvgpr_read_b32 v202, a142\n\t" \
  "v_exp_f32 v191, v191\n\t" \
  "v_accvgpr_read_b32 v203, a143\n\t" \
  "v_accvgpr_read_b32 v204, a172\n\t" \
  "v_pk_add_f32 v[184:185], v[184:185], v[20:21]\n\t" \
  "v_pk_add_f32 v[186:187], v[186:187], v[20:21]\n\t" \
  "v_pk_add_f32 v[188:189], v[188:189], v[20:21]\n\t" \
  "v_pk_add_f32 v[190:191], v[190:191], v[20:21]\n\t" \
  "v_rcp_f32 v184, v184\n\t" \
  "v_accvgpr_read_b32 v205, a173\n\t" \
  "v_rcp_f32 v185, v185\n\t" \
  "v_accvgpr_read_b32 v206, a174\n\t" \
  "v_accvgpr_read_b32 v207, a175\n\t" \
  "v_rcp_f32 v186, v186\n\t" \
  "v_pk_mul_f32 v[208:209], v[200:201], v[200:201]\n\t" \
  "v_pk_mul_f32 v[210:211], v[202:203], v[202:203]\n\t" \
  "v_rcp_f32 v187, v187\n\t" \
  "v_pk_mul_f32 v[212:213], v[204:205], v[204:205]\n\t" \
  "v_pk_mul_f32 v[214:215], v[206:207], v[206:207]\n\t" \
  "v_rcp_f32 v188, v188\n\t" \
  "v_pk_fma_f32 v[208:209], v[208:209], v[18:19], v[16:17]\n\t" \
  "v_pk_fma_f32 v[210:211], v[210:211], v[18:19], v[16:17]\n\t" \
  "v_rcp_f32 v189, v189\n\t" \
  "v_pk_fma_f32 v[212:213], v[212:213], v[18:19], v[16:17]\n\t" \
  "v_pk_fma_f32 v[214:215], v[214:215], v[18:19], v[16:17]\n\t" \
  "v_rcp_f32 v190, v190\n\t" \
  "v_pk_mul_f32 v[208:209], v[208:209], v[200:201]\n\t" \
  "v_pk_mul_f32 v[210:211], v[210:211], v[202:203]\n\t" \
  "v_rcp_f32 v191, v191\n\t" \
  "v_pk_mul_f32 v[212:213], v[212:213], v[204:205]\n\t" \
  "v_pk_mul_f32 v[214:215], v[214:215], v[206:207]\n\t" \
  "v_exp_f32 v208, v208\n\t" \
  "v_pk_mul_f32 v[184:185], v[184:185], v[176:177]\n\t" \
  "v_exp_f32 v209, v209\n\t" \
  "v_pk_mul_f32 v[186:187], v[186:187], v[178:179]\n\t" \
  "v_pk_mul_f32 v[188:189], v[188:189], v[180:181]\n\t" \
  "v_exp_f32 v210, v210\n\t" \
  "v_pk_mul_f32 v[190:191], v[190:191], v[182:183]\n\t" \
  "v_cvt_pk_bf16_f32 v192, v184, v185\n\t" \
  "v_exp_f32 v211, v211\n\t" \
  "v_cvt_pk_bf16_f32 v193, v186, v187\n\t" \
  "v_cvt_pk_bf16_f32 v194, v188, v189\n\t" \
  "v_exp_f32 v212, v212\n\t" \
  "v_cvt_pk_bf16_f32 v195, v190, v191\n\t" \
  "ds_write_b64 v22, v[192:193] offset:4416\n\t" \
  "v_exp_f32 v213, v213\n\t" \
  "ds_write_b64 v22, v[194:195] offset:4448\n\t" \
  "v_accvgpr_read_b32 v128, a204\n\t" \
  "v_exp_f32 v214, v214\n\t" \
  "v_accvgpr_read_b32 v129, a205\n\t" \
  "v_accvgpr_read_b32 v130, a206\n\t" \
  "v_exp_f32 v215, v215\n\t" \
  "v_accvgpr_read_b32 v131, a207\n\t" \
  "v_accvgpr_read_b32 v132, a236\n\t" \
  "v_pk_add_f32 v[208:209], v[208:209], v[20:21]\n\t" \
  "v_pk_add_f32 v[210:211], v[210:211], v[20:21]\n\t" \
  "v_pk_add_f32 v[212:213], v[212:213], v[20:21]\n\t" \
  "v_pk_add_f32 v[214:215], v[214:215], v[20:21]\n\t" \
  "v_rcp_f32 v208, v208\n\t" \
  "v_accvgpr_read_b32 v133, a237\n\t" \
  "v_rcp_f32 v209, v209\n\t" \
  "v_accvgpr_read_b32 v134, a238\n\t" \
  "v_accvgpr_read_b32 v135, a239\n\t" \
  "v_rcp_f32 v210, v210\n\t" \
  "v_pk_mul_f32 v[136:137], v[128:129], v[128:129]\n\t" \
  "v_pk_mul_f32 v[138:139], v[130:131], v[130:131]\n\t" \
  "v_rcp_f32 v211, v211\n\t" \
  "v_pk_mul_f32 v[140:141], v[132:133], v[132:133]\n\t" \
  "v_pk_mul_f32 v[142:143], v[134:135], v[134:135]\n\t" \
  "v_rcp_f32 v212, v212\n\t" \
  "v_pk_fma_f32 v[136:137], v[136:137], v[18:19], v[16:17]\n\t" \
  "v_pk_fma_f32 v[138:139], v[138:139], v[18:19], v[16:17]\n\t" \
  "v_rcp_f32 v213, v213\n\t" \
  "v_pk_fma_f32 v[140:141], v[140:141], v[18:19], v[16:17]\n\t" \
  "v_pk_fma_f32 v[142:143], v[142:143], v[18:19], v[16:17]\n\t" \
  "v_rcp_f32 v214, v214\n\t" \
  "v_pk_mul_f32 v[136:137], v[136:137], v[128:129]\n\t" \
  "v_pk_mul_f32 v[138:139], v[138:139], v[130:131]\n\t" \
  "v_rcp_f32 v215, v215\n\t" \
  "v_pk_mul_f32 v[140:141], v[140:141], v[132:133]\n\t" \
  "v_pk_mul_f32 v[142:143], v[142:143], v[134:135]\n\t" \
  "v_exp_f32 v136, v136\n\t" \
  "v_pk_mul_f32 v[208:209], v[208:209], v[200:201]\n\t" \
  "v_exp_f32 v137, v137\n\t" \
  "v_pk_mul_f32 v[210:211], v[210:211], v[202:203]\n\t" \
  "v_pk_mul_f32 v[212:213], v[212:213], v[204:205]\n\t" \
  "v_exp_f32 v138, v138\n\t" \
  "v_pk_mul_f32 v[214:215], v[214:215], v[206:207]\n\t" \
  "v_cvt_pk_bf16_f32 v216, v208, v209\n\t" \
  "v_exp_f32 v139, v139\n\t" \
  "v_cvt_pk_bf16_f32 v217, v210, v211\n\t" \
  "v_cvt_pk_bf16_f32 v218, v212, v213\n\t" \
  "v_exp_f32 v140, v140\n\t" \
  "v_cvt_pk_bf16_f32 v219, v214, v215\n\t" \
  "ds_write_b64 v22, v[216:217] offset:4480\n\t" \
  "v_exp_f32 v141, v141\n\t" \
  "ds_write_b64 v22, v[218:219] offset:4512\n\t" \
  "v_accvgpr_read_b32 v152, a16\n\t" \
  "v_exp_f32 v142, v142\n\t" \
  "v_accvgpr_read_b32 v153, a17\n\t" \
  "v_accvgpr_read_b32 v154, a18\n\t" \
  "v_exp_f32 v143, v143\n\t" \
  "v_accvgpr_read_b32 v155, a19\n\t" \
  "v_accvgpr_read_b32 v156, a48\n\t" \
  "v_pk_add_f32 v[136:137], v[136:137], v[20:21]\n\t" \
  "v_pk_add_f32 v[138:139], v[138:139], v[20:21]\n\t" \
  "v_pk_add_f32 v[140:141], v[140:141], v[20:21]\n\t" \
  "v_pk_add_f32 v[142:143], v[142:143], v[20:21]\n\t" \
  "v_rcp_f32 v136, v136\n\t" \
  "v_accvgpr_read_b32 v157, a49\n\t" \
  "v_rcp_f32 v137, v137\n\t" \
  "v_accvgpr_read_b32 v158, a50\n\t" \
  "v_accvgpr_read_b32 v159, a51\n\t" \
  "v_rcp_f32 v138, v138\n\t" \
  "v_pk_mul_f32 v[160:161], v[152:153], v[152:153]\n\t" \
  "v_pk_mul_f32 v[162:163], v[154:155], v[154:155]\n\t" \
  "v_rcp_f32 v139, v139\n\t" \
  "v_pk_mul_f32 v[164:165], v[156:157], v[156:157]\n\t" \
  "v_pk_mul_f32 v[166:167], v[158:159], v[158:159]\n\t" \
  "v_rcp_f32 v140, v140\n\t" \
  "v_pk_fma_f32 v[160:161], v[160:161], v[18:19], v[16:17]\n\t" \
  "v_pk_fma_f32 v[162:163], v[162:163], v[18:19], v[16:17]\n\t" \
  "v_rcp_f32 v141, v141\n\t" \
  "v_pk_fma_f32 v[164:165], v[164:165], v[18:19], v[16:17]\n\t" \
  "v_pk_fma_f32 v[166:167], v[166:167], v[18:19], v[16:17]\n\t" \
  "v_rcp_f32 v142, v142\n\t" \
  "v_pk_mul_f32 v[160:161], v[160:161], v[152:153]\n\t" \
  "v_pk_mul_f32 v[162:163], v[162:163], v[154:155]\n\t" \
  "v_rcp_f32 v143, v143\n\t" \
  "v_pk_mul_f32 v[164:165], v[164:165], v[156:157]\n\t" \
  "v_pk_mul_f32 v[166:167], v[166:167], v[158:159]\n\t" \
  "v_exp_f32 v160, v160\n\t" \
  "v_pk_mul_f32 v[136:137], v[136:137], v[128:129]\n\t" \
  "v_pk_mul_f32 v[138:139], v[138:139], v[130:131]\n\t" \
  "v_exp_f32 v161, v161\n\t" \
  "v_pk_mul_f32 v[140:141], v[140:141], v[132:133]\n\t" \
  "v_pk_mul_f32 v[142:143], v[142:143], v[134:135]\n\t" \
  "v_exp_f32 v162, v162\n\t" \
  "v_cvt_pk_bf16_f32 v144, v136, v137\n\t" \
  "v_cvt_pk_bf16_f32 v145, v138, v139\n\t" \
  "v_cvt_pk_bf16_f32 v146, v140, v141\n\t" \
  "v_exp_f32 v163, v163\n\t" \
  "v_cvt_pk_bf16_f32 v147, v142, v143\n\t" \
  "ds_write_b64 v22, v[144:145] offset:4544\n\t" \
  "v_exp_f32 v164, v164\n\t" \
  "ds_write_b64 v22, v[146:147] offset:4576\n\t" \
  "ds_read_b128 v[32:35], v23 offset:4352\n\t" \
  "v_exp_f32 v165, v165\n\t" \
  "ds_read_b128 v[36:39], v23 offset:5440\n\t" \
  "ds_read_b128 v[40:43], v23 offset:6528\n\t" \
  "ds_read_b128 v[44:47], v23 offset:7616\n\t" \
  "v_exp_f32 v166, v166\n\t" \
  "v_accvgpr_read_b32 v176, a80\n\t" \
  "v_accvgpr_read_b32 v177, a81\n\t" \
  "v_exp_f32 v167, v167\n\t" \
  "v_accvgpr_read_b32 v178, a82\n\t" \
  "v_accvgpr_read_b32 v179, a83\n\t" \
  "v_accvgpr_read_b32 v180, a112\n\t" \
  "v_pk_add_f32 v[160:161], v[160:161], v[20:21]\n\t" \
  "v_pk_add_f32 v[162:163], v[162:163], v[20:21]\n\t" \
  "v_pk_add_f32 v[164:165], v[164:165], v[20:21]\n\t" \
  "v_pk_add_f32 v[166:167], v[166:167], v[20:21]\n\t" \
  "v_rcp_f32 v160, v160\n\t" \
  "v_accvgpr_read_b32 v181, a113\n\t" \
  "v_accvgpr_read_b32 v182, a114\n\t" \
  "v_accvgpr_read_b32 v183, a115\n\t" \
  "v_rcp_f32 v161, v161\n\t" \
  "v_pk_mul_f32 v[184:185], v[176:177], v[176:177]\n\t" \
  "v_pk_mul_f32 v[186:187], v[178:179], v[178:179]\n\t" \
  "v_pk_mul_f32 v[188:189], v[180:181], v[180:181]\n\t" \
  "v_pk_mul_f32 v[190:191], v[182:183], v[182:183]\n\t" \
  "v_rcp_f32 v162, v162\n\t" \
  "v_pk_fma_f32 v[184:185], v[184:185], v[18:19], v[16:17]\n\t" \
  "v_pk_fma_f32 v[186:187], v[186:187], v[18:19], v[16:17]\n\t" \
  "v_pk_fma_f32 v[188:189], v[188:189], v[18:19], v[16:17]\n\t" \
  "v_rcp_f32 v163, v163\n\t" \
  "v_pk_fma_f32 v[190:191], v[190:191], v[18:19], v[16:17]\n\t" \
  "v_pk_mul_f32 v[184:185], v[184:185], v[176:177]\n\t" \
  "v_pk_mul_f32 v[186:187], v[186:187], v[178:179]\n\t" \
  "v_pk_mul_f32 v[188:189], v[188:189], v[180:181]\n\t" \
  "v_rcp_f32 v164, v164\n\t" \
  "v_pk_mul_f32 v[190:191], v[190:191], v[182:183]\n\t" \
  "s_waitcnt lgkmcnt(0)\n\t" \
  "global_store_dwordx4 v24, v[32:35], s[30:31] nt\n\t" \
  "v_rcp_f32 v165, v165\n\t" \
  "s_add_u32 s30, s30, 0x40000\n\t" \
  "s_addc_u32 s31, s31, 0\n\t" \
  "global_store_dwordx4 v24, v[36:39], s[30:31] nt\n\t" \
  "s_add_u32 s30, s30, 0x40000\n\t" \
  "v_rcp_f32 v166, v166\n\t" \
  "s_addc_u32 s31, s31, 0\n\t" \
  "global_store_dwordx4 v24, v[40:43], s[30:31] nt\n\t" \
  "s_add_u32 s30, s30, 0x40000\n\t" \
  "v_rcp_f32 v167, v167\n\t" \
  "s_addc_u32 s31, s31, 0\n\t" \
  "global_store_dwordx4 v24, v[44:47], s[30:31] nt\n\t" \
  "s_add_u32 s30, s30, 0x40000\n\t" \
  "s_addc_u32 s31, s31, 0\n\t" \
  "v_exp_f32 v184, v184\n\t" \
  "v_pk_mul_f32 v[160:161], v[160:161], v[152:153]\n\t" \
  "v_exp_f32 v185, v185\n\t" \
  "v_pk_mul_f32 v[162:163], v[162:163], v[154:155]\n\t" \
  "v_pk_mul_f32 v[164:165], v[164:165], v[156:157]\n\t" \
  "v_exp_f32 v186, v186\n\t" \
  "v_pk_mul_f32 v[166:167], v[166:167], v[158:159]\n\t" \
  "v_cvt_pk_bf16_f32 v168, v160, v161\n\t" \
  "v_exp_f32 v187, v187\n\t" \
  "v_cvt_pk_bf16_f32 v169, v162, v163\n\t" \
  "v_cvt_pk_bf16_f32 v170, v164, v165\n\t" \
  "v_exp_f32 v188, v188\n\t" \
  "v_cvt_pk_bf16_f32 v171, v166, v167\n\t" \
  "ds_write_b64 v22, v[168:169] offset:0\n\t" \
  "v_exp_f32 v189, v189\n\t" \
  "ds_write_b64 v22, v[170:171] offset:32\n\t" \
  "v_accvgpr_read_b32 v200, a144\n\t" \
  "v_exp_f32 v190, v190\n\t" \
  "v_accvgpr_read_b32 v201, a145\n\t" \
  "v_accvgpr_read_b32 v202, a146\n\t" \
  "v_exp_f32 v191, v191\n\t" \
  "v_accvgpr_read_b32 v203, a147\n\t" \
  "v_accvgpr_read_b32 v204, a176\n\t" \
  "v_pk_add_f32 v[184:185], v[184:185], v[20:21]\n\t" \
  "v_pk_add_f32 v[186:187], v[186:187], v[20:21]\n\t" \
  "v_pk_add_f32 v[188:189], v[188:189], v[20:21]\n\t" \
  "v_pk_add_f32 v[190:191], v[190:191], v[20:21]\n\t" \
  "v_rcp_f32 v184, v184\n\t" \
  "v_accvgpr_read_b32 v205, a177\n\t" \
  "v_rcp_f32 v185, v185\n\t" \
  "v_accvgpr_read_b32 v206, a178\n\t" \
  "v_accvgpr_read_b32 v207, a179\n\t" \
  "v_rcp_f32 v186, v186\n\t" \
  "v_pk_mul_f32 v[208:209], v[200:201], v[200:201]\n\t" \
  "v_pk_mul_f32 v[210:211], v[202:203], v[202:203]\n\t" \
  "v_rcp_f32 v187, v187\n\t" \
  "v_pk_mul_f32 v[212:213], v[204:205], v[204:205]\n\t" \
  "v_pk_mul_f32 v[214:215], v[206:207], v[206:207]\n\t" \
  "v_rcp_f32 v188, v188\n\t" \
  "v_pk_fma_f32 v[208:209], v[208:209], v[18:19], v[16:17]\n\t" \
  "v_pk_fma_f32 v[210:211], v[210:211], v[18:19], v[16:17]\n\t" \
  "v_rcp_f32 v189, v189\n\t" \
  "v_pk_fma_f32 v[212:213], v[212:213], v[18:19], v[16:17]\n\t" \
  "v_pk_fma_f32 v[214:215], v[214:215], v[18:19], v[16:17]\n\t" \
  "v_rcp_f32 v190, v190\n\t" \
  "v_pk_mul_f32 v[208:209], v[208:209], v[200:201]\n\t" \
  "v_pk_mul_f32 v[210:211], v[210:211], v[202:203]\n\t" \
  "v_rcp_f32 v191, v191\n\t" \
  "v_pk_mul_f32 v[212:213], v[212:213], v[204:205]\n\t" \
  "v_pk_mul_f32 v[214:215], v[214:215], v[206:207]\n\t" \
  "v_exp_f32 v208, v208\n\t" \
  "v_pk_mul_f32 v[184:185], v[184:185], v[176:177]\n\t" \
  "v_exp_f32 v209, v209\n\t" \
  "v_pk_mul_f32 v[186:187], v[186:187], v[178:179]\n\t" \
  "v_pk_mul_f32 v[188:189], v[188:189], v[180:181]\n\t" \
  "v_exp_f32 v210, v210\n\t" \
  "v_pk_mul_f32 v[190:191], v[190:191], v[182:183]\n\t" \
  "v_cvt_pk_bf16_f32 v192, v184, v185\n\t" \
  "v_exp_f32 v211, v211\n\t" \
  "v_cvt_pk_bf16_f32 v193, v186, v187\n\t" \
  "v_cvt_pk_bf16_f32 v194, v188, v189\n\t" \
  "v_exp_f32 v212, v212\n\t" \
  "v_cvt_pk_bf16_f32 v195, v190, v191\n\t" \
  "ds_write_b64 v22, v[192:193] offset:64\n\t" \
  "v_exp_f32 v213, v213\n\t" \
  "ds_write_b64 v22, v[194:195] offset:96\n\t" \
  "v_accvgpr_read_b32 v128, a208\n\t" \
  "v_exp_f32 v214, v214\n\t" \
  "v_accvgpr_read_b32 v129, a209\n\t" \
  "v_accvgpr_read_b32 v130, a210\n\t" \
  "v_exp_f32 v215, v215\n\t" \
  "v_accvgpr_read_b32 v131, a211\n\t" \
  "v_accvgpr_read_b32 v132, a240\n\t" \
  "v_pk_add_f32 v[208:209], v[208:209], v[20:21]\n\t" \
  "v_pk_add_f32 v[210:211], v[210:211], v[20:21]\n\t" \
  "v_pk_add_f32 v[212:213], v[212:213], v[20:21]\n\t" \
  "v_pk_add_f32 v[214:215], v[214:215], v[20:21]\n\t" \
  "v_rcp_f32 v208, v208\n\t" \
  "v_accvgpr_read_b32 v133, a241\n\t" \
  "v_rcp_f32 v209, v209\n\t" \
  "v_accvgpr_read_b32 v134, a242\n\t" \
  "v_accvgpr_read_b32 v135, a243\n\t" \
  "v_rcp_f32 v210, v210\n\t" \
  "v_pk_mul_f32 v[136:137], v[128:129], v[128:129]\n\t" \
  "v_pk_mul_f32 v[138:139], v[130:131], v[130:131]\n\t" \
  "v_rcp_f32 v211, v211\n\t" \
  "v_pk_mul_f32 v[140:141], v[132:133], v[132:133]\n\t" \
  "v_pk_mul_f32 v[142:143], v[134:135], v[134:135]\n\t" \
  "v_rcp_f32 v212, v212\n\t" \
  "v_pk_fma_f32 v[136:137], v[136:137], v[18:19], v[16:17]\n\t" \
  "v_pk_fma_f32 v[138:139], v[138:139], v[18:19], v[16:17]\n\t" \
  "v_rcp_f32 v213, v213\n\t" \
  "v_pk_fma_f32 v[140:141], v[140:141], v[18:19], v[16:17]\n\t" \
  "v_pk_fma_f32 v[142:143], v[142:143], v[18:19], v[16:17]\n\t" \
  "v_rcp_f32 v214, v214\n\t" \
  "v_pk_mul_f32 v[136:137], v[136:137], v[128:129]\n\t" \
  "v_pk_mul_f32 v[138:139], v[138:139], v[130:131]\n\t" \
  "v_rcp_f32 v215, v215\n\t" \
  "v_pk_mul_f32 v[140:141], v[140:141], v[132:133]\n\t" \
  "v_pk_mul_f32 v[142:143], v[142:143], v[134:135]\n\t" \
  "v_exp_f32 v136, v136\n\t" \
  "v_pk_mul_f32 v[208:209], v[208:209], v[200:201]\n\t" \
  "v_exp_f32 v137, v137\n\t" \
  "v_pk_mul_f32 v[210:211], v[210:211], v[202:203]\n\t" \
  "v_pk_mul_f32 v[212:213], v[212:213], v[204:205]\n\t" \
  "v_exp_f32 v138, v138\n\t" \
  "v_pk_mul_f32 v[214:215], v[214:215], v[206:207]\n\t" \
  "v_cvt_pk_bf16_f32 v216, v208, v209\n\t" \
  "v_exp_f32 v139, v139\n\t" \
  "v_cvt_pk_bf16_f32 v217, v210, v211\n\t" \
  "v_cvt_pk_bf16_f32 v218, v212, v213\n\t" \
  "v_exp_f32 v140, v140\n\t" \
  "v_cvt_pk_bf16_f32 v219, v214, v215\n\t" \
  "ds_write_b64 v22, v[216:217] offset:128\n\t" \
  "v_exp_f32 v141, v141\n\t" \
  "ds_write_b64 v22, v[218:219] offset:160\n\t" \
  "v_accvgpr_read_b32 v152, a20\n\t" \
  "v_exp_f32 v142, v142\n\t" \
  "v_accvgpr_read_b32 v153, a21\n\t" \
  "v_accvgpr_read_b32 v154, a22\n\t" \
  "v_exp_f32 v143, v143\n\t" \
  "v_accvgpr_read_b32 v155, a23\n\t" \
  "v_accvgpr_read_b32 v156, a52\n\t" \
  "v_pk_add_f32 v[136:137], v[136:137], v[20:21]\n\t" \
  "v_pk_add_f32 v[138:139], v[138:139], v[20:21]\n\t" \
  "v_pk_add_f32 v[140:141], v[140:141], v[20:21]\n\t" \
  "v_pk_add_f32 v[142:143], v[142:143], v[20:21]\n\t" \
  "v_rcp_f32 v136, v136\n\t" \
  "v_accvgpr_read_b32 v157, a53\n\t" \
  "v_rcp_f32 v137, v137\n\t" \
  "v_accvgpr_read_b32 v158, a54\n\t" \
  "v_accvgpr_read_b32 v159, a55\n\t" \
  "v_rcp_f32 v138, v138\n\t" \
  "v_pk_mul_f32 v[160:161], v[152:153], v[152:153]\n\t" \
  "v_pk_mul_f32 v[162:163], v[154:155], v[154:155]\n\t" \
  "v_rcp_f32 v139, v139\n\t" \
  "v_pk_mul_f32 v[164:165], v[156:157], v[156:157]\n\t" \
  "v_pk_mul_f32 v[166:167], v[158:159], v[158:159]\n\t" \
  "v_rcp_f32 v140, v140\n\t" \
  "v_pk_fma_f32 v[160:161], v[160:161], v[18:19], v[16:17]\n\t" \
  "v_pk_fma_f32 v[162:163], v[162:163], v[18:19], v[16:17]\n\t" \
  "v_rcp_f32 v141, v141\n\t" \
  "v_pk_fma_f32 v[164:165], v[164:165], v[18:19], v[16:17]\n\t" \
  "v_pk_fma_f32 v[166:167], v[166:167], v[18:19], v[16:17]\n\t" \
  "v_rcp_f32 v142, v142\n\t" \
  "v_pk_mul_f32 v[160:161], v[160:161], v[152:153]\n\t" \
  "v_pk_mul_f32 v[162:163], v[162:163], v[154:155]\n\t" \
  "v_rcp_f32 v143, v143\n\t" \
  "v_pk_mul_f32 v[164:165], v[164:165], v[156:157]\n\t" \
  "v_pk_mul_f32 v[166:167], v[166:167], v[158:159]\n\t" \
  "v_exp_f32 v160, v160\n\t" \
  "v_pk_mul_f32 v[136:137], v[136:137], v[128:129]\n\t" \
  "v_pk_mul_f32 v[138:139], v[138:139], v[130:131]\n\t" \
  "v_exp_f32 v161, v161\n\t" \
  "v_pk_mul_f32 v[140:141], v[140:141], v[132:133]\n\t" \
  "v_pk_mul_f32 v[142:143], v[142:143], v[134:135]\n\t" \
  "v_exp_f32 v162, v162\n\t" \
  "v_cvt_pk_bf16_f32 v144, v136, v137\n\t" \
  "v_cvt_pk_bf16_f32 v145, v138, v139\n\t" \
  "v_cvt_pk_bf16_f32 v146, v140, v141\n\t" \
  "v_exp_f32 v163, v163\n\t" \
  "v_cvt_pk_bf16_f32 v147, v142, v143\n\t" \
  "ds_write_b64 v22, v[144:145] offset:192\n\t" \
  "v_exp_f32 v164, v164\n\t" \
  "ds_write_b64 v22, v[146:147] offset:224\n\t" \
  "ds_read_b128 v[32:35], v23 offset:0\n\t" \
  "v_exp_f32 v165, v165\n\t" \
  "ds_read_b128 v[36:39], v23 offset:1088\n\t" \
  "ds_read_b128 v[40:43], v23 offset:2176\n\t" \
  "ds_read_b128 v[44:47], v23 offset:3264\n\t" \
  "v_exp_f32 v166, v166\n\t" \
  "v_accvgpr_read_b32 v176, a84\n\t" \
  "v_accvgpr_read_b32 v177, a85\n\t" \
  "v_exp_f32 v167, v167\n\t" \
  "v_accvgpr_read_b32 v178, a86\n\t" \
  "v_accvgpr_read_b32 v179, a87\n\t" \
  "v_accvgpr_read_b32 v180, a116\n\t" \
  "v_pk_add_f32 v[160:161], v[160:161], v[20:21]\n\t" \
  "v_pk_add_f32 v[162:163], v[162:163], v[20:21]\n\t" \
  "v_pk_add_f32 v[164:165], v[164:165], v[20:21]\n\t" \
  "v_pk_add_f32 v[166:167], v[166:167], v[20:21]\n\t" \
  "v_rcp_f32 v160, v160\n\t" \
  "v_accvgpr_read_b32 v181, a117\n\t" \
  "v_accvgpr_read_b32 v182, a118\n\t" \
  "v_accvgpr_read_b32 v183, a119\n\t" \
  "v_rcp_f32 v161, v161\n\t" \
  "v_pk_mul_f32 v[184:185], v[176:177], v[176:177]\n\t" \
  "v_pk_mul_f32 v[186:187], v[178:179], v[178:179]\n\t" \
  "v_pk_mul_f32 v[188:189], v[180:181], v[180:181]\n\t" \
  "v_pk_mul_f32 v[190:191], v[182:183], v[182:183]\n\t" \
  "v_rcp_f32 v162, v162\n\t" \
  "v_pk_fma_f32 v[184:185], v[184:185], v[18:19], v[16:17]\n\t" \
  "v_pk_fma_f32 v[186:187], v[186:187], v[18:19], v[16:17]\n\t" \
  "v_pk_fma_f32 v[188:189], v[188:189], v[18:19], v[16:17]\n\t" \
  "v_rcp_f32 v163, v163\n\t" \
  "v_pk_fma_f32 v[190:191], v[190:191], v[18:19], v[16:17]\n\t" \
  "v_pk_mul_f32 v[184:185], v[184:185], v[176:177]\n\t" \
  "v_pk_mul_f32 v[186:187], v[186:187], v[178:179]\n\t" \
  "v_pk_mul_f32 v[188:189], v[188:189], v[180:181]\n\t" \
  "v_rcp_f32 v164, v164\n\t" \
  "v_pk_mul_f32 v[190:191], v[190:191], v[182:183]\n\t" \
  "s_waitcnt lgkmcnt(0)\n\t" \
  "global_store_dwordx4 v24, v[32:35], s[30:31] nt\n\t" \
  "v_rcp_f32 v165, v165\n\t" \
  "s_add_u32 s30, s30, 0x40000\n\t" \
  "s_addc_u32 s31, s31, 0\n\t" \
  "global_store_dwordx4 v24, v[36:39], s[30:31] nt\n\t" \
  "s_add_u32 s30, s30, 0x40000\n\t" \
  "v_rcp_f32 v166, v166\n\t" \
  "s_addc_u32 s31, s31, 0\n\t" \
  "global_store_dwordx4 v24, v[40:43], s[30:31] nt\n\t" \
  "s_add_u32 s30, s30, 0x40000\n\t" \
  "v_rcp_f32 v167, v167\n\t" \
  "s_addc_u32 s31, s31, 0\n\t" \
  "global_store_dwordx4 v24, v[44:47], s[30:31] nt\n\t" \
  "s_add_u32 s30, s30, 0x40000\n\t" \
  "s_addc_u32 s31, s31, 0\n\t" \
  "v_exp_f32 v184, v184\n\t" \
  "v_pk_mul_f32 v[160:161], v[160:161], v[152:153]\n\t" \
  "v_exp_f32 v185, v185\n\t" \
  "v_pk_mul_f32 v[162:163], v[162:163], v[154:155]\n\t" \
  "v_pk_mul_f32 v[164:165], v[164:165], v[156:157]\n\t" \
  "v_exp_f32 v186, v186\n\t" \
  "v_pk_mul_f32 v[166:167], v[166:167], v[158:159]\n\t" \
  "v_cvt_pk_bf16_f32 v168, v160, v161\n\t" \
  "v_exp_f32 v187, v187\n\t" \
  "v_cvt_pk_bf16_f32 v169, v162, v163\n\t" \
  "v_cvt_pk_bf16_f32 v170, v164, v165\n\t" \
  "v_exp_f32 v188, v188\n\t" \
  "v_cvt_pk_bf16_f32 v171, v166, v167\n\t" \
  "ds_write_b64 v22, v[168:169] offset:4352\n\t" \
  "v_exp_f32 v189, v189\n\t" \
  "ds_write_b64 v22, v[170:171] offset:4384\n\t" \
  "v_accvgpr_read_b32 v200, a148\n\t" \
  "v_exp_f32 v190, v190\n\t" \
  "v_accvgpr_read_b32 v201, a149\n\t" \
  "v_accvgpr_read_b32 v202, a150\n\t" \
  "v_exp_f32 v191, v191\n\t" \
  "v_accvgpr_read_b32 v203, a151\n\t" \
  "v_accvgpr_read_b32 v204, a180\n\t" \
  "v_pk_add_f32 v[184:185], v[184:185], v[20:21]\n\t" \
  "v_pk_add_f32 v[186:187], v[186:187], v[20:21]\n\t" \
  "v_pk_add_f32 v[188:189], v[188:189], v[20:21]\n\t" \
  "v_pk_add_f32 v[190:191], v[190:191], v[20:21]\n\t" \
  "v_rcp_f32 v184, v184\n\t" \
  "v_accvgpr_read_b32 v205, a181\n\t" \
  "v_rcp_f32 v185, v185\n\t" \
  "v_accvgpr_read_b32 v206, a182\n\t" \
  "v_accvgpr_read_b32 v207, a183\n\t" \
  "v_rcp_f32 v186, v186\n\t" \
  "v_pk_mul_f32 v[208:209], v[200:201], v[200:201]\n\t" \
  "v_pk_mul_f32 v[210:211], v[202:203], v[202:203]\n\t" \
  "v_rcp_f32 v187, v187\n\t" \
  "v_pk_mul_f32 v[212:213], v[204:205], v[204:205]\n\t" \
  "v_pk_mul_f32 v[214:215], v[206:207], v[206:207]\n\t" \
  "v_rcp_f32 v188, v188\n\t" \
  "v_pk_fma_f32 v[208:209], v[208:209], v[18:19], v[16:17]\n\t" \
  "v_pk_fma_f32 v[210:211], v[210:211], v[18:19], v[16:17]\n\t" \
  "v_rcp_f32 v189, v189\n\t" \
  "v_pk_fma_f32 v[212:213], v[212:213], v[18:19], v[16:17]\n\t" \
  "v_pk_fma_f32 v[214:215], v[214:215], v[18:19], v[16:17]\n\t" \
  "v_rcp_f32 v190, v190\n\t" \
  "v_pk_mul_f32 v[208:209], v[208:209], v[200:201]\n\t" \
  "v_pk_mul_f32 v[210:211], v[210:211], v[202:203]\n\t" \
  "v_rcp_f32 v191, v191\n\t" \
  "v_pk_mul_f32 v[212:213], v[212:213], v[204:205]\n\t" \
  "v_pk_mul_f32 v[214:215], v[214:215], v[206:207]\n\t" \
  "v_exp_f32 v208, v208\n\t" \
  "v_pk_mul_f32 v[184:185], v[184:185], v[176:177]\n\t" \
  "v_exp_f32 v209, v209\n\t" \
  "v_pk_mul_f32 v[186:187], v[186:187], v[178:179]\n\t" \
  "v_pk_mul_f32 v[188:189], v[188:189], v[180:181]\n\t" \
  "v_exp_f32 v210, v210\n\t" \
  "v_pk_mul_f32 v[190:191], v[190:191], v[182:183]\n\t" \
  "v_cvt_pk_bf16_f32 v192, v184, v185\n\t" \
  "v_exp_f32 v211, v211\n\t" \
  "v_cvt_pk_bf16_f32 v193, v186, v187\n\t" \
  "v_cvt_pk_bf16_f32 v194, v188, v189\n\t" \
  "v_exp_f32 v212, v212\n\t" \
  "v_cvt_pk_bf16_f32 v195, v190, v191\n\t" \
  "ds_write_b64 v22, v[192:193] offset:4416\n\t" \
  "v_exp_f32 v213, v213\n\t" \
  "ds_write_b64 v22, v[194:195] offset:4448\n\t" \
  "v_accvgpr_read_b32 v128, a212\n\t" \
  "v_exp_f32 v214, v214\n\t" \
  "v_accvgpr_read_b32 v129, a213\n\t" \
  "v_accvgpr_read_b32 v130, a214\n\t" \
  "v_exp_f32 v215, v215\n\t" \
  "v_accvgpr_read_b32 v131, a215\n\t" \
  "v_accvgpr_read_b32 v132, a244\n\t" \
  "v_pk_add_f32 v[208:209], v[208:209], v[20:21]\n\t" \
  "v_pk_add_f32 v[210:211], v[210:211], v[20:21]\n\t" \
  "v_pk_add_f32 v[212:213], v[212:213], v[20:21]\n\t" \
  "v_pk_add_f32 v[214:215], v[214:215], v[20:21]\n\t" \
  "v_rcp_f32 v208, v208\n\t" \
  "v_accvgpr_read_b32 v133, a245\n\t" \
  "v_rcp_f32 v209, v209\n\t" \
  "v_accvgpr_read_b32 v134, a246\n\t" \
  "v_accvgpr_read_b32 v135, a247\n\t" \
  "v_rcp_f32 v210, v210\n\t" \
  "v_pk_mul_f32 v[136:137], v[128:129], v[128:129]\n\t" \
  "v_pk_mul_f32 v[138:139], v[130:131], v[130:131]\n\t" \
  "v_rcp_f32 v211, v211\n\t" \
  "v_pk_mul_f32 v[140:141], v[132:133], v[132:133]\n\t" \
  "v_pk_mul_f32 v[142:143], v[134:135], v[134:135]\n\t" \
  "v_rcp_f32 v212, v212\n\t" \
  "v_pk_fma_f32 v[136:137], v[136:137], v[18:19], v[16:17]\n\t" \
  "v_pk_fma_f32 v[138:139], v[138:139], v[18:19], v[16:17]\n\t" \
  "v_rcp_f32 v213, v213\n\t" \
  "v_pk_fma_f32 v[140:141], v[140:141], v[18:19], v[16:17]\n\t" \
  "v_pk_fma_f32 v[142:143], v[142:143], v[18:19], v[16:17]\n\t" \
  "v_rcp_f32 v214, v214\n\t" \
  "v_pk_mul_f32 v[136:137], v[136:137], v[128:129]\n\t" \
  "v_pk_mul_f32 v[138:139], v[138:139], v[130:131]\n\t" \
  "v_rcp_f32 v215, v215\n\t" \
  "v_pk_mul_f32 v[140:141], v[140:141], v[132:133]\n\t" \
  "v_pk_mul_f32 v[142:143], v[142:143], v[134:135]\n\t" \
  "v_exp_f32 v136, v136\n\t" \
  "v_pk_mul_f32 v[208:209], v[208:209], v[200:201]\n\t" \
  "v_exp_f32 v137, v137\n\t" \
  "v_pk_mul_f32 v[210:211], v[210:211], v[202:203]\n\t" \
  "v_pk_mul_f32 v[212:213], v[212:213], v[204:205]\n\t" \
  "v_exp_f32 v138, v138\n\t" \
  "v_pk_mul_f32 v[214:215], v[214:215], v[206:207]\n\t" \
  "v_cvt_pk_bf16_f32 v216, v208, v209\n\t" \
  "v_exp_f32 v139, v139\n\t" \
  "v_cvt_pk_bf16_f32 v217, v210, v211\n\t" \
  "v_cvt_pk_bf16_f32 v218, v212, v213\n\t" \
  "v_exp_f32 v140, v140\n\t" \
  "v_cvt_pk_bf16_f32 v219, v214, v215\n\t" \
  "ds_write_b64 v22, v[216:217] offset:4480\n\t" \
  "v_exp_f32 v141, v141\n\t" \
  "ds_write_b64 v22, v[218:219] offset:4512\n\t" \
  "v_accvgpr_read_b32 v152, a24\n\t" \
  "v_exp_f32 v142, v142\n\t" \
  "v_accvgpr_read_b32 v153, a25\n\t" \
  "v_accvgpr_read_b32 v154, a26\n\t" \
  "v_exp_f32 v143, v143\n\t" \
  "v_accvgpr_read_b32 v155, a27\n\t" \
  "v_accvgpr_read_b32 v156, a56\n\t" \
  "v_pk_add_f32 v[136:137], v[136:137], v[20:21]\n\t" \
  "v_pk_add_f32 v[138:139], v[138:139], v[20:21]\n\t" \
  "v_pk_add_f32 v[140:141], v[140:141], v[20:21]\n\t" \
  "v_pk_add_f32 v[142:143], v[142:143], v[20:21]\n\t" \
  "v_rcp_f32 v136, v136\n\t" \
  "v_accvgpr_read_b32 v157, a57\n\t" \
  "v_rcp_f32 v137, v137\n\t" \
  "v_accvgpr_read_b32 v158, a58\n\t" \
  "v_accvgpr_read_b32 v159, a59\n\t" \
  "v_rcp_f32 v138, v138\n\t" \
  "v_pk_mul_f32 v[160:161], v[152:153], v[152:153]\n\t" \
  "v_pk_mul_f32 v[162:163], v[154:155], v[154:155]\n\t" \
  "v_rcp_f32 v139, v139\n\t" \
  "v_pk_mul_f32 v[164:165], v[156:157], v[156:157]\n\t" \
  "v_pk_mul_f32 v[166:167], v[158:159], v[158:159]\n\t" \
  "v_rcp_f32 v140, v140\n\t" \
  "v_pk_fma_f32 v[160:161], v[160:161], v[18:19], v[16:17]\n\t" \
  "v_pk_fma_f32 v[162:163], v[162:163], v[18:19], v[16:17]\n\t" \
  "v_rcp_f32 v141, v141\n\t" \
  "v_pk_fma_f32 v[164:165], v[164:165], v[18:19], v[16:17]\n\t" \
  "v_pk_fma_f32 v[166:167], v[166:167], v[18:19], v[16:17]\n\t" \
  "v_rcp_f32 v142, v142\n\t" \
  "v_pk_mul_f32 v[160:161], v[160:161], v[152:153]\n\t" \
  "v_pk_mul_f32 v[162:163], v[162:163], v[154:155]\n\t" \
  "v_rcp_f32 v143, v143\n\t" \
  "v_pk_mul_f32 v[164:165], v[164:165], v[156:157]\n\t" \
  "v_pk_mul_f32 v[166:167], v[166:167], v[158:159]\n\t" \
  "v_exp_f32 v160, v160\n\t" \
  "v_pk_mul_f32 v[136:137], v[136:137], v[128:129]\n\t" \
  "v_pk_mul_f32 v[138:139], v[138:139], v[130:131]\n\t" \
  "v_exp_f32 v161, v161\n\t" \
  "v_pk_mul_f32 v[140:141], v[140:141], v[132:133]\n\t" \
  "v_pk_mul_f32 v[142:143], v[142:143], v[134:135]\n\t" \
  "v_exp_f32 v162, v162\n\t" \
  "v_cvt_pk_bf16_f32 v144, v136, v137\n\t" \
  "v_cvt_pk_bf16_f32 v145, v138, v139\n\t" \
  "v_cvt_pk_bf16_f32 v146, v140, v141\n\t" \
  "v_exp_f32 v163, v163\n\t" \
  "v_cvt_pk_bf16_f32 v147, v142, v143\n\t" \
  "ds_write_b64 v22, v[144:145] offset:4544\n\t" \
  "v_exp_f32 v164, v164\n\t" \
  "ds_write_b64 v22, v[146:147] offset:4576\n\t" \
  "ds_read_b128 v[32:35], v23 offset:4352\n\t" \
  "v_exp_f32 v165, v165\n\t" \
  "ds_read_b128 v[36:39], v23 offset:5440\n\t" \
  "ds_read_b128 v[40:43], v23 offset:6528\n\t" \
  "ds_read_b128 v[44:47], v23 offset:7616\n\t" \
  "v_exp_f32 v166, v166\n\t" \
  "v_accvgpr_read_b32 v176, a88\n\t" \
  "v_accvgpr_read_b32 v177, a89\n\t" \
  "v_exp_f32 v167, v167\n\t" \
  "v_accvgpr_read_b32 v178, a90\n\t" \
  "v_accvgpr_read_b32 v179, a91\n\t" \
  "v_accvgpr_read_b32 v180, a120\n\t" \
  "v_pk_add_f32 v[160:161], v[160:161], v[20:21]\n\t" \
  "v_pk_add_f32 v[162:163], v[162:163], v[20:21]\n\t" \
  "v_pk_add_f32 v[164:165], v[164:165], v[20:21]\n\t" \
  "v_pk_add_f32 v[166:167], v[166:167], v[20:21]\n\t" \
  "v_rcp_f32 v160, v160\n\t" \
  "v_accvgpr_read_b32 v181, a121\n\t" \
  "v_accvgpr_read_b32 v182, a122\n\t" \
  "v_accvgpr_read_b32 v183, a123\n\t" \
  "v_rcp_f32 v161, v161\n\t" \
  "v_pk_mul_f32 v[184:185], v[176:177], v[176:177]\n\t" \
  "v_pk_mul_f32 v[186:187], v[178:179], v[178:179]\n\t" \
  "v_pk_mul_f32 v[188:189], v[180:181], v[180:181]\n\t" \
  "v_pk_mul_f32 v[190:191], v[182:183], v[182:183]\n\t" \
  "v_rcp_f32 v162, v162\n\t" \
  "v_pk_fma_f32 v[184:185], v[184:185], v[18:19], v[16:17]\n\t" \
  "v_pk_fma_f32 v[186:187], v[186:187], v[18:19], v[16:17]\n\t" \
  "v_pk_fma_f32 v[188:189], v[188:189], v[18:19], v[16:17]\n\t" \
  "v_rcp_f32 v163, v163\n\t" \
  "v_pk_fma_f32 v[190:191], v[190:191], v[18:19], v[16:17]\n\t" \
  "v_pk_mul_f32 v[184:185], v[184:185], v[176:177]\n\t" \
  "v_pk_mul_f32 v[186:187], v[186:187], v[178:179]\n\t" \
  "v_pk_mul_f32 v[188:189], v[188:189], v[180:181]\n\t" \
  "v_rcp_f32 v164, v164\n\t" \
  "v_pk_mul_f32 v[190:191], v[190:191], v[182:183]\n\t" \
  "s_waitcnt lgkmcnt(0)\n\t" \
  "global_store_dwordx4 v24, v[32:35], s[30:31] nt\n\t" \
  "v_rcp_f32 v165, v165\n\t" \
  "s_add_u32 s30, s30, 0x40000\n\t" \
  "s_addc_u32 s31, s31, 0\n\t" \
  "global_store_dwordx4 v24, v[36:39], s[30:31] nt\n\t" \
  "s_add_u32 s30, s30, 0x40000\n\t" \
  "v_rcp_f32 v166, v166\n\t" \
  "s_addc_u32 s31, s31, 0\n\t" \
  "global_store_dwordx4 v24, v[40:43], s[30:31] nt\n\t" \
  "s_add_u32 s30, s30, 0x40000\n\t" \
  "v_rcp_f32 v167, v167\n\t" \
  "s_addc_u32 s31, s31, 0\n\t" \
  "global_store_dwordx4 v24, v[44:47], s[30:31] nt\n\t" \
  "s_add_u32 s30, s30, 0x40000\n\t" \
  "s_addc_u32 s31, s31, 0\n\t" \
  "v_exp_f32 v184, v184\n\t" \
  "v_pk_mul_f32 v[160:161], v[160:161], v[152:153]\n\t" \
  "v_exp_f32 v185, v185\n\t" \
  "v_pk_mul_f32 v[162:163], v[162:163], v[154:155]\n\t" \
  "v_pk_mul_f32 v[164:165], v[164:165], v[156:157]\n\t" \
  "v_exp_f32 v186, v186\n\t" \
  "v_pk_mul_f32 v[166:167], v[166:167], v[158:159]\n\t" \
  "v_cvt_pk_bf16_f32 v168, v160, v161\n\t" \
  "v_exp_f32 v187, v187\n\t" \
  "v_cvt_pk_bf16_f32 v169, v162, v163\n\t" \
  "v_cvt_pk_bf16_f32 v170, v164, v165\n\t" \
  "v_exp_f32 v188, v188\n\t" \
  "v_cvt_pk_bf16_f32 v171, v166, v167\n\t" \
  "ds_write_b64 v22, v[168:169] offset:0\n\t" \
  "v_exp_f32 v189, v189\n\t" \
  "ds_write_b64 v22, v[170:171] offset:32\n\t" \
  "v_accvgpr_read_b32 v200, a152\n\t" \
  "v_exp_f32 v190, v190\n\t" \
  "v_accvgpr_read_b32 v201, a153\n\t" \
  "v_accvgpr_read_b32 v202, a154\n\t" \
  "v_exp_f32 v191, v191\n\t" \
  "v_accvgpr_read_b32 v203, a155\n\t" \
  "v_accvgpr_read_b32 v204, a184\n\t" \
  "v_pk_add_f32 v[184:185], v[184:185], v[20:21]\n\t" \
  "v_pk_add_f32 v[186:187], v[186:187], v[20:21]\n\t" \
  "v_pk_add_f32 v[188:189], v[188:189], v[20:21]\n\t" \
  "v_pk_add_f32 v[190:191], v[190:191], v[20:21]\n\t" \
  "v_rcp_f32 v184, v184\n\t" \
  "v_accvgpr_read_b32 v205, a185\n\t" \
  "v_rcp_f32 v185, v185\n\t" \
  "v_accvgpr_read_b32 v206, a186\n\t" \
  "v_accvgpr_read_b32 v207, a187\n\t" \
  "v_rcp_f32 v186, v186\n\t" \
  "v_pk_mul_f32 v[208:209], v[200:201], v[200:201]\n\t" \
  "v_pk_mul_f32 v[210:211], v[202:203], v[202:203]\n\t" \
  "v_rcp_f32 v187, v187\n\t" \
  "v_pk_mul_f32 v[212:213], v[204:205], v[204:205]\n\t" \
  "v_pk_mul_f32 v[214:215], v[206:207], v[206:207]\n\t" \
  "v_rcp_f32 v188, v188\n\t" \
  "v_pk_fma_f32 v[208:209], v[208:209], v[18:19], v[16:17]\n\t" \
  "v_pk_fma_f32 v[210:211], v[210:211], v[18:19], v[16:17]\n\t" \
  "v_rcp_f32 v189, v189\n\t" \
  "v_pk_fma_f32 v[212:213], v[212:213], v[18:19], v[16:17]\n\t" \
  "v_pk_fma_f32 v[214:215], v[214:215], v[18:19], v[16:17]\n\t" \
  "v_rcp_f32 v190, v190\n\t" \
  "v_pk_mul_f32 v[208:209], v[208:209], v[200:201]\n\t" \
  "v_pk_mul_f32 v[210:211], v[210:211], v[202:203]\n\t" \
  "v_rcp_f32 v191, v191\n\t" \
  "v_pk_mul_f32 v[212:213], v[212:213], v[204:205]\n\t" \
  "v_pk_mul_f32 v[214:215], v[214:215], v[206:207]\n\t" \
  "v_exp_f32 v208, v208\n\t" \
  "v_pk_mul_f32 v[184:185], v[184:185], v[176:177]\n\t" \
  "v_exp_f32 v209, v209\n\t" \
  "v_pk_mul_f32 v[186:187], v[186:187], v[178:179]\n\t" \
  "v_pk_mul_f32 v[188:189], v[188:189], v[180:181]\n\t" \
  "v_exp_f32 v210, v210\n\t" \
  "v_pk_mul_f32 v[190:191], v[190:191], v[182:183]\n\t" \
  "v_cvt_pk_bf16_f32 v192, v184, v185\n\t" \
  "v_exp_f32 v211, v211\n\t" \
  "v_cvt_pk_bf16_f32 v193, v186, v187\n\t" \
  "v_cvt_pk_bf16_f32 v194, v188, v189\n\t" \
  "v_exp_f32 v212, v212\n\t" \
  "v_cvt_pk_bf16_f32 v195, v190, v191\n\t" \
  "ds_write_b64 v22, v[192:193] offset:64\n\t" \
  "v_exp_f32 v213, v213\n\t" \
  "ds_write_b64 v22, v[194:195] offset:96\n\t" \
  "v_accvgpr_read_b32 v128, a216\n\t" \
  "v_exp_f32 v214, v214\n\t" \
  "v_accvgpr_read_b32 v129, a217\n\t" \
  "v_accvgpr_read_b32 v130, a218\n\t" \
  "v_exp_f32 v215, v215\n\t" \
  "v_accvgpr_read_b32 v131, a219\n\t" \
  "v_accvgpr_read_b32 v132, a248\n\t" \
  "v_pk_add_f32 v[208:209], v[208:209], v[20:21]\n\t" \
  "v_pk_add_f32 v[210:211], v[210:211], v[20:21]\n\t" \
  "v_pk_add_f32 v[212:213], v[212:213], v[20:21]\n\t" \
  "v_pk_add_f32 v[214:215], v[214:215], v[20:21]\n\t" \
  "v_rcp_f32 v208, v208\n\t" \
  "v_accvgpr_read_b32 v133, a249\n\t" \
  "v_rcp_f32 v209, v209\n\t" \
  "v_accvgpr_read_b32 v134, a250\n\t" \
  "v_accvgpr_read_b32 v135, a251\n\t" \
  "v_rcp_f32 v210, v210\n\t" \
  "v_pk_mul_f32 v[136:137], v[128:129], v[128:129]\n\t" \
  "v_pk_mul_f32 v[138:139], v[130:131], v[130:131]\n\t" \
  "v_rcp_f32 v211, v211\n\t" \
  "v_pk_mul_f32 v[140:141], v[132:133], v[132:133]\n\t" \
  "v_pk_mul_f32 v[142:143], v[134:135], v[134:135]\n\t" \
  "v_rcp_f32 v212, v212\n\t" \
  "v_pk_fma_f32 v[136:137], v[136:137], v[18:19], v[16:17]\n\t" \
  "v_pk_fma_f32 v[138:139], v[138:139], v[18:19], v[16:17]\n\t" \
  "v_rcp_f32 v213, v213\n\t" \
  "v_pk_fma_f32 v[140:141], v[140:141], v[18:19], v[16:17]\n\t" \
  "v_pk_fma_f32 v[142:143], v[142:143], v[18:19], v[16:17]\n\t" \
  "v_rcp_f32 v214, v214\n\t" \
  "v_pk_mul_f32 v[136:137], v[136:137], v[128:129]\n\t" \
  "v_pk_mul_f32 v[138:139], v[138:139], v[130:131]\n\t" \
  "v_rcp_f32 v215, v215\n\t" \
  "v_pk_mul_f32 v[140:141], v[140:141], v[132:133]\n\t" \
  "v_pk_mul_f32 v[142:143], v[142:143], v[134:135]\n\t" \
  "v_exp_f32 v136, v136\n\t" \
  "v_pk_mul_f32 v[208:209], v[208:209], v[200:201]\n\t" \
  "v_exp_f32 v137, v137\n\t" \
  "v_pk_mul_f32 v[210:211], v[210:211], v[202:203]\n\t" \
  "v_pk_mul_f32 v[212:213], v[212:213], v[204:205]\n\t" \
  "v_exp_f32 v138, v138\n\t" \
  "v_pk_mul_f32 v[214:215], v[214:215], v[206:207]\n\t" \
  "v_cvt_pk_bf16_f32 v216, v208, v209\n\t" \
  "v_exp_f32 v139, v139\n\t" \
  "v_cvt_pk_bf16_f32 v217, v210, v211\n\t" \
  "v_cvt_pk_bf16_f32 v218, v212, v213\n\t" \
  "v_exp_f32 v140, v140\n\t" \
  "v_cvt_pk_bf16_f32 v219, v214, v215\n\t" \
  "ds_write_b64 v22, v[216:217] offset:128\n\t" \
  "v_exp_f32 v141, v141\n\t" \
  "ds_write_b64 v22, v[218:219] offset:160\n\t" \
  "v_accvgpr_read_b32 v152, a28\n\t" \
  "v_exp_f32 v142, v142\n\t" \
  "v_accvgpr_read_b32 v153, a29\n\t" \
  "v_accvgpr_read_b32 v154, a30\n\t" \
  "v_exp_f32 v143, v143\n\t" \
  "v_accvgpr_read_b32 v155, a31\n\t" \
  "v_accvgpr_read_b32 v156, a60\n\t" \
  "v_pk_add_f32 v[136:137], v[136:137], v[20:21]\n\t" \
  "v_pk_add_f32 v[138:139], v[138:139], v[20:21]\n\t" \
  "v_pk_add_f32 v[140:141], v[140:141], v[20:21]\n\t" \
  "v_pk_add_f32 v[142:143], v[142:143], v[20:21]\n\t" \
  "v_rcp_f32 v136, v136\n\t" \
  "v_accvgpr_read_b32 v157, a61\n\t" \
  "v_rcp_f32 v137, v137\n\t" \
  "v_accvgpr_read_b32 v158, a62\n\t" \
  "v_accvgpr_read_b32 v159, a63\n\t" \
  "v_rcp_f32 v138, v138\n\t" \
  "v_pk_mul_f32 v[160:161], v[152:153], v[152:153]\n\t" \
  "v_pk_mul_f32 v[162:163], v[154:155], v[154:155]\n\t" \
  "v_rcp_f32 v139, v139\n\t" \
  "v_pk_mul_f32 v[164:165], v[156:157], v[156:157]\n\t" \
  "v_pk_mul_f32 v[166:167], v[158:159], v[158:159]\n\t" \
  "v_rcp_f32 v140, v140\n\t" \
  "v_pk_fma_f32 v[160:161], v[160:161], v[18:19], v[16:17]\n\t" \
  "v_pk_fma_f32 v[162:163], v[162:163], v[18:19], v[16:17]\n\t" \
  "v_rcp_f32 v141, v141\n\t" \
  "v_pk_fma_f32 v[164:165], v[164:165], v[18:19], v[16:17]\n\t" \
  "v_pk_fma_f32 v[166:167], v[166:167], v[18:19], v[16:17]\n\t" \
  "v_rcp_f32 v142, v142\n\t" \
  "v_pk_mul_f32 v[160:161], v[160:161], v[152:153]\n\t" \
  "v_pk_mul_f32 v[162:163], v[162:163], v[154:155]\n\t" \
  "v_rcp_f32 v143, v143\n\t" \
  "v_pk_mul_f32 v[164:165], v[164:165], v[156:157]\n\t" \
  "v_pk_mul_f32 v[166:167], v[166:167], v[158:159]\n\t" \
  "v_exp_f32 v160, v160\n\t" \
  "v_pk_mul_f32 v[136:137], v[136:137], v[128:129]\n\t" \
  "v_pk_mul_f32 v[138:139], v[138:139], v[130:131]\n\t" \
  "v_exp_f32 v161, v161\n\t" \
  "v_pk_mul_f32 v[140:141], v[140:141], v[132:133]\n\t" \
  "v_pk_mul_f32 v[142:143], v[142:143], v[134:135]\n\t" \
  "v_exp_f32 v162, v162\n\t" \
  "v_cvt_pk_bf16_f32 v144, v136, v137\n\t" \
  "v_cvt_pk_bf16_f32 v145, v138, v139\n\t" \
  "v_cvt_pk_bf16_f32 v146, v140, v141\n\t" \
  "v_exp_f32 v163, v163\n\t" \
  "v_cvt_pk_bf16_f32 v147, v142, v143\n\t" \
  "ds_write_b64 v22, v[144:145] offset:192\n\t" \
  "v_exp_f32 v164, v164\n\t" \
  "ds_write_b64 v22, v[146:147] offset:224\n\t" \
  "ds_read_b128 v[32:35], v23 offset:0\n\t" \
  "v_exp_f32 v165, v165\n\t" \
  "ds_read_b128 v[36:39], v23 offset:1088\n\t" \
  "ds_read_b128 v[40:43], v23 offset:2176\n\t" \
  "ds_read_b128 v[44:47], v23 offset:3264\n\t" \
  "v_exp_f32 v166, v166\n\t" \
  "v_accvgpr_read_b32 v176, a92\n\t" \
  "v_accvgpr_read_b32 v177, a93\n\t" \
  "v_exp_f32 v167, v167\n\t" \
  "v_accvgpr_read_b32 v178, a94\n\t" \
  "v_accvgpr_read_b32 v179, a95\n\t" \
  "v_accvgpr_read_b32 v180, a124\n\t" \
  "v_pk_add_f32 v[160:161], v[160:161], v[20:21]\n\t" \
  "v_pk_add_f32 v[162:163], v[162:163], v[20:21]\n\t" \
  "v_pk_add_f32 v[164:165], v[164:165], v[20:21]\n\t" \
  "v_pk_add_f32 v[166:167], v[166:167], v[20:21]\n\t" \
  "v_rcp_f32 v160, v160\n\t" \
  "v_accvgpr_read_b32 v181, a125\n\t" \
  "v_accvgpr_read_b32 v182, a126\n\t" \
  "v_accvgpr_read_b32 v183, a127\n\t" \
  "v_rcp_f32 v161, v161\n\t" \
  "v_pk_mul_f32 v[184:185], v[176:177], v[176:177]\n\t" \
  "v_pk_mul_f32 v[186:187], v[178:179], v[178:179]\n\t" \
  "v_pk_mul_f32 v[188:189], v[180:181], v[180:181]\n\t" \
  "v_pk_mul_f32 v[190:191], v[182:183], v[182:183]\n\t" \
  "v_rcp_f32 v162, v162\n\t" \
  "v_pk_fma_f32 v[184:185], v[184:185], v[18:19], v[16:17]\n\t" \
  "v_pk_fma_f32 v[186:187], v[186:187], v[18:19], v[16:17]\n\t" \
  "v_pk_fma_f32 v[188:189], v[188:189], v[18:19], v[16:17]\n\t" \
  "v_rcp_f32 v163, v163\n\t" \
  "v_pk_fma_f32 v[190:191], v[190:191], v[18:19], v[16:17]\n\t" \
  "v_pk_mul_f32 v[184:185], v[184:185], v[176:177]\n\t" \
  "v_pk_mul_f32 v[186:187], v[186:187], v[178:179]\n\t" \
  "v_pk_mul_f32 v[188:189], v[188:189], v[180:181]\n\t" \
  "v_rcp_f32 v164, v164\n\t" \
  "v_pk_mul_f32 v[190:191], v[190:191], v[182:183]\n\t" \
  "s_waitcnt lgkmcnt(0)\n\t" \
  "global_store_dwordx4 v24, v[32:35], s[30:31] nt\n\t" \
  "v_rcp_f32 v165, v165\n\t" \
  "s_add_u32 s30, s30, 0x40000\n\t" \
  "s_addc_u32 s31, s31, 0\n\t" \
  "global_store_dwordx4 v24, v[36:39], s[30:31] nt\n\t" \
  "s_add_u32 s30, s30, 0x40000\n\t" \
  "v_rcp_f32 v166, v166\n\t" \
  "s_addc_u32 s31, s31, 0\n\t" \
  "global_store_dwordx4 v24, v[40:43], s[30:31] nt\n\t" \
  "s_add_u32 s30, s30, 0x40000\n\t" \
  "v_rcp_f32 v167, v167\n\t" \
  "s_addc_u32 s31, s31, 0\n\t" \
  "global_store_dwordx4 v24, v[44:47], s[30:31] nt\n\t" \
  "s_add_u32 s30, s30, 0x40000\n\t" \
  "s_addc_u32 s31, s31, 0\n\t" \
  "v_exp_f32 v184, v184\n\t" \
  "v_pk_mul_f32 v[160:161], v[160:161], v[152:153]\n\t" \
  "v_exp_f32 v185, v185\n\t" \
  "v_pk_mul_f32 v[162:163], v[162:163], v[154:155]\n\t" \
  "v_pk_mul_f32 v[164:165], v[164:165], v[156:157]\n\t" \
  "v_exp_f32 v186, v186\n\t" \
  "v_pk_mul_f32 v[166:167], v[166:167], v[158:159]\n\t" \
  "v_cvt_pk_bf16_f32 v168, v160, v161\n\t" \
  "v_exp_f32 v187, v187\n\t" \
  "v_cvt_pk_bf16_f32 v169, v162, v163\n\t" \
  "v_cvt_pk_bf16_f32 v170, v164, v165\n\t" \
  "v_exp_f32 v188, v188\n\t" \
  "v_cvt_pk_bf16_f32 v171, v166, v167\n\t" \
  "ds_write_b64 v22, v[168:169] offset:4352\n\t" \
  "v_exp_f32 v189, v189\n\t" \
  "ds_write_b64 v22, v[170:171] offset:4384\n\t" \
  "v_accvgpr_read_b32 v200, a156\n\t" \
  "v_exp_f32 v190, v190\n\t" \
  "v_accvgpr_read_b32 v201, a157\n\t" \
  "v_accvgpr_read_b32 v202, a158\n\t" \
  "v_exp_f32 v191, v191\n\t" \
  "v_accvgpr_read_b32 v203, a159\n\t" \
  "v_accvgpr_read_b32 v204, a188\n\t" \
  "v_pk_add_f32 v[184:185], v[184:185], v[20:21]\n\t" \
  "v_pk_add_f32 v[186:187], v[186:187], v[20:21]\n\t" \
  "v_pk_add_f32 v[188:189], v[188:189], v[20:21]\n\t" \
  "v_pk_add_f32 v[190:191], v[190:191], v[20:21]\n\t" \
  "v_rcp_f32 v184, v184\n\t" \
  "v_accvgpr_read_b32 v205, a189\n\t" \
  "v_rcp_f32 v185, v185\n\t" \
  "v_accvgpr_read_b32 v206, a190\n\t" \
  "v_accvgpr_read_b32 v207, a191\n\t" \
  "v_rcp_f32 v186, v186\n\t" \
  "v_pk_mul_f32 v[208:209], v[200:201], v[200:201]\n\t" \
  "v_pk_mul_f32 v[210:211], v[202:203], v[202:203]\n\t" \
  "v_rcp_f32 v187, v187\n\t" \
  "v_pk_mul_f32 v[212:213], v[204:205], v[204:205]\n\t" \
  "v_pk_mul_f32 v[214:215], v[206:207], v[206:207]\n\t" \
  "v_rcp_f32 v188, v188\n\t" \
  "v_pk_fma_f32 v[208:209], v[208:209], v[18:19], v[16:17]\n\t" \
  "v_pk_fma_f32 v[210:211], v[210:211], v[18:19], v[16:17]\n\t" \
  "v_rcp_f32 v189, v189\n\t" \
  "v_pk_fma_f32 v[212:213], v[212:213], v[18:19], v[16:17]\n\t" \
  "v_pk_fma_f32 v[214:215], v[214:215], v[18:19], v[16:17]\n\t" \
  "v_rcp_f32 v190, v190\n\t" \
  "v_pk_mul_f32 v[208:209], v[208:209], v[200:201]\n\t" \
  "v_pk_mul_f32 v[210:211], v[210:211], v[202:203]\n\t" \
  "v_rcp_f32 v191, v191\n\t" \
  "v_pk_mul_f32 v[212:213], v[212:213], v[204:205]\n\t" \
  "v_pk_mul_f32 v[214:215], v[214:215], v[206:207]\n\t" \
  "v_exp_f32 v208, v208\n\t" \
  "v_pk_mul_f32 v[184:185], v[184:185], v[176:177]\n\t" \
  "v_exp_f32 v209, v209\n\t" \
  "v_pk_mul_f32 v[186:187], v[186:187], v[178:179]\n\t" \
  "v_pk_mul_f32 v[188:189], v[188:189], v[180:181]\n\t" \
  "v_exp_f32 v210, v210\n\t" \
  "v_pk_mul_f32 v[190:191], v[190:191], v[182:183]\n\t" \
  "v_cvt_pk_bf16_f32 v192, v184, v185\n\t" \
  "v_exp_f32 v211, v211\n\t" \
  "v_cvt_pk_bf16_f32 v193, v186, v187\n\t" \
  "v_cvt_pk_bf16_f32 v194, v188, v189\n\t" \
  "v_exp_f32 v212, v212\n\t" \
  "v_cvt_pk_bf16_f32 v195, v190, v191\n\t" \
  "ds_write_b64 v22, v[192:193] offset:4416\n\t" \
  "v_exp_f32 v213, v213\n\t" \
  "ds_write_b64 v22, v[194:195] offset:4448\n\t" \
  "v_accvgpr_read_b32 v128, a220\n\t" \
  "v_exp_f32 v214, v214\n\t" \
  "v_accvgpr_read_b32 v129, a221\n\t" \
  "v_accvgpr_read_b32 v130, a222\n\t" \
  "v_exp_f32 v215, v215\n\t" \
  "v_accvgpr_read_b32 v131, a223\n\t" \
  "v_accvgpr_read_b32 v132, a252\n\t" \
  "v_pk_add_f32 v[208:209], v[208:209], v[20:21]\n\t" \
  "v_pk_add_f32 v[210:211], v[210:211], v[20:21]\n\t" \
  "v_pk_add_f32 v[212:213], v[212:213], v[20:21]\n\t" \
  "v_pk_add_f32 v[214:215], v[214:215], v[20:21]\n\t" \
  "v_rcp_f32 v208, v208\n\t" \
  "v_accvgpr_read_b32 v133, a253\n\t" \
  "v_rcp_f32 v209, v209\n\t" \
  "v_accvgpr_read_b32 v134, a254\n\t" \
  "v_accvgpr_read_b32 v135, a255\n\t" \
  "v_rcp_f32 v210, v210\n\t" \
  "v_pk_mul_f32 v[136:137], v[128:129], v[128:129]\n\t" \
  "v_pk_mul_f32 v[138:139], v[130:131], v[130:131]\n\t" \
  "v_rcp_f32 v211, v211\n\t" \
  "v_pk_mul_f32 v[140:141], v[132:133], v[132:133]\n\t" \
  "v_pk_mul_f32 v[142:143], v[134:135], v[134:135]\n\t" \
  "v_rcp_f32 v212, v212\n\t" \
  "v_pk_fma_f32 v[136:137], v[136:137], v[18:19], v[16:17]\n\t" \
  "v_pk_fma_f32 v[138:139], v[138:139], v[18:19], v[16:17]\n\t" \
  "v_rcp_f32 v213, v213\n\t" \
  "v_pk_fma_f32 v[140:141], v[140:141], v[18:19], v[16:17]\n\t" \
  "v_pk_fma_f32 v[142:143], v[142:143], v[18:19], v[16:17]\n\t" \
  "v_rcp_f32 v214, v214\n\t" \
  "v_pk_mul_f32 v[136:137], v[136:137], v[128:129]\n\t" \
  "v_pk_mul_f32 v[138:139], v[138:139], v[130:131]\n\t" \
  "v_rcp_f32 v215, v215\n\t" \
  "v_pk_mul_f32 v[140:141], v[140:141], v[132:133]\n\t" \
  "v_pk_mul_f32 v[142:143], v[142:143], v[134:135]\n\t" \
  "v_exp_f32 v136, v136\n\t" \
  "v_pk_mul_f32 v[208:209], v[208:209], v[200:201]\n\t" \
  "v_exp_f32 v137, v137\n\t" \
  "v_pk_mul_f32 v[210:211], v[210:211], v[202:203]\n\t" \
  "v_exp_f32 v138, v138\n\t" \
  "v_pk_mul_f32 v[212:213], v[212:213], v[204:205]\n\t" \
  "v_exp_f32 v139, v139\n\t" \
  "v_pk_mul_f32 v[214:215], v[214:215], v[206:207]\n\t" \
  "v_cvt_pk_bf16_f32 v216, v208, v209\n\t" \
  "v_exp_f32 v140, v140\n\t" \
  "v_cvt_pk_bf16_f32 v217, v210, v211\n\t" \
  "v_exp_f32 v141, v141\n\t" \
  "v_cvt_pk_bf16_f32 v218, v212, v213\n\t" \
  "v_exp_f32 v142, v142\n\t" \
  "v_cvt_pk_bf16_f32 v219, v214, v215\n\t" \
  "v_exp_f32 v143, v143\n\t" \
  "ds_write_b64 v22, v[216:217] offset:4480\n\t" \
  "ds_write_b64 v22, v[218:219] offset:4512\n\t" \
  "v_pk_add_f32 v[136:137], v[136:137], v[20:21]\n\t" \
  "v_pk_add_f32 v[138:139], v[138:139], v[20:21]\n\t" \
  "v_pk_add_f32 v[140:141], v[140:141], v[20:21]\n\t" \
  "v_pk_add_f32 v[142:143], v[142:143], v[20:21]\n\t" \
  "v_rcp_f32 v136, v136\n\t" \
  "v_rcp_f32 v137, v137\n\t" \
  "v_rcp_f32 v138, v138\n\t" \
  "v_rcp_f32 v139, v139\n\t" \
  "v_rcp_f32 v140, v140\n\t" \
  "v_rcp_f32 v141, v141\n\t" \
  "v_rcp_f32 v142, v142\n\t" \
  "v_rcp_f32 v143, v143\n\t" \
  "s_nop 0\n\t" \
  "v_pk_mul_f32 v[136:137], v[136:137], v[128:129]\n\t" \
  "v_pk_mul_f32 v[138:139], v[138:139], v[130:131]\n\t" \
  "v_pk_mul_f32 v[140:141], v[140:141], v[132:133]\n\t" \
  "v_pk_mul_f32 v[142:143], v[142:143], v[134:135]\n\t" \
  "v_cvt_pk_bf16_f32 v144, v136, v137\n\t" \
  "v_cvt_pk_bf16_f32 v145, v138, v139\n\t" \
  "v_cvt_pk_bf16_f32 v146, v140, v141\n\t" \
  "v_cvt_pk_bf16_f32 v147, v142, v143\n\t" \
  "ds_write_b64 v22, v[144:145] offset:4544\n\t" \
  "ds_write_b64 v22, v[146:147] offset:4576\n\t" \
  "ds_read_b128 v[32:35], v23 offset:4352\n\t" \
  "ds_read_b128 v[36:39], v23 offset:5440\n\t" \
  "ds_read_b128 v[40:43], v23 offset:6528\n\t" \
  "ds_read_b128 v[44:47], v23 offset:7616\n\t" \
  "s_waitcnt lgkmcnt(0)\n\t" \
  "global_store_dwordx4 v24, v[32:35], s[30:31] nt\n\t" \
  "s_add_u32 s30, s30, 0x40000\n\t" \
  "s_addc_u32 s31, s31, 0\n\t" \
  "global_store_dwordx4 v24, v[36:39], s[30:31] nt\n\t" \
  "s_add_u32 s30, s30, 0x40000\n\t" \
  "s_addc_u32 s31, s31, 0\n\t" \
  "global_store_dwordx4 v24, v[40:43], s[30:31] nt\n\t" \
  "s_add_u32 s30, s30, 0x40000\n\t" \
  "s_addc_u32 s31, s31, 0\n\t" \
  "global_store_dwordx4 v24, v[44:47], s[30:31] nt\n\t" \
  "s_add_u32 s30, s30, 0x40000\n\t" \
  "s_addc_u32 s31, s31, 0\n\t" \
  "s_branch END_%=\n\t" \
  "NANOUT_%=:\n\t" \
  "v_mov_b32 v24, %[ooff]\n\t" \
  "v_mov_b32 v32, 0x7fc07fc0\n\t" \
  "v_mov_b32 v33, 0x7fc07fc0\n\t" \
  "v_mov_b32 v34, 0x7fc07fc0\n\t" \
  "v_mov_b32 v35, 0x7fc07fc0\n\t" \
  "global_store_dwordx4 v24, v[32:35], s[30:31]\n\t" \
  "s_add_u32 s30, s30, 0x40000\n\t" \
  "s_addc_u32 s31, s31, 0\n\t" \
  "global_store_dwordx4 v24, v[32:35], s[30:31]\n\t" \
  "s_add_u32 s30, s30, 0x40000\n\t" \
  "s_addc_u32 s31, s31, 0\n\t" \
  "global_store_dwordx4 v24, v[32:35], s[30:31]\n\t" \
  "s_add_u32 s30, s30, 0x40000\n\t" \
  "s_addc_u32 s31, s31, 0\n\t" \
  "global_store_dwordx4 v24, v[32:35], s[30:31]\n\t" \
  "s_add_u32 s30, s30, 0x40000\n\t" \
  "s_addc_u32 s31, s31, 0\n\t" \
  "global_store_dwordx4 v24, v[32:35], s[30:31]\n\t" \
  "s_add_u32 s30, s30, 0x40000\n\t" \
  "s_addc_u32 s31, s31, 0\n\t" \
  "global_store_dwordx4 v24, v[32:35], s[30:31]\n\t" \
  "s_add_u32 s30, s30, 0x40000\n\t" \
  "s_addc_u32 s31, s31, 0\n\t" \
  "global_store_dwordx4 v24, v[32:35], s[30:31]\n\t" \
  "s_add_u32 s30, s30, 0x40000\n\t" \
  "s_addc_u32 s31, s31, 0\n\t" \
  "global_store_dwordx4 v24, v[32:35], s[30:31]\n\t" \
  "s_add_u32 s30, s30, 0x40000\n\t" \
  "s_addc_u32 s31, s31, 0\n\t" \
  "global_store_dwordx4 v24, v[32:35], s[30:31]\n\t" \
  "s_add_u32 s30, s30, 0x40000\n\t" \
  "s_addc_u32 s31, s31, 0\n\t" \
  "global_store_dwordx4 v24, v[32:35], s[30:31]\n\t" \
  "s_add_u32 s30, s30, 0x40000\n\t" \
  "s_addc_u32 s31, s31, 0\n\t" \
  "global_store_dwordx4 v24, v[32:35], s[30:31]\n\t" \
  "s_add_u32 s30, s30, 0x40000\n\t" \
  "s_addc_u32 s31, s31, 0\n\t" \
  "global_store_dwordx4 v24, v[32:35], s[30:31]\n\t" \
  "s_add_u32 s30, s30, 0x40000\n\t" \
  "s_addc_u32 s31, s31, 0\n\t" \
  "global_store_dwordx4 v24, v[32:35], s[30:31]\n\t" \
  "s_add_u32 s30, s30, 0x40000\n\t" \
  "s_addc_u32 s31, s31, 0\n\t" \
  "global_store_dwordx4 v24, v[32:35], s[30:31]\n\t" \
  "s_add_u32 s30, s30, 0x40000\n\t" \
  "s_addc_u32 s31, s31, 0\n\t" \
  "global_store_dwordx4 v24, v[32:35], s[30:31]\n\t" \
  "s_add_u32 s30, s30, 0x40000\n\t" \
  "s_addc_u32 s31, s31, 0\n\t" \
  "global_store_dwordx4 v24, v[32:35], s[30:31]\n\t" \
  "s_add_u32 s30, s30, 0x40000\n\t" \
  "s_addc_u32 s31, s31, 0\n\t" \
  "global_store_dwordx4 v24, v[32:35], s[30:31]\n\t" \
  "s_add_u32 s30, s30, 0x40000\n\t" \
  "s_addc_u32 s31, s31, 0\n\t" \
  "global_store_dwordx4 v24, v[32:35], s[30:31]\n\t" \
  "s_add_u32 s30, s30, 0x40000\n\t" \
  "s_addc_u32 s31, s31, 0\n\t" \
  "global_store_dwordx4 v24, v[32:35], s[30:31]\n\t" \
  "s_add_u32 s30, s30, 0x40000\n\t" \
  "s_addc_u32 s31, s31, 0\n\t" \
  "global_store_dwordx4 v24, v[32:35], s[30:31]\n\t" \
  "s_add_u32 s30, s30, 0x40000\n\t" \
  "s_addc_u32 s31, s31, 0\n\t" \
  "global_store_dwordx4 v24, v[32:35], s[30:31]\n\t" \
  "s_add_u32 s30, s30, 0x40000\n\t" \
  "s_addc_u32 s31, s31, 0\n\t" \
  "global_store_dwordx4 v24, v[32:35], s[30:31]\n\t" \
  "s_add_u32 s30, s30, 0x40000\n\t" \
  "s_addc_u32 s31, s31, 0\n\t" \
  "global_store_dwordx4 v24, v[32:35], s[30:31]\n\t" \
  "s_add_u32 s30, s30, 0x40000\n\t" \
  "s_addc_u32 s31, s31, 0\n\t" \
  "global_store_dwordx4 v24, v[32:35], s[30:31]\n\t" \
  "s_add_u32 s30, s30, 0x40000\n\t" \
  "s_addc_u32 s31, s31, 0\n\t" \
  "global_store_dwordx4 v24, v[32:35], s[30:31]\n\t" \
  "s_add_u32 s30, s30, 0x40000\n\t" \
  "s_addc_u32 s31, s31, 0\n\t" \
  "global_store_dwordx4 v24, v[32:35], s[30:31]\n\t" \
  "s_add_u32 s30, s30, 0x40000\n\t" \
  "s_addc_u32 s31, s31, 0\n\t" \
  "global_store_dwordx4 v24, v[32:35], s[30:31]\n\t" \
  "s_add_u32 s30, s30, 0x40000\n\t" \
  "s_addc_u32 s31, s31, 0\n\t" \
  "global_store_dwordx4 v24, v[32:35], s[30:31]\n\t" \
  "s_add_u32 s30, s30, 0x40000\n\t" \
  "s_addc_u32 s31, s31, 0\n\t" \
  "global_store_dwordx4 v24, v[32:35], s[30:31]\n\t" \
  "s_add_u32 s30, s30, 0x40000\n\t" \
  "s_addc_u32 s31, s31, 0\n\t" \
  "global_store_dwordx4 v24, v[32:35], s[30:31]\n\t" \
  "s_add_u32 s30, s30, 0x40000\n\t" \
  "s_addc_u32 s31, s31, 0\n\t" \
  "global_store_dwordx4 v24, v[32:35], s[30:31]\n\t" \
  "s_add_u32 s30, s30, 0x40000\n\t" \
  "s_addc_u32 s31, s31, 0\n\t" \
  "global_store_dwordx4 v24, v[32:35], s[30:31]\n\t" \
  "s_add_u32 s30, s30, 0x40000\n\t" \
  "s_addc_u32 s31, s31, 0\n\t" \
  "s_branch END_%=\n\t" \
  "OP2_%=:\n\t" \
  "s_cmp_eq_u32 s22, 0\n\t" \
  "s_cbranch_scc1 QSKIP2_%=\n\t" \
  "global_load_dwordx2 v[14:15], v54, s[90:91] sc0 sc1\n\t" \
  "QSKIP2_%=:\n\t" \
  "s_cmp_eq_u32 s23, 0\n\t" \
  "s_cbranch_scc1 RSKIP2_%=\n\t" \
  "s_lshl_b32 s57, s18, 2\n\t" \
  "v_mov_b32 v12, s57\n\t" \
  "global_load_dword v13, v12, s[14:15] sc1\n\t" \
  "RSKIP2_%=:\n\t" \
  "s_branch OPB2_%=\n\t" \
  "ASLOW2_%=:\n\t" \
  "s_memrealtime s[98:99]\n\t" \
  "s_waitcnt lgkmcnt(0)\n\t" \
  "s_lshl_b32 s57, s18, 2\n\t" \
  "v_mov_b32 v12, s57\n\t" \
  "ASPIN2_%=:\n\t" \
  "s_sleep 8\n\t" \
  "global_load_dword v13, v12, s[14:15] sc1\n\t" \
  "global_load_dword v14, v51, s[94:95] sc1\n\t" \
  "s_memrealtime s[100:101]\n\t" \
  "s_waitcnt vmcnt(0) lgkmcnt(0)\n\t" \
  "v_readfirstlane_b32 s56, v13\n\t" \
  "v_readfirstlane_b32 s55, v14\n\t" \
  "s_nop 1\n\t" \
  "s_lshr_b32 s57, s56, 8\n\t" \
  "s_cmp_eq_u32 s57, s65\n\t" \
  "s_cbranch_scc1 ATAKE2_%=\n\t" \
  "s_mov_b32 s56, s25\n\t" \
  "s_sub_u32 s100, s100, s98\n\t" \
  "s_subb_u32 s101, s101, s99\n\t" \
  "s_cmp_eq_u32 s55, s65\n\t" \
  "s_cbranch_scc1 ATAKE2_%=\n\t" \
  "s_cmp_lg_u32 s101, 0\n\t" \
  "s_cbranch_scc1 ATMO2_%=\n\t" \
  "s_cmp_lt_u32 s100, 100000000\n\t" \
  "s_cbranch_scc1 ASPIN2_%=\n\t" \
  "ATMO2_%=:\n\t" \
  "s_mov_b64 exec, 1\n\t" \
  "global_store_dword v51, v55, s[94:95] sc1\n\t" \
  "s_mov_b64 exec, -1\n\t" \
  "s_waitcnt vmcnt(0)\n\t" \
  "s_branch ATAKE2_%=\n\t" \
  "DEC2_%=:\n\t" \
  "s_memrealtime s[98:99]\n\t" \
  "s_mov_b32 s20, 0\n\t" \
  "s_and_b32 s96, s48, 0x8000\n\t" \
  "s_lshl_b32 s97, s66, 2\n\t" \
  "s_add_u32 s97, s97, s96\n\t" \
  "s_waitcnt lgkmcnt(0)\n\t" \
  "DLOOP2_%=:\n\t" \
  "v_cmp_le_u32 vcc, s65, v14\n\t" \
  "v_cmp_le_u32 s[54:55], s65, v15\n\t" \
  "s_nop 0\n\t" \
  "s_and_b64 vcc, vcc, s[54:55]\n\t" \
  "s_cmp_eq_u32 vcc_lo, -1\n\t" \
  "s_cselect_b32 s56, 1, 0\n\t" \
  "s_cmp_eq_u32 vcc_hi, -1\n\t" \
  "s_cselect_b32 s57, 2, 0\n\t" \
  "s_or_b32 s56, s56, s57\n\t" \
  "s_lshl_b32 s57, s66, 1\n\t" \
  "s_lshl_b32 s56, s56, s57\n\t" \
  "s_or_b32 s56, s56, s20\n\t" \
  "v_mov_b32 v12, s56\n\t" \
  "v_mov_b32 v52, s97\n\t" \
  "v_mov_b32 v53, s96\n\t" \
  "ds_write_b32 v52, v12\n\t" \
  "s_waitcnt lgkmcnt(0)\n\t" \
  "s_barrier\n\t" \
  "ds_read_b128 v[12:15], v53\n\t" \
  "s_waitcnt lgkmcnt(0)\n\t" \
  "v_or3_b32 v12, v12, v13, v14\n\t" \
  "v_or_b32 v12, v12, v15\n\t" \
  "v_mov_b32 v52, 0\n\t" \
  "v_mov_b32 v53, 0\n\t" \
  "v_readfirstlane_b32 s56, v12\n\t" \
  "s_barrier\n\t" \
  "s_and_b32 s57, s56, s25\n\t" \
  "s_bcnt1_i32_b32 s54, s57\n\t" \
  "s_cmp_eq_u32 s19, 0\n\t" \
  "s_cselect_b32 s55, 4, 5\n\t" \
  "s_bitcmp1_b32 s56, 9\n\t" \
  "s_cselect_b32 s55, 2, s55\n\t" \
  "s_cmp_lt_u32 s61, 8\n\t" \
  "s_cselect_b32 s55, s55, 1\n\t" \
  "s_cmp_ge_u32 s54, s55\n\t" \
  "s_cbranch_scc1 DDONE2_%=\n\t" \
  "s_cmp_lg_u32 s19, 0\n\t" \
  "s_cbranch_scc1 DRET2_%=\n\t" \
  "s_mov_b32 s57, s25\n\t" \
  "s_bitcmp1_b32 s56, 8\n\t" \
  "s_cbranch_scc1 DTMO2_%=\n\t" \
  "s_sleep 8\n\t" \
  "global_load_dwordx2 v[14:15], v54, s[90:91] sc0 sc1\n\t" \
  "global_load_dword v12, v51, s[94:95] sc1\n\t" \
  "s_memrealtime s[100:101]\n\t" \
  "s_waitcnt vmcnt(0) lgkmcnt(0)\n\t" \
  "v_readfirstlane_b32 s56, v12\n\t" \
  "s_sub_u32 s100, s100, s98\n\t" \
  "s_subb_u32 s101, s101, s99\n\t" \
  "s_cmp_eq_u32 s56, s65\n\t" \
  "s_cselect_b32 s20, 0x100, s20\n\t" \
  "s_cmp_lg_u32 s101, 0\n\t" \
  "s_cselect_b32 s20, 0x100, s20\n\t" \
  "s_cmp_gt_u32 s100, 100000000\n\t" \
  "s_cselect_b32 s20, 0x100, s20\n\t" \
  "s_cmp_gt_u32 s100, 250\n\t" \
  "s_cselect_b32 s57, 0x200, 0\n\t" \
  "s_or_b32 s20, s20, s57\n\t" \
  "s_branch DLOOP2_%=\n\t" \
  "DTMO2_%=:\n\t" \
  "s_mov_b64 exec, 1\n\t" \
  "global_store_dword v51, v55, s[94:95] sc1\n\t" \
  "s_mov_b64 exec, -1\n\t" \
  "s_waitcnt vmcnt(0)\n\t" \
  "DDONE2_%=:\n\t" \
  "s_add_u32 s12, s18, 1\n\t" \
  "s_lshl_b32 s54, s65, 8\n\t" \
  "s_or_b32 s54, s54, s57\n\t" \
  "s_lshl_b32 s55, s18, 2\n\t" \
  "v_mov_b32 v12, s54\n\t" \
  "v_mov_b32 v13, s55\n\t" \
  "s_mov_b64 exec, s[26:27]\n\t" \
  "global_store_dword v13, v12, s[14:15] sc1\n\t" \
  "s_mov_b64 exec, -1\n\t" \
  "DRET2_%=:\n\t" \
  "v_mov_b32 v13, 0\n\t" \
  "s_branch DBACK2_%=\n\t" \
  "OE2_%=:\n\t" \
  "s_cmp_eq_u32 s64, 0\n\t" \
  "s_cbranch_scc0 SLOW2_%=\n\t" \
  "BACK2_%=:\n\t" \
  "s_cmp_eq_u32 s22, 0\n\t" \
  "s_cbranch_scc1 DBACK2_%=\n\t" \
  "s_barrier\n\t" \
  "s_branch DEC2_%=\n\t" \
  "DBACK2_%=:\n\t" \
  "s_cmp_eq_u32 s23, 0\n\t" \
  "s_cbranch_scc1 ADONE2_%=\n\t" \
  "v_readfirstlane_b32 s56, v13\n\t" \
  "s_nop 3\n\t" \
  "s_lshr_b32 s57, s56, 8\n\t" \
  "s_cmp_eq_u32 s57, s65\n\t" \
  "s_cbranch_scc0 ASLOW2_%=\n\t" \
  "ATAKE2_%=:\n\t" \
  "s_and_b32 s21, s56, s25\n\t" \
  "s_andn2_b32 s25, s25, s21\n\t" \
  "s_add_u32 s18, s18, 1\n\t" \
  "s_mov_b32 s16, 5\n\t" \
  "s_lshl_b32 s54, s66, 1\n\t" \
  "s_lshr_b32 s55, s21, s54\n\t" \
  "s_bitcmp1_b32 s55, 0\n\t" \
  "s_cselect_b32 s92, 0, -1\n\t" \
  "s_bitcmp1_b32 s55, 1\n\t" \
  "s_cselect_b32 s93, 0, -1\n\t" \
  "ADONE2_%=:\n\t" \
  "s_branch OEB2_%=\n\t" \
  "OP3_%=:\n\t" \
  "s_cmp_eq_u32 s22, 0\n\t" \
  "s_cbranch_scc1 QSKIP3_%=\n\t" \
  "global_load_dwordx2 v[14:15], v54, s[90:91] sc0 sc1\n\t" \
  "QSKIP3_%=:\n\t" \
  "s_cmp_eq_u32 s23, 0\n\t" \
  "s_cbranch_scc1 RSKIP3_%=\n\t" \
  "s_lshl_b32 s57, s18, 2\n\t" \
  "v_mov_b32 v12, s57\n\t" \
  "global_load_dword v13, v12, s[14:15] sc1\n\t" \
  "RSKIP3_%=:\n\t" \
  "s_branch OPB3_%=\n\t" \
  "ASLOW3_%=:\n\t" \
  "s_memrealtime s[98:99]\n\t" \
  "s_waitcnt lgkmcnt(0)\n\t" \
  "s_lshl_b32 s57, s18, 2\n\t" \
  "v_mov_b32 v12, s57\n\t" \
  "ASPIN3_%=:\n\t" \
  "s_sleep 8\n\t" \
  "global_load_dword v13, v12, s[14:15] sc1\n\t" \
  "global_load_dword v14, v51, s[94:95] sc1\n\t" \
  "s_memrealtime s[100:101]\n\t" \
  "s_waitcnt vmcnt(0) lgkmcnt(0)\n\t" \
  "v_readfirstlane_b32 s56, v13\n\t" \
  "v_readfirstlane_b32 s55, v14\n\t" \
  "s_nop 1\n\t" \
  "s_lshr_b32 s57, s56, 8\n\t" \
  "s_cmp_eq_u32 s57, s65\n\t" \
  "s_cbranch_scc1 ATAKE3_%=\n\t" \
  "s_mov_b32 s56, s25\n\t" \
  "s_sub_u32 s100, s100, s98\n\t" \
  "s_subb_u32 s101, s101, s99\n\t" \
  "s_cmp_eq_u32 s55, s65\n\t" \
  "s_cbranch_scc1 ATAKE3_%=\n\t" \
  "s_cmp_lg_u32 s101, 0\n\t" \
  "s_cbranch_scc1 ATMO3_%=\n\t" \
  "s_cmp_lt_u32 s100, 100000000\n\t" \
  "s_cbranch_scc1 ASPIN3_%=\n\t" \
  "ATMO3_%=:\n\t" \
  "s_mov_b64 exec, 1\n\t" \
  "global_store_dword v51, v55, s[94:95] sc1\n\t" \
  "s_mov_b64 exec, -1\n\t" \
  "s_waitcnt vmcnt(0)\n\t" \
  "s_branch ATAKE3_%=\n\t" \
  "DEC3_%=:\n\t" \
  "s_memrealtime s[98:99]\n\t" \
  "s_mov_b32 s20, 0\n\t" \
  "s_and_b32 s96, s48, 0x8000\n\t" \
  "s_lshl_b32 s97, s66, 2\n\t" \
  "s_add_u32 s97, s97, s96\n\t" \
  "s_waitcnt lgkmcnt(0)\n\t" \
  "DLOOP3_%=:\n\t" \
  "v_cmp_le_u32 vcc, s65, v14\n\t" \
  "v_cmp_le_u32 s[54:55], s65, v15\n\t" \
  "s_nop 0\n\t" \
  "s_and_b64 vcc, vcc, s[54:55]\n\t" \
  "s_cmp_eq_u32 vcc_lo, -1\n\t" \
  "s_cselect_b32 s56, 1, 0\n\t" \
  "s_cmp_eq_u32 vcc_hi, -1\n\t" \
  "s_cselect_b32 s57, 2, 0\n\t" \
  "s_or_b32 s56, s56, s57\n\t" \
  "s_lshl_b32 s57, s66, 1\n\t" \
  "s_lshl_b32 s56, s56, s57\n\t" \
  "s_or_b32 s56, s56, s20\n\t" \
  "v_mov_b32 v12, s56\n\t" \
  "v_mov_b32 v52, s97\n\t" \
  "v_mov_b32 v53, s96\n\t" \
  "ds_write_b32 v52, v12\n\t" \
  "s_waitcnt lgkmcnt(0)\n\t" \
  "s_barrier\n\t" \
  "ds_read_b128 v[12:15], v53\n\t" \
  "s_waitcnt lgkmcnt(0)\n\t" \
  "v_or3_b32 v12, v12, v13, v14\n\t" \
  "v_or_b32 v12, v12, v15\n\t" \
  "v_mov_b32 v52, 0\n\t" \
  "v_mov_b32 v53, 0\n\t" \
  "v_readfirstlane_b32 s56, v12\n\t" \
  "s_barrier\n\t" \
  "s_and_b32 s57, s56, s25\n\t" \
  "s_bcnt1_i32_b32 s54, s57\n\t" \
  "s_cmp_eq_u32 s19, 0\n\t" \
  "s_cselect_b32 s55, 4, 5\n\t" \
  "s_bitcmp1_b32 s56, 9\n\t" \
  "s_cselect_b32 s55, 2, s55\n\t" \
  "s_cmp_lt_u32 s61, 8\n\t" \
  "s_cselect_b32 s55, s55, 1\n\t" \
  "s_cmp_ge_u32 s54, s55\n\t" \
  "s_cbranch_scc1 DDONE3_%=\n\t" \
  "s_cmp_lg_u32 s19, 0\n\t" \
  "s_cbranch_scc1 DRET3_%=\n\t" \
  "s_mov_b32 s57, s25\n\t" \
  "s_bitcmp1_b32 s56, 8\n\t" \
  "s_cbranch_scc1 DTMO3_%=\n\t" \
  "s_sleep 8\n\t" \
  "global_load_dwordx2 v[14:15], v54, s[90:91] sc0 sc1\n\t" \
  "global_load_dword v12, v51, s[94:95] sc1\n\t" \
  "s_memrealtime s[100:101]\n\t" \
  "s_waitcnt vmcnt(0) lgkmcnt(0)\n\t" \
  "v_readfirstlane_b32 s56, v12\n\t" \
  "s_sub_u32 s100, s100, s98\n\t" \
  "s_subb_u32 s101, s101, s99\n\t" \
  "s_cmp_eq_u32 s56, s65\n\t" \
  "s_cselect_b32 s20, 0x100, s20\n\t" \
  "s_cmp_lg_u32 s101, 0\n\t" \
  "s_cselect_b32 s20, 0x100, s20\n\t" \
  "s_cmp_gt_u32 s100, 100000000\n\t" \
  "s_cselect_b32 s20, 0x100, s20\n\t" \
  "s_cmp_gt_u32 s100, 250\n\t" \
  "s_cselect_b32 s57, 0x200, 0\n\t" \
  "s_or_b32 s20, s20, s57\n\t" \
  "s_branch DLOOP3_%=\n\t" \
  "DTMO3_%=:\n\t" \
  "s_mov_b64 exec, 1\n\t" \
  "global_store_dword v51, v55, s[94:95] sc1\n\t" \
  "s_mov_b64 exec, -1\n\t" \
  "s_waitcnt vmcnt(0)\n\t" \
  "DDONE3_%=:\n\t" \
  "s_add_u32 s12, s18, 1\n\t" \
  "s_lshl_b32 s54, s65, 8\n\t" \
  "s_or_b32 s54, s54, s57\n\t" \
  "s_lshl_b32 s55, s18, 2\n\t" \
  "v_mov_b32 v12, s54\n\t" \
  "v_mov_b32 v13, s55\n\t" \
  "s_mov_b64 exec, s[26:27]\n\t" \
  "global_store_dword v13, v12, s[14:15] sc1\n\t" \
  "s_mov_b64 exec, -1\n\t" \
  "DRET3_%=:\n\t" \
  "v_mov_b32 v13, 0\n\t" \
  "s_branch DBACK3_%=\n\t" \
  "OE3_%=:\n\t" \
  "s_cmp_eq_u32 s64, 0\n\t" \
  "s_cbranch_scc0 SLOW3_%=\n\t" \
  "BACK3_%=:\n\t" \
  "s_cmp_eq_u32 s22, 0\n\t" \
  "s_cbranch_scc1 DBACK3_%=\n\t" \
  "s_barrier\n\t" \
  "s_branch DEC3_%=\n\t" \
  "DBACK3_%=:\n\t" \
  "s_cmp_eq_u32 s23, 0\n\t" \
  "s_cbranch_scc1 ADONE3_%=\n\t" \
  "v_readfirstlane_b32 s56, v13\n\t" \
  "s_nop 3\n\t" \
  "s_lshr_b32 s57, s56, 8\n\t" \
  "s_cmp_eq_u32 s57, s65\n\t" \
  "s_cbranch_scc0 ASLOW3_%=\n\t" \
  "ATAKE3_%=:\n\t" \
  "s_and_b32 s21, s56, s25\n\t" \
  "s_andn2_b32 s25, s25, s21\n\t" \
  "s_add_u32 s18, s18, 1\n\t" \
  "s_mov_b32 s16, 5\n\t" \
  "s_lshl_b32 s54, s66, 1\n\t" \
  "s_lshr_b32 s55, s21, s54\n\t" \
  "s_bitcmp1_b32 s55, 0\n\t" \
  "s_cselect_b32 s92, 0, -1\n\t" \
  "s_bitcmp1_b32 s55, 1\n\t" \
  "s_cselect_b32 s93, 0, -1\n\t" \
  "ADONE3_%=:\n\t" \
  "s_cmp_lg_u32 s50, 0\n\t" \
  "s_cselect_b64 vcc, -1, 0\n\t" \
  "s_branch OEB3_%=\n\t" \
  "OP4_%=:\n\t" \
  "s_cmp_eq_u32 s22, 0\n\t" \
  "s_cbranch_scc1 QSKIP4_%=\n\t" \
  "global_load_dwordx2 v[14:15], v54, s[90:91] sc0 sc1\n\t" \
  "QSKIP4_%=:\n\t" \
  "s_cmp_eq_u32 s23, 0\n\t" \
  "s_cbranch_scc1 RSKIP4_%=\n\t" \
  "s_lshl_b32 s57, s18, 2\n\t" \
  "v_mov_b32 v12, s57\n\t" \
  "global_load_dword v13, v12, s[14:15] sc1\n\t" \
  "RSKIP4_%=:\n\t" \
  "s_branch OPB4_%=\n\t" \
  "ASLOW4_%=:\n\t" \
  "s_memrealtime s[98:99]\n\t" \
  "s_waitcnt lgkmcnt(0)\n\t" \
  "s_lshl_b32 s57, s18, 2\n\t" \
  "v_mov_b32 v12, s57\n\t" \
  "ASPIN4_%=:\n\t" \
  "s_sleep 8\n\t" \
  "global_load_dword v13, v12, s[14:15] sc1\n\t" \
  "global_load_dword v14, v51, s[94:95] sc1\n\t" \
  "s_memrealtime s[100:101]\n\t" \
  "s_waitcnt vmcnt(0) lgkmcnt(0)\n\t" \
  "v_readfirstlane_b32 s56, v13\n\t" \
  "v_readfirstlane_b32 s55, v14\n\t" \
  "s_nop 1\n\t" \
  "s_lshr_b32 s57, s56, 8\n\t" \
  "s_cmp_eq_u32 s57, s65\n\t" \
  "s_cbranch_scc1 ATAKE4_%=\n\t" \
  "s_mov_b32 s56, s25\n\t" \
  "s_sub_u32 s100, s100, s98\n\t" \
  "s_subb_u32 s101, s101, s99\n\t" \
  "s_cmp_eq_u32 s55, s65\n\t" \
  "s_cbranch_scc1 ATAKE4_%=\n\t" \
  "s_cmp_lg_u32 s101, 0\n\t" \
  "s_cbranch_scc1 ATMO4_%=\n\t" \
  "s_cmp_lt_u32 s100, 100000000\n\t" \
  "s_cbranch_scc1 ASPIN4_%=\n\t" \
  "ATMO4_%=:\n\t" \
  "s_mov_b64 exec, 1\n\t" \
  "global_store_dword v51, v55, s[94:95] sc1\n\t" \
  "s_mov_b64 exec, -1\n\t" \
  "s_waitcnt vmcnt(0)\n\t" \
  "s_branch ATAKE4_%=\n\t" \
  "DEC4_%=:\n\t" \
  "s_memrealtime s[98:99]\n\t" \
  "s_mov_b32 s20, 0\n\t" \
  "s_and_b32 s96, s48, 0x8000\n\t" \
  "s_lshl_b32 s97, s66, 2\n\t" \
  "s_add_u32 s97, s97, s96\n\t" \
  "s_waitcnt lgkmcnt(0)\n\t" \
  "DLOOP4_%=:\n\t" \
  "v_cmp_le_u32 vcc, s65, v14\n\t" \
  "v_cmp_le_u32 s[54:55], s65, v15\n\t" \
  "s_nop 0\n\t" \
  "s_and_b64 vcc, vcc, s[54:55]\n\t" \
  "s_cmp_eq_u32 vcc_lo, -1\n\t" \
  "s_cselect_b32 s56, 1, 0\n\t" \
  "s_cmp_eq_u32 vcc_hi, -1\n\t" \
  "s_cselect_b32 s57, 2, 0\n\t" \
  "s_or_b32 s56, s56, s57\n\t" \
  "s_lshl_b32 s57, s66, 1\n\t" \
  "s_lshl_b32 s56, s56, s57\n\t" \
  "s_or_b32 s56, s56, s20\n\t" \
  "v_mov_b32 v12, s56\n\t" \
  "v_mov_b32 v52, s97\n\t" \
  "v_mov_b32 v53, s96\n\t" \
  "ds_write_b32 v52, v12\n\t" \
  "s_waitcnt lgkmcnt(0)\n\t" \
  "s_barrier\n\t" \
  "ds_read_b128 v[12:15], v53\n\t" \
  "s_waitcnt lgkmcnt(0)\n\t" \
  "v_or3_b32 v12, v12, v13, v14\n\t" \
  "v_or_b32 v12, v12, v15\n\t" \
  "v_mov_b32 v52, 0\n\t" \
  "v_mov_b32 v53, 0\n\t" \
  "v_readfirstlane_b32 s56, v12\n\t" \
  "s_barrier\n\t" \
  "s_and_b32 s57, s56, s25\n\t" \
  "s_bcnt1_i32_b32 s54, s57\n\t" \
  "s_cmp_eq_u32 s19, 0\n\t" \
  "s_cselect_b32 s55, 4, 5\n\t" \
  "s_bitcmp1_b32 s56, 9\n\t" \
  "s_cselect_b32 s55, 2, s55\n\t" \
  "s_cmp_lt_u32 s61, 8\n\t" \
  "s_cselect_b32 s55, s55, 1\n\t" \
  "s_cmp_ge_u32 s54, s55\n\t" \
  "s_cbranch_scc1 DDONE4_%=\n\t" \
  "s_cmp_lg_u32 s19, 0\n\t" \
  "s_cbranch_scc1 DRET4_%=\n\t" \
  "s_mov_b32 s57, s25\n\t" \
  "s_bitcmp1_b32 s56, 8\n\t" \
  "s_cbranch_scc1 DTMO4_%=\n\t" \
  "s_sleep 8\n\t" \
  "global_load_dwordx2 v[14:15], v54, s[90:91] sc0 sc1\n\t" \
  "global_load_dword v12, v51, s[94:95] sc1\n\t" \
  "s_memrealtime s[100:101]\n\t" \
  "s_waitcnt vmcnt(0) lgkmcnt(0)\n\t" \
  "v_readfirstlane_b32 s56, v12\n\t" \
  "s_sub_u32 s100, s100, s98\n\t" \
  "s_subb_u32 s101, s101, s99\n\t" \
  "s_cmp_eq_u32 s56, s65\n\t" \
  "s_cselect_b32 s20, 0x100, s20\n\t" \
  "s_cmp_lg_u32 s101, 0\n\t" \
  "s_cselect_b32 s20, 0x100, s20\n\t" \
  "s_cmp_gt_u32 s100, 100000000\n\t" \
  "s_cselect_b32 s20, 0x100, s20\n\t" \
  "s_cmp_gt_u32 s100, 250\n\t" \
  "s_cselect_b32 s57, 0x200, 0\n\t" \
  "s_or_b32 s20, s20, s57\n\t" \
  "s_branch DLOOP4_%=\n\t" \
  "DTMO4_%=:\n\t" \
  "s_mov_b64 exec, 1\n\t" \
  "global_store_dword v51, v55, s[94:95] sc1\n\t" \
  "s_mov_b64 exec, -1\n\t" \
  "s_waitcnt vmcnt(0)\n\t" \
  "DDONE4_%=:\n\t" \
  "s_add_u32 s12, s18, 1\n\t" \
  "s_lshl_b32 s54, s65, 8\n\t" \
  "s_or_b32 s54, s54, s57\n\t" \
  "s_lshl_b32 s55, s18, 2\n\t" \
  "v_mov_b32 v12, s54\n\t" \
  "v_mov_b32 v13, s55\n\t" \
  "s_mov_b64 exec, s[26:27]\n\t" \
  "global_store_dword v13, v12, s[14:15] sc1\n\t" \
  "s_mov_b64 exec, -1\n\t" \
  "DRET4_%=:\n\t" \
  "v_mov_b32 v13, 0\n\t" \
  "s_branch DBACK4_%=\n\t" \
  "OE4_%=:\n\t" \
  "s_cmp_eq_u32 s64, 0\n\t" \
  "s_cbranch_scc0 SLOW4_%=\n\t" \
  "BACK4_%=:\n\t" \
  "s_cmp_eq_u32 s22, 0\n\t" \
  "s_cbranch_scc1 DBACK4_%=\n\t" \
  "s_barrier\n\t" \
  "s_branch DEC4_%=\n\t" \
  "DBACK4_%=:\n\t" \
  "s_cmp_eq_u32 s23, 0\n\t" \
  "s_cbranch_scc1 ADONE4_%=\n\t" \
  "v_readfirstlane_b32 s56, v13\n\t" \
  "s_nop 3\n\t" \
  "s_lshr_b32 s57, s56, 8\n\t" \
  "s_cmp_eq_u32 s57, s65\n\t" \
  "s_cbranch_scc0 ASLOW4_%=\n\t" \
  "ATAKE4_%=:\n\t" \
  "s_and_b32 s21, s56, s25\n\t" \
  "s_andn2_b32 s25, s25, s21\n\t" \
  "s_add_u32 s18, s18, 1\n\t" \
  "s_mov_b32 s16, 5\n\t" \
  "s_lshl_b32 s54, s66, 1\n\t" \
  "s_lshr_b32 s55, s21, s54\n\t" \
  "s_bitcmp1_b32 s55, 0\n\t" \
  "s_cselect_b32 s92, 0, -1\n\t" \
  "s_bitcmp1_b32 s55, 1\n\t" \
  "s_cselect_b32 s93, 0, -1\n\t" \
  "ADONE4_%=:\n\t" \
  "s_branch OEB4_%=\n\t" \
  "OP5_%=:\n\t" \
  "s_cmp_eq_u32 s22, 0\n\t" \
  "s_cbranch_scc1 QSKIP5_%=\n\t" \
  "global_load_dwordx2 v[14:15], v54, s[90:91] sc0 sc1\n\t" \
  "QSKIP5_%=:\n\t" \
  "s_cmp_eq_u32 s23, 0\n\t" \
  "s_cbranch_scc1 RSKIP5_%=\n\t" \
  "s_lshl_b32 s57, s18, 2\n\t" \
  "v_mov_b32 v12, s57\n\t" \
  "global_load_dword v13, v12, s[14:15] sc1\n\t" \
  "RSKIP5_%=:\n\t" \
  "s_branch OPB5_%=\n\t" \
  "ASLOW5_%=:\n\t" \
  "s_memrealtime s[98:99]\n\t" \
  "s_waitcnt lgkmcnt(0)\n\t" \
  "s_lshl_b32 s57, s18, 2\n\t" \
  "v_mov_b32 v12, s57\n\t" \
  "ASPIN5_%=:\n\t" \
  "s_sleep 8\n\t" \
  "global_load_dword v13, v12, s[14:15] sc1\n\t" \
  "global_load_dword v14, v51, s[94:95] sc1\n\t" \
  "s_memrealtime s[100:101]\n\t" \
  "s_waitcnt vmcnt(0) lgkmcnt(0)\n\t" \
  "v_readfirstlane_b32 s56, v13\n\t" \
  "v_readfirstlane_b32 s55, v14\n\t" \
  "s_nop 1\n\t" \
  "s_lshr_b32 s57, s56, 8\n\t" \
  "s_cmp_eq_u32 s57, s65\n\t" \
  "s_cbranch_scc1 ATAKE5_%=\n\t" \
  "s_mov_b32 s56, s25\n\t" \
  "s_sub_u32 s100, s100, s98\n\t" \
  "s_subb_u32 s101, s101, s99\n\t" \
  "s_cmp_eq_u32 s55, s65\n\t" \
  "s_cbranch_scc1 ATAKE5_%=\n\t" \
  "s_cmp_lg_u32 s101, 0\n\t" \
  "s_cbranch_scc1 ATMO5_%=\n\t" \
  "s_cmp_lt_u32 s100, 100000000\n\t" \
  "s_cbranch_scc1 ASPIN5_%=\n\t" \
  "ATMO5_%=:\n\t" \
  "s_mov_b64 exec, 1\n\t" \
  "global_store_dword v51, v55, s[94:95] sc1\n\t" \
  "s_mov_b64 exec, -1\n\t" \
  "s_waitcnt vmcnt(0)\n\t" \
  "s_branch ATAKE5_%=\n\t" \
  "DEC5_%=:\n\t" \
  "s_memrealtime s[98:99]\n\t" \
  "s_mov_b32 s20, 0\n\t" \
  "s_and_b32 s96, s48, 0x8000\n\t" \
  "s_lshl_b32 s97, s66, 2\n\t" \
  "s_add_u32 s97, s97, s96\n\t" \
  "s_waitcnt lgkmcnt(0)\n\t" \
  "DLOOP5_%=:\n\t" \
  "v_cmp_le_u32 vcc, s65, v14\n\t" \
  "v_cmp_le_u32 s[54:55], s65, v15\n\t" \
  "s_nop 0\n\t" \
  "s_and_b64 vcc, vcc, s[54:55]\n\t" \
  "s_cmp_eq_u32 vcc_lo, -1\n\t" \
  "s_cselect_b32 s56, 1, 0\n\t" \
  "s_cmp_eq_u32 vcc_hi, -1\n\t" \
  "s_cselect_b32 s57, 2, 0\n\t" \
  "s_or_b32 s56, s56, s57\n\t" \
  "s_lshl_b32 s57, s66, 1\n\t" \
  "s_lshl_b32 s56, s56, s57\n\t" \
  "s_or_b32 s56, s56, s20\n\t" \
  "v_mov_b32 v12, s56\n\t" \
  "v_mov_b32 v52, s97\n\t" \
  "v_mov_b32 v53, s96\n\t" \
  "ds_write_b32 v52, v12\n\t" \
  "s_waitcnt lgkmcnt(0)\n\t" \
  "s_barrier\n\t" \
  "ds_read_b128 v[12:15], v53\n\t" \
  "s_waitcnt lgkmcnt(0)\n\t" \
  "v_or3_b32 v12, v12, v13, v14\n\t" \
  "v_or_b32 v12, v12, v15\n\t" \
  "v_mov_b32 v52, 0\n\t" \
  "v_mov_b32 v53, 0\n\t" \
  "v_readfirstlane_b32 s56, v12\n\t" \
  "s_barrier\n\t" \
  "s_and_b32 s57, s56, s25\n\t" \
  "s_bcnt1_i32_b32 s54, s57\n\t" \
  "s_cmp_eq_u32 s19, 0\n\t" \
  "s_cselect_b32 s55, 4, 5\n\t" \
  "s_bitcmp1_b32 s56, 9\n\t" \
  "s_cselect_b32 s55, 2, s55\n\t" \
  "s_cmp_lt_u32 s61, 8\n\t" \
  "s_cselect_b32 s55, s55, 1\n\t" \
  "s_cmp_ge_u32 s54, s55\n\t" \
  "s_cbranch_scc1 DDONE5_%=\n\t" \
  "s_cmp_lg_u32 s19, 0\n\t" \
  "s_cbranch_scc1 DRET5_%=\n\t" \
  "s_mov_b32 s57, s25\n\t" \
  "s_bitcmp1_b32 s56, 8\n\t" \
  "s_cbranch_scc1 DTMO5_%=\n\t" \
  "s_sleep 8\n\t" \
  "global_load_dwordx2 v[14:15], v54, s[90:91] sc0 sc1\n\t" \
  "global_load_dword v12, v51, s[94:95] sc1\n\t" \
  "s_memrealtime s[100:101]\n\t" \
  "s_waitcnt vmcnt(0) lgkmcnt(0)\n\t" \
  "v_readfirstlane_b32 s56, v12\n\t" \
  "s_sub_u32 s100, s100, s98\n\t" \
  "s_subb_u32 s101, s101, s99\n\t" \
  "s_cmp_eq_u32 s56, s65\n\t" \
  "s_cselect_b32 s20, 0x100, s20\n\t" \
  "s_cmp_lg_u32 s101, 0\n\t" \
  "s_cselect_b32 s20, 0x100, s20\n\t" \
  "s_cmp_gt_u32 s100, 100000000\n\t" \
  "s_cselect_b32 s20, 0x100, s20\n\t" \
  "s_cmp_gt_u32 s100, 250\n\t" \
  "s_cselect_b32 s57, 0x200, 0\n\t" \
  "s_or_b32 s20, s20, s57\n\t" \
  "s_branch DLOOP5_%=\n\t" \
  "DTMO5_%=:\n\t" \
  "s_mov_b64 exec, 1\n\t" \
  "global_store_dword v51, v55, s[94:95] sc1\n\t" \
  "s_mov_b64 exec, -1\n\t" \
  "s_waitcnt vmcnt(0)\n\t" \
  "DDONE5_%=:\n\t" \
  "s_add_u32 s12, s18, 1\n\t" \
  "s_lshl_b32 s54, s65, 8\n\t" \
  "s_or_b32 s54, s54, s57\n\t" \
  "s_lshl_b32 s55, s18, 2\n\t" \
  "v_mov_b32 v12, s54\n\t" \
  "v_mov_b32 v13, s55\n\t" \
  "s_mov_b64 exec, s[26:27]\n\t" \
  "global_store_dword v13, v12, s[14:15] sc1\n\t" \
  "s_mov_b64 exec, -1\n\t" \
  "DRET5_%=:\n\t" \
  "v_mov_b32 v13, 0\n\t" \
  "s_branch DBACK5_%=\n\t" \
  "OE5_%=:\n\t" \
  "s_cmp_eq_u32 s64, 0\n\t" \
  "s_cbranch_scc0 SLOW5_%=\n\t" \
  "BACK5_%=:\n\t" \
  "s_cmp_eq_u32 s22, 0\n\t" \
  "s_cbranch_scc1 DBACK5_%=\n\t" \
  "s_barrier\n\t" \
  "s_branch DEC5_%=\n\t" \
  "DBACK5_%=:\n\t" \
  "s_cmp_eq_u32 s23, 0\n\t" \
  "s_cbranch_scc1 ADONE5_%=\n\t" \
  "v_readfirstlane_b32 s56, v13\n\t" \
  "s_nop 3\n\t" \
  "s_lshr_b32 s57, s56, 8\n\t" \
  "s_cmp_eq_u32 s57, s65\n\t" \
  "s_cbranch_scc0 ASLOW5_%=\n\t" \
  "ATAKE5_%=:\n\t" \
  "s_and_b32 s21, s56, s25\n\t" \
  "s_andn2_b32 s25, s25, s21\n\t" \
  "s_add_u32 s18, s18, 1\n\t" \
  "s_mov_b32 s16, 5\n\t" \
  "s_lshl_b32 s54, s66, 1\n\t" \
  "s_lshr_b32 s55, s21, s54\n\t" \
  "s_bitcmp1_b32 s55, 0\n\t" \
  "s_cselect_b32 s92, 0, -1\n\t" \
  "s_bitcmp1_b32 s55, 1\n\t" \
  "s_cselect_b32 s93, 0, -1\n\t" \
  "ADONE5_%=:\n\t" \
  "s_cmp_lg_u32 s50, 0\n\t" \
  "s_cselect_b64 vcc, -1, 0\n\t" \
  "s_branch OEB5_%=\n\t" \
  "SLOW2_%=:\n\t" \
  "s_memrealtime s[98:99]\n\t" \
  "s_lshl_b32 s57, s53, 11\n\t" \
  "s_add_u32 s96, s90, s57\n\t" \
  "s_addc_u32 s97, s91, 0\n\t" \
  "s_waitcnt lgkmcnt(0)\n\t" \
  "SPIN2_%=:\n\t" \
  "global_load_dwordx2 v[12:13], v54, s[96:97] sc0 sc1\n\t" \
  "s_waitcnt vmcnt(0)\n\t" \
  "v_cmp_le_u32 vcc, s65, v12\n\t" \
  "v_cmp_le_u32 s[54:55], s65, v13\n\t" \
  "s_nop 0\n\t" \
  "s_and_b64 vcc, vcc, s[54:55]\n\t" \
  "s_or_b64 vcc, vcc, s[92:93]\n\t" \
  "s_cmp_eq_u64 vcc, -1\n\t" \
  "s_cbranch_scc1 SDONE2_%=\n\t" \
  "global_load_dword v12, v51, s[94:95] sc1\n\t" \
  "s_memrealtime s[100:101]\n\t" \
  "s_waitcnt vmcnt(0) lgkmcnt(0)\n\t" \
  "v_readfirstlane_b32 s56, v12\n\t" \
  "s_sub_u32 s100, s100, s98\n\t" \
  "s_subb_u32 s101, s101, s99\n\t" \
  "s_cmp_eq_u32 s56, s65\n\t" \
  "s_cbranch_scc1 SDONE2_%=\n\t" \
  "s_cmp_lg_u32 s101, 0\n\t" \
  "s_cbranch_scc1 TMO2_%=\n\t" \
  "s_sleep 8\n\t" \
  "s_cmp_lt_u32 s100, 100000000\n\t" \
  "s_cbranch_scc1 SPIN2_%=\n\t" \
  "TMO2_%=:\n\t" \
  "s_mov_b64 exec, 1\n\t" \
  "global_store_dword v51, v55, s[94:95] sc1\n\t" \
  "s_mov_b64 exec, -1\n\t" \
  "s_waitcnt vmcnt(0)\n\t" \
  "SDONE2_%=:\n\t" \
  "v_mov_b32 v13, 0\n\t" \
  "s_branch BACK2_%=\n\t" \
  "SLOW3_%=:\n\t" \
  "s_memrealtime s[98:99]\n\t" \
  "s_lshl_b32 s57, s53, 11\n\t" \
  "s_add_u32 s96, s90, s57\n\t" \
  "s_addc_u32 s97, s91, 0\n\t" \
  "s_waitcnt lgkmcnt(0)\n\t" \
  "SPIN3_%=:\n\t" \
  "global_load_dwordx2 v[12:13], v54, s[96:97] sc0 sc1\n\t" \
  "s_waitcnt vmcnt(0)\n\t" \
  "v_cmp_le_u32 vcc, s65, v12\n\t" \
  "v_cmp_le_u32 s[54:55], s65, v13\n\t" \
  "s_nop 0\n\t" \
  "s_and_b64 vcc, vcc, s[54:55]\n\t" \
  "s_or_b64 vcc, vcc, s[92:93]\n\t" \
  "s_cmp_eq_u64 vcc, -1\n\t" \
  "s_cbranch_scc1 SDONE3_%=\n\t" \
  "global_load_dword v12, v51, s[94:95] sc1\n\t" \
  "s_memrealtime s[100:101]\n\t" \
  "s_waitcnt vmcnt(0) lgkmcnt(0)\n\t" \
  "v_readfirstlane_b32 s56, v12\n\t" \
  "s_sub_u32 s100, s100, s98\n\t" \
  "s_subb_u32 s101, s101, s99\n\t" \
  "s_cmp_eq_u32 s56, s65\n\t" \
  "s_cbranch_scc1 SDONE3_%=\n\t" \
  "s_cmp_lg_u32 s101, 0\n\t" \
  "s_cbranch_scc1 TMO3_%=\n\t" \
  "s_sleep 8\n\t" \
  "s_cmp_lt_u32 s100, 100000000\n\t" \
  "s_cbranch_scc1 SPIN3_%=\n\t" \
  "TMO3_%=:\n\t" \
  "s_mov_b64 exec, 1\n\t" \
  "global_store_dword v51, v55, s[94:95] sc1\n\t" \
  "s_mov_b64 exec, -1\n\t" \
  "s_waitcnt vmcnt(0)\n\t" \
  "SDONE3_%=:\n\t" \
  "v_mov_b32 v13, 0\n\t" \
  "s_branch BACK3_%=\n\t" \
  "SLOW4_%=:\n\t" \
  "s_memrealtime s[98:99]\n\t" \
  "s_lshl_b32 s57, s53, 11\n\t" \
  "s_add_u32 s96, s90, s57\n\t" \
  "s_addc_u32 s97, s91, 0\n\t" \
  "s_waitcnt lgkmcnt(0)\n\t" \
  "SPIN4_%=:\n\t" \
  "global_load_dwordx2 v[12:13], v54, s[96:97] sc0 sc1\n\t" \
  "s_waitcnt vmcnt(0)\n\t" \
  "v_cmp_le_u32 vcc, s65, v12\n\t" \
  "v_cmp_le_u32 s[54:55], s65, v13\n\t" \
  "s_nop 0\n\t" \
  "s_and_b64 vcc, vcc, s[54:55]\n\t" \
  "s_or_b64 vcc, vcc, s[92:93]\n\t" \
  "s_cmp_eq_u64 vcc, -1\n\t" \
  "s_cbranch_scc1 SDONE4_%=\n\t" \
  "global_load_dword v12, v51, s[94:95] sc1\n\t" \
  "s_memrealtime s[100:101]\n\t" \
  "s_waitcnt vmcnt(0) lgkmcnt(0)\n\t" \
  "v_readfirstlane_b32 s56, v12\n\t" \
  "s_sub_u32 s100, s100, s98\n\t" \
  "s_subb_u32 s101, s101, s99\n\t" \
  "s_cmp_eq_u32 s56, s65\n\t" \
  "s_cbranch_scc1 SDONE4_%=\n\t" \
  "s_cmp_lg_u32 s101, 0\n\t" \
  "s_cbranch_scc1 TMO4_%=\n\t" \
  "s_sleep 8\n\t" \
  "s_cmp_lt_u32 s100, 100000000\n\t" \
  "s_cbranch_scc1 SPIN4_%=\n\t" \
  "TMO4_%=:\n\t" \
  "s_mov_b64 exec, 1\n\t" \
  "global_store_dword v51, v55, s[94:95] sc1\n\t" \
  "s_mov_b64 exec, -1\n\t" \
  "s_waitcnt vmcnt(0)\n\t" \
  "SDONE4_%=:\n\t" \
  "v_mov_b32 v13, 0\n\t" \
  "s_branch BACK4_%=\n\t" \
  "SLOW5_%=:\n\t" \
  "s_memrealtime s[98:99]\n\t" \
  "s_lshl_b32 s57, s53, 11\n\t" \
  "s_add_u32 s96, s90, s57\n\t" \
  "s_addc_u32 s97, s91, 0\n\t" \
  "s_waitcnt lgkmcnt(0)\n\t" \
  "SPIN5_%=:\n\t" \
  "global_load_dwordx2 v[12:13], v54, s[96:97] sc0 sc1\n\t" \
  "s_waitcnt vmcnt(0)\n\t" \
  "v_cmp_le_u32 vcc, s65, v12\n\t" \
  "v_cmp_le_u32 s[54:55], s65, v13\n\t" \
  "s_nop 0\n\t" \
  "s_and_b64 vcc, vcc, s[54:55]\n\t" \
  "s_or_b64 vcc, vcc, s[92:93]\n\t" \
  "s_cmp_eq_u64 vcc, -1\n\t" \
  "s_cbranch_scc1 SDONE5_%=\n\t" \
  "global_load_dword v12, v51, s[94:95] sc1\n\t" \
  "s_memrealtime s[100:101]\n\t" \
  "s_waitcnt vmcnt(0) lgkmcnt(0)\n\t" \
  "v_readfirstlane_b32 s56, v12\n\t" \
  "s_sub_u32 s100, s100, s98\n\t" \
  "s_subb_u32 s101, s101, s99\n\t" \
  "s_cmp_eq_u32 s56, s65\n\t" \
  "s_cbranch_scc1 SDONE5_%=\n\t" \
  "s_cmp_lg_u32 s101, 0\n\t" \
  "s_cbranch_scc1 TMO5_%=\n\t" \
  "s_sleep 8\n\t" \
  "s_cmp_lt_u32 s100, 100000000\n\t" \
  "s_cbranch_scc1 SPIN5_%=\n\t" \
  "TMO5_%=:\n\t" \
  "s_mov_b64 exec, 1\n\t" \
  "global_store_dword v51, v55, s[94:95] sc1\n\t" \
  "s_mov_b64 exec, -1\n\t" \
  "s_waitcnt vmcnt(0)\n\t" \
  "SDONE5_%=:\n\t" \
  "v_mov_b32 v13, 0\n\t" \
  "s_branch BACK5_%=\n\t" \
  "END_%=:\n\t" \
  ""
#define GEMM_ASM_CLOBBERS "v10", "v11", "v12", "v13", "v14", "v15", "v16", "v17", "v18", "v19", "v20", "v21", "v22", "v23", "v24", "v25", "v26", "v27", "v28", "v29", "v30", "v31", "v32", "v33", "v34", "v35", "v36", "v37", "v38", "v39", "v40", "v41", "v42", "v43", "v44", "v45", "v46", "v47", "v48", "v49", "v50", "v51", "v52", "v53", "v54", "v55", "v56", "v57", "v58", "v59", "v60", "v61", "v62", "v63", "v64", "v65", "v66", "v67", "v68", "v69", "v70", "v71", "v72", "v73", "v74", "v75", "v76", "v77", "v78", "v79", "v80", "v81", "v82", "v83", "v84", "v85", "v86", "v87", "v88", "v89", "v90", "v91", "v92", "v93", "v94", "v95", "v96", "v97", "v98", "v99", "v100", "v101", "v102", "v103", "v104", "v105", "v106", "v107", "v108", "v109", "v110", "v111", "v112", "v113", "v114", "v115", "v116", "v117", "v118", "v119", "v120", "v121", "v122", "v123", "v124", "v125", "v126", "v127", "v128", "v129", "v130", "v131", "v132", "v133", "v134", "v135", "v136", "v137", "v138", "v139", "v140", "v141", "v142", "v143", "v144", "v145", "v146", "v147", "v148", "v149", "v150", "v151", "v152", "v153", "v154", "v155", "v156", "v157", "v158", "v159", "v160", "v161", "v162", "v163", "v164", "v165", "v166", "v167", "v168", "v169", "v170", "v171", "v172", "v173", "v174", "v175", "v176", "v177", "v178", "v179", "v180", "v181", "v182", "v183", "v184", "v185", "v186", "v187", "v188", "v189", "v190", "v191", "v192", "v193", "v194", "v195", "v196", "v197", "v198", "v199", "v200", "v201", "v202", "v203", "v204", "v205", "v206", "v207", "v208", "v209", "v210", "v211", "v212", "v213", "v214", "v215", "v216", "v217", "v218", "v219", "v220", "v221", "v222", "v223", "v224", "v225", "v226", "v227", "v228", "v229", "v230", "v231", "v232", "v233", "v234", "v235", "v236", "v237", "v238", "v239", "v240", "v241", "v242", "v243", "v244", "v245", "v246", "v247", "v248", "v249", "v250", "v251", "v252", "v253", "v254", "v255", "s12", "s13", "s14", "s15", "s16", "s17", "s18", "s19", "s20", "s21", "s22", "s23", "s24", "s25", "s26", "s27", "s28", "s29", "s30", "s31", "s32", "s33", "s34", "s35", "s36", "s37", "s38", "s39", "s40", "s41", "s42", "s43", "s44", "s45", "s46", "s47", "s48", "s49", "s50", "s51", "s52", "s53", "s54", "s55", "s56", "s57", "s58", "s59", "s60", "s61", "s62", "s63", "s64", "s65", "s66", "s67", "s68", "s69", "s70", "s71", "s72", "s73", "s74", "s75", "s76", "s77", "s78", "s79", "s80", "s81", "s82", "s83", "s84", "s85", "s86", "s87", "s88", "s89", "s90", "s91", "s92", "s93", "s94", "s95", "s96", "s97", "s98", "s99", "s100", "s101", "a0", "a1", "a2", "a3", "a4", "a5", "a6", "a7", "a8", "a9", "a10", "a11", "a12", "a13", "a14", "a15", "a16", "a17", "a18", "a19", "a20", "a21", "a22", "a23", "a24", "a25", "a26", "a27", "a28", "a29", "a30", "a31", "a32", "a33", "a34", "a35", "a36", "a37", "a38", "a39", "a40", "a41", "a42", "a43", "a44", "a45", "a46", "a47", "a48", "a49", "a50", "a51", "a52", "a53", "a54", "a55", "a56", "a57", "a58", "a59", "a60", "a61", "a62", "a63", "a64", "a65", "a66", "a67", "a68", "a69", "a70", "a71", "a72", "a73", "a74", "a75", "a76", "a77", "a78", "a79", "a80", "a81", "a82", "a83", "a84", "a85", "a86", "a87", "a88", "a89", "a90", "a91", "a92", "a93", "a94", "a95", "a96", "a97", "a98", "a99", "a100", "a101", "a102", "a103", "a104", "a105", "a106", "a107", "a108", "a109", "a110", "a111", "a112", "a113", "a114", "a115", "a116", "a117", "a118", "a119", "a120", "a121", "a122", "a123", "a124", "a125", "a126", "a127", "a128", "a129", "a130", "a131", "a132", "a133", "a134", "a135", "a136", "a137", "a138", "a139", "a140", "a141", "a142", "a143", "a144", "a145", "a146", "a147", "a148", "a149", "a150", "a151", "a152", "a153", "a154", "a155", "a156", "a157", "a158", "a159", "a160", "a161", "a162", "a163", "a164", "a165", "a166", "a167", "a168", "a169", "a170", "a171", "a172", "a173", "a174", "a175", "a176", "a177", "a178", "a179", "a180", "a181", "a182", "a183", "a184", "a185", "a186", "a187", "a188", "a189", "a190", "a191", "a192", "a193", "a194", "a195", "a196", "a197", "a198", "a199", "a200", "a201", "a202", "a203", "a204", "a205", "a206", "a207", "a208", "a209", "a210", "a211", "a212", "a213", "a214", "a215", "a216", "a217", "a218", "a219", "a220", "a221", "a222", "a223", "a224", "a225", "a226", "a227", "a228", "a229", "a230", "a231", "a232", "a233", "a234", "a235", "a236", "a237", "a238", "a239", "a240", "a241", "a242", "a243", "a244", "a245", "a246", "a247", "a248", "a249", "a250", "a251", "a252", "a253", "a254", "a255", "m0", "scc", "vcc", "memory"


typedef unsigned short u16;
constexpr int NRANK = 8;
constexpr int LDS_BYTES = 163840;
constexpr unsigned WS_CNT = 0;
constexpr unsigned WS_TMO = 32768;
constexpr unsigned WS_DEC = 36864;
constexpr unsigned WS_FLAGS = 65536;
constexpr unsigned WS_DATA = 1u << 20;
constexpr size_t WS_TOTAL = (size_t)WS_DATA + 2u * 8u * 512u * 512u * 2u;

struct Params {
  const u16* x; const u16* w; u16* out; char* myws;
  char* ws[NRANK];
  int rank; int pad;
};

__global__ __launch_bounds__(256, 1) void a2a_gemm_k(Params p) {
  extern __shared__ __attribute__((aligned(16))) char lds[];
  const unsigned tid = threadIdx.x, l = tid & 63u;
  const unsigned w = __builtin_amdgcn_readfirstlane(tid >> 6);
  const unsigned wm = w & 1u, wn = w >> 1;
  const unsigned ldsb = (unsigned)(size_t)lds;
  const unsigned voffA = (8u * w + (l >> 3)) * 1024u + (((l & 7u) ^ ((l >> 4) | ((w & 1u) << 2))) * 16u);
  const unsigned voffB = (((w >> 1) * 8u + (w & 1u) * 2u + (l >> 5)) * 65536u) + (((l & 31u) ^ (((l >> 5) | (w << 1)) << 1)) * 16u);
  const unsigned r = l & 15u, g = l >> 4, q = (l >> 2) & 3u, pp = l & 3u;
  const unsigned rdA0 = ldsb + (((r >> 3) * 8u + 4u * wm) * 1024u) + (r & 7u) * 128u + ((g ^ ((r >> 1) & 7u)) * 16u);
  const unsigned f16 = ((q | ((g & 1u) << 2)) << 1) * 16u;
  const unsigned row0 = ((((g & 1u) << 1) | (q >> 1)) * 4u + ((g >> 1) << 1)) * 2u + (q & 1u);
  const unsigned rdB = ldsb + 65536u + row0 * 512u + ((pp >> 1) + 16u * wn) * 16u + (pp & 1u) * 8u;
  const unsigned ooff = (wm * 128u + (l >> 4)) * 65536u + wn * 256u + (l & 15u) * 16u;
  const unsigned long long karg = (unsigned long long)__builtin_amdgcn_kernarg_segment_ptr();
  asm volatile(GEMM_ASM_TEXT
    :
    : [voffA] "v"(voffA), [voffB] "v"(voffB), [rdA0] "v"(rdA0), [rdB] "v"(rdB), [f16] "v"(f16), [ooff] "v"(ooff),
      [karg] "s"(karg), [bid] "s"(blockIdx.x), [wave] "s"(w),
      [wsdata] "i"(WS_DATA), [wsflags] "i"(WS_FLAGS), [wstmo] "i"(WS_TMO), [wsdec] "i"(WS_DEC)
    : GEMM_ASM_CLOBBERS);
}

extern "C" void kernel_launch(int rank, int nranks, ncclComm_t comm, hipStream_t stream,
                              void* const* d_in, const int* in_sizes, int n_in,
                              void* d_out, int out_size, const int* in_dtypes, int out_dtype,
                              void* const* d_out_all, void* const* d_ws_all, size_t ws_bytes) {
  if (nranks != NRANK || n_in != 2 || ws_bytes < WS_TOTAL || (unsigned)rank >= (unsigned)NRANK ||
      in_sizes[0] != 4096 * 512 || in_sizes[1] != 4096 * 32768 || out_size != 512 * 32768) { fprintf(stderr, "unexpected configuration\n"); return; }
  static bool init = false;
  if (!init) { HIP_CHECK(hipFuncSetAttribute((const void*)a2a_gemm_k, hipFuncAttributeMaxDynamicSharedMemorySize, LDS_BYTES)); init = true; }
  Params p;
  p.x = (const u16*)d_in[0]; p.w = (const u16*)d_in[1]; p.out = (u16*)d_out;
  for (int i = 0; i < NRANK; ++i) p.ws[i] = (char*)d_ws_all[i];
  p.myws = p.ws[rank]; p.rank = rank; p.pad = 0;
  hipLaunchKernelGGL(a2a_gemm_k, dim3(256), dim3(256), LDS_BYTES, stream, p);
  HIP_CHECK(hipGetLastError());
}
```
